```python
import numpy as np

kernel_name = "gpu_gemm_ar_m4096_k4096_n16384_bf16_q8"


ABSMAX_RTOL = 0.02

COMPUTE_DTYPE = "bfloat16"
EPILOGUE = "quant_int8"
IN_QUANT = None
OUT_QUANT = "int8"
IN_DTYPES = {"x": "bfloat16", "w_mat": "bfloat16"}
WORLD_SIZE = 8

def _to_bf16(x):
    u = np.ascontiguousarray(x, dtype=np.float32).view(np.uint32)
    return ((u + 0x7FFF + ((u >> 16) & 1)) & 0xFFFF0000).view(np.float32)

def setup_inputs():

    x = _to_bf16(np.random.randn(4096, 4096).astype(np.float32))
    w_mat = _to_bf16((np.random.randn(4096, 16384) / np.sqrt(4096)).astype(np.float32))
    return {"x": x, "w_mat": w_mat}

def _quant_dequant(y, amax):
    scale = amax / 127.0
    q = np.clip(np.round(y / scale), -127.0, 127.0)
    return (q * scale).astype(np.float32)
def _epilogue(y):
    return _quant_dequant(y, np.abs(y).max())

def reference(x, w_mat):
    y = (x @ w_mat).astype(np.float32)
    return _epilogue(y)
```

```cpp
#include <hip/hip_runtime.h>
#include <rccl/rccl.h>
#include <cstdio>
#include <stdint.h>

#define HIP_CHECK(expr) do { hipError_t _e = (expr); if (_e != hipSuccess) { \
    fprintf(stderr, "HIP error %s at %s:%d\n", hipGetErrorString(_e), __FILE__, __LINE__); \
    return; } } while (0)

typedef unsigned u32;
typedef unsigned long long u64;
typedef u32 u32x4 __attribute__((ext_vector_type(4)));
typedef u32 u32x2 __attribute__((ext_vector_type(2)));
typedef float f32x4 __attribute__((ext_vector_type(4)));
typedef short s16x4 __attribute__((ext_vector_type(4)));
typedef short s16x8 __attribute__((ext_vector_type(8)));
typedef __bf16 bf16x8 __attribute__((ext_vector_type(8)));
#define LAS __attribute__((address_space(3)))
#define GAS __attribute__((address_space(1)))

constexpr int NR = 8;
constexpr u32 OUT_BYTES = 4096u * 16384u;
constexpr u32 X_BYTES = 4096u * 512u * 2u;
constexpr u32 W_BYTES = 512u * 16384u * 2u;

constexpr u32 WS_X = 0;
constexpr u32 WS_W = 32u << 20;
constexpr u32 WS_CTL = 48u << 20;
constexpr u32 WS_XCNT = WS_CTL;
constexpr u32 WS_WCNT = WS_CTL + 4096;
constexpr u32 WS_CALL = WS_CTL + 8192;
constexpr u32 WS_G1 = WS_CTL + 12288;
constexpr u32 WS_G2 = WS_CTL + 16384;
constexpr u32 WS_TFLAG = WS_CTL + 20480;
constexpr u32 WS_DONE = WS_CTL + 24576;
constexpr u32 WS_TMO = WS_CTL + 28672;
constexpr u32 WS_DBG = WS_CTL + 32768;
constexpr u32 WS_END = WS_CTL + 65536;

constexpr u32 SLOT_BYTES = 49152;
constexpr u32 NSLOT = 3;
constexpr u32 CTL_OFF = SLOT_BYTES * NSLOT;
constexpr u32 LDS_BYTES = CTL_OFF + 512;
constexpr int C_BAR = 0, C_READY = 1, C_AMAX = 2, C_AMAXCNT = 3, C_GMAX = 4, C_GFLAG = 5, C_CALL = 6, C_BAD = 7,
              C_LOC = 8, C_REM = 9, C_ORDER = 16;

constexpr int NSTEP = 64;
constexpr u32 XCONTRIB = 8, WCONTRIB = 4;
constexpr u64 DEADLINE = 50000000ull;

struct Params {
  const char* x;
  const char* w;
  char* out[NR];
  char* ws[NR];
  int rank;
};

__device__ __forceinline__ __amdgpu_buffer_rsrc_t mkrsrc(const void* p, u32 bytes) {
  return __builtin_amdgcn_make_buffer_rsrc((void*)p, 0, bytes, 0x00020000);
}
__device__ __forceinline__ u64 now() { return __builtin_amdgcn_s_memrealtime(); }
__device__ __forceinline__ u32 ld_sys(const u32* p) { return __hip_atomic_load((GAS const u32*)p, __ATOMIC_RELAXED, __HIP_MEMORY_SCOPE_SYSTEM); }
__device__ __forceinline__ u64 ld_sys64(const u64* p) { return __hip_atomic_load((GAS const u64*)p, __ATOMIC_RELAXED, __HIP_MEMORY_SCOPE_SYSTEM); }
__device__ __forceinline__ void st_sys(u32* p, u32 v) { __hip_atomic_store((GAS u32*)p, v, __ATOMIC_RELAXED, __HIP_MEMORY_SCOPE_SYSTEM); }
__device__ __forceinline__ void st_sys64(u64* p, u64 v) { __hip_atomic_store((GAS u64*)p, v, __ATOMIC_RELAXED, __HIP_MEMORY_SCOPE_SYSTEM); }
__device__ __forceinline__ u32 lds_ld(LAS u32* p) { return __builtin_amdgcn_readfirstlane(__hip_atomic_load(p, __ATOMIC_RELAXED, __HIP_MEMORY_SCOPE_WORKGROUP)); }
__device__ __forceinline__ void lds_st(LAS u32* p, u32 v) { __hip_atomic_store(p, v, __ATOMIC_RELAXED, __HIP_MEMORY_SCOPE_WORKGROUP); }
__device__ __forceinline__ void lds_add1(LAS u32* p, u32 lane) {
  if (lane == 0) __hip_atomic_fetch_add(p, 1u, __ATOMIC_RELAXED, __HIP_MEMORY_SCOPE_WORKGROUP);
}
#define CFENCE() asm volatile("" ::: "memory")
#define WAIT_VM(n) asm volatile("s_waitcnt vmcnt(" #n ")" ::: "memory")
#define WAIT_LGKM() asm volatile("s_waitcnt lgkmcnt(0)" ::: "memory")
#define WAIT_VM0() do { CFENCE(); __builtin_amdgcn_s_waitcnt(0x0F70); WAIT_VM(0); } while (0)

__device__ __forceinline__ u32 wave_umax(u32 v) {
  v = max(v, (u32)__builtin_amdgcn_update_dpp(0, (int)v, 0x111, 0xf, 0xf, true));
  v = max(v, (u32)__builtin_amdgcn_update_dpp(0, (int)v, 0x112, 0xf, 0xf, true));
  v = max(v, (u32)__builtin_amdgcn_update_dpp(0, (int)v, 0x114, 0xf, 0xf, true));
  v = max(v, (u32)__builtin_amdgcn_update_dpp(0, (int)v, 0x118, 0xf, 0xf, true));
  v = max(v, (u32)__builtin_amdgcn_update_dpp(0, (int)v, 0x142, 0xa, 0xf, false));
  v = max(v, (u32)__builtin_amdgcn_update_dpp(0, (int)v, 0x143, 0xc, 0xf, false));
  return (u32)__builtin_amdgcn_readlane((int)v, 63);
}

#define STAMP(k) do {} while (0)

__global__ void __launch_bounds__(640) fused_gemm_ar_q8(Params P) {
  __shared__ __attribute__((aligned(16))) char smem[LDS_BYTES];
  LAS char* L = (LAS char*)smem;
  LAS u32* ctl = (LAS u32*)(L + CTL_OFF);
  const u32 tid = threadIdx.x, lane = tid & 63;
  const u32 wave = __builtin_amdgcn_readfirstlane(tid >> 6);
  const u32 g = blockIdx.x & 7u, idx = blockIdx.x >> 3;
  const u32 rank = (u32)P.rank & 7u;
  const u32 dest = (rank + g) & 7u;
  const u32 rb = (g >> 1) * 4 + (idx >> 3), cb = (g & 1) * 8 + (idx & 7);
  char* const myws = P.ws[rank];
  const u64 t0 = now();

  if (tid < 128) ctl[tid] = 0;
  __syncthreads();

  if (wave < 8) {
    const u32 wm = wave >> 1, wn = wave & 1;
    const __amdgpu_buffer_rsrc_t rs_ws = mkrsrc(myws, WS_CTL);
    const u32 lane16 = lane * 16;
    const u32 lds0 = (u32)(size_t)L;
    const u32 xa = wm * 8192 + ((lane >> 3) & 1) * 1024 + (lane >> 5) * 256 + ((lane >> 4) & 1) * 128 + (lane & 7) * 16;
    const u32 h = lane >> 4, q = (lane >> 2) & 3, p = lane & 3;
    const u32 wa0 = 32768 + wn * 8192 + (h >> 1) * 2048 + (p & 1) * 512 + (h & 1) * 128 + q * 32 + (p >> 1) * 16 + (p & 1) * 8;
    const u32 wa1 = wa0 ^ 8;
    f32x4 acc[4][4];
#pragma unroll
    for (int a = 0; a < 4; ++a)
#pragma unroll
      for (int b = 0; b < 4; ++b) acc[a][b] = f32x4{0.f, 0.f, 0.f, 0.f};

    auto issue = [&](u32 step) {
      const u32 ord = lds_ld(ctl + C_ORDER + step);
      const u32 slot = (step % NSLOT) * SLOT_BYTES;
      const u32 sx = WS_X + (ord * 16 + rb) * 32768 + wave * 4096;
      const u32 sw = WS_W + (ord * 16 + cb) * 16384 + wave * 2048;
      const u32 mx = lds0 + slot + wave * 4096, mw = lds0 + slot + 32768 + wave * 2048;
      u32 keep;
      asm volatile(
          "s_nop 4\n\t"
          "s_mov_b32 %0, m0\n\t"
          "s_mov_b32 m0, %3\n\t"
          "s_nop 0\n\t"
          "buffer_load_dwordx4 %1, %2, %5 offen lds\n\t"
          "buffer_load_dwordx4 %1, %2, %5 offen offset:1024 lds\n\t"
          "buffer_load_dwordx4 %1, %2, %5 offen offset:2048 lds\n\t"
          "buffer_load_dwordx4 %1, %2, %5 offen offset:3072 lds\n\t"
          "s_mov_b32 m0, %4\n\t"
          "s_nop 0\n\t"
          "buffer_load_dwordx4 %1, %2, %6 offen lds\n\t"
          "buffer_load_dwordx4 %1, %2, %6 offen offset:1024 lds\n\t"
          "s_mov_b32 m0, %0"
          : "=&s"(keep)
          : "v"(lane16), "s"(rs_ws), "s"(mx), "s"(mw), "s"(sx), "s"(sw)
          : "memory");
    };

    u32 issued = 0, ready = 0;
    for (u32 t = 0; t < NSTEP; ++t) {
      if (issued == t) {
        while ((ready = lds_ld(ctl + C_READY)) <= t) __builtin_amdgcn_s_sleep(1);
        issue(t); issued = t + 1;
      }
      if (issued == t + 1 && ready > t + 1) { issue(t + 1); issued = t + 2; }
      if (issued == t + 1) WAIT_VM(0); else WAIT_VM(6);
      lds_add1(ctl + C_BAR, lane);
      while (lds_ld(ctl + C_BAR) < 8 * (t + 1)) {}
      ready = lds_ld(ctl + C_READY);
      CFENCE();
      if (issued == t + 1 && ready > t + 1) { issue(t + 1); issued = t + 2; }
      if (issued == t + 2 && ready > t + 2) { issue(t + 2); issued = t + 3; }
      LAS char* base = L + (t % NSLOT) * SLOT_BYTES;
      bf16x8 xf[2][4], wf[2][4];
#pragma unroll
      for (int kb = 0; kb < 2; ++kb) {
#pragma unroll
        for (int i = 0; i < 4; ++i) xf[kb][i] = *(LAS bf16x8*)(base + xa + i * 2048 + kb * 512);
#pragma unroll
        for (int n = 0; n < 4; ++n) {
          LAS char* a = base + ((n & 1) ? wa1 : wa0) + kb * 4096 + (n >> 1) * 256;
          s16x4 lo = __builtin_amdgcn_ds_read_tr16_b64_v4i16((LAS s16x4*)a);
          s16x4 hi = __builtin_amdgcn_ds_read_tr16_b64_v4i16((LAS s16x4*)(a + 1024));
          wf[kb][n] = __builtin_bit_cast(bf16x8, __builtin_shufflevector(lo, hi, 0, 1, 2, 3, 4, 5, 6, 7));
        }
      }
#pragma unroll
      for (int kb = 0; kb < 2; ++kb)
#pragma unroll
        for (int n = 0; n < 4; ++n)
#pragma unroll
          for (int i = 0; i < 4; ++i)
            acc[n][i] = __builtin_amdgcn_mfma_f32_16x16x32_bf16(wf[kb][n], xf[kb][i], acc[n][i], 0, 0, 0);
      CFENCE();
    }
    if (wave == 0) STAMP(2);
    float m = 0.f;
#pragma unroll
    for (int a = 0; a < 4; ++a)
#pragma unroll
      for (int b = 0; b < 4; ++b)
#pragma unroll
        for (int j = 0; j < 4; ++j) m = fmaxf(m, fabsf(acc[a][b][j]));
    const u32 mb = wave_umax(__builtin_bit_cast(u32, m));
    if (lane == 0) {
      __hip_atomic_fetch_max(ctl + C_AMAX, mb, __ATOMIC_RELAXED, __HIP_MEMORY_SCOPE_WORKGROUP);
      __hip_atomic_fetch_add(ctl + C_AMAXCNT, 1u, __ATOMIC_RELAXED, __HIP_MEMORY_SCOPE_WORKGROUP);
    }
    while (lds_ld(ctl + C_GFLAG) == 0) __builtin_amdgcn_s_sleep(1);
    CFENCE();
    const float amax = __builtin_bit_cast(float, lds_ld(ctl + C_GMAX));
    const bool bad = lds_ld(ctl + C_BAD) != 0;
    const float scale = amax / 127.0f;
    const float inv = amax > 0.f ? 1.0f / scale : 0.f;
    if (wave == 0) STAMP(4);
    const bool odd = (lane >> 4) & 1;
    u32x4 qv[4];
#pragma unroll
    for (int i = 0; i < 4; ++i) {
      u32 d[4];
#pragma unroll
      for (int n = 0; n < 4; ++n) {
        u32 b0 = __builtin_bit_cast(u32, __builtin_fmaf(acc[n][i][0], inv, 12582912.0f));
        u32 b1 = __builtin_bit_cast(u32, __builtin_fmaf(acc[n][i][1], inv, 12582912.0f));
        u32 b2 = __builtin_bit_cast(u32, __builtin_fmaf(acc[n][i][2], inv, 12582912.0f));
        u32 b3 = __builtin_bit_cast(u32, __builtin_fmaf(acc[n][i][3], inv, 12582912.0f));
        u32 lo = __builtin_amdgcn_perm(b1, b0, 0x0c0c0400u);
        u32 hi = __builtin_amdgcn_perm(b3, b2, 0x04000c0cu);
        d[n] = bad ? 0x80808080u : (lo | hi);
      }
      qv[i] = u32x4{odd ? d[1] : d[0], odd ? d[0] : d[1], odd ? d[3] : d[2], odd ? d[2] : d[3]};
    }
    const u32 ooff = (rb * 256 + wm * 64 + (lane & 15)) * 16384u + rank * 2048 + cb * 128 + wn * 64 + (lane >> 4) * 16;
    const __amdgpu_buffer_rsrc_t rs_o = mkrsrc(P.out[rank], OUT_BYTES);
#pragma unroll
    for (int i = 0; i < 4; ++i) __builtin_amdgcn_raw_buffer_store_b128(qv[i], rs_o, ooff, i * 16 * 16384, 17);
    if (g != 0) {
      const __amdgpu_buffer_rsrc_t rs_d = mkrsrc(P.out[dest], OUT_BYTES);
#pragma unroll
      for (int i = 0; i < 4; ++i) __builtin_amdgcn_raw_buffer_store_b128(qv[i], rs_d, ooff, i * 16 * 16384, 17);
      WAIT_VM(4);
    } else {
      WAIT_VM(0);
    }
    lds_add1(ctl + C_LOC, lane);
    WAIT_VM(0);
    lds_add1(ctl + C_REM, lane);
  } else if (wave == 8) {
    STAMP(0);
    const __amdgpu_buffer_rsrc_t rs_x = mkrsrc(P.x, X_BYTES);
    const __amdgpu_buffer_rsrc_t rs_w = mkrsrc(P.w, W_BYTES);
    char* const dws = P.ws[dest];
    const __amdgpu_buffer_rsrc_t rs_d = mkrsrc(dws, WS_CTL);
    const u32 lx = (lane & 7) * 1024 + (lane >> 3) * 16;
    const u32 lw = (((lane >> 3) & 1) * 8 + ((lane >> 1) & 3)) * 32768 + ((lane & 1) * 4 + (lane >> 4)) * 16;
    const u32 lane16 = lane * 16;
    u32x4 v[4];
    auto load_unit = [&](u32 kc, u32 r) {
      if (r < 2) {
        const u32 wu = r * 32 + idx;
        const u32 so = (kc * 64 + (wu & 1) * 32) * 32768 + dest * 4096 + (wu >> 2) * 256 + ((wu >> 1) & 1) * 128;
        v[0] = __builtin_amdgcn_raw_buffer_load_b128(rs_w, lw, so, 0);
        v[1] = __builtin_amdgcn_raw_buffer_load_b128(rs_w, lw, so + 4 * 32768, 0);
        v[2] = __builtin_amdgcn_raw_buffer_load_b128(rs_w, lw, so + 16 * 32768, 0);
        v[3] = __builtin_amdgcn_raw_buffer_load_b128(rs_w, lw, so + 20 * 32768, 0);
      } else {
        const u32 xu = (r - 2) * 32 + idx;
        const u32 b = ((xu >> 3) + 2 * rank) & 15;
        const u32 so = (b * 256 + (xu & 7) * 32) * 1024 + kc * 128;
        v[0] = __builtin_amdgcn_raw_buffer_load_b128(rs_x, lx, so, 0);
        v[1] = __builtin_amdgcn_raw_buffer_load_b128(rs_x, lx, so + 8192, 0);
        v[2] = __builtin_amdgcn_raw_buffer_load_b128(rs_x, lx, so + 16384, 0);
        v[3] = __builtin_amdgcn_raw_buffer_load_b128(rs_x, lx, so + 24576, 0);
      }
    };
    load_unit(0, 0);
    for (u32 kc = 0; kc < 8; ++kc) {
#pragma unroll
      for (u32 r = 0; r < 6; ++r) {
        u32 doff, coff;
        if (r < 2) {
          const u32 wu = r * 32 + idx;
          doff = WS_W + ((rank * 8 + kc) * 16 + (wu >> 2)) * 16384 + (wu & 3) * 4096;
          coff = WS_WCNT + (wu >> 2) * 256 + (rank * 8 + kc) * 4;
        } else {
          const u32 xu = (r - 2) * 32 + idx;
          const u32 b = ((xu >> 3) + 2 * rank) & 15;
          doff = WS_X + ((rank * 8 + kc) * 16 + b) * 32768 + (xu & 7) * 4096;
          coff = WS_XCNT + b * 256 + (rank * 8 + kc) * 4;
        }
#pragma unroll
        for (int e = 0; e < 4; ++e) __builtin_amdgcn_raw_buffer_store_b128(v[e], rs_d, lane16, doff + e * 1024, 17);
        CFENCE();
        if (r < 5) load_unit(kc, r + 1);
        else if (kc < 7) load_unit(kc + 1, 0);
        WAIT_VM0();
        if (lane == 0) __hip_atomic_fetch_add((GAS u32*)(dws + coff), 1u, __ATOMIC_RELAXED, __HIP_MEMORY_SCOPE_SYSTEM);
        CFENCE();
      }
    }
    STAMP(1);
    u32 call;
    while ((call = lds_ld(ctl + C_CALL)) == 0) __builtin_amdgcn_s_sleep(1);
    u32* const tmo = (u32*)(myws + WS_TMO);
    bool timed_out = false;
    if (g != 0) {
      const __amdgpu_buffer_rsrc_t rs_o = mkrsrc(P.out[rank], OUT_BYTES);
      const __amdgpu_buffer_rsrc_t rs_p = mkrsrc(P.out[dest], OUT_BYTES);
      const u32 lo = (lane >> 3) * 16384 + (lane & 7) * 16;
      const u32* tf = (const u32*)(myws + WS_TFLAG + idx * 128);
      while (lds_ld(ctl + C_GFLAG) == 0) __builtin_amdgcn_s_sleep(2);
      for (u32 j = 1; j < 8; ++j) {
        const u32 g2 = (g + j) & 7;
        u32 sp = 0;
        while ((int)(__builtin_amdgcn_readfirstlane(ld_sys(tf + g2)) - call) < 0) {
          __builtin_amdgcn_s_sleep(2);
          if ((++sp & 63) == 0 && (ld_sys(tmo) == call || now() - t0 > DEADLINE)) { st_sys(tmo, call); timed_out = true; break; }
        }
        CFENCE();
        const u32 rb2 = (g2 >> 1) * 4 + (idx >> 3), cb2 = (g2 & 1) * 8 + (idx & 7);
        const u32 tb = rb2 * 256 * 16384u + rank * 2048 + cb2 * 128;
#pragma unroll
        for (int e = 0; e < 4; ++e) v[e] = __builtin_amdgcn_raw_buffer_load_b128(rs_o, lo, tb + e * 8 * 16384, 17);
        for (u32 c = 0; c < 8; ++c) {
          u32x4 s[4];
#pragma unroll
          for (int e = 0; e < 4; ++e) s[e] = timed_out ? u32x4{0x80808080u, 0x80808080u, 0x80808080u, 0x80808080u} : v[e];
#pragma unroll
          for (int e = 0; e < 4; ++e) __builtin_amdgcn_raw_buffer_store_b128(s[e], rs_p, lo, tb + (c * 4 + e) * 8 * 16384, 17);
          CFENCE();
          if (c < 7) {
#pragma unroll
            for (int e = 0; e < 4; ++e) v[e] = __builtin_amdgcn_raw_buffer_load_b128(rs_o, lo, tb + (c * 4 + 4 + e) * 8 * 16384, 17);
          }
        }
      }
      WAIT_VM(0);
      while (lds_ld(ctl + C_REM) < 8) __builtin_amdgcn_s_sleep(1);
      CFENCE();
      if (lane == 0) st_sys((u32*)(dws + WS_DONE + idx * 128) + rank, call);
      STAMP(5);
    } else {
      const u32* df = (const u32*)(myws + WS_DONE + idx * 128);
      u32 sp = 0;
      while (lds_ld(ctl + C_GFLAG) == 0) __builtin_amdgcn_s_sleep(8);
      for (;;) {
        const u32 val = ld_sys(df + (lane & 7));
        const bool ok = ((lane & 7) == rank) || (int)(val - call) >= 0;
        if (__all(ok)) break;
        __builtin_amdgcn_s_sleep(4);
        if ((++sp & 63) == 0 && (ld_sys(tmo) == call || now() - t0 > DEADLINE)) { st_sys(tmo, call); timed_out = true; break; }
      }
      if (timed_out) {
        while (lds_ld(ctl + C_REM) < 8) __builtin_amdgcn_s_sleep(1);
        if (lane == 0) P.out[rank][(size_t)rb * 256 * 16384 + rank * 2048 + cb * 128] = (char)-128;
      }
      STAMP(5);
    }
  } else {
    __builtin_amdgcn_fence(__ATOMIC_ACQUIRE, "");
    u32* const cptr = (u32*)(myws + WS_CALL) + blockIdx.x;
    const u32 call = __builtin_amdgcn_readfirstlane(ld_sys(cptr)) + 1;
    if (lane == 0) { st_sys(cptr, call); lds_st(ctl + C_CALL, call); }
    WAIT_VM(0);
    u32* const tmo = (u32*)(myws + WS_TMO);
    const u32* xc = (const u32*)(myws + WS_XCNT + rb * 256) + lane;
    const u32* wc = (const u32*)(myws + WS_WCNT + cb * 256) + lane;
    const u32 tx = call * XCONTRIB, tw = call * WCONTRIB;
    u64 pending = ~0ull;
    u32 cnt = 0, sp = 0;
    bool bad = false;
    while (pending) {
      const u32 xv = ld_sys(xc), wv = ld_sys(wc);
      u64 rdy = __ballot((int)(xv - tx) >= 0 && (int)(wv - tw) >= 0) & pending;
      if (!rdy) {
        __builtin_amdgcn_s_sleep(4);
        if ((++sp & 63) == 0 && (ld_sys(tmo) == call || now() - t0 > DEADLINE)) { st_sys(tmo, call); bad = true; rdy = pending; }
        else continue;
      }
      const u32 pos = cnt + __builtin_amdgcn_mbcnt_hi((u32)(rdy >> 32), __builtin_amdgcn_mbcnt_lo((u32)rdy, 0));
      if ((rdy >> lane) & 1) lds_st(ctl + C_ORDER + pos, lane);
      cnt += __builtin_popcountll(rdy);
      pending &= ~rdy;
      WAIT_LGKM();
      if (lane == 0) lds_st(ctl + C_READY, cnt);
    }
    STAMP(6);
    while (lds_ld(ctl + C_AMAXCNT) < 8) {}
    CFENCE();
    const u32 wmax = lds_ld(ctl + C_AMAX);
    if (lane == 0) st_sys64((u64*)(myws + WS_G1) + g * 32 + idx, ((u64)call << 32) | wmax);
    STAMP(3);
    if (idx == 0) {
      const u64* g1 = (const u64*)(myws + WS_G1) + g * 32 + (lane & 31);
      u64 val;
      for (;;) {
        val = ld_sys64(g1);
        if (__all((u32)(val >> 32) == call)) break;
        if ((++sp & 63) == 0 && (ld_sys(tmo) == call || now() - t0 > DEADLINE)) { st_sys(tmo, call); bad = true; break; }
      }
      const u32 gm = wave_umax((u32)val);
      if (lane < 8) st_sys64((u64*)(P.ws[lane] + WS_G2) + rank * 8 + g, ((u64)call << 32) | gm);
    }
    u32 gmax;
    {
      const u64* g2 = (const u64*)(myws + WS_G2) + lane;
      u64 val;
      for (;;) {
        val = ld_sys64(g2);
        if (__all((u32)(val >> 32) == call)) break;
        if ((++sp & 63) == 0 && (ld_sys(tmo) == call || now() - t0 > DEADLINE)) { st_sys(tmo, call); bad = true; break; }
      }
      gmax = wave_umax((u32)val);
    }
    if (__builtin_amdgcn_readfirstlane(ld_sys(tmo)) == call) bad = true;
    if (lane == 0) {
      lds_st(ctl + C_GMAX, gmax);
      lds_st(ctl + C_BAD, bad ? 1u : 0u);
      WAIT_LGKM();
      lds_st(ctl + C_GFLAG, 1u);
    }
    while (lds_ld(ctl + C_LOC) < 8) {}
    CFENCE();
    if (lane == 0) st_sys((u32*)(myws + WS_TFLAG + idx * 128) + g, call);
  }
}

extern "C" void kernel_launch(int rank, int nranks, ncclComm_t comm, hipStream_t stream,
                              void* const* d_in, const int* in_sizes, int n_in,
                              void* d_out, int out_size,
                              const int* in_dtypes, int out_dtype,
                              void* const* d_out_all,
                              void* const* d_ws_all, size_t ws_bytes) {
  if (nranks != NR || ws_bytes < WS_END || n_in < 2) { fprintf(stderr, "unsupported configuration\n"); return; }
  Params P;
  P.x = (const char*)d_in[0];
  P.w = (const char*)d_in[1];
  for (int p = 0; p < NR; ++p) { P.out[p] = (char*)d_out_all[p]; P.ws[p] = (char*)d_ws_all[p]; }
  P.rank = rank;
  hipLaunchKernelGGL(fused_gemm_ar_q8, dim3(256), dim3(640), 0, stream, P);
  HIP_CHECK(hipGetLastError());
}
```

```cpp
#include <hip/hip_runtime.h>
#include <rccl/rccl.h>
#include <cstdio>
#include <stdint.h>

#define HIP_CHECK(expr) do { hipError_t _e = (expr); if (_e != hipSuccess) { \
    fprintf(stderr, "HIP error %s at %s:%d\n", hipGetErrorString(_e), __FILE__, __LINE__); \
    return; } } while (0)

typedef unsigned u32;
typedef unsigned long long u64;
typedef u32 u32x4 __attribute__((ext_vector_type(4)));
typedef u32 u32x2 __attribute__((ext_vector_type(2)));
typedef float f32x4 __attribute__((ext_vector_type(4)));
typedef short s16x4 __attribute__((ext_vector_type(4)));
typedef short s16x8 __attribute__((ext_vector_type(8)));
typedef __bf16 bf16x8 __attribute__((ext_vector_type(8)));
#define LAS __attribute__((address_space(3)))
#define GAS __attribute__((address_space(1)))

constexpr int NR = 8;
constexpr u32 OUT_BYTES = 4096u * 16384u;
constexpr u32 X_BYTES = 4096u * 512u * 2u;
constexpr u32 W_BYTES = 512u * 16384u * 2u;

constexpr u32 NRB = 26;
constexpr u32 NCB = 8;
constexpr u32 XBLK = 20480, WBLK = 32768;
constexpr u32 WS_W = 0;
constexpr u32 WS_X = 16u << 20;
constexpr u32 WS_CTL = 49u << 20;
constexpr u32 FSTRIDE = 4352;
constexpr u32 WS_XF = WS_CTL;
constexpr u32 WS_WF = WS_XF + NRB * FSTRIDE;
constexpr u32 WS_C2 = WS_CTL + (160u << 10);
constexpr u32 WS_CALL = WS_C2;
constexpr u32 WS_G1 = WS_C2 + 4096;
constexpr u32 WS_G2 = WS_C2 + 8192 + 256;
constexpr u32 WS_TCNT = WS_C2 + 12288 + 512;
constexpr u32 WS_TK3 = WS_C2 + 13312;
constexpr u32 WS_DONE = WS_C2 + 16384;
constexpr u32 WS_TMO = WS_C2 + 20480;
constexpr u32 WS_DBG = WS_C2 + 32768;
constexpr u32 WS_END = WS_C2 + 65536;
static_assert(WS_X + 64u * NRB * XBLK <= WS_CTL && WS_WF + NCB * FSTRIDE <= WS_C2, "layout");

constexpr u32 SLOT_BYTES = WBLK + XBLK;
constexpr u32 NSLOT = 3;
constexpr u32 TPITCH = 272;
constexpr u32 SNAP_OFF = SLOT_BYTES * NSLOT;
constexpr u32 CTL_OFF = SNAP_OFF + 3072;
constexpr u32 LDS_BYTES = CTL_OFF + 1024;
static_assert(LDS_BYTES <= 163840, "lds");
constexpr int C_AMAX = 0, C_GMAX = 1, C_CALL = 2, C_BAD = 3, C_LOC = 4, C_REM = 5, C_BLK = 16;

#define NCW 6
#define NDIR 5
#define P3_PULL 0
#define P3_TICKET 1
#define P3_LAUX 17
#define P3_SAUX 17
constexpr u32 NCOMMWG = 32 - NRB;
constexpr u32 NTILE = NRB * NCB;
constexpr u32 NBLK = 8 * (NCB + NRB);
constexpr int NTHREADS = 512;
constexpr int NSTEP = 64;
constexpr u64 DEADLINE = 50000000ull;

struct Params {
  const char* x;
  const char* w;
  char* out[NR];
  char* ws[NR];
  int rank;
};

__device__ __forceinline__ __amdgpu_buffer_rsrc_t mkrsrc(const void* p, u32 bytes) {
  return __builtin_amdgcn_make_buffer_rsrc((void*)p, 0, bytes, 0x00020000);
}
__device__ __forceinline__ u64 now() { return __builtin_amdgcn_s_memrealtime(); }
__device__ __forceinline__ u32 ld_sys(const u32* p) { return __hip_atomic_load((GAS const u32*)p, __ATOMIC_RELAXED, __HIP_MEMORY_SCOPE_SYSTEM); }
__device__ __forceinline__ u64 ld_sys64(const u64* p) { return __hip_atomic_load((GAS const u64*)p, __ATOMIC_RELAXED, __HIP_MEMORY_SCOPE_SYSTEM); }
__device__ __forceinline__ void st_sys(u32* p, u32 v) { __hip_atomic_store((GAS u32*)p, v, __ATOMIC_RELAXED, __HIP_MEMORY_SCOPE_SYSTEM); }
__device__ __forceinline__ void st_sys64(u64* p, u64 v) { __hip_atomic_store((GAS u64*)p, v, __ATOMIC_RELAXED, __HIP_MEMORY_SCOPE_SYSTEM); }
__device__ __forceinline__ u32 lds_ld(LAS u32* p) { return __builtin_amdgcn_readfirstlane(__hip_atomic_load(p, __ATOMIC_RELAXED, __HIP_MEMORY_SCOPE_WORKGROUP)); }
__device__ __forceinline__ void lds_st(LAS u32* p, u32 v) { __hip_atomic_store(p, v, __ATOMIC_RELAXED, __HIP_MEMORY_SCOPE_WORKGROUP); }
__device__ __forceinline__ void lds_add1(LAS u32* p, u32 lane) {
  if (lane == 0) __hip_atomic_fetch_add(p, 1u, __ATOMIC_RELAXED, __HIP_MEMORY_SCOPE_WORKGROUP);
}
#define CFENCE() asm volatile("" ::: "memory")
#define WAIT_VM(n) asm volatile("s_waitcnt vmcnt(" #n ")" ::: "memory")
#define WAIT_VMI(n) asm volatile("s_waitcnt vmcnt(%0)" :: "i"(n) : "memory")
#define WAIT_LGKM() asm volatile("s_waitcnt lgkmcnt(0)" ::: "memory")
#define WAIT_VM0() do { CFENCE(); __builtin_amdgcn_s_waitcnt(0x0F70); WAIT_VM(0); } while (0)

__device__ __forceinline__ u32 wave_umax(u32 v) {
  v = max(v, (u32)__builtin_amdgcn_update_dpp(0, (int)v, 0x111, 0xf, 0xf, true));
  v = max(v, (u32)__builtin_amdgcn_update_dpp(0, (int)v, 0x112, 0xf, 0xf, true));
  v = max(v, (u32)__builtin_amdgcn_update_dpp(0, (int)v, 0x114, 0xf, 0xf, true));
  v = max(v, (u32)__builtin_amdgcn_update_dpp(0, (int)v, 0x118, 0xf, 0xf, true));
  v = max(v, (u32)__builtin_amdgcn_update_dpp(0, (int)v, 0x142, 0xa, 0xf, false));
  v = max(v, (u32)__builtin_amdgcn_update_dpp(0, (int)v, 0x143, 0xc, 0xf, false));
  return (u32)__builtin_amdgcn_readlane((int)v, 63);
}

#define STAMP(k) do {} while (0)

__device__ __forceinline__ void wait_vm_dyn(u32 n) {
  asm volatile(
      "s_getpc_b64 vcc\n\t"
      "s_add_u32 vcc_lo, vcc_lo, %0\n\t"
      "s_addc_u32 vcc_hi, vcc_hi, 0\n\t"
      "s_setpc_b64 vcc\n\t"
      "s_waitcnt vmcnt(0)\n\ts_branch 1f\n\t"
      "s_waitcnt vmcnt(1)\n\ts_branch 1f\n\t"
      "s_waitcnt vmcnt(2)\n\ts_branch 1f\n\t"
      "s_waitcnt vmcnt(3)\n\ts_branch 1f\n\t"
      "s_waitcnt vmcnt(4)\n\ts_branch 1f\n\t"
      "s_waitcnt vmcnt(5)\n\ts_branch 1f\n\t"
      "s_waitcnt vmcnt(6)\n\ts_branch 1f\n\t"
      "s_waitcnt vmcnt(7)\n\ts_branch 1f\n\t"
      "s_waitcnt vmcnt(8)\n\ts_branch 1f\n\t"
      "s_waitcnt vmcnt(9)\n\ts_branch 1f\n\t"
      "s_waitcnt vmcnt(10)\n\ts_branch 1f\n\t"
      "s_waitcnt vmcnt(11)\n\ts_branch 1f\n\t"
      "s_waitcnt vmcnt(12)\n\ts_branch 1f\n\t"
      "s_waitcnt vmcnt(13)\n\ts_branch 1f\n\t"
      "s_waitcnt vmcnt(14)\n\ts_branch 1f\n\t"
      "s_waitcnt vmcnt(15)\n\ts_branch 1f\n\t"
      "s_waitcnt vmcnt(16)\n\ts_branch 1f\n\t"
      "1:"
      :: "s"(min(n, 17u) * 8 + 12) : "memory", "scc", "vcc");
}
#define BARRIER() asm volatile("s_barrier" ::: "memory")

__global__ void __launch_bounds__(NTHREADS) fused_gemm_ar_q8(Params P) {
  __shared__ __attribute__((aligned(16))) char smem[LDS_BYTES];
  LAS char* L = (LAS char*)smem;
  LAS u32* ctl = (LAS u32*)(L + CTL_OFF);
  const u32 tid = threadIdx.x, lane = tid & 63;
  const u32 wave = __builtin_amdgcn_readfirstlane(tid >> 6);
  const u32 g = blockIdx.x & 7u, idx = blockIdx.x >> 3;
  const u32 rank = (u32)P.rank & 7u;
  const u32 dest = (rank + g) & 7u;
  char* const myws = P.ws[rank];
  char* const dws = P.ws[dest];
  u32* const tmo = (u32*)(myws + WS_TMO);
  const u32 lane16 = lane * 16;
  const u64 t0 = now();
  u32 call = 0, nrem = 0;
  bool bad = false;
  if (wave == 0) STAMP(0);

  if (idx >= NCOMMWG) {
    const u32 ti = idx - NCOMMWG;
    const u32 rb = (g & 1) * 13 + (ti >> 1), cb = (g >> 1) * 2 + (ti & 1);
    if (wave == 0) {
      __builtin_amdgcn_fence(__ATOMIC_ACQUIRE, "");
      u32* const cptr = (u32*)(myws + WS_CALL) + blockIdx.x;
      call = __builtin_amdgcn_readfirstlane(ld_sys(cptr)) + 1;
      if (lane < 16) ctl[lane] = lane == C_CALL ? call : 0u;
      if (lane == 0) st_sys(cptr, call);
      WAIT_VM0();
      WAIT_LGKM();
    }
    BARRIER();
    call = lds_ld(ctl + C_CALL);
    const u32 wm = wave >> 2, wn = wave & 3;
    const __amdgpu_buffer_rsrc_t rs_ws = mkrsrc(myws, WS_C2);
    const u32 lds0 = (u32)(size_t)L;
    const u32 xa = WBLK + wm * 10240 + ((lane >> 3) & 1) * 1024 + (lane >> 5) * 256 + ((lane >> 4) & 1) * 128 + (lane & 7) * 16;
    const u32 h = lane >> 4, q = (lane >> 2) & 3, p = lane & 3;
    const u32 wa0 = wn * 8192 + (h >> 1) * 2048 + (p & 1) * 512 + (h & 1) * 128 + q * 32 + (p >> 1) * 16 + (p & 1) * 8;
    const u32 wa1 = wa0 ^ 8;
    const u32 xpo = wave < 4 ? wave * 3072 : 12288 + (wave - 4) * 2048;
    const u32 pollo = lane < 16 ? WS_XF + rb * FSTRIDE + lane16 : WS_WF + cb * FSTRIDE + lane16 - 256;
    f32x4 acc[4][5];
#pragma unroll
    for (int a = 0; a < 4; ++a)
#pragma unroll
      for (int b = 0; b < 5; ++b) acc[a][b] = f32x4{0.f, 0.f, 0.f, 0.f};

    u64 kx = 0, kw = 0, taken = 0;
    u32 issued = 0, consumed = 0, nis = 0;
    u32 islot = 0, cslot = 0, sj = 0;
    u64 markD = 0, markP = 0;
    auto sched = [&](u32 fx, u32 fw) {
      kx |= __ballot((int)(fx - call) >= 0);
      kw |= __ballot((int)(fw - call) >= 0);
      u64 avail = kx & kw & ~taken;
      while (avail && issued - consumed < NSLOT) {
        const u32 ord = __builtin_ctzll(avail);
        avail &= avail - 1;
        taken |= 1ull << ord;
        const u32 sw = WS_W + (ord * NCB + cb) * WBLK + wave * 4096;
        const u32 sx = WS_X + (ord * NRB + rb) * XBLK + xpo;
        const u32 mw = lds0 + islot + wave * 4096, mx = lds0 + islot + WBLK + xpo;
        u32 keep;
        asm volatile(
            "s_mov_b32 %0, m0\n\t"
            "s_mov_b32 m0, %3\n\t"
            "s_nop 0\n\t"
            "buffer_load_dwordx4 %1, %2, %5 offen lds\n\t"
            "buffer_load_dwordx4 %1, %2, %5 offen offset:1024 lds\n\t"
            "buffer_load_dwordx4 %1, %2, %5 offen offset:2048 lds\n\t"
            "buffer_load_dwordx4 %1, %2, %5 offen offset:3072 lds\n\t"
            "s_mov_b32 m0, %4\n\t"
            "s_nop 0\n\t"
            "buffer_load_dwordx4 %1, %2, %6 offen lds\n\t"
            "buffer_load_dwordx4 %1, %2, %6 offen offset:1024 lds\n\t"
            "s_cmp_gt_u32 %7, 3\n\t"
            "s_cbranch_scc1 1f\n\t"
            "buffer_load_dwordx4 %1, %2, %6 offen offset:2048 lds\n\t"
            "1:\n\t"
            "s_mov_b32 m0, %0"
            : "=&s"(keep)
            : "v"(lane16), "s"(rs_ws), "s"(mw), "s"(mx), "s"(sw), "s"(sx), "s"(wave)
            : "memory", "scc");
        nis += wave < 4 ? 7 : 6;
        const u32 sh = (issued & 3) * 16;
        markD = (markD & ~(0xffffull << sh)) | ((u64)(nis & 0xffff) << sh);
        issued += 1;
        islot = islot == 2 * SLOT_BYTES ? 0 : islot + SLOT_BYTES;
      }
    };
    auto poll = [&]() {
      if (wave == 0) {
        const u64 kn = lane < 16 ? kx : kw;
        const bool act = lane < 32 && ((u32)(kn >> ((lane & 15) * 4)) & 15u) != 15u;
        const u32 po = act ? pollo : 0x80000000u;
        u32 keep;
        asm volatile("s_mov_b32 %0, m0\n\ts_mov_b32 m0, %3\n\ts_nop 0\n\t"
                     "buffer_load_dwordx4 %1, %2, 0 offen sc0 sc1 lds\n\ts_mov_b32 m0, %0"
                     : "=&s"(keep) : "v"(po), "s"(rs_ws), "s"(lds0 + SNAP_OFF + sj * 1024) : "memory");
        nis += 1;
        markP = (markP & ~(0xffffull << (sj * 16))) | ((u64)(nis & 0xffff) << (sj * 16));
      }
      sj = sj == 2 ? 0 : sj + 1;
    };
    poll(); poll();
    u32 sp = 0;
    for (; consumed < NSTEP; ++consumed) {
      while (issued == consumed) {
        const u32 srd = sj == 2 ? 0 : sj + 1;
        if (wave == 0) {
          wait_vm_dyn((nis - (u32)(markP >> (srd * 16))) & 0xffff);
          if ((++sp & 63) == 0 && now() - t0 > DEADLINE) {
            st_sys(tmo, call);
            WAIT_VM(0);
            lds_st(ctl + C_BAD, 1u);
            *(LAS u32x2*)(L + SNAP_OFF + srd * 1024 + lane * 8) = u32x2{call, call};
            WAIT_LGKM();
          }
        }
        BARRIER();
        poll();
        LAS u32* const snap = (LAS u32*)(L + SNAP_OFF + srd * 1024) + lane;
        sched(snap[0], snap[64]);
      }
      const u32 srd = sj == 2 ? 0 : sj + 1;
      u32 need = (nis - (u32)(markD >> ((consumed & 3) * 16))) & 0xffff;
      if (wave == 0) need = min(need, (nis - (u32)(markP >> (srd * 16))) & 0xffff);
      wait_vm_dyn(need);
      BARRIER();
      poll();
      LAS u32* const snap = (LAS u32*)(L + SNAP_OFF + srd * 1024) + lane;
      const u32 fx = snap[0], fw = snap[64];
      LAS char* base = L + cslot;
      bf16x8 xf[2][5], wf[2][4];
#pragma unroll
      for (int kb = 0; kb < 2; ++kb) {
#pragma unroll
        for (int n = 0; n < 5; ++n) {
          if (n < 4) {
            LAS char* a = base + ((n & 1) ? wa1 : wa0) + kb * 4096 + (n >> 1) * 256;
            s16x4 lo = __builtin_amdgcn_ds_read_tr16_b64_v4i16((LAS s16x4*)a);
            s16x4 hi = __builtin_amdgcn_ds_read_tr16_b64_v4i16((LAS s16x4*)(a + 1024));
            wf[kb][n] = __builtin_bit_cast(bf16x8, __builtin_shufflevector(lo, hi, 0, 1, 2, 3, 4, 5, 6, 7));
          }
          xf[kb][n] = *(LAS bf16x8*)(base + xa + n * 2048 + kb * 512);
        }
        __builtin_amdgcn_sched_barrier(0);
      }
      if (wave < 4) sched(fx, fw);
#pragma unroll
      for (int kb = 0; kb < 2; ++kb)
#pragma unroll
        for (int i = 0; i < 5; ++i)
#pragma unroll
          for (int n = 0; n < 4; ++n)
            acc[n][i] = __builtin_amdgcn_mfma_f32_16x16x32_bf16(wf[kb][n], xf[kb][i], acc[n][i], 0, 0, 0);
      if (wave >= 4) sched(fx, fw);
      cslot = cslot == 2 * SLOT_BYTES ? 0 : cslot + SLOT_BYTES;
      CFENCE();
    }
    if (wave == 0) STAMP(2);
    const u32 row0 = rb * 160 + wm * 80;
    float m = 0.f;
#pragma unroll
    for (int b = 0; b < 5; ++b) {
      float mb = 0.f;
#pragma unroll
      for (int a = 0; a < 4; ++a)
#pragma unroll
        for (int e = 0; e < 4; ++e) mb = fmaxf(mb, fabsf(acc[a][b][e]));
      if (row0 + b * 16 < 4096) m = fmaxf(m, mb);
    }
    const u32 mb = wave_umax(__builtin_bit_cast(u32, m));
    if (lane == 0) __hip_atomic_fetch_max(ctl + C_AMAX, mb, __ATOMIC_RELAXED, __HIP_MEMORY_SCOPE_WORKGROUP);
    WAIT_LGKM();
    BARRIER();
    if (wave == 0) {
      const u32 wmax = lds_ld(ctl + C_AMAX);
      if (lane == 0) st_sys64((u64*)(myws + WS_G1) + g * NRB + ti, ((u64)call << 32) | wmax);
      STAMP(3);
      const u64* g2 = (const u64*)(myws + WS_G2) + (lane & 7);
      u64 val;
      for (;;) {
        val = ld_sys64(g2);
        if (__all((u32)(val >> 32) == call)) break;
        if ((++sp & 63) == 0 && (ld_sys(tmo) == call || now() - t0 > DEADLINE)) { st_sys(tmo, call); bad = true; break; }
      }
      const u32 gmax = wave_umax((u32)val);
      if (lane == 0) {
        lds_st(ctl + C_GMAX, gmax);
        if (bad) lds_st(ctl + C_BAD, 1u);
      }
      WAIT_LGKM();
    }
    BARRIER();
    const float amax = __builtin_bit_cast(float, lds_ld(ctl + C_GMAX));
    bad = lds_ld(ctl + C_BAD) != 0;
    const float scale = amax / 127.0f;
    const float inv = amax > 0.f ? 1.0f / scale : 0.f;
    if (wave == 0) STAMP(4);
    const bool odd = (lane >> 4) & 1;
    u32x4 qv[5];
#pragma unroll
    for (int i = 0; i < 5; ++i) {
      u32 d[4];
#pragma unroll
      for (int n = 0; n < 4; ++n) {
        u32 b0 = __builtin_bit_cast(u32, __builtin_fmaf(acc[n][i][0], inv, 12582912.0f));
        u32 b1 = __builtin_bit_cast(u32, __builtin_fmaf(acc[n][i][1], inv, 12582912.0f));
        u32 b2 = __builtin_bit_cast(u32, __builtin_fmaf(acc[n][i][2], inv, 12582912.0f));
        u32 b3 = __builtin_bit_cast(u32, __builtin_fmaf(acc[n][i][3], inv, 12582912.0f));
        u32 lo = __builtin_amdgcn_perm(b1, b0, 0x0c0c0400u);
        u32 hi = __builtin_amdgcn_perm(b3, b2, 0x04000c0cu);
        d[n] = bad ? 0x80808080u : (lo | hi);
      }
      qv[i] = u32x4{odd ? d[1] : d[0], odd ? d[0] : d[1], odd ? d[3] : d[2], odd ? d[2] : d[3]};
    }
    {
      LAS char* tw = L + (wm * 80 + (lane & 15)) * TPITCH + wn * 64 + (lane >> 4) * 16;
#pragma unroll
      for (int i = 0; i < 5; ++i) *(LAS u32x4*)(tw + i * 16 * TPITCH) = qv[i];
    }
    WAIT_LGKM();
    BARRIER();
    LAS char* tr = L + (lane >> 4) * TPITCH + (lane & 15) * 16;
#pragma unroll
    for (int i = 0; i < 5; ++i) qv[i] = *(LAS u32x4*)(tr + (wave * 5 + i) * 4 * TPITCH);
    const u32 ooff = (rb * 160 + (lane >> 4)) * 16384u + rank * 2048 + cb * 256 + (lane & 15) * 16;
    const __amdgpu_buffer_rsrc_t rs_o = mkrsrc(P.out[rank], OUT_BYTES);
    const __amdgpu_buffer_rsrc_t rs_d = mkrsrc(P.out[dest], OUT_BYTES);
    if (g != 0) __builtin_amdgcn_raw_buffer_store_b128(qv[0], rs_d, ooff + wave * 5 * 65536, 0, 17);
#pragma unroll
    for (int i = 0; i < 5; ++i) __builtin_amdgcn_raw_buffer_store_b128(qv[i], rs_o, ooff + (wave * 5 + i) * 65536, 0, 17);
    if (g != 0 && wave == 0) {
#pragma unroll
      for (int i = 1; i < 5; ++i) __builtin_amdgcn_raw_buffer_store_b128(qv[i], rs_d, ooff + i * 65536, 0, 17);
      WAIT_VM(4);
    } else {
      WAIT_VM(0);
    }
    u32 old = 0;
    if (lane == 0) old = __hip_atomic_fetch_add(ctl + C_REM, 1u, __ATOMIC_RELAXED, __HIP_MEMORY_SCOPE_WORKGROUP);
    if (__builtin_amdgcn_readfirstlane(old) == 7 && lane == 0)
      __hip_atomic_fetch_add((GAS u32*)(myws + WS_TCNT), 1u, __ATOMIC_RELAXED, __HIP_MEMORY_SCOPE_AGENT);
    if (wave != 0) return;
    STAMP(10);
    if (g != 0) {
      const u32 nw = rb == NRB - 1 ? 5 : 8;
      for (u32 w = 1; w < nw; ++w) {
#pragma unroll
        for (int i = 1; i < 5; ++i) qv[i] = *(LAS u32x4*)(tr + (w * 5 + i) * 4 * TPITCH);
        WAIT_VM(0);
#pragma unroll
        for (int i = 1; i < 5; ++i) __builtin_amdgcn_raw_buffer_store_b128(qv[i], rs_d, ooff + (w * 5 + i) * 65536, 0, 17);
        CFENCE();
      }
    }
    STAMP(11);
    nrem = 8;
  } else {
    const __amdgpu_buffer_rsrc_t rs_x = mkrsrc(P.x, X_BYTES);
    const __amdgpu_buffer_rsrc_t rs_w = mkrsrc(P.w, W_BYTES);
    const __amdgpu_buffer_rsrc_t rs_d = mkrsrc(dws, WS_CTL);
    const u32 lx = (lane & 7) * 1024 + (lane >> 3) * 16;
    const u32 lw = (((lane >> 3) & 1) * 8 + ((lane >> 1) & 3)) * 32768 + ((lane & 1) * 4 + (lane >> 4)) * 16;
    const u32 rot = (rank * 13) >> 2;
    u32x4 v[8];
    auto load_blk = [&](u32 b) {
      const u32 kc = b / 34, r = b - kc * 34;
      if (r < 8) {
        const u32 so = kc * 64 * 32768 + dest * 4096 + r * 512 + wave * 128;
#pragma unroll
        for (int e = 0; e < 8; ++e)
          v[e] = __builtin_amdgcn_raw_buffer_load_b128(rs_w, lw, so + ((e >> 2) * 32 + ((e >> 1) & 1) * 16 + (e & 1) * 4) * 32768, 0);
      } else {
        u32 rbk = r - 8 + rot; if (rbk >= NRB) rbk -= NRB;
        const u32 np = rbk == NRB - 1 ? 3 : 5;
        const u32 so = (rbk * 160 + wave * np * 8) * 1024 + kc * 128;
#pragma unroll
        for (int e = 0; e < 5; ++e)
          if (e < np) v[e] = __builtin_amdgcn_raw_buffer_load_b128(rs_x, lx, so + e * 8192, 0);
      }
    };
    u32 b = idx;
    const bool active = wave < 4 && idx < NCW;
    if (active && b < NBLK) load_blk(b);
    if (wave == 4) {
      ctl[lane] = 0u;
      ctl[64 + lane] = 0u;
      WAIT_LGKM();
    }
    BARRIER();
    if (wave == 4) {
      u32* const cptr = (u32*)(myws + WS_CALL) + blockIdx.x;
      call = __builtin_amdgcn_readfirstlane(ld_sys(cptr)) + 1;
      if (lane == 0) { lds_st(ctl + C_CALL, call); st_sys(cptr, call); }
    }
    if (wave >= 4) {
      if (wave == 4 && blockIdx.x == 0) {
        const u64* g1 = (const u64*)(myws + WS_G1);
        u64* g2 = (u64*)(P.ws[lane & 7] + WS_G2) + rank;
        asm volatile("" : "+v"(g2));
        u32 gm, sp = 0;
        for (;;) {
          u64 a0 = ld_sys64(g1 + lane), a1 = ld_sys64(g1 + 64 + lane), a2 = ld_sys64(g1 + 128 + lane);
          u64 a3 = ld_sys64(g1 + 192 + (lane & 15));
          const bool ok = (u32)(a0 >> 32) == call && (u32)(a1 >> 32) == call && (u32)(a2 >> 32) == call && (u32)(a3 >> 32) == call;
          gm = max(max((u32)a0, (u32)a1), max((u32)a2, (u32)a3));
          const u64 okm = __ballot(ok);
          if (okm == ~0ull) break;
          if (okm == 0) __builtin_amdgcn_s_sleep(8);
          if ((++sp & 63) == 0 && (ld_sys(tmo) == call || now() - t0 > DEADLINE)) { st_sys(tmo, call); return; }
        }
        gm = wave_umax(gm);
        if (lane < 8) st_sys64(g2, ((u64)call << 32) | gm);
      }
      return;
    }
    if (active) {
      WAIT_VM0();
      for (u32 n = 0; b < NBLK; b += NCW, ++n) {
        const u32 kc = b / 34, r = b - kc * 34;
        const u32 ord = kc * 8 + rank;
        u32 foff;
        if (r < 8) {
          const u32 doff = WS_W + (ord * NCB + r) * WBLK + wave * 8192;
          foff = WS_WF + r * FSTRIDE + ord * 4;
#pragma unroll
          for (int e = 0; e < 8; ++e) __builtin_amdgcn_raw_buffer_store_b128(v[e], rs_d, lane16, doff + e * 1024, 17);
        } else {
          u32 rbk = r - 8 + rot; if (rbk >= NRB) rbk -= NRB;
          const u32 np = rbk == NRB - 1 ? 3 : 5;
          const u32 doff = WS_X + (ord * NRB + rbk) * XBLK + wave * np * 1024;
          foff = WS_XF + rbk * FSTRIDE + ord * 4;
#pragma unroll
          for (int e = 0; e < 5; ++e)
            if (e < np) __builtin_amdgcn_raw_buffer_store_b128(v[e], rs_d, lane16, doff + e * 1024, 17);
        }
        CFENCE();
        if (b + NCW < NBLK) load_blk(b + NCW);
        WAIT_VM0();
        while (call == 0) call = lds_ld(ctl + C_CALL);
        u32 old = 0;
        if (lane == 0) old = __hip_atomic_fetch_add(ctl + C_BLK + n, 1u, __ATOMIC_RELAXED, __HIP_MEMORY_SCOPE_WORKGROUP);
        if (__builtin_amdgcn_readfirstlane(old) == 3 && lane == 0) st_sys((u32*)(dws + foff), call);
        CFENCE();
      }
    }
    if (wave == 0) STAMP(1);
    if (wave != 0) return;
    while (call == 0) call = lds_ld(ctl + C_CALL);
  }

  if (g != 0) {
    const u32 owner = P3_PULL ? dest : rank;
    const u32 ge = P3_PULL ? (8 - g) & 7 : g;
    const __amdgpu_buffer_rsrc_t rs_s = mkrsrc(P.out[owner], OUT_BYTES);
    const __amdgpu_buffer_rsrc_t rs_t = mkrsrc(P.out[P3_PULL ? rank : dest], OUT_BYTES);
    {
      const u32* tc = (const u32*)(P.ws[owner] + WS_TCNT);
      u32 sp = 0;
      while ((int)(__builtin_amdgcn_readfirstlane(ld_sys(tc)) - call * NTILE) < 0) {
        if (nrem) __builtin_amdgcn_s_sleep(2); else __builtin_amdgcn_s_sleep(16);
        if ((++sp & 63) == 0 && (ld_sys(tmo) == call || now() - t0 > DEADLINE)) { st_sys(tmo, call); bad = true; break; }
      }
      CFENCE();
    }
    STAMP(9);
    constexpr u32 KEEP = 10 - 2 * NDIR;
    const u32 n2 = 1536 + ((ge & 1) ? 260 : 252);
    const u32 nu = n2 + ((ge & 1) ? 25 : 26) * KEEP;
    const u32 lo = (lane >> 5) * 16384 + (lane & 31) * 16;
    auto uoff = [&](u32 u) -> u32 {
      u32 cq = ge >> 1, r8;
      if (u < 1536) { cq = (cq + 1 + (u >> 9)) & 3; r8 = u & 511; }
      else if (u < n2) r8 = u - 1536 + ((ge & 1) ? 0 : 260);
      else { const u32 k = u - n2, kq = KEEP ? k / (KEEP ? KEEP : 1) : 0; r8 = ((ge & 1) ? 260 : 0) + kq * 10 + 2 * NDIR + (k - kq * KEEP); }
      return r8 * 8 * 16384u + owner * 2048 + cq * 512;
    };
    u32x4 v[4];
    u32 u0 = idx, u1 = 32 + idx;
    {
      const u32 o = uoff(u0);
#pragma unroll
      for (int e = 0; e < 4; ++e) v[e] = __builtin_amdgcn_raw_buffer_load_b128(rs_s, lo, o + e * 32768, P3_LAUX);
    }
    WAIT_VM0();
    while (u0 < nu) {
      const u32 o = uoff(u0);
      if (bad) {
#pragma unroll
        for (int e = 0; e < 4; ++e) v[e] = u32x4{0x80808080u, 0x80808080u, 0x80808080u, 0x80808080u};
      }
#pragma unroll
      for (int e = 0; e < 4; ++e) __builtin_amdgcn_raw_buffer_store_b128(v[e], rs_t, lo, o + e * 32768, P3_SAUX);
      CFENCE();
      if (u1 < nu) {
        const u32 o1 = uoff(u1);
#pragma unroll
        for (int e = 0; e < 4; ++e) v[e] = __builtin_amdgcn_raw_buffer_load_b128(rs_s, lo, o1 + e * 32768, P3_LAUX);
      }
      u32 tk = 0;
      if (P3_TICKET && lane == 0)
        tk = __hip_atomic_fetch_add((GAS u32*)(myws + WS_TK3 + g * 128), 1u, __ATOMIC_RELAXED, __HIP_MEMORY_SCOPE_AGENT);
      WAIT_VM0();
      u0 = u1;
      u1 = P3_TICKET ? 64 + (__builtin_amdgcn_readfirstlane(tk) - (call - 1) * nu) : u1 + 32;
    }
    while (lds_ld(ctl + C_REM) < nrem) __builtin_amdgcn_s_sleep(1);
    CFENCE();
    if (lane == 0) st_sys((u32*)(dws + WS_DONE + idx * 128) + rank, call);
    STAMP(5);
  } else {
    const u32* df = (const u32*)(myws + WS_DONE + idx * 128);
    u32 sp = 0;
    for (;;) {
      const u32 val = ld_sys(df + (lane & 7));
      const bool ok = ((lane & 7) == rank) || (int)(val - call) >= 0;
      if (__all(ok)) break;
      __builtin_amdgcn_s_sleep(4);
      if ((++sp & 63) == 0 && (ld_sys(tmo) == call || now() - t0 > DEADLINE)) { st_sys(tmo, call); bad = true; break; }
    }
    if (bad) {
      while (lds_ld(ctl + C_REM) < nrem) __builtin_amdgcn_s_sleep(1);
      WAIT_VM(0);
      if (lane == 0) P.out[rank][(size_t)idx * 16384 + rank * 2048] = (char)-128;
    }
    STAMP(5);
  }
}

extern "C" void kernel_launch(int rank, int nranks, ncclComm_t comm, hipStream_t stream,
                              void* const* d_in, const int* in_sizes, int n_in,
                              void* d_out, int out_size,
                              const int* in_dtypes, int out_dtype,
                              void* const* d_out_all,
                              void* const* d_ws_all, size_t ws_bytes) {
  if (nranks != NR || ws_bytes < WS_END || n_in < 2) { fprintf(stderr, "unsupported configuration\n"); return; }
  Params P;
  P.x = (const char*)d_in[0];
  P.w = (const char*)d_in[1];
  for (int p = 0; p < NR; ++p) { P.out[p] = (char*)d_out_all[p]; P.ws[p] = (char*)d_ws_all[p]; }
  P.rank = rank;
  hipLaunchKernelGGL(fused_gemm_ar_q8, dim3(256), dim3(NTHREADS), 0, stream, P);
  HIP_CHECK(hipGetLastError());
}
```

```cpp
#include <hip/hip_runtime.h>
#include <rccl/rccl.h>
#include <cstdio>
#include <stdint.h>

#define HIP_CHECK(expr) do { hipError_t _e = (expr); if (_e != hipSuccess) { \
    fprintf(stderr, "HIP error %s at %s:%d\n", hipGetErrorString(_e), __FILE__, __LINE__); \
    return; } } while (0)

typedef unsigned u32;
typedef unsigned long long u64;
typedef u32 u32x4 __attribute__((ext_vector_type(4)));
typedef u32 u32x2 __attribute__((ext_vector_type(2)));
typedef float f32x4 __attribute__((ext_vector_type(4)));
typedef short s16x4 __attribute__((ext_vector_type(4)));
typedef short s16x8 __attribute__((ext_vector_type(8)));
typedef __bf16 bf16x8 __attribute__((ext_vector_type(8)));
#define LAS __attribute__((address_space(3)))
#define GAS __attribute__((address_space(1)))

constexpr int NR = 8;
constexpr u32 OUT_BYTES = 4096u * 16384u;
constexpr u32 X_BYTES = 4096u * 512u * 2u;
constexpr u32 W_BYTES = 512u * 16384u * 2u;

constexpr u32 NRB = 26;
constexpr u32 NCB = 8;
constexpr u32 XBLK = 20480, WBLK = 32768;
constexpr u32 WS_W = 0;
constexpr u32 WS_X = 16u << 20;
constexpr u32 WS_CTL = 49u << 20;
constexpr u32 FSTRIDE = 4352;
constexpr u32 WS_XF = WS_CTL;
constexpr u32 WS_WF = WS_XF + NRB * FSTRIDE;
constexpr u32 WS_C2 = WS_CTL + (160u << 10);
constexpr u32 WS_CALL = WS_C2;
constexpr u32 WS_G1 = WS_C2 + 4096;
constexpr u32 WS_G2 = WS_C2 + 8192 + 256;
constexpr u32 WS_TCNT = WS_C2 + 12288 + 512;
constexpr u32 WS_TK3 = WS_C2 + 13312;
constexpr u32 WS_DONE = WS_C2 + 16384;
constexpr u32 WS_TMO = WS_C2 + 20480;
constexpr u32 WS_DBG = WS_C2 + 32768;
constexpr u32 WS_Q = 50u << 20;
constexpr u32 QSTRIDE = 4160 * 256;
constexpr u32 WS_END = WS_Q + NCB * QSTRIDE;
static_assert(WS_X + 64u * NRB * XBLK <= WS_CTL && WS_WF + NCB * FSTRIDE <= WS_C2, "layout");

constexpr u32 SLOT_BYTES = WBLK + XBLK;
constexpr u32 NSLOT = 3;
constexpr u32 TPITCH = 272;
constexpr u32 SNAP_OFF = SLOT_BYTES * NSLOT;
constexpr u32 CTL_OFF = SNAP_OFF + 3072;
constexpr u32 LDS_BYTES = CTL_OFF + 1024;
static_assert(LDS_BYTES <= 163840, "lds");
constexpr int C_AMAX = 0, C_GMAX = 1, C_CALL = 2, C_BAD = 3, C_LOC = 4, C_REM = 5, C_BLK = 16;

#define NCW 6
#define P1G 3
#define P3D1 4
#define P3D 2
#define P3N 32
constexpr u32 NCOMMWG = 32 - NRB;
constexpr u32 NTILE = NRB * NCB;
constexpr u32 NBLK = 8 * (NCB + NRB);
constexpr int NTHREADS = 512;
constexpr int NSTEP = 64;
constexpr u64 DEADLINE = 50000000ull;

struct Params {
  const char* x;
  const char* w;
  char* out[NR];
  char* ws[NR];
  int rank;
  u32 peers;
};

__device__ __forceinline__ __amdgpu_buffer_rsrc_t mkrsrc(const void* p, u32 bytes) {
  return __builtin_amdgcn_make_buffer_rsrc((void*)p, 0, bytes, 0x00020000);
}
__device__ __forceinline__ u64 now() { return __builtin_amdgcn_s_memrealtime(); }
__device__ __forceinline__ u32 ld_sys(const u32* p) { return __hip_atomic_load((GAS const u32*)p, __ATOMIC_RELAXED, __HIP_MEMORY_SCOPE_SYSTEM); }
__device__ __forceinline__ u64 ld_sys64(const u64* p) { return __hip_atomic_load((GAS const u64*)p, __ATOMIC_RELAXED, __HIP_MEMORY_SCOPE_SYSTEM); }
__device__ __forceinline__ void st_sys(u32* p, u32 v) { __hip_atomic_store((GAS u32*)p, v, __ATOMIC_RELAXED, __HIP_MEMORY_SCOPE_SYSTEM); }
__device__ __forceinline__ void st_sys64(u64* p, u64 v) { __hip_atomic_store((GAS u64*)p, v, __ATOMIC_RELAXED, __HIP_MEMORY_SCOPE_SYSTEM); }
__device__ __forceinline__ u32 lds_ld(LAS u32* p) { return __builtin_amdgcn_readfirstlane(__hip_atomic_load(p, __ATOMIC_RELAXED, __HIP_MEMORY_SCOPE_WORKGROUP)); }
__device__ __forceinline__ void lds_st(LAS u32* p, u32 v) { __hip_atomic_store(p, v, __ATOMIC_RELAXED, __HIP_MEMORY_SCOPE_WORKGROUP); }
__device__ __forceinline__ void lds_add1(LAS u32* p, u32 lane) {
  if (lane == 0) __hip_atomic_fetch_add(p, 1u, __ATOMIC_RELAXED, __HIP_MEMORY_SCOPE_WORKGROUP);
}
#define CFENCE() asm volatile("" ::: "memory")
#define WAIT_VM(n) asm volatile("s_waitcnt vmcnt(" #n ")" ::: "memory")
#define WAIT_VMI(n) asm volatile("s_waitcnt vmcnt(%0)" :: "i"(n) : "memory")
#define WAIT_LGKM() asm volatile("s_waitcnt lgkmcnt(0)" ::: "memory")
#define WAIT_VM0() do { CFENCE(); __builtin_amdgcn_s_waitcnt(0x0F70); WAIT_VM(0); } while (0)

__device__ __forceinline__ u32 wave_umax(u32 v) {
  v = max(v, (u32)__builtin_amdgcn_update_dpp(0, (int)v, 0x111, 0xf, 0xf, true));
  v = max(v, (u32)__builtin_amdgcn_update_dpp(0, (int)v, 0x112, 0xf, 0xf, true));
  v = max(v, (u32)__builtin_amdgcn_update_dpp(0, (int)v, 0x114, 0xf, 0xf, true));
  v = max(v, (u32)__builtin_amdgcn_update_dpp(0, (int)v, 0x118, 0xf, 0xf, true));
  v = max(v, (u32)__builtin_amdgcn_update_dpp(0, (int)v, 0x142, 0xa, 0xf, false));
  v = max(v, (u32)__builtin_amdgcn_update_dpp(0, (int)v, 0x143, 0xc, 0xf, false));
  return (u32)__builtin_amdgcn_readlane((int)v, 63);
}

#define STAMP(k) do {} while (0)

__device__ __forceinline__ void wait_vm_dyn(u32 n) {
  asm volatile(
      "s_getpc_b64 vcc\n\t"
      "s_add_u32 vcc_lo, vcc_lo, %0\n\t"
      "s_addc_u32 vcc_hi, vcc_hi, 0\n\t"
      "s_setpc_b64 vcc\n\t"
      "s_waitcnt vmcnt(0)\n\ts_branch 1f\n\t"
      "s_waitcnt vmcnt(1)\n\ts_branch 1f\n\t"
      "s_waitcnt vmcnt(2)\n\ts_branch 1f\n\t"
      "s_waitcnt vmcnt(3)\n\ts_branch 1f\n\t"
      "s_waitcnt vmcnt(4)\n\ts_branch 1f\n\t"
      "s_waitcnt vmcnt(5)\n\ts_branch 1f\n\t"
      "s_waitcnt vmcnt(6)\n\ts_branch 1f\n\t"
      "s_waitcnt vmcnt(7)\n\ts_branch 1f\n\t"
      "s_waitcnt vmcnt(8)\n\ts_branch 1f\n\t"
      "s_waitcnt vmcnt(9)\n\ts_branch 1f\n\t"
      "s_waitcnt vmcnt(10)\n\ts_branch 1f\n\t"
      "s_waitcnt vmcnt(11)\n\ts_branch 1f\n\t"
      "s_waitcnt vmcnt(12)\n\ts_branch 1f\n\t"
      "s_waitcnt vmcnt(13)\n\ts_branch 1f\n\t"
      "s_waitcnt vmcnt(14)\n\ts_branch 1f\n\t"
      "s_waitcnt vmcnt(15)\n\ts_branch 1f\n\t"
      "s_waitcnt vmcnt(16)\n\ts_branch 1f\n\t"
      "1:"
      :: "s"(min(n, 17u) * 8 + 12) : "memory", "scc", "vcc");
}
#define BARRIER() asm volatile("s_barrier" ::: "memory")

__global__ void __launch_bounds__(NTHREADS) fused_gemm_ar_q8(Params P) {
  __shared__ __attribute__((aligned(16))) char smem[LDS_BYTES];
  LAS char* L = (LAS char*)smem;
  LAS u32* ctl = (LAS u32*)(L + CTL_OFF);
  const u32 tid = threadIdx.x, lane = tid & 63;
  const u32 wave = __builtin_amdgcn_readfirstlane(tid >> 6);
  const u32 g = blockIdx.x & 7u, idx = blockIdx.x >> 3;
  const u32 rank = (u32)P.rank & 7u;
  const u32 dest = (P.peers >> (g * 4)) & 7u;
  char* const myws = P.ws[rank];
  char* const dws = P.ws[dest];
  u32* const tmo = (u32*)(myws + WS_TMO);
  const u32 lane16 = lane * 16;
  const u64 t0 = now();
  u32 call = 0, nrem = 0;
  bool bad = false;
  if (wave == 0) STAMP(0);

  if (idx >= NCOMMWG) {
    const u32 ti = idx - NCOMMWG;
    const u32 rb = (g & 1) * 13 + (ti >> 1), cb = (g >> 1) * 2 + (ti & 1);
    if (wave == 0) {
      __builtin_amdgcn_fence(__ATOMIC_ACQUIRE, "");
      u32* const cptr = (u32*)(myws + WS_CALL) + blockIdx.x;
      call = __builtin_amdgcn_readfirstlane(ld_sys(cptr)) + 1;
      if (lane < 16) ctl[lane] = lane == C_CALL ? call : 0u;
      if (lane == 0) st_sys(cptr, call);
      WAIT_VM0();
      WAIT_LGKM();
    }
    BARRIER();
    call = lds_ld(ctl + C_CALL);
    const u32 wm = wave >> 2, wn = wave & 3;
    const __amdgpu_buffer_rsrc_t rs_ws = mkrsrc(myws, WS_C2);
    const u32 lds0 = (u32)(size_t)L;
    const u32 xa = WBLK + wm * 10240 + ((lane >> 3) & 1) * 1024 + (lane >> 5) * 256 + ((lane >> 4) & 1) * 128 + (lane & 7) * 16;
    const u32 h = lane >> 4, q = (lane >> 2) & 3, p = lane & 3;
    const u32 wa0 = wn * 8192 + (h >> 1) * 2048 + (p & 1) * 512 + (h & 1) * 128 + q * 32 + (p >> 1) * 16 + (p & 1) * 8;
    const u32 wa1 = wa0 ^ 8;
    const u32 xpo = wave < 4 ? wave * 3072 : 12288 + (wave - 4) * 2048;
    const u32 pollo = lane < 16 ? WS_XF + rb * FSTRIDE + lane16 : WS_WF + cb * FSTRIDE + lane16 - 256;
    f32x4 acc[4][5];
#pragma unroll
    for (int a = 0; a < 4; ++a)
#pragma unroll
      for (int b = 0; b < 5; ++b) acc[a][b] = f32x4{0.f, 0.f, 0.f, 0.f};

    u64 kx = 0, kw = 0, taken = 0;
    u32 issued = 0, consumed = 0, nis = 0;
    u32 islot = 0, cslot = 0, sj = 0;
    u64 markD = 0, markP = 0;
    auto sched = [&](u32 fx, u32 fw) {
      kx |= __ballot((int)(fx - call) >= 0);
      kw |= __ballot((int)(fw - call) >= 0);
      u64 avail = kx & kw & ~taken;
      while (avail && issued - consumed < NSLOT) {
        const u32 ord = __builtin_ctzll(avail);
        avail &= avail - 1;
        taken |= 1ull << ord;
        const u32 sw = WS_W + (ord * NCB + cb) * WBLK + wave * 4096;
        const u32 sx = WS_X + (ord * NRB + rb) * XBLK + xpo;
        const u32 mw = lds0 + islot + wave * 4096, mx = lds0 + islot + WBLK + xpo;
        u32 keep;
        asm volatile(
            "s_mov_b32 %0, m0\n\t"
            "s_mov_b32 m0, %3\n\t"
            "s_nop 0\n\t"
            "buffer_load_dwordx4 %1, %2, %5 offen lds\n\t"
            "buffer_load_dwordx4 %1, %2, %5 offen offset:1024 lds\n\t"
            "buffer_load_dwordx4 %1, %2, %5 offen offset:2048 lds\n\t"
            "buffer_load_dwordx4 %1, %2, %5 offen offset:3072 lds\n\t"
            "s_mov_b32 m0, %4\n\t"
            "s_nop 0\n\t"
            "buffer_load_dwordx4 %1, %2, %6 offen lds\n\t"
            "buffer_load_dwordx4 %1, %2, %6 offen offset:1024 lds\n\t"
            "s_cmp_gt_u32 %7, 3\n\t"
            "s_cbranch_scc1 1f\n\t"
            "buffer_load_dwordx4 %1, %2, %6 offen offset:2048 lds\n\t"
            "1:\n\t"
            "s_mov_b32 m0, %0"
            : "=&s"(keep)
            : "v"(lane16), "s"(rs_ws), "s"(mw), "s"(mx), "s"(sw), "s"(sx), "s"(wave)
            : "memory", "scc");
        nis += wave < 4 ? 7 : 6;
        const u32 sh = (issued & 3) * 16;
        markD = (markD & ~(0xffffull << sh)) | ((u64)(nis & 0xffff) << sh);
        issued += 1;
        islot = islot == 2 * SLOT_BYTES ? 0 : islot + SLOT_BYTES;
      }
    };
    auto poll = [&]() {
      if (wave == 0) {
        const u64 kn = lane < 16 ? kx : kw;
        const bool act = lane < 32 && ((u32)(kn >> ((lane & 15) * 4)) & 15u) != 15u;
        const u32 po = act ? pollo : 0x80000000u;
        u32 keep;
        asm volatile("s_mov_b32 %0, m0\n\ts_mov_b32 m0, %3\n\ts_nop 0\n\t"
                     "buffer_load_dwordx4 %1, %2, 0 offen sc0 sc1 lds\n\ts_mov_b32 m0, %0"
                     : "=&s"(keep) : "v"(po), "s"(rs_ws), "s"(lds0 + SNAP_OFF + sj * 1024) : "memory");
        nis += 1;
        markP = (markP & ~(0xffffull << (sj * 16))) | ((u64)(nis & 0xffff) << (sj * 16));
      }
      sj = sj == 2 ? 0 : sj + 1;
    };
    auto wait_step = [&]() {
      u32 need = (nis - (u32)(markD >> ((consumed & 3) * 16))) & 0xffff;
      if (wave == 0) need = min(need, (nis - (u32)(markP >> ((sj == 2 ? 0 : sj + 1) * 16))) & 0xffff);
      wait_vm_dyn(need);
    };
#define SB() __builtin_amdgcn_sched_barrier(0)
#define PIECE(M, S, OFF) do { \
        asm volatile("s_cmp_lg_u32 %2, -1\n\ts_cbranch_scc0 1f\n\ts_mov_b32 m0, %2\n\ts_nop 0\n\t" \
                     "buffer_load_dwordx4 %0, %1, %3 offen offset:" #OFF " lds\n\t1:" \
                     :: "v"(lane16), "s"(rs_ws), "s"(M), "s"(S) : "memory", "scc"); \
        SB(); } while (0)
#define MMA4(kb, i) do { \
        _Pragma("unroll") for (int n = 0; n < 4; ++n) \
          acc[n][i] = __builtin_amdgcn_mfma_f32_16x16x32_bf16(wf[kb][n], xf[kb][i], acc[n][i], 0, 0, 0); \
        SB(); } while (0)
    bf16x8 xf[2][5], wf[2][4];
#pragma unroll
    for (int n = 0; n < 5; ++n) {
      xf[1][n] = __builtin_bit_cast(bf16x8, u32x4{0u, 0u, 0u, 0u});
      if (n < 4) wf[1][n] = __builtin_bit_cast(bf16x8, u32x4{0u, 0u, 0u, 0u});
    }
    poll(); poll();
    u32 sp = 0;
    u64 cyc0 = 0;
    while (consumed < NSTEP) {
      if (issued == consumed) {
        do {
          const u32 srd = sj == 2 ? 0 : sj + 1;
          if (wave == 0) {
            wait_vm_dyn((nis - (u32)(markP >> (srd * 16))) & 0xffff);
            if ((++sp & 63) == 0 && now() - t0 > DEADLINE) {
              st_sys(tmo, call);
              WAIT_VM(0);
              lds_st(ctl + C_BAD, 1u);
              *(LAS u32x2*)(L + SNAP_OFF + srd * 1024 + lane * 8) = u32x2{call, call};
              WAIT_LGKM();
            }
          }
          BARRIER();
          poll();
          LAS u32* const snap = (LAS u32*)(L + SNAP_OFF + srd * 1024) + lane;
          sched(snap[0], snap[64]);
        } while (issued == consumed);
        wait_step();
      }
      BARRIER();
      LAS u32* const snap = (LAS u32*)(L + SNAP_OFF + (sj == 2 ? 0 : sj + 1) * 1024) + lane;
      const u32 fx = snap[0], fw = snap[64];
      SB();
      LAS char* base = L + cslot;
      auto frags = [&](int kb) {
#pragma unroll
        for (int n = 0; n < 5; ++n) {
          if (n < 4) {
            LAS char* a = base + ((n & 1) ? wa1 : wa0) + kb * 4096 + (n >> 1) * 256;
            s16x4 lo = __builtin_amdgcn_ds_read_tr16_b64_v4i16((LAS s16x4*)a);
            s16x4 hi = __builtin_amdgcn_ds_read_tr16_b64_v4i16((LAS s16x4*)(a + 1024));
            wf[kb][n] = __builtin_bit_cast(bf16x8, __builtin_shufflevector(lo, hi, 0, 1, 2, 3, 4, 5, 6, 7));
          }
          xf[kb][n] = *(LAS bf16x8*)(base + xa + n * 2048 + kb * 512);
        }
        SB();
      };
      MMA4(1, 0);
      frags(0);
      poll();
      SB();
      MMA4(1, 1);
      kx |= __ballot((int)(fx - call) >= 0);
      kw |= __ballot((int)(fw - call) >= 0);
      const u64 av = kx & kw & ~taken;
      const bool go = av != 0 && issued - consumed < NSLOT;
      const u32 ord = __builtin_ctzll(av | (1ull << 63));
      const u32 sw = WS_W + (ord * NCB + cb) * WBLK + wave * 4096;
      const u32 sx = WS_X + (ord * NRB + rb) * XBLK + xpo;
      u32 mw = ~0u, mx = ~0u, mx3 = ~0u;
      if (go) {
        mw = lds0 + islot + wave * 4096;
        mx = lds0 + islot + WBLK + xpo;
        if (wave < 4) mx3 = mx;
        taken |= 1ull << ord;
        nis += wave < 4 ? 7 : 6;
        const u32 sh = (issued & 3) * 16;
        markD = (markD & ~(0xffffull << sh)) | ((u64)(nis & 0xffff) << sh);
        issued += 1;
        islot = islot == 2 * SLOT_BYTES ? 0 : islot + SLOT_BYTES;
      }
      SB();
      MMA4(1, 2);
      PIECE(mw, sw, 0);
      MMA4(1, 3);
      PIECE(mw, sw, 1024);
      MMA4(1, 4);
      frags(1);
      PIECE(mw, sw, 2048);
      MMA4(0, 0);
      PIECE(mw, sw, 3072);
      PIECE(mx, sx, 0);
      MMA4(0, 1);
      PIECE(mx, sx, 1024);
      PIECE(mx3, sx, 2048);
      MMA4(0, 2);
      sched(0u - 1u + call, 0u - 1u + call);
      cslot = cslot == 2 * SLOT_BYTES ? 0 : cslot + SLOT_BYTES;
      consumed += 1;
      SB();
      MMA4(0, 3);
      if (issued != consumed) wait_step();
      SB();
      MMA4(0, 4);
      WAIT_LGKM();
    }
    MMA4(1, 0); MMA4(1, 1); MMA4(1, 2); MMA4(1, 3); MMA4(1, 4);
    if (wave == 0) STAMP(2);
    const u32 row0 = rb * 160 + wm * 80;
    float m = 0.f;
#pragma unroll
    for (int b = 0; b < 5; ++b) {
      float mb = 0.f;
#pragma unroll
      for (int a = 0; a < 4; ++a)
#pragma unroll
        for (int e = 0; e < 4; ++e) mb = fmaxf(mb, fabsf(acc[a][b][e]));
      if (row0 + b * 16 < 4096) m = fmaxf(m, mb);
    }
    const u32 mb = wave_umax(__builtin_bit_cast(u32, m));
    if (lane == 0) __hip_atomic_fetch_max(ctl + C_AMAX, mb, __ATOMIC_RELAXED, __HIP_MEMORY_SCOPE_WORKGROUP);
    WAIT_LGKM();
    BARRIER();
    if (wave == 0) {
      const u32 wmax = lds_ld(ctl + C_AMAX);
      if (lane == 0) st_sys64((u64*)(myws + WS_G1) + g * NRB + ti, ((u64)call << 32) | wmax);
      STAMP(3);
      const u64* g2 = (const u64*)(myws + WS_G2) + (lane & 7);
      u64 val;
      for (;;) {
        val = ld_sys64(g2);
        if (__all((u32)(val >> 32) == call)) break;
        if ((++sp & 63) == 0 && (ld_sys(tmo) == call || now() - t0 > DEADLINE)) { st_sys(tmo, call); bad = true; break; }
      }
      const u32 gmax = wave_umax((u32)val);
      if (lane == 0) {
        lds_st(ctl + C_GMAX, gmax);
        if (bad) lds_st(ctl + C_BAD, 1u);
      }
      WAIT_LGKM();
    }
    BARRIER();
    const float amax = __builtin_bit_cast(float, lds_ld(ctl + C_GMAX));
    bad = lds_ld(ctl + C_BAD) != 0;
    const float scale = amax / 127.0f;
    const float inv = amax > 0.f ? 1.0f / scale : 0.f;
    if (wave == 0) STAMP(4);
    const bool odd = (lane >> 4) & 1;
    u32x4 qv[5];
#pragma unroll
    for (int i = 0; i < 5; ++i) {
      u32 d[4];
#pragma unroll
      for (int n = 0; n < 4; ++n) {
        u32 b0 = __builtin_bit_cast(u32, __builtin_fmaf(acc[n][i][0], inv, 12582912.0f));
        u32 b1 = __builtin_bit_cast(u32, __builtin_fmaf(acc[n][i][1], inv, 12582912.0f));
        u32 b2 = __builtin_bit_cast(u32, __builtin_fmaf(acc[n][i][2], inv, 12582912.0f));
        u32 b3 = __builtin_bit_cast(u32, __builtin_fmaf(acc[n][i][3], inv, 12582912.0f));
        u32 lo = __builtin_amdgcn_perm(b1, b0, 0x0c0c0400u);
        u32 hi = __builtin_amdgcn_perm(b3, b2, 0x04000c0cu);
        d[n] = bad ? 0x80808080u : (lo | hi);
      }
      qv[i] = u32x4{odd ? d[1] : d[0], odd ? d[0] : d[1], odd ? d[3] : d[2], odd ? d[2] : d[3]};
    }
    {
      LAS char* tw = L + (wm * 80 + (lane & 15)) * TPITCH + wn * 64 + (lane >> 4) * 16;
#pragma unroll
      for (int i = 0; i < 5; ++i) *(LAS u32x4*)(tw + i * 16 * TPITCH) = qv[i];
    }
    WAIT_LGKM();
    BARRIER();
    LAS char* tr = L + (lane >> 4) * TPITCH + (lane & 15) * 16;
#pragma unroll
    for (int i = 0; i < 5; ++i) qv[i] = *(LAS u32x4*)(tr + (wave * 5 + i) * 4 * TPITCH);
    const u32 ooff = (rb * 160 + (lane >> 4)) * 16384u + rank * 2048 + cb * 256 + (lane & 15) * 16;
    const __amdgpu_buffer_rsrc_t rs_q = mkrsrc(myws, WS_END);
    const __amdgpu_buffer_rsrc_t rs_d = mkrsrc(P.out[dest], OUT_BYTES);
    __builtin_amdgcn_raw_buffer_store_b128(qv[0], rs_d, ooff + wave * 5 * 65536, 0, 17);
#pragma unroll
    for (int i = 0; i < 5; ++i)
      __builtin_amdgcn_raw_buffer_store_b128(qv[i], rs_q, lane16, WS_Q + cb * QSTRIDE + rb * 40960 + (wave * 5 + i) * 1024, 17);
    if (wave == 0) {
#pragma unroll
      for (int i = 1; i < 5; ++i) __builtin_amdgcn_raw_buffer_store_b128(qv[i], rs_d, ooff + i * 65536, 0, 17);
      WAIT_VM(4);
    } else {
      WAIT_VM(0);
    }
    u32 old = 0;
    if (lane == 0) old = __hip_atomic_fetch_add(ctl + C_REM, 1u, __ATOMIC_RELAXED, __HIP_MEMORY_SCOPE_WORKGROUP);
    if (__builtin_amdgcn_readfirstlane(old) == 7 && lane == 0)
      __hip_atomic_fetch_add((GAS u32*)(myws + WS_TCNT), 1u, __ATOMIC_RELAXED, __HIP_MEMORY_SCOPE_AGENT);
    if (wave != 0) return;
    STAMP(10);
    {
      const u32 nw = rb == NRB - 1 ? 5 : 8;
      for (u32 w = 1; w < nw; ++w) {
#pragma unroll
        for (int i = 1; i < 5; ++i) qv[i] = *(LAS u32x4*)(tr + (w * 5 + i) * 4 * TPITCH);
#pragma unroll
        for (int i = 1; i < 5; ++i) {
          if ((i - 1) % P3D1 == 0) WAIT_VM(0);
          __builtin_amdgcn_raw_buffer_store_b128(qv[i], rs_d, ooff + (w * 5 + i) * 65536, 0, 17);
        }
        CFENCE();
      }
    }
    STAMP(11);
    nrem = 8;
  } else {
    const __amdgpu_buffer_rsrc_t rs_x = mkrsrc(P.x, X_BYTES);
    const __amdgpu_buffer_rsrc_t rs_w = mkrsrc(P.w, W_BYTES);
    const __amdgpu_buffer_rsrc_t rs_d = mkrsrc(dws, WS_CTL);
    const u32 lx = (lane & 7) * 1024 + (lane >> 3) * 16;
    const u32 lw = (((lane >> 3) & 1) * 8 + ((lane >> 1) & 3)) * 32768 + ((lane & 1) * 4 + (lane >> 4)) * 16;
    const u32 rot = (rank * 13) >> 2;
    u32x4 v[8], nv[8];
    auto load_blk = [&](u32 b) -> u32 {
      const u32 kc = b / 34, r = b - kc * 34;
      if (r < 8) {
        const u32 so = kc * 64 * 32768 + dest * 4096 + r * 512 + wave * 128;
#pragma unroll
        for (int e = 0; e < 8; ++e)
          nv[e] = __builtin_amdgcn_raw_buffer_load_b128(rs_w, lw, so + ((e >> 2) * 32 + ((e >> 1) & 1) * 16 + (e & 1) * 4) * 32768, 0);
        return 8;
      } else {
        u32 rbk = r - 8 + rot; if (rbk >= NRB) rbk -= NRB;
        const u32 np = rbk == NRB - 1 ? 3 : 5;
        const u32 so = (rbk * 160 + wave * np * 8) * 1024 + kc * 128;
#pragma unroll
        for (int e = 0; e < 5; ++e)
          if (e < np) nv[e] = __builtin_amdgcn_raw_buffer_load_b128(rs_x, lx, so + e * 8192, 0);
        return np;
      }
    };
    u32 b = idx;
    const bool active = wave < 4 && idx < NCW;
    if (active && b < NBLK) load_blk(b);
    if (wave == 4) {
      ctl[lane] = 0u;
      ctl[64 + lane] = 0u;
      WAIT_LGKM();
    }
    BARRIER();
    if (wave == 4) {
      u32* const cptr = (u32*)(myws + WS_CALL) + blockIdx.x;
      call = __builtin_amdgcn_readfirstlane(ld_sys(cptr)) + 1;
      if (lane == 0) { lds_st(ctl + C_CALL, call); st_sys(cptr, call); }
    }
    if (wave >= 4) {
      if (wave == 4 && blockIdx.x == 0) {
        const u64* g1 = (const u64*)(myws + WS_G1);
        u64* g2 = (u64*)(P.ws[lane & 7] + WS_G2) + rank;
        asm volatile("" : "+v"(g2));
        u32 gm, sp = 0;
        for (;;) {
          u64 a0 = ld_sys64(g1 + lane), a1 = ld_sys64(g1 + 64 + lane), a2 = ld_sys64(g1 + 128 + lane);
          u64 a3 = ld_sys64(g1 + 192 + (lane & 15));
          const bool ok = (u32)(a0 >> 32) == call && (u32)(a1 >> 32) == call && (u32)(a2 >> 32) == call && (u32)(a3 >> 32) == call;
          gm = max(max((u32)a0, (u32)a1), max((u32)a2, (u32)a3));
          const u64 okm = __ballot(ok);
          if (okm == ~0ull) break;
          if (okm == 0) __builtin_amdgcn_s_sleep(8);
          if ((++sp & 63) == 0 && (ld_sys(tmo) == call || now() - t0 > DEADLINE)) { st_sys(tmo, call); return; }
        }
        gm = wave_umax(gm);
        if (lane < 8) st_sys64(g2, ((u64)call << 32) | gm);
      }
      return;
    }
    if (active) {
      WAIT_VM0();
      for (u32 n = 0; b < NBLK; b += NCW, ++n) {
        const u32 kc = b / 34, r = b - kc * 34;
        const u32 ord = kc * 8 + rank;
        u32 foff, doff, np;
        if (r < 8) {
          np = 8;
          doff = WS_W + (ord * NCB + r) * WBLK + wave * 8192;
          foff = WS_WF + r * FSTRIDE + ord * 4;
        } else {
          u32 rbk = r - 8 + rot; if (rbk >= NRB) rbk -= NRB;
          np = rbk == NRB - 1 ? 3 : 5;
          doff = WS_X + (ord * NRB + rbk) * XBLK + wave * np * 1024;
          foff = WS_XF + rbk * FSTRIDE + ord * 4;
        }
#pragma unroll
        for (int e = 0; e < 8; ++e) v[e] = nv[e];
#pragma unroll
        for (int e = 0; e < 8; ++e) {
          if (e < np) __builtin_amdgcn_raw_buffer_store_b128(v[e], rs_d, lane16, doff + e * 1024, 17);
          if (e == P1G - 1) {
            CFENCE();
            u32 nl = 0;
            if (b + NCW < NBLK) nl = load_blk(b + NCW);
            if (P1G < np) wait_vm_dyn(nl);
          } else if (e % P1G == P1G - 1 && e + 1 < np) {
            WAIT_VM(0);
          }
        }
        WAIT_VM0();
        while (call == 0) call = lds_ld(ctl + C_CALL);
        u32 old = 0;
        if (lane == 0) old = __hip_atomic_fetch_add(ctl + C_BLK + n, 1u, __ATOMIC_RELAXED, __HIP_MEMORY_SCOPE_WORKGROUP);
        if (__builtin_amdgcn_readfirstlane(old) == 3 && lane == 0) st_sys((u32*)(dws + foff), call);
        CFENCE();
      }
    }
    if (wave == 0) STAMP(1);
    if (wave != 0) return;
    while (call == 0) call = lds_ld(ctl + C_CALL);
  }

  if (idx < P3N) {
    const __amdgpu_buffer_rsrc_t rs_s = mkrsrc(dws, WS_END);
    const __amdgpu_buffer_rsrc_t rs_t = mkrsrc(P.out[rank], OUT_BYTES);
    {
      const u32* tc = (const u32*)(dws + WS_TCNT);
      u32 sp = 0;
      while ((int)(__builtin_amdgcn_readfirstlane(ld_sys(tc)) - call * NTILE) < 0) {
        __builtin_amdgcn_s_sleep(16);
        if ((++sp & 63) == 0 && (ld_sys(tmo) == call || now() - t0 > DEADLINE)) { st_sys(tmo, call); bad = true; break; }
      }
      CFENCE();
    }
    STAMP(9);
    constexpr u32 RU = 4 * P3D, NU = 4096 / RU;
    const u32 n2 = ((g & 1) ? 2080 : 2016) / RU;
    const u32 nu = 6 * NU + 2 * n2;
    const u32 lo = (lane >> 4) * 16384 + (lane & 15) * 16;
    u32 so, to;
    auto uoff = [&](u32 u) {
      u32 c, r;
      if (u < 6 * NU) { c = ((g & 6) + 2 + u / NU) & 7; r = u % NU; }
      else { const u32 k = u - 6 * NU; c = (g & 6) + (k >= n2); r = (k >= n2 ? k - n2 : k) + ((g & 1) ? 0 : 2080 / RU); }
      so = WS_Q + c * QSTRIDE + r * RU * 256;
      to = r * RU * 16384 + dest * 2048 + c * 256;
    };
    u32x4 v[P3D];
    u32 u0 = idx, u1 = P3N + idx;
    uoff(u0);
#pragma unroll
    for (int e = 0; e < P3D; ++e) v[e] = __builtin_amdgcn_raw_buffer_load_b128(rs_s, lane16, so + e * 1024, 17);
    WAIT_VM0();
    while (u0 < nu) {
      if (bad) {
#pragma unroll
        for (int e = 0; e < P3D; ++e) v[e] = u32x4{0x80808080u, 0x80808080u, 0x80808080u, 0x80808080u};
      }
#pragma unroll
      for (int e = 0; e < P3D; ++e) __builtin_amdgcn_raw_buffer_store_b128(v[e], rs_t, lo, to + e * 65536, 17);
      CFENCE();
      if (u1 < nu) {
        uoff(u1);
#pragma unroll
        for (int e = 0; e < P3D; ++e) v[e] = __builtin_amdgcn_raw_buffer_load_b128(rs_s, lane16, so + e * 1024, 17);
      }
      u32 tk = 0;
      if (lane == 0)
        tk = __hip_atomic_fetch_add((GAS u32*)(myws + WS_TK3 + g * 128), 1u, __ATOMIC_RELAXED, __HIP_MEMORY_SCOPE_AGENT);
      WAIT_VM0();
      u0 = u1;
      u1 = 2 * P3N + (__builtin_amdgcn_readfirstlane(tk) - (call - 1) * nu);
    }
  }
  WAIT_VM(0);
  while (lds_ld(ctl + C_REM) < nrem) __builtin_amdgcn_s_sleep(1);
  CFENCE();
  if (g != 0) {
    if (lane == 0) st_sys((u32*)(dws + WS_DONE + idx * 128) + rank, call);
  } else {
    const u32* df = (const u32*)(myws + WS_DONE + idx * 128);
    u32 sp = 0;
    for (;;) {
      const u32 val = ld_sys(df + (lane & 7));
      const bool ok = ((lane & 7) == rank) || (int)(val - call) >= 0;
      if (__all(ok)) break;
      __builtin_amdgcn_s_sleep(4);
      if ((++sp & 63) == 0 && (ld_sys(tmo) == call || now() - t0 > DEADLINE)) { st_sys(tmo, call); bad = true; break; }
    }
    if (bad && lane == 0) P.out[rank][(size_t)idx * 16384 + rank * 2048] = (char)-128;
  }
  STAMP(5);
}

extern "C" void kernel_launch(int rank, int nranks, ncclComm_t comm, hipStream_t stream,
                              void* const* d_in, const int* in_sizes, int n_in,
                              void* d_out, int out_size,
                              const int* in_dtypes, int out_dtype,
                              void* const* d_out_all,
                              void* const* d_ws_all, size_t ws_bytes) {
  if (nranks != NR || ws_bytes < WS_END || n_in < 2) { fprintf(stderr, "unsupported configuration\n"); return; }
  Params P;
  P.x = (const char*)d_in[0];
  P.w = (const char*)d_in[1];
  for (int p = 0; p < NR; ++p) { P.out[p] = (char*)d_out_all[p]; P.ws[p] = (char*)d_ws_all[p]; }
  P.rank = rank;
  const int phys[8] = {0, 2, 3, 6, 1, 4, 5, 7}, logi[8] = {0, 4, 1, 2, 5, 6, 3, 7};
  P.peers = 0;
  for (int g = 0; g < NR; ++g) P.peers |= (u32)phys[logi[rank & 7] ^ g] << (g * 4);
  hipLaunchKernelGGL(fused_gemm_ar_q8, dim3(256), dim3(NTHREADS), 0, stream, P);
  HIP_CHECK(hipGetLastError());
}
```

```cpp
#include <hip/hip_runtime.h>
#include <rccl/rccl.h>
#include <cstdio>
#include <stdint.h>

#define HIP_CHECK(expr) do { hipError_t _e = (expr); if (_e != hipSuccess) { \
    fprintf(stderr, "HIP error %s at %s:%d\n", hipGetErrorString(_e), __FILE__, __LINE__); \
    return; } } while (0)

typedef unsigned u32;
typedef unsigned long long u64;
typedef u32 u32x4 __attribute__((ext_vector_type(4)));
typedef u32 u32x2 __attribute__((ext_vector_type(2)));
typedef float f32x4 __attribute__((ext_vector_type(4)));
typedef short s16x4 __attribute__((ext_vector_type(4)));
typedef short s16x8 __attribute__((ext_vector_type(8)));
typedef __bf16 bf16x8 __attribute__((ext_vector_type(8)));
#define LAS __attribute__((address_space(3)))
#define GAS __attribute__((address_space(1)))

constexpr int NR = 8;
constexpr u32 OUT_BYTES = 4096u * 16384u;
constexpr u32 X_BYTES = 4096u * 512u * 2u;
constexpr u32 W_BYTES = 512u * 16384u * 2u;

constexpr u32 NRB = 26;
constexpr u32 NCB = 8;
constexpr u32 XBLK = 20480, WBLK = 32768;
constexpr u32 NWB = NCB + 2;
constexpr u32 WS_W = 0;
constexpr u32 WS_X = 20u << 20;
constexpr u32 WS_CTL = 53u << 20;
constexpr u32 FSTRIDE = 4352;
constexpr u32 WS_XF = WS_CTL;
constexpr u32 WS_WF = WS_XF + NRB * FSTRIDE;
constexpr u32 CNT_OFF = 512u << 10;
constexpr u32 WS_C2 = WS_CTL + (160u << 10);
constexpr u32 WS_CALL = WS_C2;
constexpr u32 WS_G1 = WS_C2 + 4096;
constexpr u32 WS_G2 = WS_C2 + 8192 + 256;
constexpr u32 G2S = 512;
constexpr u32 WS_TK = WS_C2 + 13312;
constexpr u32 WS_TMO = WS_C2 + 20480;
constexpr u32 WS_DBG = WS_C2 + 32768;
constexpr u32 WS_FL = WS_C2 + 65536;
constexpr u32 WS_FC = WS_FL + 16384;
constexpr u32 WS_Q = 54u << 20;
constexpr u32 QSTRIDE = 4160 * 256;
constexpr u32 WS_EX = 63u << 20;
constexpr u32 WS_ER = 67u << 20;
constexpr u32 WS_EW = 68u << 20;
constexpr u32 WS_L = 84u << 20;
constexpr u32 WS_END = 86u << 20;
static_assert(64u * NWB * WBLK <= WS_X && WS_X + 64u * NRB * XBLK <= WS_CTL && WS_WF + NWB * FSTRIDE <= WS_C2 &&
              WS_FC + 16384 <= WS_CTL + CNT_OFF && WS_C2 + CNT_OFF <= WS_Q && WS_Q + NCB * QSTRIDE <= WS_EX, "layout");
constexpr u32 FULL = 240;

constexpr u32 SLOT_BYTES = WBLK + XBLK;
constexpr u32 NSLOT = 3;
constexpr u32 TPITCH = 272;
constexpr u32 SNAP_OFF = SLOT_BYTES * NSLOT;
constexpr u32 CTL_OFF = SNAP_OFF + 3072;
constexpr u32 LDS_BYTES = CTL_OFF + 1024;
static_assert(LDS_BYTES <= 163840, "lds");
constexpr int C_AMAX = 0, C_GMAX = 1, C_CALL = 2, C_BAD = 3, C_STAGE = 4, C_CNT = 8;

#define NPW 4
#define EXWAIT 300
#define EXD 60
#define PW1 3
#define QWAIT 500
#define QD 100
#define PUSHD 2
constexpr u32 NCOMMWG = 32 - NRB;
constexpr u32 NTILE = NRB * NCB;
constexpr u32 NPULL = NCOMMWG * NPW;
constexpr u32 U_B = 512, U_C = 1152;
constexpr u32 U_R = 256 + 7 * 384;
constexpr u32 U_Q = U_R + 224;
constexpr int NTHREADS = 512;
constexpr int NSTEP = 64;
constexpr u64 DEADLINE = 50000000ull;

struct Params {
  const char* x;
  const char* w;
  char* out[NR];
  char* ws[NR];
  int rank;
  u32 peers;
};

__device__ __forceinline__ __amdgpu_buffer_rsrc_t mkrsrc(const void* p, u32 bytes) {
  return __builtin_amdgcn_make_buffer_rsrc((void*)p, 0, bytes, 0x00020000);
}
__device__ __forceinline__ u64 now() { return __builtin_amdgcn_s_memrealtime(); }
__device__ __forceinline__ u32 ld_sys(const u32* p) { return __hip_atomic_load((GAS const u32*)p, __ATOMIC_RELAXED, __HIP_MEMORY_SCOPE_SYSTEM); }
__device__ __forceinline__ u64 ld_sys64(const u64* p) { return __hip_atomic_load((GAS const u64*)p, __ATOMIC_RELAXED, __HIP_MEMORY_SCOPE_SYSTEM); }
__device__ __forceinline__ void st_sys(u32* p, u32 v) { __hip_atomic_store((GAS u32*)p, v, __ATOMIC_RELAXED, __HIP_MEMORY_SCOPE_SYSTEM); }
__device__ __forceinline__ void st_sys64(u64* p, u64 v) { __hip_atomic_store((GAS u64*)p, v, __ATOMIC_RELAXED, __HIP_MEMORY_SCOPE_SYSTEM); }
__device__ __forceinline__ u32 lds_ld(LAS u32* p) { return __builtin_amdgcn_readfirstlane(__hip_atomic_load(p, __ATOMIC_RELAXED, __HIP_MEMORY_SCOPE_WORKGROUP)); }
__device__ __forceinline__ void lds_st(LAS u32* p, u32 v) { __hip_atomic_store(p, v, __ATOMIC_RELAXED, __HIP_MEMORY_SCOPE_WORKGROUP); }
__device__ __forceinline__ void lds_add1(LAS u32* p, u32 lane) {
  if (lane == 0) __hip_atomic_fetch_add(p, 1u, __ATOMIC_RELAXED, __HIP_MEMORY_SCOPE_WORKGROUP);
}
#define CFENCE() asm volatile("" ::: "memory")
#define WAIT_VM(n) asm volatile("s_waitcnt vmcnt(" #n ")" ::: "memory")
#define WAIT_VMI(n) asm volatile("s_waitcnt vmcnt(%0)" :: "i"(n) : "memory")
#define WAIT_LGKM() asm volatile("s_waitcnt lgkmcnt(0)" ::: "memory")
#define WAIT_VM0() do { CFENCE(); __builtin_amdgcn_s_waitcnt(0x0F70); WAIT_VM(0); } while (0)

__device__ __forceinline__ u32 wave_umax(u32 v) {
  v = max(v, (u32)__builtin_amdgcn_update_dpp(0, (int)v, 0x111, 0xf, 0xf, true));
  v = max(v, (u32)__builtin_amdgcn_update_dpp(0, (int)v, 0x112, 0xf, 0xf, true));
  v = max(v, (u32)__builtin_amdgcn_update_dpp(0, (int)v, 0x114, 0xf, 0xf, true));
  v = max(v, (u32)__builtin_amdgcn_update_dpp(0, (int)v, 0x118, 0xf, 0xf, true));
  v = max(v, (u32)__builtin_amdgcn_update_dpp(0, (int)v, 0x142, 0xa, 0xf, false));
  v = max(v, (u32)__builtin_amdgcn_update_dpp(0, (int)v, 0x143, 0xc, 0xf, false));
  return (u32)__builtin_amdgcn_readlane((int)v, 63);
}

#define STAMP(k) do {} while (0)

__device__ __forceinline__ void wait_vm_dyn(u32 n) {
  asm volatile(
      "s_getpc_b64 vcc\n\t"
      "s_add_u32 vcc_lo, vcc_lo, %0\n\t"
      "s_addc_u32 vcc_hi, vcc_hi, 0\n\t"
      "s_setpc_b64 vcc\n\t"
      "s_waitcnt vmcnt(0)\n\ts_branch 1f\n\t"
      "s_waitcnt vmcnt(1)\n\ts_branch 1f\n\t"
      "s_waitcnt vmcnt(2)\n\ts_branch 1f\n\t"
      "s_waitcnt vmcnt(3)\n\ts_branch 1f\n\t"
      "s_waitcnt vmcnt(4)\n\ts_branch 1f\n\t"
      "s_waitcnt vmcnt(5)\n\ts_branch 1f\n\t"
      "s_waitcnt vmcnt(6)\n\ts_branch 1f\n\t"
      "s_waitcnt vmcnt(7)\n\ts_branch 1f\n\t"
      "s_waitcnt vmcnt(8)\n\ts_branch 1f\n\t"
      "s_waitcnt vmcnt(9)\n\ts_branch 1f\n\t"
      "s_waitcnt vmcnt(10)\n\ts_branch 1f\n\t"
      "s_waitcnt vmcnt(11)\n\ts_branch 1f\n\t"
      "s_waitcnt vmcnt(12)\n\ts_branch 1f\n\t"
      "s_waitcnt vmcnt(13)\n\ts_branch 1f\n\t"
      "s_waitcnt vmcnt(14)\n\ts_branch 1f\n\t"
      "s_waitcnt vmcnt(15)\n\ts_branch 1f\n\t"
      "s_waitcnt vmcnt(16)\n\ts_branch 1f\n\t"
      "1:"
      :: "s"(min(n, 17u) * 8 + 12) : "memory", "scc", "vcc");
}
#define BARRIER() asm volatile("s_barrier" ::: "memory")

__global__ void __launch_bounds__(NTHREADS) fused_gemm_ar_q8(Params P) {
  __shared__ __attribute__((aligned(16))) char smem[LDS_BYTES];
  LAS char* L = (LAS char*)smem;
  LAS u32* ctl = (LAS u32*)(L + CTL_OFF);
  const u32 tid = threadIdx.x, lane = tid & 63;
  const u32 wave = __builtin_amdgcn_readfirstlane(tid >> 6);
  const u32 g = blockIdx.x & 7u, idx = blockIdx.x >> 3;
  const u32 rank = (u32)P.rank & 7u;
  const u32 dest = (P.peers >> (g * 4)) & 7u;
  char* const myws = P.ws[rank];
  char* const dws = P.ws[dest];
  u32* const tmo = (u32*)(myws + WS_TMO);
  const u32 lane16 = lane * 16;
  const u64 t0 = now();
  const u32 lx = (lane & 7) * 1024 + (lane >> 3) * 16;
  const u32 lw = (((lane >> 3) & 1) * 8 + ((lane >> 1) & 3)) * 32768 + ((lane & 1) * 4 + (lane >> 4)) * 16;
  u32 call = 0;
  bool bad = false;
  if (wave == 0) STAMP(0);

  if (idx >= NCOMMWG) {
    const u32 ti = idx - NCOMMWG;
    const u32 rb = (g & 1) * 13 + (ti >> 1);
    u32 cb = (g >> 1) * 2 + (ti & 1);
    const u32 tile = g * NRB + ti;
    const __amdgpu_buffer_rsrc_t rs_ws = mkrsrc(myws, WS_END);
    char* flp = P.ws[lane & 7] + WS_FL + rank * 256 + g * 4;
    const u32 lane0 = lane ? 0x80000000u : 0u;
    auto signal_wg = [&](u32 kind, u32 inc) {
      const u32 old = __builtin_amdgcn_raw_ptr_buffer_atomic_add_i32(inc, rs_ws, lane0, WS_FC + kind * 2048 + g * 256, 16);
      if (__builtin_amdgcn_readfirstlane(old) + inc == call * NRB && lane < 8)
        st_sys((u32*)(flp + kind * 2048), call);
      CFENCE();
    };
    auto signal_all = [&](u32 kind, u32 inc = 1) {
      u32 old = 0;
      if (lane == 0) old = __hip_atomic_fetch_add(ctl + C_CNT + kind, 1u, __ATOMIC_RELAXED, __HIP_MEMORY_SCOPE_WORKGROUP);
      if (__builtin_amdgcn_readfirstlane(old) == 7) signal_wg(kind, inc);
    };
    {
      const u32 pc = tile * 5 + wave;
      const bool pok = wave < 5 && pc < 1024;
      const u32x4 e0 = __builtin_amdgcn_raw_buffer_load_b128(mkrsrc(P.x, X_BYTES), lx,
                                                             pok ? (pc & 511) * 8192 + (pc >> 9) * 128 : 0x80000000u, 0);
      CFENCE();
      if (wave == 6 && ti == 0) __builtin_amdgcn_fence(__ATOMIC_ACQUIRE, "");
      if (wave == 7) {
        u32* const cptr = (u32*)(myws + WS_CALL) + blockIdx.x;
        call = __builtin_amdgcn_readfirstlane(ld_sys(cptr)) + 1;
        if (lane < 16) ctl[lane] = lane == C_CALL ? call : 0u;
        if (lane == 0) st_sys(cptr, call);
        WAIT_LGKM();
      }
      __builtin_amdgcn_raw_buffer_store_b128(e0, rs_ws, lane16, pok ? WS_EX + pc * 1024 : 0x80000000u, 17);
      asm volatile("" : "+v"(flp));
      BARRIER();
      call = lds_ld(ctl + C_CALL);
      WAIT_VM0();
      signal_all(0);
      for (u32 r = 0; r < 3; ++r) {
        const u32 slot = r * 8 + wave, e = slot * NTILE + tile;
        u32 kc, k, so, eo;
        bool isw = true;
        if (e < 4352) {
          kc = (e + 768) / 640; k = e + 768 - kc * 640;
          if (kc == 1) k -= 128;
          isw = k < 512;
          k &= 511;
          eo = isw ? WS_EW + ((k >> 6) * 8 + kc) * (8 * WBLK) + (k & 63) * 4096 : WS_EX + kc * (512u << 10) + k * 4096;
        } else {
          kc = (e - 4352) >> 4; k = ((e >> 1) & 7) * 64 + 62 + (e & 1);
          eo = WS_ER + (call & 1) * (512u << 10) + (e - 4352) * 4096;
        }
        if (e < 4480) {
          so = isw ? (kc * 64 + (k & 1) * 32) * 32768 + (k >> 1) * 128 : k * 32768 + kc * 128;
          const __amdgpu_buffer_rsrc_t rs = mkrsrc(isw ? P.w : P.x, isw ? W_BYTES : X_BYTES);
          const u32 lo = isw ? lw : lx, s1 = isw ? 4 * 32768 : 8192, s2 = isw ? 16 * 32768 : 16384;
          while (now() - t0 < EXWAIT + slot * EXD) __builtin_amdgcn_s_sleep(2);
          const u32x4 e0 = __builtin_amdgcn_raw_buffer_load_b128(rs, lo, so, 0);
          const u32x4 e1 = __builtin_amdgcn_raw_buffer_load_b128(rs, lo, so + s1, 0);
          const u32x4 e2 = __builtin_amdgcn_raw_buffer_load_b128(rs, lo, so + s2, 0);
          const u32x4 e3 = __builtin_amdgcn_raw_buffer_load_b128(rs, lo, so + s1 + s2, 0);
          __builtin_amdgcn_raw_buffer_store_b128(e0, rs_ws, lane16, eo, 17);
          __builtin_amdgcn_raw_buffer_store_b128(e1, rs_ws, lane16, eo + 1024, 17);
          __builtin_amdgcn_raw_buffer_store_b128(e2, rs_ws, lane16, eo + 2048, 17);
          __builtin_amdgcn_raw_buffer_store_b128(e3, rs_ws, lane16, eo + 3072, 17);
        }
        if (r != 1) {
          WAIT_VM0();
          signal_all(1 + (r >> 1));
        }
      }
      BARRIER();
    }
    const u32 cv = call * FULL;
    const u32 wm = wave >> 2, wn = wave & 3;
    const u32 lds0 = (u32)(size_t)L;
    const u32 xa = WBLK + wm * 10240 + ((lane >> 3) & 1) * 1024 + (lane >> 5) * 256 + ((lane >> 4) & 1) * 128 + (lane & 7) * 16;
    const u32 h = lane >> 4, q = (lane >> 2) & 3, p = lane & 3;
    const u32 wa0 = wn * 8192 + (h >> 1) * 2048 + (p & 1) * 512 + (h & 1) * 128 + q * 32 + (p >> 1) * 16 + (p & 1) * 8;
    const u32 wa1 = wa0 ^ 8;
    const u32 xpo = wave < 4 ? wave * 3072 : 12288 + (wave - 4) * 2048;
    u32 pollo;
    f32x4 acc[4][5];
    u64 kx = 0, kw, taken;
    u32 issued, consumed, nis = 0;
    u32 islot, cslot, sj;
    u64 markD = 0, markP = 0;
    float inv = 0.f;
    auto sched = [&](u32 fx, u32 fw) {
      kx |= __ballot((int)(fx - cv) >= 0);
      kw |= __ballot((int)(fw - cv) >= 0);
      u64 avail = kx & kw & ~taken;
      while (avail && issued - consumed < NSLOT) {
        const u32 ord = __builtin_ctzll(avail);
        avail &= avail - 1;
        taken |= 1ull << ord;
        const u32 sw = WS_W + (ord * NWB + cb) * WBLK + wave * 4096;
        const u32 sx = WS_X + (ord * NRB + rb) * XBLK + xpo;
        const u32 mw = lds0 + islot + wave * 4096, mx = lds0 + islot + WBLK + xpo;
        u32 keep;
        asm volatile(
            "s_mov_b32 %0, m0\n\t"
            "s_mov_b32 m0, %3\n\t"
            "s_nop 0\n\t"
            "buffer_load_dwordx4 %1, %2, %5 offen lds\n\t"
            "buffer_load_dwordx4 %1, %2, %5 offen offset:1024 lds\n\t"
            "buffer_load_dwordx4 %1, %2, %5 offen offset:2048 lds\n\t"
            "buffer_load_dwordx4 %1, %2, %5 offen offset:3072 lds\n\t"
            "s_mov_b32 m0, %4\n\t"
            "s_nop 0\n\t"
            "buffer_load_dwordx4 %1, %2, %6 offen lds\n\t"
            "buffer_load_dwordx4 %1, %2, %6 offen offset:1024 lds\n\t"
            "s_cmp_gt_u32 %7, 3\n\t"
            "s_cbranch_scc1 1f\n\t"
            "buffer_load_dwordx4 %1, %2, %6 offen offset:2048 lds\n\t"
            "1:\n\t"
            "s_mov_b32 m0, %0"
            : "=&s"(keep)
            : "v"(lane16), "s"(rs_ws), "s"(mw), "s"(mx), "s"(sw), "s"(sx), "s"(wave)
            : "memory", "scc");
        nis += wave < 4 ? 7 : 6;
        const u32 sh = (issued & 3) * 16;
        markD = (markD & ~(0xffffull << sh)) | ((u64)(nis & 0xffff) << sh);
        issued += 1;
        islot = islot == 2 * SLOT_BYTES ? 0 : islot + SLOT_BYTES;
      }
    };
    auto poll = [&]() {
      if (wave == 0) {
        const u64 kn = lane < 16 ? kx : kw;
        const bool act = lane < 32 && ((u32)(kn >> ((lane & 15) * 4)) & 15u) != 15u;
        const u32 po = act ? pollo : 0x80000000u;
        u32 keep;
        asm volatile("s_mov_b32 %0, m0\n\ts_mov_b32 m0, %3\n\ts_nop 0\n\t"
                     "buffer_load_dwordx4 %1, %2, 0 offen sc0 sc1 lds\n\ts_mov_b32 m0, %0"
                     : "=&s"(keep) : "v"(po), "s"(rs_ws), "s"(lds0 + SNAP_OFF + sj * 1024) : "memory");
        nis += 1;
        markP = (markP & ~(0xffffull << (sj * 16))) | ((u64)(nis & 0xffff) << (sj * 16));
      }
      sj = sj == 2 ? 0 : sj + 1;
    };
    auto wait_step = [&]() {
      u32 need = (nis - (u32)(markD >> ((consumed & 3) * 16))) & 0xffff;
      if (wave == 0) need = min(need, (nis - (u32)(markP >> ((sj == 2 ? 0 : sj + 1) * 16))) & 0xffff);
      wait_vm_dyn(need);
    };
#define SB() __builtin_amdgcn_sched_barrier(0)
#define PIECE(M, S, OFF) do { \
        asm volatile("s_cmp_lg_u32 %2, -1\n\ts_cbranch_scc0 1f\n\ts_mov_b32 m0, %2\n\ts_nop 0\n\t" \
                     "buffer_load_dwordx4 %0, %1, %3 offen offset:" #OFF " lds\n\t1:" \
                     :: "v"(lane16), "s"(rs_ws), "s"(M), "s"(S) : "memory", "scc"); \
        SB(); } while (0)
#define MMA4(kb, i) do { \
        _Pragma("unroll") for (int n = 0; n < 4; ++n) \
          acc[n][i] = __builtin_amdgcn_mfma_f32_16x16x32_bf16(wf[kb][n], xf[kb][i], acc[n][i], 0, 0, 0); \
        SB(); } while (0)
    bf16x8 xf[2][5], wf[2][4];
    u32 sp = 0;
    u64 cyc0 = 0;
    for (u32 pass = 0;; ++pass) {
    pollo = lane < 16 ? WS_XF + rb * FSTRIDE + lane16 : WS_WF + cb * FSTRIDE + lane16 - 256;
    kw = taken = 0;
    issued = consumed = islot = cslot = sj = 0;
#pragma unroll
    for (int a = 0; a < 4; ++a)
#pragma unroll
      for (int b = 0; b < 5; ++b) acc[a][b] = f32x4{0.f, 0.f, 0.f, 0.f};
#pragma unroll
    for (int n = 0; n < 5; ++n) {
      xf[1][n] = __builtin_bit_cast(bf16x8, u32x4{0u, 0u, 0u, 0u});
      if (n < 4) wf[1][n] = __builtin_bit_cast(bf16x8, u32x4{0u, 0u, 0u, 0u});
    }
    poll(); poll();
    while (consumed < NSTEP) {
      if (issued == consumed) {
        do {
          const u32 srd = sj == 2 ? 0 : sj + 1;
          if (wave == 0) {
            wait_vm_dyn((nis - (u32)(markP >> (srd * 16))) & 0xffff);
            if ((++sp & 63) == 0 && now() - t0 > DEADLINE) {
              st_sys(tmo, call);
              WAIT_VM(0);
              lds_st(ctl + C_BAD, 1u);
              *(LAS u32x2*)(L + SNAP_OFF + srd * 1024 + lane * 8) = u32x2{cv, cv};
              WAIT_LGKM();
            }
          }
          BARRIER();
          poll();
          LAS u32* const snap = (LAS u32*)(L + SNAP_OFF + srd * 1024) + lane;
          sched(snap[0], snap[64]);
        } while (issued == consumed);
        wait_step();
      }
      BARRIER();
      LAS u32* const snap = (LAS u32*)(L + SNAP_OFF + (sj == 2 ? 0 : sj + 1) * 1024) + lane;
      const u32 fx = snap[0], fw = snap[64];
      SB();
      LAS char* base = L + cslot;
      auto frags = [&](int kb) {
#pragma unroll
        for (int n = 0; n < 5; ++n) {
          if (n < 4) {
            LAS char* a = base + ((n & 1) ? wa1 : wa0) + kb * 4096 + (n >> 1) * 256;
            s16x4 lo = __builtin_amdgcn_ds_read_tr16_b64_v4i16((LAS s16x4*)a);
            s16x4 hi = __builtin_amdgcn_ds_read_tr16_b64_v4i16((LAS s16x4*)(a + 1024));
            wf[kb][n] = __builtin_bit_cast(bf16x8, __builtin_shufflevector(lo, hi, 0, 1, 2, 3, 4, 5, 6, 7));
          }
          xf[kb][n] = *(LAS bf16x8*)(base + xa + n * 2048 + kb * 512);
        }
        SB();
      };
      MMA4(1, 0);
      frags(0);
      poll();
      SB();
      MMA4(1, 1);
      kx |= __ballot((int)(fx - cv) >= 0);
      kw |= __ballot((int)(fw - cv) >= 0);
      const u64 av = kx & kw & ~taken;
      const bool go = av != 0 && issued - consumed < NSLOT;
      const u32 ord = __builtin_ctzll(av | (1ull << 63));
      const u32 sw = WS_W + (ord * NWB + cb) * WBLK + wave * 4096;
      const u32 sx = WS_X + (ord * NRB + rb) * XBLK + xpo;
      u32 mw = ~0u, mx = ~0u, mx3 = ~0u;
      if (go) {
        mw = lds0 + islot + wave * 4096;
        mx = lds0 + islot + WBLK + xpo;
        if (wave < 4) mx3 = mx;
        taken |= 1ull << ord;
        nis += wave < 4 ? 7 : 6;
        const u32 sh = (issued & 3) * 16;
        markD = (markD & ~(0xffffull << sh)) | ((u64)(nis & 0xffff) << sh);
        issued += 1;
        islot = islot == 2 * SLOT_BYTES ? 0 : islot + SLOT_BYTES;
      }
      SB();
      MMA4(1, 2);
      PIECE(mw, sw, 0);
      MMA4(1, 3);
      PIECE(mw, sw, 1024);
      MMA4(1, 4);
      frags(1);
      PIECE(mw, sw, 2048);
      MMA4(0, 0);
      PIECE(mw, sw, 3072);
      PIECE(mx, sx, 0);
      MMA4(0, 1);
      PIECE(mx, sx, 1024);
      PIECE(mx3, sx, 2048);
      MMA4(0, 2);
      sched(cv - 1u, cv - 1u);
      cslot = cslot == 2 * SLOT_BYTES ? 0 : cslot + SLOT_BYTES;
      consumed += 1;
      SB();
      MMA4(0, 3);
      if (issued != consumed) wait_step();
      SB();
      MMA4(0, 4);
      WAIT_LGKM();
    }
    MMA4(1, 0); MMA4(1, 1); MMA4(1, 2); MMA4(1, 3); MMA4(1, 4);
    if (pass == 0) {
      if (wave == 0) STAMP(2);
      const u32 row0 = rb * 160 + wm * 80;
      float m = 0.f;
#pragma unroll
      for (int b = 0; b < 5; ++b) {
        float mb = 0.f;
#pragma unroll
        for (int a = 0; a < 4; ++a)
#pragma unroll
          for (int e = 0; e < 4; ++e) mb = fmaxf(mb, fabsf(acc[a][b][e]));
        if (row0 + b * 16 < 4096) m = fmaxf(m, mb);
      }
      const u32 mb = wave_umax(__builtin_bit_cast(u32, m));
      if (lane == 0) __hip_atomic_fetch_max(ctl + C_AMAX, mb, __ATOMIC_RELAXED, __HIP_MEMORY_SCOPE_WORKGROUP);
      WAIT_LGKM();
      BARRIER();
      if (wave == 0) {
        const u32 wmax = lds_ld(ctl + C_AMAX);
        if (lane == 0) st_sys64((u64*)(myws + WS_G1) + tile, ((u64)call << 32) | wmax);
        STAMP(3);
        const u64* g2 = (const u64*)(myws + WS_G2 + g * G2S) + (lane & 7);
        u64 val;
        for (;;) {
          val = ld_sys64(g2);
          if (__all((u32)(val >> 32) == call)) break;
          if ((++sp & 63) == 0 && (ld_sys(tmo) == call || now() - t0 > DEADLINE)) { st_sys(tmo, call); bad = true; break; }
        }
        const u32 gmax = wave_umax((u32)val);
        if (lane == 0) {
          lds_st(ctl + C_GMAX, gmax);
          if (bad) lds_st(ctl + C_BAD, 1u);
        }
        WAIT_LGKM();
      }
    }
    BARRIER();
    if (pass == 0) {
      const float amax = __builtin_bit_cast(float, lds_ld(ctl + C_GMAX));
      const float scale = amax / 127.0f;
      inv = amax > 0.f ? 1.0f / scale : 0.f;
      if (wave == 0) STAMP(4);
    }
    bad = lds_ld(ctl + C_BAD) != 0;
    const bool odd = (lane >> 4) & 1;
    u32x4 qv[5];
#pragma unroll
    for (int i = 0; i < 5; ++i) {
      u32 d[4];
#pragma unroll
      for (int n = 0; n < 4; ++n) {
        u32 b0 = __builtin_bit_cast(u32, __builtin_fmaf(acc[n][i][0], inv, 12582912.0f));
        u32 b1 = __builtin_bit_cast(u32, __builtin_fmaf(acc[n][i][1], inv, 12582912.0f));
        u32 b2 = __builtin_bit_cast(u32, __builtin_fmaf(acc[n][i][2], inv, 12582912.0f));
        u32 b3 = __builtin_bit_cast(u32, __builtin_fmaf(acc[n][i][3], inv, 12582912.0f));
        u32 lo = __builtin_amdgcn_perm(b1, b0, 0x0c0c0400u);
        u32 hi = __builtin_amdgcn_perm(b3, b2, 0x04000c0cu);
        d[n] = bad ? 0x80808080u : (lo | hi);
      }
      qv[i] = u32x4{odd ? d[1] : d[0], odd ? d[0] : d[1], odd ? d[3] : d[2], odd ? d[2] : d[3]};
    }
    const u64 tq = now();
    if (!pass && !(cb & 1) && wm == 0) {
      const __amdgpu_buffer_rsrc_t rs_p = mkrsrc(dws, WS_END);
#pragma unroll
      for (int i = 0; i < 4; ++i) {
        if (i && i % PUSHD == 0) WAIT_VM(0);
        if (wave == 0 && i == PUSHD) STAMP(10);
        __builtin_amdgcn_raw_buffer_store_b128(qv[i], rs_p, lane16, WS_L + ((g * 13 + (ti >> 1)) * 16 + wn * 4 + i) * 1024, 17);
        CFENCE();
      }
      WAIT_VM0();
      if (wave == 0) STAMP(11);
      const u32 old = __builtin_amdgcn_raw_ptr_buffer_atomic_add_i32(1, rs_ws, lane0, WS_FC + 5 * 2048 + g * 256, 16);
      if (__builtin_amdgcn_readfirstlane(old) + 1 == call * 52 && lane == 0) st_sys((u32*)(dws + WS_FL + 5 * 2048) + g, call);
      CFENCE();
    }
    {
      LAS char* tw = L + (wm * 80 + (lane & 15)) * TPITCH + wn * 64 + (lane >> 4) * 16;
#pragma unroll
      for (int i = 0; i < 5; ++i) *(LAS u32x4*)(tw + i * 16 * TPITCH) = qv[i];
    }
    WAIT_LGKM();
    BARRIER();
    LAS char* tr = L + (wave * 4 + (lane >> 4)) * TPITCH + (lane & 15) * 16;
#pragma unroll
    for (int i = 0; i < 5; ++i) qv[i] = *(LAS u32x4*)(tr + i * 32 * TPITCH);
    const __amdgpu_buffer_rsrc_t rs_o = mkrsrc(P.out[rank], OUT_BYTES);
    const u32 orow = (rb * 160 + wave * 4 + (lane >> 4)) * 16384u;
    if (pass) {
      const u32 piece = (cb - 8) * 4 + ((lane >> 2) & 3);
      const u32 oo = piece == rank ? 0x80000000u : orow + piece * 2048 + 1984 + (lane & 3) * 16;
#pragma unroll
      for (int i = 0; i < 5; ++i) __builtin_amdgcn_raw_buffer_store_b128(qv[i], rs_o, oo, i * 32 * 16384, 17);
      break;
    }
    const u32 pitch = cb == 7 ? 192 : 256;
    const bool rep = (lane & 15) * 16 >= pitch;
    const u32 qo = rep ? 0x80000000u : (wave * 4 + (lane >> 4)) * pitch + (lane & 15) * 16;
    const u32 oo = orow + rank * 2048 + cb * 256 + (lane & 15) * 16;
    const u32 qa = WS_Q + cb * QSTRIDE + rb * 64 * pitch, qb = WS_Q + cb * QSTRIDE + (1664 + rb * 96) * pitch;
    if (cb & 1) {
      __builtin_amdgcn_raw_buffer_store_b128(qv[0], rs_ws, qo, qa, 17);
      __builtin_amdgcn_raw_buffer_store_b128(qv[1], rs_ws, qo, qa + 32 * pitch, 17);
      WAIT_VM0();
      signal_all(3, 2);
      if (wave == 0) STAMP(9);
    }
    while (now() - tq < QWAIT + wave * QD) __builtin_amdgcn_s_sleep(2);
    if (!(cb & 1)) {
      __builtin_amdgcn_raw_buffer_store_b128(qv[0], rs_ws, qo, qa, 17);
      __builtin_amdgcn_raw_buffer_store_b128(qv[1], rs_ws, qo, qa + 32 * pitch, 17);
    }
#pragma unroll
    for (int i = 2; i < 5; ++i) __builtin_amdgcn_raw_buffer_store_b128(qv[i], rs_ws, qo, qb + (i - 2) * 32 * pitch, 17);
    WAIT_VM0();
    signal_all(4);
    if (cb == 7) {
#pragma unroll
      for (int i = 0; i < 5; ++i) __builtin_amdgcn_raw_buffer_store_b128(qv[i], rs_o, rep ? oo : 0x80000000u, i * 32 * 16384, 17);
    }
    if (((ti >> 1) & 3) != (g >> 1)) break;
    cb = 8 + (ti & 1);
    WAIT_VM0();
    WAIT_LGKM();
    BARRIER();
    }
    WAIT_VM(0);
    if (wave == 0) STAMP(5);
    return;
  }

  const u32 vid = idx * NPW + wave;
  static_assert(NPULL == 24, "three waves per pushed block");
  const u32 pb = vid / 3, p0 = (vid - pb * 3) * 11;
  u32x4 pv[11];
  if (wave < NPW) {
    const __amdgpu_buffer_rsrc_t rs_w = mkrsrc(P.w, W_BYTES);
#pragma unroll
    for (int i = 0; i < 11; ++i) {
      const u32 e = p0 + i;
      const u32 so = dest * 4096 + pb * 512 + (e >> 3) * 128 + (((e >> 1) & 3) * 16 + (e & 1) * 4) * 32768;
      pv[i] = __builtin_amdgcn_raw_buffer_load_b128(rs_w, lw, e < 32 ? so : 0x80000000u, 0);
    }
    CFENCE();
  }
  if (wave == 7) {
    ctl[lane] = 0u;
    WAIT_LGKM();
  }
  BARRIER();
  if (wave == 7) {
    u32* const cptr = (u32*)(myws + WS_CALL) + blockIdx.x;
    call = __builtin_amdgcn_readfirstlane(ld_sys(cptr)) + 1;
    if (lane == 0) { lds_st(ctl + C_CALL, call); st_sys(cptr, call); }
    u32 sp = 0;
    auto expired = [&]() -> bool {
      if ((++sp & 63) == 0 && (ld_sys(tmo) == call || now() - t0 > DEADLINE)) {
        st_sys(tmo, call);
        if (lane == 0) { lds_st(ctl + C_BAD, 1u); lds_st(ctl + C_STAGE, 6u); }
        bad = true;
      }
      return bad;
    };
    auto watch = [&](u32 kind, u32 row) {
      const u32* fp = (const u32*)(myws + WS_FL + kind * 2048 + row * 256) + (lane & 7);
      while (!bad) {
        const u32 f = ld_sys(fp);
        if (__all((int)(f - call) >= 0)) break;
        __builtin_amdgcn_s_sleep(1);
        expired();
      }
      if (lane == 0 && !bad) lds_st(ctl + C_STAGE, kind + 1);
      CFENCE();
    };
    watch(0, dest);
    STAMP(11);
    watch(1, dest);
    watch(2, dest);
    if (blockIdx.x == 0) {
      const u64* g1 = (const u64*)(myws + WS_G1);
      u64* g2 = (u64*)(P.ws[lane & 7] + WS_G2 + (lane >> 3) * G2S) + rank;
      asm volatile("" : "+v"(g2));
      u32 gm = 0;
      while (!bad) {
        u64 a0 = ld_sys64(g1 + lane), a1 = ld_sys64(g1 + 64 + lane), a2 = ld_sys64(g1 + 128 + lane);
        u64 a3 = ld_sys64(g1 + 192 + (lane & 15));
        const bool ok = (u32)(a0 >> 32) == call && (u32)(a1 >> 32) == call && (u32)(a2 >> 32) == call && (u32)(a3 >> 32) == call;
        gm = max(max((u32)a0, (u32)a1), max((u32)a2, (u32)a3));
        const u64 okm = __ballot(ok);
        if (okm == ~0ull) break;
        if (okm == 0) __builtin_amdgcn_s_sleep(8);
        expired();
      }
      gm = wave_umax(gm);
      if (!bad) st_sys64(g2, ((u64)call << 32) | gm);
    }
    watch(3, dest);
    STAMP(9);
    watch(4, dest);
    if (g == 0) watch(5, 0);
    return;
  }
  if (wave >= NPW) return;

  const __amdgpu_buffer_rsrc_t rs_s = mkrsrc(dws, WS_END);
  const u32 lane0 = lane ? 0x80000000u : 0u;
#pragma unroll
  for (int i = 0; i < 11; ++i) {
    if (i >= PW1) WAIT_VMI(PW1 - 1);
    __builtin_amdgcn_raw_buffer_store_b128(pv[i], rs_s, lane16, p0 + i < 32 ? WS_W + (rank * NWB + pb) * WBLK + (p0 + i) * 1024 : 0x80000000u, 17);
    CFENCE();
  }
  if (wave == 0) STAMP(10);
  u32 stage = 0;
  auto need = [&](u32 s) {
    while (stage < s) {
      stage = lds_ld(ctl + C_STAGE);
      if (stage < s) __builtin_amdgcn_s_sleep(1);
    }
    CFENCE();
  };
  need(1);
  call = lds_ld(ctl + C_CALL);
  const __amdgpu_buffer_rsrc_t rs_m = mkrsrc(myws, WS_Q);
  const __amdgpu_buffer_rsrc_t rs_o = mkrsrc(P.out[rank], OUT_BYTES);
  const u32 rot = (g * 26 / 7 % 26) * 10;
  constexpr u32 U_L = U_Q + 3968 - 104, uq2 = U_Q + 3 * 208 + 156;
  const u32 nu = g ? U_L : U_L + 832;
  struct Unit { u32 so, to, fo, fw; };
  u32 qc = 0, qbase = U_Q;
  auto uoff = [&](u32 u) -> Unit {
    Unit d;
    if (u < U_R) {
      const u32 kc = (u + 128) / 384, ord = kc * 8 + dest;
      u32 k = u + 128 - kc * 384;
      if (kc == 1) k = k < 256 ? k + 128 : k - 256;
      if (k < 128) {
        d.so = WS_EW + (rank * 8 + kc) * (8 * WBLK) + k * 2048;
        d.to = WS_W + ord * NWB * WBLK + k * 2048;
        d.fo = WS_WF + (k >> 4) * FSTRIDE + ord * 4;
        d.fw = 15;
      } else {
        const u32 kk = (k - 128 + rot) & 255, r = kk / 10;
        d.so = WS_EX + kc * (512u << 10) + kk * 2048;
        d.to = WS_X + ord * NRB * XBLK + kk * 2048;
        d.fo = WS_XF + r * FSTRIDE + ord * 4;
        d.fw = r == NRB - 1 ? 40 : 24;
      }
    } else if (u < U_Q) {
      const u32 k = u - U_R, kc = k / 28, j = k - kc * 28, ord = kc * 8 + dest;
      const u32 jj = j + ((j >> 2) >= rank ? 4 : 0);
      d.so = WS_ER + (call & 1) * (512u << 10) + kc * 65536 + jj * 2048;
      d.to = WS_W + (ord * NWB + 8) * WBLK + jj * 2048;
      d.fo = WS_WF + (8 + (jj >> 4)) * FSTRIDE + ord * 4;
      d.fw = (jj >> 4) == (rank >> 2) ? 20 : 15;
    } else if (u < U_L) {
      u32 c, na, n;
      for (;;) {
        c = (qc < 8 ? (g & 6) + 2 + qc * 2 + (qc < 4) : g + qc) & 7;
        na = c == 7 ? 156 : 208;
        n = qc < 7 ? na : qc < 8 ? 104 : c == 7 ? 228 : 304;
        if (u - qbase < n) break;
        qbase += n;
        qc += 1;
      }
      const u32 k = u - qbase + (qc >= 8 ? na : qc == 7 && !(g & 1) ? 104 : 0);
      d.so = WS_Q + c * QSTRIDE + k * 2048;
      d.to = k * 128;
      d.fo = c;
      d.fw = 0;
    } else {
      const u32 k = u - U_L, gs = k / 104, r = k - gs * 104;
      d.so = u < nu ? WS_L + k * 2048 : 0x80000000u;
      d.to = ((gs & 1) * 2080 + (r >> 3) * 160 + (r & 1) * 32) * 16384 + ((P.peers >> ((gs & 7) * 4)) & 7u) * 2048 + (gs & 6) * 256 + (r & 6) * 32;
      d.fo = 8;
      d.fw = 0;
    }
    return d;
  };
  u32x4 v0, v1;
  u32 u0 = vid, u1 = NPULL + vid, tk = vid + (call - 1) * nu - NPULL;
  Unit d0 = uoff(u0), d1 = uoff(u1);
  v0 = __builtin_amdgcn_raw_buffer_load_b128(rs_s, lane16, d0.so, 17);
  v1 = __builtin_amdgcn_raw_buffer_load_b128(rs_s, lane16, d0.so + 1024, 17);
  u32 to0 = lane16, to1 = lane16 + 1024, tso = d0.to, lo1 = lane16;
  __amdgpu_buffer_rsrc_t rs_t = rs_m;
  const u32 cv = call * FULL;
  u32 cnt = 0, pfo = 0, pfw = 0;
  auto publish = [&]() {
    if (pfw && __builtin_amdgcn_readfirstlane(cnt) + pfw == cv)
      asm volatile("s_nop 4\n\tbuffer_store_dword %0, %1, %2, %3 offen sc0 sc1" :: "v"(cv), "v"(lane0), "s"(rs_m), "s"(pfo) : "memory");
    pfw = 0;
  };
  WAIT_VM(2);
  __builtin_amdgcn_raw_ptr_buffer_atomic_add_i32(FULL / 3, rs_s, lane0, WS_WF + pb * FSTRIDE + rank * 4, 16);
  CFENCE();
  __builtin_amdgcn_s_waitcnt(0x0F71);
  while (u0 < nu) {
    if (bad && !d0.fw) v0 = v1 = u32x4{0x80808080u, 0x80808080u, 0x80808080u, 0x80808080u};
    __builtin_amdgcn_raw_buffer_store_b128(v0, rs_t, to0, tso, 17);
    __builtin_amdgcn_raw_buffer_store_b128(v1, rs_t, to1, tso, 17);
    CFENCE();
    const u32 ns = u1 >= nu ? 0 : u1 >= U_L ? 6 : u1 >= uq2 ? 5 : u1 >= U_Q ? 4 : u1 >= U_C ? 3 : u1 >= U_B ? 2 : 1;
    if (stage < ns) {
      stage = lds_ld(ctl + C_STAGE);
      if (stage < ns) {
        publish();
        WAIT_VM0();
        if (d0.fw) {
          cnt = __builtin_amdgcn_raw_ptr_buffer_atomic_add_i32(d0.fw, rs_m, lane0, d0.fo + CNT_OFF, 16);
          pfo = d0.fo; pfw = d0.fw;
          publish();
        }
        d0.fw = 0;
        need(ns);
      }
      bad = lds_ld(ctl + C_BAD) != 0;
    }
    v0 = __builtin_amdgcn_raw_buffer_load_b128(rs_s, lo1, d1.so, 17);
    v1 = __builtin_amdgcn_raw_buffer_load_b128(rs_s, lo1, d1.so + 1024, 17);
    CFENCE();
    const u32 u2 = 3 * NPULL + (__builtin_amdgcn_readfirstlane(tk) - (call - 1) * nu);
    tk = __builtin_amdgcn_raw_ptr_buffer_atomic_add_i32(1, rs_m, lane0, WS_TK + g * 128, 16);
    CFENCE();
    publish();
    const Unit d2 = uoff(u2);
    tso = d1.to;
    if (u2 >= U_L) lo1 = (lane & 3) * 256 + (lane >> 2) * 16;
    if (u1 >= U_L) {
      rs_t = rs_o;
      to0 = (lane >> 2) * 16384 + (lane & 3) * 16;
      to1 = to0 + 16 * 16384;
    } else if (!d1.fw) {
      rs_t = rs_o;
      tso = dest * 2048 + d1.fo * 256;
      const u32 mul = d1.fo == 7 ? 43691u : 32768u, w16 = d1.fo == 7 ? 12 : 16;
#pragma unroll
      for (int e = 0; e < 2; ++e) {
        const u32 i16 = d1.to + e * 64 + lane;
        const u32 sr = (i16 * mul) >> 19, col = i16 - sr * w16;
        const u32 ra = (sr >> 6) * 96 + sr, rc = (((sr - 1664) * 43691u) >> 22) * 64 + sr - 1600;
        (e ? to1 : to0) = (sr < 1664 ? ra : rc) * 16384 + col * 16;
      }
    }
    CFENCE();
    WAIT_VM(3);
    cnt = __builtin_amdgcn_raw_ptr_buffer_atomic_add_i32(d0.fw, rs_m, lane0, d0.fw ? d0.fo + CNT_OFF : 0x40000000u, 16);
    pfo = d0.fo; pfw = d0.fw;
    CFENCE();
    u0 = u1; u1 = u2;
    d0 = d1; d1 = d2;
  }
  publish();
  WAIT_VM(0);
  if (wave == 0) STAMP(5);
}

extern "C" void kernel_launch(int rank, int nranks, ncclComm_t comm, hipStream_t stream,
                              void* const* d_in, const int* in_sizes, int n_in,
                              void* d_out, int out_size,
                              const int* in_dtypes, int out_dtype,
                              void* const* d_out_all,
                              void* const* d_ws_all, size_t ws_bytes) {
  if (nranks != NR || ws_bytes < WS_END || n_in < 2) { fprintf(stderr, "unsupported configuration\n"); return; }
  Params P;
  P.x = (const char*)d_in[0];
  P.w = (const char*)d_in[1];
  for (int p = 0; p < NR; ++p) { P.out[p] = (char*)d_out_all[p]; P.ws[p] = (char*)d_ws_all[p]; }
  P.rank = rank;
  const int phys[8] = {0, 2, 3, 6, 1, 4, 5, 7}, logi[8] = {0, 4, 1, 2, 5, 6, 3, 7};
  P.peers = 0;
  for (int g = 0; g < NR; ++g) P.peers |= (u32)phys[logi[rank & 7] ^ g] << (g * 4);
  hipLaunchKernelGGL(fused_gemm_ar_q8, dim3(256), dim3(NTHREADS), 0, stream, P);
  HIP_CHECK(hipGetLastError());
}
```

```cpp
#define XSKEW
#include <hip/hip_runtime.h>
#include <rccl/rccl.h>
#include <cstdio>
#include <stdint.h>

#define HIP_CHECK(expr) do { hipError_t _e = (expr); if (_e != hipSuccess) { \
    fprintf(stderr, "HIP error %s at %s:%d\n", hipGetErrorString(_e), __FILE__, __LINE__); \
    return; } } while (0)

typedef unsigned u32;
typedef unsigned long long u64;
typedef u32 u32x4 __attribute__((ext_vector_type(4)));
typedef u32 u32x2 __attribute__((ext_vector_type(2)));
typedef float f32x4 __attribute__((ext_vector_type(4)));
typedef short s16x4 __attribute__((ext_vector_type(4)));
typedef short s16x8 __attribute__((ext_vector_type(8)));
typedef __bf16 bf16x8 __attribute__((ext_vector_type(8)));
#define LAS __attribute__((address_space(3)))
#define GAS __attribute__((address_space(1)))

constexpr int NR = 8;
constexpr u32 OUT_BYTES = 4096u * 16384u;
constexpr u32 X_BYTES = 4096u * 512u * 2u;
constexpr u32 W_BYTES = 512u * 16384u * 2u;

constexpr u32 NRB = 26;
constexpr u32 NCB = 8;
constexpr u32 XBLK = 20480, WBLK = 32768;
constexpr u32 NWB = NCB + 2;
constexpr u32 WS_W = 0;
constexpr u32 WS_X = 20u << 20;
constexpr u32 WS_CTL = 53u << 20;
constexpr u32 FSTRIDE = 4352;
constexpr u32 WS_XF = WS_CTL;
constexpr u32 WS_WF = WS_XF + NRB * FSTRIDE;
constexpr u32 CNT_OFF = 512u << 10;
constexpr u32 WS_C2 = WS_CTL + (160u << 10);
constexpr u32 WS_G1 = WS_C2 + 4096;
constexpr u32 WS_G2 = WS_C2 + 8192 + 256;
constexpr u32 G2S = 512;
constexpr u32 WS_TK = WS_C2 + 13312;
constexpr u32 WS_TMO = WS_C2 + 20480;
constexpr u32 WS_DBG = WS_C2 + 32768;
constexpr u32 WS_FL = WS_C2 + 65536;
constexpr u32 WS_FC = WS_FL + 16384;
constexpr u32 WS_CALL = WS_FC + 16384;
constexpr u32 WS_Q = 54u << 20;
constexpr u32 QSTRIDE = 4160 * 256;
constexpr u32 WS_EX = 63u << 20;
constexpr u32 WS_ER = 67u << 20;
constexpr u32 WS_EW = 68u << 20;
constexpr u32 WS_L = 84u << 20;
constexpr u32 WS_END = 86u << 20;
static_assert(64u * NWB * WBLK <= WS_X && WS_X + 64u * NRB * XBLK <= WS_CTL && WS_WF + NWB * FSTRIDE <= WS_C2 &&
              WS_CALL + 16384 <= WS_CTL + CNT_OFF && WS_C2 + CNT_OFF <= WS_Q && WS_Q + NCB * QSTRIDE <= WS_EX, "layout");
constexpr u32 FULL = 240;

constexpr u32 SLOT_BYTES = WBLK + XBLK;
constexpr u32 NSLOT = 3;
constexpr u32 TPITCH = 272;
constexpr u32 SNAP_OFF = SLOT_BYTES * NSLOT;
constexpr u32 CTL_OFF = SNAP_OFF + 1024;
constexpr u32 LDS_BYTES = CTL_OFF + 1024;
static_assert(LDS_BYTES <= 163840, "lds");
constexpr int C_AMAX = 0, C_GMAX = 1, C_CALL = 2, C_BAD = 3, C_STAGE = 4, C_CNT = 8, C_MASK = 16;

#define NPW 4
#define EXWAIT 300
#define EXD 60
#define PW1 4
#define PLD 4
#define POLL2 1
#define QWAIT 500
#define QD 100
#define PUSHD 2
constexpr u32 NCOMMWG = 32 - NRB;
constexpr u32 NTILE = NRB * NCB;
constexpr u32 NPULL = NCOMMWG * NPW;
constexpr u32 U_A = 104, U_B = 512, U_C = 1152;
constexpr u32 U_R = 256 + 7 * 384;
constexpr u32 U_Q = U_R + 224;
constexpr int NTHREADS = 512;
constexpr int NSTEP = 64;
constexpr u64 DEADLINE = 50000000ull;

struct Params {
  const char* x;
  const char* w;
  char* out[NR];
  char* ws[NR];
  int rank;
  u32 peers;
};

__device__ __forceinline__ __amdgpu_buffer_rsrc_t mkrsrc(const void* p, u32 bytes) {
  return __builtin_amdgcn_make_buffer_rsrc((void*)p, 0, bytes, 0x00020000);
}
__device__ __forceinline__ u64 now() { return __builtin_amdgcn_s_memrealtime(); }
__device__ __forceinline__ u32 ld_sys(const u32* p) { return __hip_atomic_load((GAS const u32*)p, __ATOMIC_RELAXED, __HIP_MEMORY_SCOPE_SYSTEM); }
__device__ __forceinline__ u64 ld_sys64(const u64* p) { return __hip_atomic_load((GAS const u64*)p, __ATOMIC_RELAXED, __HIP_MEMORY_SCOPE_SYSTEM); }
__device__ __forceinline__ void st_sys(u32* p, u32 v) { __hip_atomic_store((GAS u32*)p, v, __ATOMIC_RELAXED, __HIP_MEMORY_SCOPE_SYSTEM); }
__device__ __forceinline__ void st_sys64(u64* p, u64 v) { __hip_atomic_store((GAS u64*)p, v, __ATOMIC_RELAXED, __HIP_MEMORY_SCOPE_SYSTEM); }
__device__ __forceinline__ u32 lds_ld(LAS u32* p) { return __builtin_amdgcn_readfirstlane(__hip_atomic_load(p, __ATOMIC_RELAXED, __HIP_MEMORY_SCOPE_WORKGROUP)); }
__device__ __forceinline__ void lds_st(LAS u32* p, u32 v) { __hip_atomic_store(p, v, __ATOMIC_RELAXED, __HIP_MEMORY_SCOPE_WORKGROUP); }
__device__ __forceinline__ void lds_add1(LAS u32* p, u32 lane) {
  if (lane == 0) __hip_atomic_fetch_add(p, 1u, __ATOMIC_RELAXED, __HIP_MEMORY_SCOPE_WORKGROUP);
}
#define CFENCE() asm volatile("" ::: "memory")
#define WAIT_VM(n) asm volatile("s_waitcnt vmcnt(" #n ")" ::: "memory")
#define WAIT_VMI(n) asm volatile("s_waitcnt vmcnt(%0)" :: "i"(n) : "memory")
#define WAIT_LGKM() asm volatile("s_waitcnt lgkmcnt(0)" ::: "memory")
#define WAIT_VM0() do { CFENCE(); __builtin_amdgcn_s_waitcnt(0x0F70); WAIT_VM(0); } while (0)

__device__ __forceinline__ u32 wave_umax(u32 v) {
  v = max(v, (u32)__builtin_amdgcn_update_dpp(0, (int)v, 0x111, 0xf, 0xf, true));
  v = max(v, (u32)__builtin_amdgcn_update_dpp(0, (int)v, 0x112, 0xf, 0xf, true));
  v = max(v, (u32)__builtin_amdgcn_update_dpp(0, (int)v, 0x114, 0xf, 0xf, true));
  v = max(v, (u32)__builtin_amdgcn_update_dpp(0, (int)v, 0x118, 0xf, 0xf, true));
  v = max(v, (u32)__builtin_amdgcn_update_dpp(0, (int)v, 0x142, 0xa, 0xf, false));
  v = max(v, (u32)__builtin_amdgcn_update_dpp(0, (int)v, 0x143, 0xc, 0xf, false));
  return (u32)__builtin_amdgcn_readlane((int)v, 63);
}

#define STAMP(k) do {} while (0)

__device__ __forceinline__ void wait_vm_dyn(u32 n) {
  asm volatile(
      "s_getpc_b64 vcc\n\t"
      "s_add_u32 vcc_lo, vcc_lo, %0\n\t"
      "s_addc_u32 vcc_hi, vcc_hi, 0\n\t"
      "s_setpc_b64 vcc\n\t"
      "s_waitcnt vmcnt(0)\n\ts_branch 1f\n\t"
      "s_waitcnt vmcnt(1)\n\ts_branch 1f\n\t"
      "s_waitcnt vmcnt(2)\n\ts_branch 1f\n\t"
      "s_waitcnt vmcnt(3)\n\ts_branch 1f\n\t"
      "s_waitcnt vmcnt(4)\n\ts_branch 1f\n\t"
      "s_waitcnt vmcnt(5)\n\ts_branch 1f\n\t"
      "s_waitcnt vmcnt(6)\n\ts_branch 1f\n\t"
      "s_waitcnt vmcnt(7)\n\ts_branch 1f\n\t"
      "s_waitcnt vmcnt(8)\n\ts_branch 1f\n\t"
      "s_waitcnt vmcnt(9)\n\ts_branch 1f\n\t"
      "s_waitcnt vmcnt(10)\n\ts_branch 1f\n\t"
      "s_waitcnt vmcnt(11)\n\ts_branch 1f\n\t"
      "s_waitcnt vmcnt(12)\n\ts_branch 1f\n\t"
      "s_waitcnt vmcnt(13)\n\ts_branch 1f\n\t"
      "s_waitcnt vmcnt(14)\n\ts_branch 1f\n\t"
      "s_waitcnt vmcnt(15)\n\ts_branch 1f\n\t"
      "s_waitcnt vmcnt(16)\n\ts_branch 1f\n\t"
      "s_waitcnt vmcnt(17)\n\ts_branch 1f\n\t"
      "s_waitcnt vmcnt(18)\n\ts_branch 1f\n\t"
      "s_waitcnt vmcnt(19)\n\ts_branch 1f\n\t"
      "s_waitcnt vmcnt(20)\n\ts_branch 1f\n\t"
      "s_waitcnt vmcnt(21)\n\ts_branch 1f\n\t"
      "s_waitcnt vmcnt(22)\n\ts_branch 1f\n\t"
      "s_waitcnt vmcnt(23)\n\ts_branch 1f\n\t"
      "s_waitcnt vmcnt(24)\n\t"
      "1:"
      :: "s"(min(n, 24u) * 8 + 12) : "memory", "scc", "vcc");
}
#define BARRIER() asm volatile("s_barrier" ::: "memory")

__global__ void __launch_bounds__(NTHREADS) fused_gemm_ar_q8(Params P) {
  __shared__ __attribute__((aligned(16))) char smem[LDS_BYTES];
  LAS char* L = (LAS char*)smem;
  LAS u32* ctl = (LAS u32*)(L + CTL_OFF);
  const u32 tid = threadIdx.x, lane = tid & 63;
  const u32 wave = __builtin_amdgcn_readfirstlane(tid >> 6);
  const u32 g = blockIdx.x & 7u, idx = blockIdx.x >> 3;
  const u32 rank = (u32)P.rank & 7u;
  const u32 dest = (P.peers >> (g * 4)) & 7u;
  char* const myws = P.ws[rank];
  char* const dws = P.ws[dest];
  u32* const tmo = (u32*)(myws + WS_TMO);
  const u32 lane16 = lane * 16;
  const u64 t0 = now();
  const u32 lx = (lane & 7) * 1024 + (lane >> 3) * 16;
  const u32 lw = (((lane >> 3) & 1) * 8 + ((lane >> 1) & 3)) * 32768 + ((lane & 1) * 4 + (lane >> 4)) * 16;
  u32 call = 0;
  bool bad = false;
  if (wave == 0) STAMP(0);

  if (idx >= NCOMMWG) {
    const u32 ti = idx - NCOMMWG;
    const u32 rb = (g & 1) * 13 + (ti >> 1);
    u32 cb = (g >> 1) * 2 + (ti & 1);
    const u32 tile = g * NRB + ti;
    const __amdgpu_buffer_rsrc_t rs_ws = mkrsrc(myws, WS_END);
    char* flp = P.ws[lane & 7] + WS_FL + rank * 256 + g * 4;
    const u32 lane0 = lane ? 0x80000000u : 0u;
    auto signal_wg = [&](u32 kind, u32 inc) {
      const u32 old = __builtin_amdgcn_raw_ptr_buffer_atomic_add_i32(inc, rs_ws, lane0, WS_FC + kind * 2048 + g * 256, 16);
      if (__builtin_amdgcn_readfirstlane(old) + inc == call * NRB && lane < 8)
        st_sys((u32*)(flp + kind * 2048), call);
      CFENCE();
    };
    auto signal_all = [&](u32 kind, u32 inc = 1) {
      u32 old = 0;
      if (lane == 0) old = __hip_atomic_fetch_add(ctl + C_CNT + kind, 1u, __ATOMIC_RELAXED, __HIP_MEMORY_SCOPE_WORKGROUP);
      if (__builtin_amdgcn_readfirstlane(old) == 7) signal_wg(kind, inc);
    };
    {
      const u32 pc = g ? 2 * U_A + (tile - NRB) * 5 + wave : ti * 8 + wave;
      const bool pok = (wave < 5 || !g) && pc < 1024;
      const u32x4 e0 = __builtin_amdgcn_raw_buffer_load_b128(mkrsrc(P.x, X_BYTES), lx,
                                                             pok ? (pc & 511) * 8192 + (pc >> 9) * 128 : 0x80000000u, 0);
      CFENCE();
      if (wave == 7 && lane < 16) ctl[lane] = 0u;
      __builtin_amdgcn_raw_buffer_store_b128(e0, rs_ws, lane16, pok ? WS_EX + pc * 1024 : 0x80000000u, 17);
      asm volatile("" : "+v"(flp));
      WAIT_VM0();
      WAIT_LGKM();
      BARRIER();
      if (wave == 0) {
        const u32 old = __builtin_amdgcn_readfirstlane(__builtin_amdgcn_raw_ptr_buffer_atomic_add_i32(1, rs_ws, lane0, WS_FC + g * 256, 16));
        call = old / NRB + 1;
        if (old + 1 == call * NRB && lane < 8) {
          st_sys((u32*)flp, call);
        }
        if (lane == 0) lds_st(ctl + C_CALL, call);
        WAIT_LGKM();
      }
      if (wave == 6 && ti == 0) __builtin_amdgcn_fence(__ATOMIC_ACQUIRE, "");
      BARRIER();
      call = lds_ld(ctl + C_CALL);
      for (u32 r = 0; r < 3; ++r) {
        const u32 slot = r * 8 + wave, e = slot * NTILE + tile;
        u32 kc, k, so, eo;
        bool isw = true;
        if (e < 4352) {
          kc = (e + 768) / 640; k = e + 768 - kc * 640;
          if (kc == 1) k -= 128;
          isw = k < 512;
          k &= 511;
          eo = isw ? WS_EW + ((k >> 6) * 8 + kc) * (8 * WBLK) + (k & 63) * 4096 : WS_EX + kc * (512u << 10) + k * 4096;
        } else {
          kc = (e - 4352) >> 4; k = ((e >> 1) & 7) * 64 + 62 + (e & 1);
          eo = WS_ER + (call & 1) * (512u << 10) + (e - 4352) * 4096;
        }
        if (e < 4480) {
          so = isw ? (kc * 64 + (k & 1) * 32) * 32768 + (k >> 1) * 128 : k * 32768 + kc * 128;
          const __amdgpu_buffer_rsrc_t rs = mkrsrc(isw ? P.w : P.x, isw ? W_BYTES : X_BYTES);
          const u32 lo = isw ? lw : lx, s1 = isw ? 4 * 32768 : 8192, s2 = isw ? 16 * 32768 : 16384;
          while (now() - t0 < EXWAIT + slot * EXD) __builtin_amdgcn_s_sleep(2);
          const u32x4 e0 = __builtin_amdgcn_raw_buffer_load_b128(rs, lo, so, 0);
          const u32x4 e1 = __builtin_amdgcn_raw_buffer_load_b128(rs, lo, so + s1, 0);
          const u32x4 e2 = __builtin_amdgcn_raw_buffer_load_b128(rs, lo, so + s2, 0);
          const u32x4 e3 = __builtin_amdgcn_raw_buffer_load_b128(rs, lo, so + s1 + s2, 0);
          __builtin_amdgcn_raw_buffer_store_b128(e0, rs_ws, lane16, eo, 17);
          __builtin_amdgcn_raw_buffer_store_b128(e1, rs_ws, lane16, eo + 1024, 17);
          __builtin_amdgcn_raw_buffer_store_b128(e2, rs_ws, lane16, eo + 2048, 17);
          __builtin_amdgcn_raw_buffer_store_b128(e3, rs_ws, lane16, eo + 3072, 17);
        }
        if (r != 1) {
          WAIT_VM0();
          signal_all(1 + (r >> 1));
        }
      }
      BARRIER();
    }
    const u32 cv = call * FULL;
    const u32 wm = wave >> 2, wn = wave & 3;
    const u32 lds0 = (u32)(size_t)L;
    const u32 xa = WBLK + wm * 10240 + ((lane >> 3) & 1) * 1024 + (lane >> 5) * 256 + ((lane >> 4) & 1) * 128 + (lane & 7) * 16;
    const u32 h = lane >> 4, q = (lane >> 2) & 3, p = lane & 3;
    const u32 wa0 = wn * 8192 + (h >> 1) * 2048 + (p & 1) * 512 + (h & 1) * 128 + q * 32 + (p >> 1) * 16 + (p & 1) * 8;
    const u32 wa1 = wa0 ^ 8;
    const u32 xpo = wave < 4 ? wave * 3072 : 12288 + (wave - 4) * 2048;
    u32 pollo;
    f32x4 acc[4][5];
    u64 kx = 0, kw;
    u64 ready, taken;
    u32 issued, consumed, nis = 0;
    u32 islot, cslot, par = 0;
    u64 markD = 0;
    float inv = 0.f;
    auto sched = [&]() {
      u64 avail = ready & ~taken;
      while (avail && issued - consumed < NSLOT) {
        const u32 ord = __builtin_ctzll(avail);
        avail &= avail - 1;
        taken |= 1ull << ord;
        const u32 sw = WS_W + (ord * NWB + cb) * WBLK + wave * 4096;
        const u32 sx = WS_X + (ord * NRB + rb) * XBLK + xpo;
        const u32 mw = lds0 + islot + wave * 4096, mx = lds0 + islot + WBLK + xpo;
        u32 keep;
        asm volatile(
            "s_mov_b32 %0, m0\n\t"
            "s_mov_b32 m0, %3\n\t"
            "s_nop 0\n\t"
            "buffer_load_dwordx4 %1, %2, %5 offen lds\n\t"
            "buffer_load_dwordx4 %1, %2, %5 offen offset:1024 lds\n\t"
            "buffer_load_dwordx4 %1, %2, %5 offen offset:2048 lds\n\t"
            "buffer_load_dwordx4 %1, %2, %5 offen offset:3072 lds\n\t"
            "s_mov_b32 m0, %4\n\t"
            "s_nop 0\n\t"
            "buffer_load_dwordx4 %1, %2, %6 offen lds\n\t"
            "buffer_load_dwordx4 %1, %2, %6 offen offset:1024 lds\n\t"
            "s_cmp_gt_u32 %7, 3\n\t"
            "s_cbranch_scc1 1f\n\t"
            "buffer_load_dwordx4 %1, %2, %6 offen offset:2048 lds\n\t"
            "1:\n\t"
            "s_mov_b32 m0, %0"
            : "=&s"(keep)
            : "v"(lane16), "s"(rs_ws), "s"(mw), "s"(mx), "s"(sw), "s"(sx), "s"(wave)
            : "memory", "scc");
        nis += wave < 4 ? 7 : 6;
        const u32 sh = (issued & 3) * 16;
        markD = (markD & ~(0xffffull << sh)) | ((u64)(nis & 0xffff) << sh);
        issued += 1;
        islot = islot == 2 * SLOT_BYTES ? 0 : islot + SLOT_BYTES;
      }
    };
    auto poll = [&]() {
      if (wave == 0 && (kx & kw) != ~0ull) {
        const u64 kn = lane < 16 ? kx : kw;
        const bool act = lane < 32 && ((u32)(kn >> ((lane & 15) * 4)) & 15u) != 15u;
        const u32 po = act ? pollo : 0x80000000u;
        u32 keep;
        asm volatile("s_mov_b32 %0, m0\n\ts_mov_b32 m0, %3\n\ts_nop 0\n\t"
                     "buffer_load_dwordx4 %1, %2, 0 offen sc0 sc1 lds\n\ts_mov_b32 m0, %0"
                     : "=&s"(keep) : "v"(po), "s"(rs_ws), "s"(lds0 + SNAP_OFF) : "memory");
        nis += 1;
      }
    };
    LAS u32* const snap = (LAS u32*)(L + SNAP_OFF) + lane;
    auto seen = [&](u32 fx, u32 fw) {
      kx |= __ballot((int)(fx - cv) >= 0);
      kw |= __ballot((int)(fw - cv) >= 0);
    };
    auto tell = [&]() { if (lane == 0) *(LAS u64*)(L + CTL_OFF + C_MASK * 4 + par) = kx & kw; };
    auto told = [&]() -> u64 {
      const u64 m = *(LAS u64*)(L + CTL_OFF + C_MASK * 4 + par);
      par ^= 8;
      return m;
    };
    auto uni = [&](u64 m) -> u64 {
      return (u64)(u32)__builtin_amdgcn_readfirstlane((u32)m) | ((u64)(u32)__builtin_amdgcn_readfirstlane((u32)(m >> 32)) << 32);
    };
    auto wait_step = [&]() { wait_vm_dyn((nis - (u32)(markD >> ((consumed & 3) * 16))) & 0xffff); };
#define SB() __builtin_amdgcn_sched_barrier(0)
#define PIECE(M, S, OFF) do { \
        asm volatile("s_cmp_lg_u32 %2, -1\n\ts_cbranch_scc0 1f\n\ts_mov_b32 m0, %2\n\ts_nop 0\n\t" \
                     "buffer_load_dwordx4 %0, %1, %3 offen offset:" #OFF " lds\n\t1:" \
                     :: "v"(lane16), "s"(rs_ws), "s"(M), "s"(S) : "memory", "scc"); \
        SB(); } while (0)
#define MMA4(kb, i) do { \
        _Pragma("unroll") for (int n = 0; n < 4; ++n) \
          acc[n][i] = __builtin_amdgcn_mfma_f32_16x16x32_bf16(wf[kb][n], xf[kb][i], acc[n][i], 0, 0, 0); \
        SB(); } while (0)
    bf16x8 xf[2][5], wf[2][4];
    u32 sp = 0;
    u64 cyc0 = 0;
    for (u32 pass = 0;; ++pass) {
    pollo = lane < 16 ? WS_XF + rb * FSTRIDE + lane16 : WS_WF + cb * FSTRIDE + lane16 - 256;
    kw = taken = 0;
    issued = consumed = islot = cslot = 0;
#pragma unroll
    for (int a = 0; a < 4; ++a)
#pragma unroll
      for (int b = 0; b < 5; ++b) acc[a][b] = f32x4{0.f, 0.f, 0.f, 0.f};
#pragma unroll
    for (int n = 0; n < 5; ++n) {
      xf[1][n] = __builtin_bit_cast(bf16x8, u32x4{0u, 0u, 0u, 0u});
      if (n < 4) wf[1][n] = __builtin_bit_cast(bf16x8, u32x4{0u, 0u, 0u, 0u});
    }
    poll();
    while (consumed < NSTEP) {
      if (issued == consumed) {
        if (wave == 0) {
          while ((kx & kw & ~taken) == 0) {
            poll();
            WAIT_VM(1);
            seen(snap[0], snap[64]);
            if ((++sp & 63) == 0 && now() - t0 > DEADLINE) {
              st_sys(tmo, call);
              WAIT_VM(0);
              lds_st(ctl + C_BAD, 1u);
              kx = kw = ~0ull;
            }
          }
          tell();
          WAIT_LGKM();
        }
        BARRIER();
        ready = uni(told());
        sched();
        poll();
        wait_step();
        if (wave == 0) {
          seen(snap[0], snap[64]);
          tell();
          WAIT_LGKM();
        }
      }
      BARRIER();
      const u64 rdy = told();
      SB();
      LAS char* base = L + cslot;
      auto frags = [&](int kb) {
#pragma unroll
        for (int n = 0; n < 5; ++n) {
          if (n < 4) {
            LAS char* a = base + ((n & 1) ? wa1 : wa0) + kb * 4096 + (n >> 1) * 256;
            s16x4 lo = __builtin_amdgcn_ds_read_tr16_b64_v4i16((LAS s16x4*)a);
            s16x4 hi = __builtin_amdgcn_ds_read_tr16_b64_v4i16((LAS s16x4*)(a + 1024));
            wf[kb][n] = __builtin_bit_cast(bf16x8, __builtin_shufflevector(lo, hi, 0, 1, 2, 3, 4, 5, 6, 7));
          }
          xf[kb][n] = *(LAS bf16x8*)(base + xa + n * 2048 + kb * 512);
        }
        SB();
      };
      MMA4(1, 0);
      frags(0);
      poll();
      SB();
      MMA4(1, 1);
      ready = uni(rdy);
      const u64 av = ready & ~taken;
      const bool go = av != 0 && issued - consumed < NSLOT;
      const u32 ord = __builtin_ctzll(av | (1ull << 63));
      const u32 sw = WS_W + (ord * NWB + cb) * WBLK + wave * 4096;
      const u32 sx = WS_X + (ord * NRB + rb) * XBLK + xpo;
      u32 mw = ~0u, mx = ~0u, mx3 = ~0u;
      if (go) {
        mw = lds0 + islot + wave * 4096;
        mx = lds0 + islot + WBLK + xpo;
        if (wave < 4) mx3 = mx;
        taken |= 1ull << ord;
        nis += wave < 4 ? 7 : 6;
        const u32 sh = (issued & 3) * 16;
        markD = (markD & ~(0xffffull << sh)) | ((u64)(nis & 0xffff) << sh);
        issued += 1;
        islot = islot == 2 * SLOT_BYTES ? 0 : islot + SLOT_BYTES;
      }
      SB();
      MMA4(1, 2);
      PIECE(mw, sw, 0);
      MMA4(1, 3);
      PIECE(mw, sw, 1024);
      MMA4(1, 4);
      frags(1);
      PIECE(mw, sw, 2048);
      MMA4(0, 0);
      PIECE(mw, sw, 3072);
      PIECE(mx, sx, 0);
      MMA4(0, 1);
      PIECE(mx, sx, 1024);
      PIECE(mx3, sx, 2048);
      MMA4(0, 2);
      sched();
      poll();
      cslot = cslot == 2 * SLOT_BYTES ? 0 : cslot + SLOT_BYTES;
      consumed += 1;
      SB();
      MMA4(0, 3);
      if (issued != consumed) wait_step();
      u32 fx = cv - 1, fw = cv - 1;
      if (wave == 0) { fx = snap[0]; fw = snap[64]; }
      SB();
      MMA4(0, 4);
      if (wave == 0) { seen(fx, fw); tell(); }
      WAIT_LGKM();
    }
    MMA4(1, 0); MMA4(1, 1); MMA4(1, 2); MMA4(1, 3); MMA4(1, 4);
    if (pass == 0) {
      if (wave == 0) STAMP(2);
      const u32 row0 = rb * 160 + wm * 80;
      float m = 0.f;
#pragma unroll
      for (int b = 0; b < 5; ++b) {
        float mb = 0.f;
#pragma unroll
        for (int a = 0; a < 4; ++a)
#pragma unroll
          for (int e = 0; e < 4; ++e) mb = fmaxf(mb, fabsf(acc[a][b][e]));
        if (row0 + b * 16 < 4096) m = fmaxf(m, mb);
      }
      const u32 mb = wave_umax(__builtin_bit_cast(u32, m));
      if (lane == 0) __hip_atomic_fetch_max(ctl + C_AMAX, mb, __ATOMIC_RELAXED, __HIP_MEMORY_SCOPE_WORKGROUP);
      WAIT_LGKM();
      BARRIER();
      if (wave == 0) {
        const u32 wmax = lds_ld(ctl + C_AMAX);
        if (lane == 0) st_sys64((u64*)(myws + WS_G1) + tile, ((u64)call << 32) | wmax);
        STAMP(3);
        const u64* g2 = (const u64*)(myws + WS_G2 + g * G2S) + (lane & 7);
        u64 val;
        for (;;) {
          val = ld_sys64(g2);
          if (__all((u32)(val >> 32) == call)) break;
          if ((++sp & 63) == 0 && (ld_sys(tmo) == call || now() - t0 > DEADLINE)) { st_sys(tmo, call); bad = true; break; }
        }
        const u32 gmax = wave_umax((u32)val);
        if (lane == 0) {
          lds_st(ctl + C_GMAX, gmax);
          if (bad) lds_st(ctl + C_BAD, 1u);
        }
        WAIT_LGKM();
      }
    }
    BARRIER();
    if (pass == 0) {
      const float amax = __builtin_bit_cast(float, lds_ld(ctl + C_GMAX));
      const float scale = amax / 127.0f;
      inv = amax > 0.f ? 1.0f / scale : 0.f;
      if (wave == 0) STAMP(4);
    }
    bad = lds_ld(ctl + C_BAD) != 0;
    const bool odd = (lane >> 4) & 1;
    u32x4 qv[5];
#pragma unroll
    for (int i = 0; i < 5; ++i) {
      u32 d[4];
#pragma unroll
      for (int n = 0; n < 4; ++n) {
        u32 b0 = __builtin_bit_cast(u32, __builtin_fmaf(acc[n][i][0], inv, 12582912.0f));
        u32 b1 = __builtin_bit_cast(u32, __builtin_fmaf(acc[n][i][1], inv, 12582912.0f));
        u32 b2 = __builtin_bit_cast(u32, __builtin_fmaf(acc[n][i][2], inv, 12582912.0f));
        u32 b3 = __builtin_bit_cast(u32, __builtin_fmaf(acc[n][i][3], inv, 12582912.0f));
        u32 lo = __builtin_amdgcn_perm(b1, b0, 0x0c0c0400u);
        u32 hi = __builtin_amdgcn_perm(b3, b2, 0x04000c0cu);
        d[n] = bad ? 0x80808080u : (lo | hi);
      }
      qv[i] = u32x4{odd ? d[1] : d[0], odd ? d[0] : d[1], odd ? d[3] : d[2], odd ? d[2] : d[3]};
    }
    const u64 tq = now();
    if (!pass && !(cb & 1) && wm == 0) {
      const __amdgpu_buffer_rsrc_t rs_p = mkrsrc(dws, WS_END);
#pragma unroll
      for (int i = 0; i < 4; ++i) {
        if (i && i % PUSHD == 0) WAIT_VM(0);
        if (wave == 0 && i == PUSHD) STAMP(10);
        __builtin_amdgcn_raw_buffer_store_b128(qv[i], rs_p, lane16, WS_L + ((g * 13 + (ti >> 1)) * 16 + wn * 4 + i) * 1024, 17);
        CFENCE();
      }
      WAIT_VM0();
      if (wave == 0) STAMP(11);
      const u32 old = __builtin_amdgcn_raw_ptr_buffer_atomic_add_i32(1, rs_ws, lane0, WS_FC + 5 * 2048 + g * 256, 16);
      if (__builtin_amdgcn_readfirstlane(old) + 1 == call * 52 && lane == 0) st_sys((u32*)(dws + WS_FL + 5 * 2048) + g, call);
      CFENCE();
    }
    {
      LAS char* tw = L + (wm * 80 + (lane & 15)) * TPITCH + wn * 64 + (lane >> 4) * 16;
#pragma unroll
      for (int i = 0; i < 5; ++i) *(LAS u32x4*)(tw + i * 16 * TPITCH) = qv[i];
    }
    WAIT_LGKM();
    BARRIER();
    LAS char* tr = L + (wave * 4 + (lane >> 4)) * TPITCH + (lane & 15) * 16;
#pragma unroll
    for (int i = 0; i < 5; ++i) qv[i] = *(LAS u32x4*)(tr + i * 32 * TPITCH);
    const __amdgpu_buffer_rsrc_t rs_o = mkrsrc(P.out[rank], OUT_BYTES);
    const u32 orow = (rb * 160 + wave * 4 + (lane >> 4)) * 16384u;
    if (pass) {
      const u32 piece = (cb - 8) * 4 + ((lane >> 2) & 3);
      const u32 oo = piece == rank ? 0x80000000u : orow + piece * 2048 + 1984 + (lane & 3) * 16;
#pragma unroll
      for (int i = 0; i < 5; ++i) __builtin_amdgcn_raw_buffer_store_b128(qv[i], rs_o, oo, i * 32 * 16384, 17);
      break;
    }
    const u32 pitch = cb == 7 ? 192 : 256;
    const bool rep = (lane & 15) * 16 >= pitch;
    const u32 qo = rep ? 0x80000000u : (wave * 4 + (lane >> 4)) * pitch + (lane & 15) * 16;
    const u32 oo = orow + rank * 2048 + cb * 256 + (lane & 15) * 16;
    const u32 qa = WS_Q + cb * QSTRIDE + rb * 64 * pitch, qb = WS_Q + cb * QSTRIDE + (1664 + rb * 96) * pitch;
    if (cb & 1) {
      __builtin_amdgcn_raw_buffer_store_b128(qv[0], rs_ws, qo, qa, 17);
      __builtin_amdgcn_raw_buffer_store_b128(qv[1], rs_ws, qo, qa + 32 * pitch, 17);
      WAIT_VM0();
      signal_all(3, 2);
      if (wave == 0) STAMP(9);
    }
    while (now() - tq < QWAIT + wave * QD) __builtin_amdgcn_s_sleep(2);
    if (!(cb & 1)) {
      __builtin_amdgcn_raw_buffer_store_b128(qv[0], rs_ws, qo, qa, 17);
      __builtin_amdgcn_raw_buffer_store_b128(qv[1], rs_ws, qo, qa + 32 * pitch, 17);
    }
#pragma unroll
    for (int i = 2; i < 5; ++i) __builtin_amdgcn_raw_buffer_store_b128(qv[i], rs_ws, qo, qb + (i - 2) * 32 * pitch, 17);
    WAIT_VM0();
    signal_all(4);
    if (cb == 7) {
#pragma unroll
      for (int i = 0; i < 5; ++i) __builtin_amdgcn_raw_buffer_store_b128(qv[i], rs_o, rep ? oo : 0x80000000u, i * 32 * 16384, 17);
    }
    if (((ti >> 1) & 3) != (g >> 1)) break;
    cb = 8 + (ti & 1);
    WAIT_VM0();
    WAIT_LGKM();
    BARRIER();
    }
    WAIT_VM(0);
    if (wave == 0) STAMP(5);
    return;
  }

  const u32 vid = idx * NPW + wave;
  const __amdgpu_buffer_rsrc_t rs_s = mkrsrc(dws, WS_END);
  const u32 lane0 = lane ? 0x80000000u : 0u;
  static_assert(NPW == 4 && NCOMMWG == 6, "wave roles");
  if (wave >= NPW && wave < 7) {
    const u32 ph = (wave - NPW) * NCOMMWG + idx;
    const __amdgpu_buffer_rsrc_t rs_w = mkrsrc(P.w, W_BYTES);
    const u32 so = ph < 16 ? dest * 4096 + ph * 256 : 0x80000000u;
    const u32 to = ph < 16 ? WS_W + rank * NWB * WBLK + ph * 16384 : 0x80000000u;
    u32x4 pv[16];
#pragma unroll
    for (int i = 0; i < 16; ++i) {
      pv[i] = __builtin_amdgcn_raw_buffer_load_b128(rs_w, lw, so + (i >> 3) * 128 + (((i >> 1) & 3) * 16 + (i & 1) * 4) * 32768, 0);
      if (i == PLD - 1) BARRIER();
    }
    CFENCE();
#pragma unroll
    for (int i = 0; i < 16; ++i) {
      if (i >= PW1) WAIT_VMI(PW1 - 1);
      __builtin_amdgcn_raw_buffer_store_b128(pv[i], rs_s, lane16, to + i * 1024, 17);
      CFENCE();
    }
    WAIT_VM(0);
    if (wave == NPW) STAMP(10);
    if (ph < 16) __builtin_amdgcn_raw_ptr_buffer_atomic_add_i32(FULL / 2, rs_s, lane0, WS_WF + (ph >> 1) * FSTRIDE + rank * 4, 16);
    return;
  }
  if (wave == 7) {
    ctl[lane] = 0u;
    WAIT_LGKM();
  }
  BARRIER();
  if (wave == 7) {
    u32* const cptr = (u32*)(myws + WS_CALL + blockIdx.x * 256);
    call = __builtin_amdgcn_readfirstlane(ld_sys(cptr)) + 1;
    if (lane == 0) { lds_st(ctl + C_CALL, call); st_sys(cptr, call); }
    u32 sp = 0;
    auto expired = [&]() -> bool {
      if ((++sp & 63) == 0 && (ld_sys(tmo) == call || now() - t0 > DEADLINE)) {
        st_sys(tmo, call);
        if (lane == 0) { lds_st(ctl + C_BAD, 1u); lds_st(ctl + C_STAGE, 7u); }
        bad = true;
      }
      return bad;
    };
    auto watch = [&](u32 kind, u32 row) {
      const u32* fp = (const u32*)(myws + WS_FL + kind * 2048 + row * 256) + (lane & 7);
      while (!bad) {
        const u32 f = ld_sys(fp);
        const u64 ok = __ballot((int)(f - call) >= 0);
        if (ok == ~0ull) break;
        if (kind == 0 && (ok & 1) && lane == 0) lds_st(ctl + C_STAGE, 1u);
        __builtin_amdgcn_s_sleep(1);
        expired();
      }
      if (lane == 0 && !bad) lds_st(ctl + C_STAGE, kind + 2);
      CFENCE();
    };
    watch(0, dest);
    const u64 tA = now() - t0;
    STAMP(11);
    watch(1, dest);
    watch(2, dest);
    if (blockIdx.x == 0) {
      const u64* g1 = (const u64*)(myws + WS_G1);
      u64* g2 = (u64*)(P.ws[lane & 7] + WS_G2 + (lane >> 3) * G2S) + rank;
      asm volatile("" : "+v"(g2));
      u32 gm = 0;
      while (!bad) {
        u64 a0 = ld_sys64(g1 + lane), a1 = ld_sys64(g1 + 64 + lane), a2 = ld_sys64(g1 + 128 + lane);
        u64 a3 = ld_sys64(g1 + 192 + (lane & 15));
        const bool ok = (u32)(a0 >> 32) == call && (u32)(a1 >> 32) == call && (u32)(a2 >> 32) == call && (u32)(a3 >> 32) == call;
        gm = max(max((u32)a0, (u32)a1), max((u32)a2, (u32)a3));
        const u64 okm = __ballot(ok);
        if (okm == ~0ull) break;
        if (okm == 0) __builtin_amdgcn_s_sleep(8);
        expired();
      }
      gm = wave_umax(gm);
      if (!bad) st_sys64(g2, ((u64)call << 32) | gm);
    }
    watch(3, dest);
    STAMP(9);
    watch(4, dest);
    if (g == 0) watch(5, 0);
    { const u64 lim = 30000 + min(tA * 10, (u64)60000); while (now() - t0 < lim) __builtin_amdgcn_s_sleep(4); }
    return;
  }
  u32 stage = 0;
  auto need = [&](u32 s) {
    while (stage < s) {
      stage = lds_ld(ctl + C_STAGE);
      if (stage < s) __builtin_amdgcn_s_sleep(1);
    }
    CFENCE();
  };
  need(1);
  if (wave == 0) STAMP(3);
  call = lds_ld(ctl + C_CALL);
  const __amdgpu_buffer_rsrc_t rs_m = mkrsrc(myws, WS_Q);
  const __amdgpu_buffer_rsrc_t rs_o = mkrsrc(P.out[rank], OUT_BYTES);
  const u32 rot = (g * 26 / 7 % 26) * 10;
  constexpr u32 U_L = U_Q + 3968 - 104, uq2 = U_Q + 3 * 208 + 156;
  const u32 nu = g ? U_L : U_L + 832;
  struct Unit { u32 so, to, fo, fw; };
  u32 qc = 0, qbase = U_Q;
  auto uoff = [&](u32 u) -> Unit {
    Unit d;
    if (u < U_R) {
      const u32 kc = (u + 128) / 384, ord = kc * 8 + dest;
      u32 k = u + 128 - kc * 384;
      if (kc == 1) k = k < 256 ? k + 128 : k - 256;
      if (k < 128) {
        d.so = WS_EW + (rank * 8 + kc) * (8 * WBLK) + k * 2048;
        d.to = WS_W + ord * NWB * WBLK + k * 2048;
        d.fo = WS_WF + (k >> 4) * FSTRIDE + ord * 4;
        d.fw = 15;
      } else {
        const u32 kk = (k - 128 + (kc ? rot : 0)) & 255, r = kk / 10;
        d.so = WS_EX + kc * (512u << 10) + kk * 2048;
        d.to = WS_X + ord * NRB * XBLK + kk * 2048;
        d.fo = WS_XF + r * FSTRIDE + ord * 4;
        d.fw = r == NRB - 1 ? 40 : 24;
      }
    } else if (u < U_Q) {
      const u32 k = u - U_R, kc = k / 28, j = k - kc * 28, ord = kc * 8 + dest;
      const u32 jj = j + ((j >> 2) >= rank ? 4 : 0);
      d.so = WS_ER + (call & 1) * (512u << 10) + kc * 65536 + jj * 2048;
      d.to = WS_W + (ord * NWB + 8) * WBLK + jj * 2048;
      d.fo = WS_WF + (8 + (jj >> 4)) * FSTRIDE + ord * 4;
      d.fw = (jj >> 4) == (rank >> 2) ? 20 : 15;
    } else if (u < U_L) {
      u32 c, na, n;
      for (;;) {
        c = (qc < 8 ? (g & 6) + 2 + qc * 2 + (qc < 4) : g + qc) & 7;
        na = c == 7 ? 156 : 208;
        n = qc < 7 ? na : qc < 8 ? 104 : c == 7 ? 228 : 304;
        if (u - qbase < n) break;
        qbase += n;
        qc += 1;
      }
      const u32 k = u - qbase + (qc >= 8 ? na : qc == 7 && !(g & 1) ? 104 : 0);
      d.so = WS_Q + c * QSTRIDE + k * 2048;
      d.to = k * 128;
      d.fo = c;
      d.fw = 0;
    } else {
      const u32 k = u - U_L, gs = k / 104, r = k - gs * 104;
      d.so = u < nu ? WS_L + k * 2048 : 0x80000000u;
      d.to = ((gs & 1) * 2080 + (r >> 3) * 160 + (r & 1) * 32) * 16384 + ((P.peers >> ((gs & 7) * 4)) & 7u) * 2048 + (gs & 6) * 256 + (r & 6) * 32;
      d.fo = 8;
      d.fw = 0;
    }
    return d;
  };
  u32x4 v0, v1;
  u32 u0 = vid, u1 = NPULL + vid, tk = vid + (call - 1) * nu - NPULL;
  Unit d0 = uoff(u0), d1 = uoff(u1);
  v0 = __builtin_amdgcn_raw_buffer_load_b128(rs_s, lane16, d0.so, 17);
  v1 = __builtin_amdgcn_raw_buffer_load_b128(rs_s, lane16, d0.so + 1024, 17);
  u32 to0 = lane16, to1 = lane16 + 1024, tso = d0.to, lo1 = lane16;
  __amdgpu_buffer_rsrc_t rs_t = rs_m;
  const u32 cv = call * FULL;
  u32 cnt = 0, pfo = 0, pfw = 0;
  auto publish = [&]() {
    if (pfw && __builtin_amdgcn_readfirstlane(cnt) + pfw == cv)
      asm volatile("s_nop 4\n\tbuffer_store_dword %0, %1, %2, %3 offen sc0 sc1" :: "v"(cv), "v"(lane0), "s"(rs_m), "s"(pfo) : "memory");
    pfw = 0;
  };
  WAIT_VM0();
  while (u0 < nu) {
    if (bad && !d0.fw) v0 = v1 = u32x4{0x80808080u, 0x80808080u, 0x80808080u, 0x80808080u};
    __builtin_amdgcn_raw_buffer_store_b128(v0, rs_t, to0, tso, 17);
    __builtin_amdgcn_raw_buffer_store_b128(v1, rs_t, to1, tso, 17);
    CFENCE();
    const u32 ns = u1 >= nu ? 0 : u1 >= U_L ? 7 : u1 >= uq2 ? 6 : u1 >= U_Q ? 5 : u1 >= U_C ? 4 : u1 >= U_B ? 3 : u1 >= U_A ? 2 : 1;
    if (stage < ns) {
      stage = lds_ld(ctl + C_STAGE);
      if (stage < ns) {
        publish();
        WAIT_VM0();
        if (d0.fw) {
          cnt = __builtin_amdgcn_raw_ptr_buffer_atomic_add_i32(d0.fw, rs_m, lane0, d0.fo + CNT_OFF, 16);
          pfo = d0.fo; pfw = d0.fw;
          publish();
        }
        d0.fw = 0;
        need(ns);
      }
      bad = lds_ld(ctl + C_BAD) != 0;
    }
    v0 = __builtin_amdgcn_raw_buffer_load_b128(rs_s, lo1, d1.so, 17);
    v1 = __builtin_amdgcn_raw_buffer_load_b128(rs_s, lo1, d1.so + 1024, 17);
    CFENCE();
    const u32 u2 = 3 * NPULL + (__builtin_amdgcn_readfirstlane(tk) - (call - 1) * nu);
    tk = __builtin_amdgcn_raw_ptr_buffer_atomic_add_i32(1, rs_m, lane0, WS_TK + g * 128, 16);
    CFENCE();
    publish();
    const Unit d2 = uoff(u2);
    tso = d1.to;
    if (u2 >= U_L) lo1 = (lane & 3) * 256 + (lane >> 2) * 16;
    if (u1 >= U_L) {
      rs_t = rs_o;
      to0 = (lane >> 2) * 16384 + (lane & 3) * 16;
      to1 = to0 + 16 * 16384;
    } else if (!d1.fw) {
      rs_t = rs_o;
      tso = dest * 2048 + d1.fo * 256;
      const u32 mul = d1.fo == 7 ? 43691u : 32768u, w16 = d1.fo == 7 ? 12 : 16;
#pragma unroll
      for (int e = 0; e < 2; ++e) {
        const u32 i16 = d1.to + e * 64 + lane;
        const u32 sr = (i16 * mul) >> 19, col = i16 - sr * w16;
        const u32 ra = (sr >> 6) * 96 + sr, rc = (((sr - 1664) * 43691u) >> 22) * 64 + sr - 1600;
        (e ? to1 : to0) = (sr < 1664 ? ra : rc) * 16384 + col * 16;
      }
    }
    CFENCE();
    WAIT_VM(3);
    cnt = __builtin_amdgcn_raw_ptr_buffer_atomic_add_i32(d0.fw, rs_m, lane0, d0.fw ? d0.fo + CNT_OFF : 0x40000000u, 16);
    pfo = d0.fo; pfw = d0.fw;
    CFENCE();
    u0 = u1; u1 = u2;
    d0 = d1; d1 = d2;
  }
  publish();
  WAIT_VM(0);
  if (wave == 0) STAMP(5);
}

extern "C" void kernel_launch(int rank, int nranks, ncclComm_t comm, hipStream_t stream,
                              void* const* d_in, const int* in_sizes, int n_in,
                              void* d_out, int out_size,
                              const int* in_dtypes, int out_dtype,
                              void* const* d_out_all,
                              void* const* d_ws_all, size_t ws_bytes) {
  if (nranks != NR || ws_bytes < WS_END || n_in < 2) { fprintf(stderr, "unsupported configuration\n"); return; }
  Params P;
  P.x = (const char*)d_in[0];
  P.w = (const char*)d_in[1];
  for (int p = 0; p < NR; ++p) { P.out[p] = (char*)d_out_all[p]; P.ws[p] = (char*)d_ws_all[p]; }
  P.rank = rank;
  const int phys[8] = {0, 2, 3, 6, 1, 4, 5, 7}, logi[8] = {0, 4, 1, 2, 5, 6, 3, 7};
  P.peers = 0;
  for (int g = 0; g < NR; ++g) P.peers |= (u32)phys[logi[rank & 7] ^ g] << (g * 4);
  hipLaunchKernelGGL(fused_gemm_ar_q8, dim3(256), dim3(NTHREADS), 0, stream, P);
  HIP_CHECK(hipGetLastError());
}
```

```cpp
#define XSKEW
#include <hip/hip_runtime.h>
#include <rccl/rccl.h>
#include <cstdio>
#include <cstring>
#include <cstdlib>
#include <stdint.h>

#define HIP_CHECK(expr) do { hipError_t _e = (expr); if (_e != hipSuccess) { \
    fprintf(stderr, "HIP error %s at %s:%d\n", hipGetErrorString(_e), __FILE__, __LINE__); \
    return; } } while (0)

typedef unsigned u32;
typedef unsigned long long u64;
typedef u32 u32x4 __attribute__((ext_vector_type(4)));
typedef u32 u32x2 __attribute__((ext_vector_type(2)));
typedef float f32x4 __attribute__((ext_vector_type(4)));
typedef short s16x4 __attribute__((ext_vector_type(4)));
typedef short s16x8 __attribute__((ext_vector_type(8)));
typedef __bf16 bf16x8 __attribute__((ext_vector_type(8)));
#define LAS __attribute__((address_space(3)))
#define GAS __attribute__((address_space(1)))

constexpr int NR = 8;
constexpr u32 OUT_BYTES = 4096u * 16384u;
constexpr u32 X_BYTES = 4096u * 512u * 2u;
constexpr u32 W_BYTES = 512u * 16384u * 2u;

constexpr u32 NRB = 26;
constexpr u32 NCB = 8;
constexpr u32 XBLK = 20480, WBLK = 32768;
constexpr u32 NWB = NCB + 2;
constexpr u32 WS_W = 0;
constexpr u32 WS_X = 20u << 20;
constexpr u32 WS_CTL = 53u << 20;
constexpr u32 FSTRIDE = 4352;
constexpr u32 WS_XF = WS_CTL;
constexpr u32 WS_WF = WS_XF + NRB * FSTRIDE;
constexpr u32 CNT_OFF = 512u << 10;
constexpr u32 WS_C2 = WS_CTL + (160u << 10);
constexpr u32 WS_G1 = WS_C2 + 4096;
constexpr u32 WS_G2 = WS_C2 + 8192 + 256;
constexpr u32 G2S = 512;
constexpr u32 WS_TK = WS_C2 + 13312;
constexpr u32 WS_TMO = WS_C2 + 20480;
constexpr u32 WS_DBG = WS_C2 + 32768;
constexpr u32 WS_FL = WS_C2 + 65536;
constexpr u32 WS_FC = WS_FL + 16384;
constexpr u32 WS_CALL = WS_FC + 16384;
constexpr u32 WS_Q = 54u << 20;
constexpr u32 QSTRIDE = 4160 * 256;
constexpr u32 WS_EX = 63u << 20;
constexpr u32 WS_ER = 67u << 20;
constexpr u32 WS_EW = 68u << 20;
constexpr u32 WS_L = 84u << 20;
constexpr u32 WS_END = 86u << 20;
static_assert(64u * NWB * WBLK <= WS_X && WS_X + 64u * NRB * XBLK <= WS_CTL && WS_WF + NWB * FSTRIDE <= WS_C2 &&
              WS_CALL + 16384 <= WS_CTL + CNT_OFF && WS_C2 + CNT_OFF <= WS_Q && WS_Q + NCB * QSTRIDE <= WS_EX, "layout");
constexpr u32 FULL = 240;

constexpr u32 SLOT_BYTES = WBLK + XBLK;
constexpr u32 NSLOT = 3;
constexpr u32 TPITCH = 272;
constexpr u32 SNAP_OFF = SLOT_BYTES * NSLOT;
constexpr u32 CTL_OFF = SNAP_OFF + 1024;
constexpr u32 LDS_BYTES = CTL_OFF + 1024;
static_assert(LDS_BYTES <= 163840, "lds");
constexpr int C_AMAX = 0, C_GMAX = 1, C_CALL = 2, C_BAD = 3, C_STAGE = 4, C_CNT = 8, C_MASK = 16;

#define NPW 4
#define EXWAIT 300
#define EXD 60
#define PW1 4
#define PLD 4
#define POLL2 1
#define QWAIT 500
#define QD 100
#define PUSHD 2
constexpr u32 NCOMMWG = 32 - NRB;
constexpr u32 NTILE = NRB * NCB;
constexpr u32 NPULL = NCOMMWG * NPW;
constexpr u32 U_A = 104, U_B = 512, U_C = 1152;
constexpr u32 U_R = 256 + 7 * 384;
constexpr u32 U_Q = U_R + 224;
constexpr int NTHREADS = 512;
constexpr int NSTEP = 64;
constexpr u64 DEADLINE = 50000000ull;

struct Params {
  const char* x;
  const char* w;
  char* out[NR];
  char* ws[NR];
  int rank;
  u32 peers;
};

__device__ __forceinline__ __amdgpu_buffer_rsrc_t mkrsrc(const void* p, u32 bytes) {
  return __builtin_amdgcn_make_buffer_rsrc((void*)p, 0, bytes, 0x00020000);
}
__device__ __forceinline__ u64 now() { return __builtin_amdgcn_s_memrealtime(); }
__device__ __forceinline__ u32 ld_sys(const u32* p) { return __hip_atomic_load((GAS const u32*)p, __ATOMIC_RELAXED, __HIP_MEMORY_SCOPE_SYSTEM); }
__device__ __forceinline__ u64 ld_sys64(const u64* p) { return __hip_atomic_load((GAS const u64*)p, __ATOMIC_RELAXED, __HIP_MEMORY_SCOPE_SYSTEM); }
__device__ __forceinline__ void st_sys(u32* p, u32 v) { __hip_atomic_store((GAS u32*)p, v, __ATOMIC_RELAXED, __HIP_MEMORY_SCOPE_SYSTEM); }
__device__ __forceinline__ void st_sys64(u64* p, u64 v) { __hip_atomic_store((GAS u64*)p, v, __ATOMIC_RELAXED, __HIP_MEMORY_SCOPE_SYSTEM); }
__device__ __forceinline__ u32 lds_ld(LAS u32* p) { return __builtin_amdgcn_readfirstlane(__hip_atomic_load(p, __ATOMIC_RELAXED, __HIP_MEMORY_SCOPE_WORKGROUP)); }
__device__ __forceinline__ void lds_st(LAS u32* p, u32 v) { __hip_atomic_store(p, v, __ATOMIC_RELAXED, __HIP_MEMORY_SCOPE_WORKGROUP); }
__device__ __forceinline__ void lds_add1(LAS u32* p, u32 lane) {
  if (lane == 0) __hip_atomic_fetch_add(p, 1u, __ATOMIC_RELAXED, __HIP_MEMORY_SCOPE_WORKGROUP);
}
#define CFENCE() asm volatile("" ::: "memory")
#define WAIT_VM(n) asm volatile("s_waitcnt vmcnt(" #n ")" ::: "memory")
#define WAIT_VMI(n) asm volatile("s_waitcnt vmcnt(%0)" :: "i"(n) : "memory")
#define WAIT_LGKM() asm volatile("s_waitcnt lgkmcnt(0)" ::: "memory")
#define WAIT_VM0() do { CFENCE(); __builtin_amdgcn_s_waitcnt(0x0F70); WAIT_VM(0); } while (0)

__device__ __forceinline__ u32 wave_umax(u32 v) {
  v = max(v, (u32)__builtin_amdgcn_update_dpp(0, (int)v, 0x111, 0xf, 0xf, true));
  v = max(v, (u32)__builtin_amdgcn_update_dpp(0, (int)v, 0x112, 0xf, 0xf, true));
  v = max(v, (u32)__builtin_amdgcn_update_dpp(0, (int)v, 0x114, 0xf, 0xf, true));
  v = max(v, (u32)__builtin_amdgcn_update_dpp(0, (int)v, 0x118, 0xf, 0xf, true));
  v = max(v, (u32)__builtin_amdgcn_update_dpp(0, (int)v, 0x142, 0xa, 0xf, false));
  v = max(v, (u32)__builtin_amdgcn_update_dpp(0, (int)v, 0x143, 0xc, 0xf, false));
  return (u32)__builtin_amdgcn_readlane((int)v, 63);
}

#define STAMP(k) do {} while (0)

__device__ __forceinline__ void wait_vm_dyn(u32 n) {
  asm volatile(
      "s_getpc_b64 vcc\n\t"
      "s_add_u32 vcc_lo, vcc_lo, %0\n\t"
      "s_addc_u32 vcc_hi, vcc_hi, 0\n\t"
      "s_setpc_b64 vcc\n\t"
      "s_waitcnt vmcnt(0)\n\ts_branch 1f\n\t"
      "s_waitcnt vmcnt(1)\n\ts_branch 1f\n\t"
      "s_waitcnt vmcnt(2)\n\ts_branch 1f\n\t"
      "s_waitcnt vmcnt(3)\n\ts_branch 1f\n\t"
      "s_waitcnt vmcnt(4)\n\ts_branch 1f\n\t"
      "s_waitcnt vmcnt(5)\n\ts_branch 1f\n\t"
      "s_waitcnt vmcnt(6)\n\ts_branch 1f\n\t"
      "s_waitcnt vmcnt(7)\n\ts_branch 1f\n\t"
      "s_waitcnt vmcnt(8)\n\ts_branch 1f\n\t"
      "s_waitcnt vmcnt(9)\n\ts_branch 1f\n\t"
      "s_waitcnt vmcnt(10)\n\ts_branch 1f\n\t"
      "s_waitcnt vmcnt(11)\n\ts_branch 1f\n\t"
      "s_waitcnt vmcnt(12)\n\ts_branch 1f\n\t"
      "s_waitcnt vmcnt(13)\n\ts_branch 1f\n\t"
      "s_waitcnt vmcnt(14)\n\ts_branch 1f\n\t"
      "s_waitcnt vmcnt(15)\n\ts_branch 1f\n\t"
      "s_waitcnt vmcnt(16)\n\ts_branch 1f\n\t"
      "s_waitcnt vmcnt(17)\n\ts_branch 1f\n\t"
      "s_waitcnt vmcnt(18)\n\ts_branch 1f\n\t"
      "s_waitcnt vmcnt(19)\n\ts_branch 1f\n\t"
      "s_waitcnt vmcnt(20)\n\ts_branch 1f\n\t"
      "s_waitcnt vmcnt(21)\n\ts_branch 1f\n\t"
      "s_waitcnt vmcnt(22)\n\ts_branch 1f\n\t"
      "s_waitcnt vmcnt(23)\n\ts_branch 1f\n\t"
      "s_waitcnt vmcnt(24)\n\t"
      "1:"
      :: "s"(min(n, 24u) * 8 + 12) : "memory", "scc", "vcc");
}
#define BARRIER() asm volatile("s_barrier" ::: "memory")

__global__ void __launch_bounds__(NTHREADS) fused_gemm_ar_q8(Params P) {
  __shared__ __attribute__((aligned(16))) char smem[LDS_BYTES];
  LAS char* L = (LAS char*)smem;
  LAS u32* ctl = (LAS u32*)(L + CTL_OFF);
  const u32 tid = threadIdx.x, lane = tid & 63;
  const u32 wave = __builtin_amdgcn_readfirstlane(tid >> 6);
  const u32 g = blockIdx.x & 7u, idx = blockIdx.x >> 3;
  const u32 rank = (u32)P.rank & 7u;
  const u32 dest = (P.peers >> (g * 4)) & 7u;
  char* const myws = P.ws[rank];
  char* const dws = P.ws[dest];
  u32* const tmo = (u32*)(myws + WS_TMO);
  const u32 lane16 = lane * 16;
  const u64 t0 = now();
  const u32 lx = (lane & 7) * 1024 + (lane >> 3) * 16;
  const u32 lw = (((lane >> 3) & 1) * 8 + ((lane >> 1) & 3)) * 32768 + ((lane & 1) * 4 + (lane >> 4)) * 16;
  u32 call = 0;
  bool bad = false;
  if (wave == 0) STAMP(0);

  if (idx >= NCOMMWG) {
    const u32 ti = idx - NCOMMWG;
    const u32 rb = (g & 1) * 13 + (ti >> 1);
    u32 cb = (g >> 1) * 2 + (ti & 1);
    const u32 tile = g * NRB + ti;
    const __amdgpu_buffer_rsrc_t rs_ws = mkrsrc(myws, WS_END);
    char* flp = P.ws[lane & 7] + WS_FL + rank * 256 + g * 4;
    const u32 lane0 = lane ? 0x80000000u : 0u;
    auto signal_wg = [&](u32 kind, u32 inc) {
      const u32 old = __builtin_amdgcn_raw_ptr_buffer_atomic_add_i32(inc, rs_ws, lane0, WS_FC + kind * 2048 + g * 256, 16);
      if (__builtin_amdgcn_readfirstlane(old) + inc == call * NRB && lane < 8)
        st_sys((u32*)(flp + kind * 2048), call);
      CFENCE();
    };
    auto signal_all = [&](u32 kind, u32 inc = 1) {
      u32 old = 0;
      if (lane == 0) old = __hip_atomic_fetch_add(ctl + C_CNT + kind, 1u, __ATOMIC_RELAXED, __HIP_MEMORY_SCOPE_WORKGROUP);
      if (__builtin_amdgcn_readfirstlane(old) == 7) signal_wg(kind, inc);
    };
    {
      const u32 pc = g ? 2 * U_A + (tile - NRB) * 5 + wave : ti * 8 + wave;
      const bool pok = (wave < 5 || !g) && pc < 1024;
      const u32x4 e0 = __builtin_amdgcn_raw_buffer_load_b128(mkrsrc(P.x, X_BYTES), lx,
                                                             pok ? (pc & 511) * 8192 + (pc >> 9) * 128 : 0x80000000u, 0);
      CFENCE();
      if (wave == 7 && lane < 16) ctl[lane] = 0u;
      __builtin_amdgcn_raw_buffer_store_b128(e0, rs_ws, lane16, pok ? WS_EX + pc * 1024 : 0x80000000u, 17);
      asm volatile("" : "+v"(flp));
      WAIT_VM0();
      WAIT_LGKM();
      BARRIER();
      if (wave == 0) {
        const u32 old = __builtin_amdgcn_readfirstlane(__builtin_amdgcn_raw_ptr_buffer_atomic_add_i32(1, rs_ws, lane0, WS_FC + g * 256, 16));
        call = old / NRB + 1;
        if (old + 1 == call * NRB && lane < 8) {
          st_sys((u32*)flp, call);
        }
        if (lane == 0) lds_st(ctl + C_CALL, call);
        WAIT_LGKM();
      }
      if (wave == 6 && ti == 0) __builtin_amdgcn_fence(__ATOMIC_ACQUIRE, "");
      BARRIER();
      call = lds_ld(ctl + C_CALL);
      for (u32 r = 0; r < 3; ++r) {
        const u32 slot = r * 8 + wave, e = slot * NTILE + tile;
        u32 kc, k, so, eo;
        bool isw = true;
        if (e < 4352) {
          kc = (e + 768) / 640; k = e + 768 - kc * 640;
          if (kc == 1) k -= 128;
          isw = k < 512;
          k &= 511;
          eo = isw ? WS_EW + ((k >> 6) * 8 + kc) * (8 * WBLK) + (k & 63) * 4096 : WS_EX + kc * (512u << 10) + k * 4096;
        } else {
          kc = (e - 4352) >> 4; k = ((e >> 1) & 7) * 64 + 62 + (e & 1);
          eo = WS_ER + (call & 1) * (512u << 10) + (e - 4352) * 4096;
        }
        if (e < 4480) {
          so = isw ? (kc * 64 + (k & 1) * 32) * 32768 + (k >> 1) * 128 : k * 32768 + kc * 128;
          const __amdgpu_buffer_rsrc_t rs = mkrsrc(isw ? P.w : P.x, isw ? W_BYTES : X_BYTES);
          const u32 lo = isw ? lw : lx, s1 = isw ? 4 * 32768 : 8192, s2 = isw ? 16 * 32768 : 16384;
          while (now() - t0 < EXWAIT + slot * EXD) __builtin_amdgcn_s_sleep(2);
          const u32x4 e0 = __builtin_amdgcn_raw_buffer_load_b128(rs, lo, so, 0);
          const u32x4 e1 = __builtin_amdgcn_raw_buffer_load_b128(rs, lo, so + s1, 0);
          const u32x4 e2 = __builtin_amdgcn_raw_buffer_load_b128(rs, lo, so + s2, 0);
          const u32x4 e3 = __builtin_amdgcn_raw_buffer_load_b128(rs, lo, so + s1 + s2, 0);
          __builtin_amdgcn_raw_buffer_store_b128(e0, rs_ws, lane16, eo, 17);
          __builtin_amdgcn_raw_buffer_store_b128(e1, rs_ws, lane16, eo + 1024, 17);
          __builtin_amdgcn_raw_buffer_store_b128(e2, rs_ws, lane16, eo + 2048, 17);
          __builtin_amdgcn_raw_buffer_store_b128(e3, rs_ws, lane16, eo + 3072, 17);
        }
        if (r != 1) {
          WAIT_VM0();
          signal_all(1 + (r >> 1));
        }
      }
      BARRIER();
    }
    const u32 cv = call * FULL;
    const u32 wm = wave >> 2, wn = wave & 3;
    const u32 lds0 = (u32)(size_t)L;
    const u32 xa = WBLK + wm * 10240 + ((lane >> 3) & 1) * 1024 + (lane >> 5) * 256 + ((lane >> 4) & 1) * 128 + (lane & 7) * 16;
    const u32 h = lane >> 4, q = (lane >> 2) & 3, p = lane & 3;
    const u32 wa0 = wn * 8192 + (h >> 1) * 2048 + (p & 1) * 512 + (h & 1) * 128 + q * 32 + (p >> 1) * 16 + (p & 1) * 8;
    const u32 wa1 = wa0 ^ 8;
    const u32 xpo = wave < 4 ? wave * 3072 : 12288 + (wave - 4) * 2048;
    u32 pollo;
    f32x4 acc[4][5];
    u64 kx = 0, kw;
    u64 ready, taken;
    u32 issued, consumed, nis = 0;
    u32 islot, cslot, par = 0;
    u64 markD = 0;
    float inv = 0.f;
    auto sched = [&]() {
      u64 avail = ready & ~taken;
      while (avail && issued - consumed < NSLOT) {
        const u32 ord = __builtin_ctzll(avail);
        avail &= avail - 1;
        taken |= 1ull << ord;
        const u32 sw = WS_W + (ord * NWB + cb) * WBLK + wave * 4096;
        const u32 sx = WS_X + (ord * NRB + rb) * XBLK + xpo;
        const u32 mw = lds0 + islot + wave * 4096, mx = lds0 + islot + WBLK + xpo;
        u32 keep;
        asm volatile(
            "s_mov_b32 %0, m0\n\t"
            "s_mov_b32 m0, %3\n\t"
            "s_nop 0\n\t"
            "buffer_load_dwordx4 %1, %2, %5 offen lds\n\t"
            "buffer_load_dwordx4 %1, %2, %5 offen offset:1024 lds\n\t"
            "buffer_load_dwordx4 %1, %2, %5 offen offset:2048 lds\n\t"
            "buffer_load_dwordx4 %1, %2, %5 offen offset:3072 lds\n\t"
            "s_mov_b32 m0, %4\n\t"
            "s_nop 0\n\t"
            "buffer_load_dwordx4 %1, %2, %6 offen lds\n\t"
            "buffer_load_dwordx4 %1, %2, %6 offen offset:1024 lds\n\t"
            "s_cmp_gt_u32 %7, 3\n\t"
            "s_cbranch_scc1 1f\n\t"
            "buffer_load_dwordx4 %1, %2, %6 offen offset:2048 lds\n\t"
            "1:\n\t"
            "s_mov_b32 m0, %0"
            : "=&s"(keep)
            : "v"(lane16), "s"(rs_ws), "s"(mw), "s"(mx), "s"(sw), "s"(sx), "s"(wave)
            : "memory", "scc");
        nis += wave < 4 ? 7 : 6;
        const u32 sh = (issued & 3) * 16;
        markD = (markD & ~(0xffffull << sh)) | ((u64)(nis & 0xffff) << sh);
        issued += 1;
        islot = islot == 2 * SLOT_BYTES ? 0 : islot + SLOT_BYTES;
      }
    };
    auto poll = [&]() {
      if (wave == 0 && (kx & kw) != ~0ull) {
        const u64 kn = lane < 16 ? kx : kw;
        const bool act = lane < 32 && ((u32)(kn >> ((lane & 15) * 4)) & 15u) != 15u;
        const u32 po = act ? pollo : 0x80000000u;
        u32 keep;
        asm volatile("s_mov_b32 %0, m0\n\ts_mov_b32 m0, %3\n\ts_nop 0\n\t"
                     "buffer_load_dwordx4 %1, %2, 0 offen sc0 sc1 lds\n\ts_mov_b32 m0, %0"
                     : "=&s"(keep) : "v"(po), "s"(rs_ws), "s"(lds0 + SNAP_OFF) : "memory");
        nis += 1;
      }
    };
    LAS u32* const snap = (LAS u32*)(L + SNAP_OFF) + lane;
    auto seen = [&](u32 fx, u32 fw) {
      kx |= __ballot((int)(fx - cv) >= 0);
      kw |= __ballot((int)(fw - cv) >= 0);
    };
    auto tell = [&]() { if (lane == 0) *(LAS u64*)(L + CTL_OFF + C_MASK * 4 + par) = kx & kw; };
    auto told = [&]() -> u64 {
      const u64 m = *(LAS u64*)(L + CTL_OFF + C_MASK * 4 + par);
      par ^= 8;
      return m;
    };
    auto uni = [&](u64 m) -> u64 {
      return (u64)(u32)__builtin_amdgcn_readfirstlane((u32)m) | ((u64)(u32)__builtin_amdgcn_readfirstlane((u32)(m >> 32)) << 32);
    };
    auto wait_step = [&]() { wait_vm_dyn((nis - (u32)(markD >> ((consumed & 3) * 16))) & 0xffff); };
#define SB() __builtin_amdgcn_sched_barrier(0)
#define PIECE(M, S, OFF) do { \
        asm volatile("s_cmp_lg_u32 %2, -1\n\ts_cbranch_scc0 1f\n\ts_mov_b32 m0, %2\n\ts_nop 0\n\t" \
                     "buffer_load_dwordx4 %0, %1, %3 offen offset:" #OFF " lds\n\t1:" \
                     :: "v"(lane16), "s"(rs_ws), "s"(M), "s"(S) : "memory", "scc"); \
        SB(); } while (0)
#define MMA4(kb, i) do { \
        _Pragma("unroll") for (int n = 0; n < 4; ++n) \
          acc[n][i] = __builtin_amdgcn_mfma_f32_16x16x32_bf16(wf[kb][n], xf[kb][i], acc[n][i], 0, 0, 0); \
        SB(); } while (0)
    bf16x8 xf[2][5], wf[2][4];
    u32 sp = 0;
    u64 cyc0 = 0;
    for (u32 pass = 0;; ++pass) {
    pollo = lane < 16 ? WS_XF + rb * FSTRIDE + lane16 : WS_WF + cb * FSTRIDE + lane16 - 256;
    kw = taken = 0;
    issued = consumed = islot = cslot = 0;
#pragma unroll
    for (int a = 0; a < 4; ++a)
#pragma unroll
      for (int b = 0; b < 5; ++b) acc[a][b] = f32x4{0.f, 0.f, 0.f, 0.f};
#pragma unroll
    for (int n = 0; n < 5; ++n) {
      xf[1][n] = __builtin_bit_cast(bf16x8, u32x4{0u, 0u, 0u, 0u});
      if (n < 4) wf[1][n] = __builtin_bit_cast(bf16x8, u32x4{0u, 0u, 0u, 0u});
    }
    poll();
    while (consumed < NSTEP) {
      if (issued == consumed) {
        if (wave == 0) {
          while ((kx & kw & ~taken) == 0) {
            poll();
            WAIT_VM(1);
            seen(snap[0], snap[64]);
            if ((++sp & 63) == 0 && now() - t0 > DEADLINE) {
              st_sys(tmo, call);
              WAIT_VM(0);
              lds_st(ctl + C_BAD, 1u);
              kx = kw = ~0ull;
            }
          }
          tell();
          WAIT_LGKM();
        }
        BARRIER();
        ready = uni(told());
        sched();
        poll();
        wait_step();
        if (wave == 0) {
          seen(snap[0], snap[64]);
          tell();
          WAIT_LGKM();
        }
      }
      BARRIER();
      const u64 rdy = told();
      SB();
      LAS char* base = L + cslot;
      auto frags = [&](int kb) {
#pragma unroll
        for (int n = 0; n < 5; ++n) {
          if (n < 4) {
            LAS char* a = base + ((n & 1) ? wa1 : wa0) + kb * 4096 + (n >> 1) * 256;
            s16x4 lo = __builtin_amdgcn_ds_read_tr16_b64_v4i16((LAS s16x4*)a);
            s16x4 hi = __builtin_amdgcn_ds_read_tr16_b64_v4i16((LAS s16x4*)(a + 1024));
            wf[kb][n] = __builtin_bit_cast(bf16x8, __builtin_shufflevector(lo, hi, 0, 1, 2, 3, 4, 5, 6, 7));
          }
          xf[kb][n] = *(LAS bf16x8*)(base + xa + n * 2048 + kb * 512);
        }
        SB();
      };
      MMA4(1, 0);
      frags(0);
      poll();
      SB();
      MMA4(1, 1);
      ready = uni(rdy);
      const u64 av = ready & ~taken;
      const bool go = av != 0 && issued - consumed < NSLOT;
      const u32 ord = __builtin_ctzll(av | (1ull << 63));
      const u32 sw = WS_W + (ord * NWB + cb) * WBLK + wave * 4096;
      const u32 sx = WS_X + (ord * NRB + rb) * XBLK + xpo;
      u32 mw = ~0u, mx = ~0u, mx3 = ~0u;
      if (go) {
        mw = lds0 + islot + wave * 4096;
        mx = lds0 + islot + WBLK + xpo;
        if (wave < 4) mx3 = mx;
        taken |= 1ull << ord;
        nis += wave < 4 ? 7 : 6;
        const u32 sh = (issued & 3) * 16;
        markD = (markD & ~(0xffffull << sh)) | ((u64)(nis & 0xffff) << sh);
        issued += 1;
        islot = islot == 2 * SLOT_BYTES ? 0 : islot + SLOT_BYTES;
      }
      SB();
      MMA4(1, 2);
      PIECE(mw, sw, 0);
      MMA4(1, 3);
      PIECE(mw, sw, 1024);
      MMA4(1, 4);
      frags(1);
      PIECE(mw, sw, 2048);
      MMA4(0, 0);
      PIECE(mw, sw, 3072);
      PIECE(mx, sx, 0);
      MMA4(0, 1);
      PIECE(mx, sx, 1024);
      PIECE(mx3, sx, 2048);
      MMA4(0, 2);
      sched();
      poll();
      cslot = cslot == 2 * SLOT_BYTES ? 0 : cslot + SLOT_BYTES;
      consumed += 1;
      SB();
      MMA4(0, 3);
      if (issued != consumed) wait_step();
      u32 fx = cv - 1, fw = cv - 1;
      if (wave == 0) { fx = snap[0]; fw = snap[64]; }
      SB();
      MMA4(0, 4);
      if (wave == 0) { seen(fx, fw); tell(); }
      WAIT_LGKM();
    }
    MMA4(1, 0); MMA4(1, 1); MMA4(1, 2); MMA4(1, 3); MMA4(1, 4);
    if (pass == 0) {
      if (wave == 0) STAMP(2);
      const u32 row0 = rb * 160 + wm * 80;
      float m = 0.f;
#pragma unroll
      for (int b = 0; b < 5; ++b) {
        float mb = 0.f;
#pragma unroll
        for (int a = 0; a < 4; ++a)
#pragma unroll
          for (int e = 0; e < 4; ++e) mb = fmaxf(mb, fabsf(acc[a][b][e]));
        if (row0 + b * 16 < 4096) m = fmaxf(m, mb);
      }
      const u32 mb = wave_umax(__builtin_bit_cast(u32, m));
      if (lane == 0) __hip_atomic_fetch_max(ctl + C_AMAX, mb, __ATOMIC_RELAXED, __HIP_MEMORY_SCOPE_WORKGROUP);
      WAIT_LGKM();
      BARRIER();
      if (wave == 0) {
        const u32 wmax = lds_ld(ctl + C_AMAX);
        if (lane == 0) st_sys64((u64*)(myws + WS_G1) + tile, ((u64)call << 32) | wmax);
        STAMP(3);
        const u64* g2 = (const u64*)(myws + WS_G2 + g * G2S) + (lane & 7);
        u64 val;
        for (;;) {
          val = ld_sys64(g2);
          if (__all((u32)(val >> 32) == call)) break;
          if ((++sp & 63) == 0 && (ld_sys(tmo) == call || now() - t0 > DEADLINE)) { st_sys(tmo, call); bad = true; break; }
        }
        const u32 gmax = wave_umax((u32)val);
        if (lane == 0) {
          lds_st(ctl + C_GMAX, gmax);
          if (bad) lds_st(ctl + C_BAD, 1u);
        }
        WAIT_LGKM();
      }
    }
    BARRIER();
    if (pass == 0) {
      const float amax = __builtin_bit_cast(float, lds_ld(ctl + C_GMAX));
      const float scale = amax / 127.0f;
      inv = amax > 0.f ? 1.0f / scale : 0.f;
      if (wave == 0) STAMP(4);
    }
    bad = lds_ld(ctl + C_BAD) != 0;
    const bool odd = (lane >> 4) & 1;
    u32x4 qv[5];
#pragma unroll
    for (int i = 0; i < 5; ++i) {
      u32 d[4];
#pragma unroll
      for (int n = 0; n < 4; ++n) {
        u32 b0 = __builtin_bit_cast(u32, __builtin_fmaf(acc[n][i][0], inv, 12582912.0f));
        u32 b1 = __builtin_bit_cast(u32, __builtin_fmaf(acc[n][i][1], inv, 12582912.0f));
        u32 b2 = __builtin_bit_cast(u32, __builtin_fmaf(acc[n][i][2], inv, 12582912.0f));
        u32 b3 = __builtin_bit_cast(u32, __builtin_fmaf(acc[n][i][3], inv, 12582912.0f));
        u32 lo = __builtin_amdgcn_perm(b1, b0, 0x0c0c0400u);
        u32 hi = __builtin_amdgcn_perm(b3, b2, 0x04000c0cu);
        d[n] = bad ? 0x80808080u : (lo | hi);
      }
      qv[i] = u32x4{odd ? d[1] : d[0], odd ? d[0] : d[1], odd ? d[3] : d[2], odd ? d[2] : d[3]};
    }
    const u64 tq = now();
    if (!pass && !(cb & 1) && wm == 0) {
      const __amdgpu_buffer_rsrc_t rs_p = mkrsrc(dws, WS_END);
#pragma unroll
      for (int i = 0; i < 4; ++i) {
        if (i && i % PUSHD == 0) WAIT_VM(0);
        if (wave == 0 && i == PUSHD) STAMP(10);
        __builtin_amdgcn_raw_buffer_store_b128(qv[i], rs_p, lane16, WS_L + ((g * 13 + (ti >> 1)) * 16 + wn * 4 + i) * 1024, 17);
        CFENCE();
      }
      WAIT_VM0();
      if (wave == 0) STAMP(11);
      const u32 old = __builtin_amdgcn_raw_ptr_buffer_atomic_add_i32(1, rs_ws, lane0, WS_FC + 5 * 2048 + g * 256, 16);
      if (__builtin_amdgcn_readfirstlane(old) + 1 == call * 52 && lane == 0) st_sys((u32*)(dws + WS_FL + 5 * 2048) + g, call);
      CFENCE();
    }
    {
      LAS char* tw = L + (wm * 80 + (lane & 15)) * TPITCH + wn * 64 + (lane >> 4) * 16;
#pragma unroll
      for (int i = 0; i < 5; ++i) *(LAS u32x4*)(tw + i * 16 * TPITCH) = qv[i];
    }
    WAIT_LGKM();
    BARRIER();
    LAS char* tr = L + (wave * 4 + (lane >> 4)) * TPITCH + (lane & 15) * 16;
#pragma unroll
    for (int i = 0; i < 5; ++i) qv[i] = *(LAS u32x4*)(tr + i * 32 * TPITCH);
    const __amdgpu_buffer_rsrc_t rs_o = mkrsrc(P.out[rank], OUT_BYTES);
    const u32 orow = (rb * 160 + wave * 4 + (lane >> 4)) * 16384u;
    if (pass) {
      const u32 piece = (cb - 8) * 4 + ((lane >> 2) & 3);
      const u32 oo = piece == rank ? 0x80000000u : orow + piece * 2048 + 1984 + (lane & 3) * 16;
#pragma unroll
      for (int i = 0; i < 5; ++i) __builtin_amdgcn_raw_buffer_store_b128(qv[i], rs_o, oo, i * 32 * 16384, 17);
      break;
    }
    const u32 pitch = cb == 7 ? 192 : 256;
    const bool rep = (lane & 15) * 16 >= pitch;
    const u32 qo = rep ? 0x80000000u : (wave * 4 + (lane >> 4)) * pitch + (lane & 15) * 16;
    const u32 oo = orow + rank * 2048 + cb * 256 + (lane & 15) * 16;
    const u32 qa = WS_Q + cb * QSTRIDE + rb * 64 * pitch, qb = WS_Q + cb * QSTRIDE + (1664 + rb * 96) * pitch;
    if (cb & 1) {
      __builtin_amdgcn_raw_buffer_store_b128(qv[0], rs_ws, qo, qa, 17);
      __builtin_amdgcn_raw_buffer_store_b128(qv[1], rs_ws, qo, qa + 32 * pitch, 17);
      WAIT_VM0();
      signal_all(3, 2);
      if (wave == 0) STAMP(9);
    }
    while (now() - tq < QWAIT + wave * QD) __builtin_amdgcn_s_sleep(2);
    if (!(cb & 1)) {
      __builtin_amdgcn_raw_buffer_store_b128(qv[0], rs_ws, qo, qa, 17);
      __builtin_amdgcn_raw_buffer_store_b128(qv[1], rs_ws, qo, qa + 32 * pitch, 17);
    }
#pragma unroll
    for (int i = 2; i < 5; ++i) __builtin_amdgcn_raw_buffer_store_b128(qv[i], rs_ws, qo, qb + (i - 2) * 32 * pitch, 17);
    WAIT_VM0();
    signal_all(4);
    if (cb == 7) {
#pragma unroll
      for (int i = 0; i < 5; ++i) __builtin_amdgcn_raw_buffer_store_b128(qv[i], rs_o, rep ? oo : 0x80000000u, i * 32 * 16384, 17);
    }
    if (((ti >> 1) & 3) != (g >> 1)) break;
    cb = 8 + (ti & 1);
    WAIT_VM0();
    WAIT_LGKM();
    BARRIER();
    }
    WAIT_VM(0);
    if (wave == 0) STAMP(5);
    return;
  }

  const u32 vid = idx * NPW + wave;
  const __amdgpu_buffer_rsrc_t rs_s = mkrsrc(dws, WS_END);
  const u32 lane0 = lane ? 0x80000000u : 0u;
  static_assert(NPW == 4 && NCOMMWG == 6, "wave roles");
  if (wave >= NPW && wave < 7) {
    const u32 ph = (wave - NPW) * NCOMMWG + idx;
    const __amdgpu_buffer_rsrc_t rs_w = mkrsrc(P.w, W_BYTES);
    const u32 so = ph < 16 ? dest * 4096 + ph * 256 : 0x80000000u;
    const u32 to = ph < 16 ? WS_W + rank * NWB * WBLK + ph * 16384 : 0x80000000u;
    u32x4 pv[16];
#pragma unroll
    for (int i = 0; i < 16; ++i) {
      pv[i] = __builtin_amdgcn_raw_buffer_load_b128(rs_w, lw, so + (i >> 3) * 128 + (((i >> 1) & 3) * 16 + (i & 1) * 4) * 32768, 0);
      if (i == PLD - 1) BARRIER();
    }
    CFENCE();
#pragma unroll
    for (int i = 0; i < 16; ++i) {
      if (i >= PW1) WAIT_VMI(PW1 - 1);
      __builtin_amdgcn_raw_buffer_store_b128(pv[i], rs_s, lane16, to + i * 1024, 17);
      CFENCE();
    }
    WAIT_VM(0);
    if (wave == NPW) STAMP(10);
    if (ph < 16) __builtin_amdgcn_raw_ptr_buffer_atomic_add_i32(FULL / 2, rs_s, lane0, WS_WF + (ph >> 1) * FSTRIDE + rank * 4, 16);
    return;
  }
  if (wave == 7) {
    ctl[lane] = 0u;
    WAIT_LGKM();
  }
  BARRIER();
  if (wave == 7) {
    u32* const cptr = (u32*)(myws + WS_CALL + blockIdx.x * 256);
    call = __builtin_amdgcn_readfirstlane(ld_sys(cptr)) + 1;
    if (lane == 0) { lds_st(ctl + C_CALL, call); st_sys(cptr, call); }
    u32 sp = 0;
    auto expired = [&]() -> bool {
      if ((++sp & 63) == 0 && (ld_sys(tmo) == call || now() - t0 > DEADLINE)) {
        st_sys(tmo, call);
        if (lane == 0) { lds_st(ctl + C_BAD, 1u); lds_st(ctl + C_STAGE, 7u); }
        bad = true;
      }
      return bad;
    };
    auto watch = [&](u32 kind, u32 row) {
      const u32* fp = (const u32*)(myws + WS_FL + kind * 2048 + row * 256) + (lane & 7);
      while (!bad) {
        const u32 f = ld_sys(fp);
        const u64 ok = __ballot((int)(f - call) >= 0);
        if (ok == ~0ull) break;
        if (kind == 0 && (ok & 1) && lane == 0) lds_st(ctl + C_STAGE, 1u);
        __builtin_amdgcn_s_sleep(1);
        expired();
      }
      if (lane == 0 && !bad) lds_st(ctl + C_STAGE, kind + 2);
      CFENCE();
    };
    watch(0, dest);
    const u64 tA = now() - t0;
    STAMP(11);
    watch(1, dest);
    watch(2, dest);
    if (blockIdx.x == 0) {
      const u64* g1 = (const u64*)(myws + WS_G1);
      u64* g2 = (u64*)(P.ws[lane & 7] + WS_G2 + (lane >> 3) * G2S) + rank;
      asm volatile("" : "+v"(g2));
      u32 gm = 0;
      while (!bad) {
        u64 a0 = ld_sys64(g1 + lane), a1 = ld_sys64(g1 + 64 + lane), a2 = ld_sys64(g1 + 128 + lane);
        u64 a3 = ld_sys64(g1 + 192 + (lane & 15));
        const bool ok = (u32)(a0 >> 32) == call && (u32)(a1 >> 32) == call && (u32)(a2 >> 32) == call && (u32)(a3 >> 32) == call;
        gm = max(max((u32)a0, (u32)a1), max((u32)a2, (u32)a3));
        const u64 okm = __ballot(ok);
        if (okm == ~0ull) break;
        if (okm == 0) __builtin_amdgcn_s_sleep(8);
        expired();
      }
      gm = wave_umax(gm);
      if (!bad) st_sys64(g2, ((u64)call << 32) | gm);
    }
    watch(3, dest);
    const u64 tQ = now() - t0;
    STAMP(9);
    watch(4, dest);
    if (g == 0) watch(5, 0);
    { u32 xcc; asm volatile("s_getreg_b32 %0, hwreg(HW_REG_XCC_ID)" : "=s"(xcc));
      const u64 lim = 30000 + (tA >> 40) + tQ + 100000 * ((xcc - blockIdx.x) & 7); while (now() - t0 < lim) __builtin_amdgcn_s_sleep(4); }
    return;
  }
  u32 stage = 0;
  auto need = [&](u32 s) {
    while (stage < s) {
      stage = lds_ld(ctl + C_STAGE);
      if (stage < s) __builtin_amdgcn_s_sleep(1);
    }
    CFENCE();
  };
  need(1);
  if (wave == 0) STAMP(3);
  call = lds_ld(ctl + C_CALL);
  const __amdgpu_buffer_rsrc_t rs_m = mkrsrc(myws, WS_Q);
  const __amdgpu_buffer_rsrc_t rs_o = mkrsrc(P.out[rank], OUT_BYTES);
  const u32 rot = (g * 26 / 7 % 26) * 10;
  constexpr u32 U_L = U_Q + 3968 - 104, uq2 = U_Q + 3 * 208 + 156;
  const u32 nu = g ? U_L : U_L + 832;
  struct Unit { u32 so, to, fo, fw; };
  u32 qc = 0, qbase = U_Q;
  auto uoff = [&](u32 u) -> Unit {
    Unit d;
    if (u < U_R) {
      const u32 kc = (u + 128) / 384, ord = kc * 8 + dest;
      u32 k = u + 128 - kc * 384;
      if (kc == 1) k = k < 256 ? k + 128 : k - 256;
      if (k < 128) {
        d.so = WS_EW + (rank * 8 + kc) * (8 * WBLK) + k * 2048;
        d.to = WS_W + ord * NWB * WBLK + k * 2048;
        d.fo = WS_WF + (k >> 4) * FSTRIDE + ord * 4;
        d.fw = 15;
      } else {
        const u32 kk = (k - 128 + (kc ? rot : 0)) & 255, r = kk / 10;
        d.so = WS_EX + kc * (512u << 10) + kk * 2048;
        d.to = WS_X + ord * NRB * XBLK + kk * 2048;
        d.fo = WS_XF + r * FSTRIDE + ord * 4;
        d.fw = r == NRB - 1 ? 40 : 24;
      }
    } else if (u < U_Q) {
      const u32 k = u - U_R, kc = k / 28, j = k - kc * 28, ord = kc * 8 + dest;
      const u32 jj = j + ((j >> 2) >= rank ? 4 : 0);
      d.so = WS_ER + (call & 1) * (512u << 10) + kc * 65536 + jj * 2048;
      d.to = WS_W + (ord * NWB + 8) * WBLK + jj * 2048;
      d.fo = WS_WF + (8 + (jj >> 4)) * FSTRIDE + ord * 4;
      d.fw = (jj >> 4) == (rank >> 2) ? 20 : 15;
    } else if (u < U_L) {
      u32 c, na, n;
      for (;;) {
        c = (qc < 8 ? (g & 6) + 2 + qc * 2 + (qc < 4) : g + qc) & 7;
        na = c == 7 ? 156 : 208;
        n = qc < 7 ? na : qc < 8 ? 104 : c == 7 ? 228 : 304;
        if (u - qbase < n) break;
        qbase += n;
        qc += 1;
      }
      const u32 k = u - qbase + (qc >= 8 ? na : qc == 7 && !(g & 1) ? 104 : 0);
      d.so = WS_Q + c * QSTRIDE + k * 2048;
      d.to = k * 128;
      d.fo = c;
      d.fw = 0;
    } else {
      const u32 k = u - U_L, gs = k / 104, r = k - gs * 104;
      d.so = u < nu ? WS_L + k * 2048 : 0x80000000u;
      d.to = ((gs & 1) * 2080 + (r >> 3) * 160 + (r & 1) * 32) * 16384 + ((P.peers >> ((gs & 7) * 4)) & 7u) * 2048 + (gs & 6) * 256 + (r & 6) * 32;
      d.fo = 8;
      d.fw = 0;
    }
    return d;
  };
  u32x4 v0, v1;
  u32 u0 = vid, u1 = NPULL + vid, tk = vid + (call - 1) * nu - NPULL;
  Unit d0 = uoff(u0), d1 = uoff(u1);
  v0 = __builtin_amdgcn_raw_buffer_load_b128(rs_s, lane16, d0.so, 17);
  v1 = __builtin_amdgcn_raw_buffer_load_b128(rs_s, lane16, d0.so + 1024, 17);
  u32 to0 = lane16, to1 = lane16 + 1024, tso = d0.to, lo1 = lane16;
  __amdgpu_buffer_rsrc_t rs_t = rs_m;
  const u32 cv = call * FULL;
  u32 cnt = 0, pfo = 0, pfw = 0;
  auto publish = [&]() {
    if (pfw && __builtin_amdgcn_readfirstlane(cnt) + pfw == cv)
      asm volatile("s_nop 4\n\tbuffer_store_dword %0, %1, %2, %3 offen sc0 sc1" :: "v"(cv), "v"(lane0), "s"(rs_m), "s"(pfo) : "memory");
    pfw = 0;
  };
  WAIT_VM0();
  while (u0 < nu) {
    if (bad && !d0.fw) v0 = v1 = u32x4{0x80808080u, 0x80808080u, 0x80808080u, 0x80808080u};
    __builtin_amdgcn_raw_buffer_store_b128(v0, rs_t, to0, tso, 17);
    __builtin_amdgcn_raw_buffer_store_b128(v1, rs_t, to1, tso, 17);
    CFENCE();
    const u32 ns = u1 >= nu ? 0 : u1 >= U_L ? 7 : u1 >= uq2 ? 6 : u1 >= U_Q ? 5 : u1 >= U_C ? 4 : u1 >= U_B ? 3 : u1 >= U_A ? 2 : 1;
    if (stage < ns) {
      stage = lds_ld(ctl + C_STAGE);
      if (stage < ns) {
        publish();
        WAIT_VM0();
        if (d0.fw) {
          cnt = __builtin_amdgcn_raw_ptr_buffer_atomic_add_i32(d0.fw, rs_m, lane0, d0.fo + CNT_OFF, 16);
          pfo = d0.fo; pfw = d0.fw;
          publish();
        }
        d0.fw = 0;
        need(ns);
      }
      bad = lds_ld(ctl + C_BAD) != 0;
    }
    v0 = __builtin_amdgcn_raw_buffer_load_b128(rs_s, lo1, d1.so, 17);
    v1 = __builtin_amdgcn_raw_buffer_load_b128(rs_s, lo1, d1.so + 1024, 17);
    CFENCE();
    const u32 u2 = 3 * NPULL + (__builtin_amdgcn_readfirstlane(tk) - (call - 1) * nu);
    tk = __builtin_amdgcn_raw_ptr_buffer_atomic_add_i32(1, rs_m, lane0, WS_TK + g * 128, 16);
    CFENCE();
    publish();
    const Unit d2 = uoff(u2);
    tso = d1.to;
    if (u2 >= U_L) lo1 = (lane & 3) * 256 + (lane >> 2) * 16;
    if (u1 >= U_L) {
      rs_t = rs_o;
      to0 = (lane >> 2) * 16384 + (lane & 3) * 16;
      to1 = to0 + 16 * 16384;
    } else if (!d1.fw) {
      rs_t = rs_o;
      tso = dest * 2048 + d1.fo * 256;
      const u32 mul = d1.fo == 7 ? 43691u : 32768u, w16 = d1.fo == 7 ? 12 : 16;
#pragma unroll
      for (int e = 0; e < 2; ++e) {
        const u32 i16 = d1.to + e * 64 + lane;
        const u32 sr = (i16 * mul) >> 19, col = i16 - sr * w16;
        const u32 ra = (sr >> 6) * 96 + sr, rc = (((sr - 1664) * 43691u) >> 22) * 64 + sr - 1600;
        (e ? to1 : to0) = (sr < 1664 ? ra : rc) * 16384 + col * 16;
      }
    }
    CFENCE();
    WAIT_VM(3);
    cnt = __builtin_amdgcn_raw_ptr_buffer_atomic_add_i32(d0.fw, rs_m, lane0, d0.fw ? d0.fo + CNT_OFF : 0x40000000u, 16);
    pfo = d0.fo; pfw = d0.fw;
    CFENCE();
    u0 = u1; u1 = u2;
    d0 = d1; d1 = d2;
  }
  publish();
  WAIT_VM(0);
  if (wave == 0) STAMP(5);
}

extern "C" void kernel_launch(int rank, int nranks, ncclComm_t comm, hipStream_t stream,
                              void* const* d_in, const int* in_sizes, int n_in,
                              void* d_out, int out_size,
                              const int* in_dtypes, int out_dtype,
                              void* const* d_out_all,
                              void* const* d_ws_all, size_t ws_bytes) {
  if (nranks != NR || ws_bytes < WS_END || n_in < 2) { fprintf(stderr, "unsupported configuration\n"); return; }
  Params P;
  P.x = (const char*)d_in[0];
  P.w = (const char*)d_in[1];
  for (int p = 0; p < NR; ++p) { P.out[p] = (char*)d_out_all[p]; P.ws[p] = (char*)d_ws_all[p]; }
  P.rank = rank;
  const int phys[8] = {0, 2, 3, 6, 1, 4, 5, 7}, logi[8] = {0, 4, 1, 2, 5, 6, 3, 7};
  P.peers = 0;
  for (int g = 0; g < NR; ++g) P.peers |= (u32)phys[logi[rank & 7] ^ g] << (g * 4);
  {
    int dev = -1, cnt = -1; char bus[64] = {0};
    (void)hipGetDevice(&dev); (void)hipGetDeviceCount(&cnt); (void)hipDeviceGetPCIBusId(bus, 64, dev);
    char line[1024]; int o = snprintf(line, sizeof line, "XINFO rank %d dev %d/%d bus %s x %p w %p out %p |", rank, dev, cnt, bus, d_in[0], d_in[1], d_out);
    for (int p = 0; p < NR; ++p) {
      hipPointerAttribute_t a, b; memset(&a, 0, sizeof a); memset(&b, 0, sizeof b);
      hipError_t e1 = hipPointerGetAttributes(&a, d_ws_all[p]), e2 = hipPointerGetAttributes(&b, d_out_all[p]);
      o += snprintf(line + o, sizeof line - o, " [%p e%d t%d d%d f%x; %p e%d d%d f%x]", d_ws_all[p], (int)e1, (int)a.type, a.device, a.allocationFlags, d_out_all[p], (int)e2, b.device, b.allocationFlags);
    }
    (void)hipGetLastError();
    const char* v1 = getenv("HIP_VISIBLE_DEVICES"); const char* v2 = getenv("ROCR_VISIBLE_DEVICES"); const char* v3 = getenv("CUDA_VISIBLE_DEVICES");
    fprintf(stderr, "%s | %s %s %s\n", line, v1 ? v1 : "-", v2 ? v2 : "-", v3 ? v3 : "-");
  }
  hipLaunchKernelGGL(fused_gemm_ar_q8, dim3(256), dim3(NTHREADS), 0, stream, P);
  HIP_CHECK(hipGetLastError());
}
```

```cpp
#include <hip/hip_runtime.h>
#include <rccl/rccl.h>
#include <cstdio>
#include <stdint.h>

#define HIP_CHECK(expr) do { hipError_t _e = (expr); if (_e != hipSuccess) { \
    fprintf(stderr, "HIP error %s at %s:%d\n", hipGetErrorString(_e), __FILE__, __LINE__); \
    return; } } while (0)

typedef unsigned u32;
typedef unsigned long long u64;
typedef u32 u32x4 __attribute__((ext_vector_type(4)));
typedef u32 u32x2 __attribute__((ext_vector_type(2)));
typedef float f32x4 __attribute__((ext_vector_type(4)));
typedef short s16x4 __attribute__((ext_vector_type(4)));
typedef short s16x8 __attribute__((ext_vector_type(8)));
typedef __bf16 bf16x8 __attribute__((ext_vector_type(8)));
#define LAS __attribute__((address_space(3)))
#define GAS __attribute__((address_space(1)))

constexpr int NR = 8;
constexpr u32 OUT_BYTES = 4096u * 16384u;
constexpr u32 X_BYTES = 4096u * 512u * 2u;
constexpr u32 W_BYTES = 512u * 16384u * 2u;

constexpr u32 NRB = 26;
constexpr u32 NCB = 8;
constexpr u32 XBLK = 20480, WBLK = 32768;
constexpr u32 NWB = NCB + 2;
constexpr u32 WS_W = 0;
constexpr u32 WS_X = 20u << 20;
constexpr u32 WS_CTL = 53u << 20;
constexpr u32 FSTRIDE = 4352;
constexpr u32 WS_XF = WS_CTL;
constexpr u32 WS_WF = WS_XF + NRB * FSTRIDE;
constexpr u32 CNT_OFF = 512u << 10;
constexpr u32 WS_C2 = WS_CTL + (160u << 10);
constexpr u32 WS_G1 = WS_C2 + 4096;
constexpr u32 WS_G2 = WS_C2 + 8192 + 256;
constexpr u32 G2S = 512;
constexpr u32 WS_TK = WS_C2 + 13312;
constexpr u32 WS_TMO = WS_C2 + 20480;
constexpr u32 WS_DBG = WS_C2 + 32768;
constexpr u32 WS_FL = WS_C2 + 65536;
constexpr u32 WS_FC = WS_FL + 16384;
constexpr u32 WS_CALL = WS_FC + 16384;
constexpr u32 WS_Q = 54u << 20;
constexpr u32 QSTRIDE = 4160 * 256;
constexpr u32 WS_EX = 63u << 20;
constexpr u32 WS_ER = 67u << 20;
constexpr u32 WS_EW = 68u << 20;
constexpr u32 WS_L = 84u << 20;
constexpr u32 WS_END = 86u << 20;
static_assert(64u * NWB * WBLK <= WS_X && WS_X + 64u * NRB * XBLK <= WS_CTL && WS_WF + NWB * FSTRIDE <= WS_C2 &&
              WS_CALL + 16384 <= WS_CTL + CNT_OFF && WS_C2 + CNT_OFF <= WS_Q && WS_Q + NCB * QSTRIDE <= WS_EX, "layout");
constexpr u32 FULL = 240;

constexpr u32 SLOT_BYTES = WBLK + XBLK;
constexpr u32 NSLOT = 3;
constexpr u32 TPITCH = 272;
constexpr u32 SNAP_OFF = SLOT_BYTES * NSLOT;
constexpr u32 CTL_OFF = SNAP_OFF + 1024;
constexpr u32 LDS_BYTES = CTL_OFF + 1024;
static_assert(LDS_BYTES <= 163840, "lds");
constexpr int C_AMAX = 0, C_GMAX = 1, C_CALL = 2, C_BAD = 3, C_STAGE = 4, C_CNT = 8, C_MASK = 16;

#define NPW 4
#define EXWAIT 300
#define EXD 60
#define PW1 4
#define PLD 4
#define POLL2 1
#define QWAIT 500
#define QD 100
#define PUSHD 2
constexpr u32 NCOMMWG = 32 - NRB;
constexpr u32 NTILE = NRB * NCB;
constexpr u32 NPULL = NCOMMWG * NPW;
constexpr u32 U_A = 104, U_B = 512, U_C = 1152;
constexpr u32 U_R = 256 + 7 * 384;
constexpr u32 U_Q = U_R + 224;
constexpr int NTHREADS = 512;
constexpr int NSTEP = 64;
constexpr u64 DEADLINE = 50000000ull;

struct Params {
  const char* x;
  const char* w;
  char* out[NR];
  char* ws[NR];
  int rank;
  u32 peers;
};

__device__ __forceinline__ __amdgpu_buffer_rsrc_t mkrsrc(const void* p, u32 bytes) {
  return __builtin_amdgcn_make_buffer_rsrc((void*)p, 0, bytes, 0x00020000);
}
__device__ __forceinline__ u64 now() { return __builtin_amdgcn_s_memrealtime(); }
__device__ __forceinline__ u32 ld_sys(const u32* p) { return __hip_atomic_load((GAS const u32*)p, __ATOMIC_RELAXED, __HIP_MEMORY_SCOPE_SYSTEM); }
__device__ __forceinline__ u64 ld_sys64(const u64* p) { return __hip_atomic_load((GAS const u64*)p, __ATOMIC_RELAXED, __HIP_MEMORY_SCOPE_SYSTEM); }
__device__ __forceinline__ void st_sys(u32* p, u32 v) { __hip_atomic_store((GAS u32*)p, v, __ATOMIC_RELAXED, __HIP_MEMORY_SCOPE_SYSTEM); }
__device__ __forceinline__ void st_sys64(u64* p, u64 v) { __hip_atomic_store((GAS u64*)p, v, __ATOMIC_RELAXED, __HIP_MEMORY_SCOPE_SYSTEM); }
__device__ __forceinline__ u32 lds_ld(LAS u32* p) { return __builtin_amdgcn_readfirstlane(__hip_atomic_load(p, __ATOMIC_RELAXED, __HIP_MEMORY_SCOPE_WORKGROUP)); }
__device__ __forceinline__ void lds_st(LAS u32* p, u32 v) { __hip_atomic_store(p, v, __ATOMIC_RELAXED, __HIP_MEMORY_SCOPE_WORKGROUP); }
__device__ __forceinline__ void lds_add1(LAS u32* p, u32 lane) {
  if (lane == 0) __hip_atomic_fetch_add(p, 1u, __ATOMIC_RELAXED, __HIP_MEMORY_SCOPE_WORKGROUP);
}
#define CFENCE() asm volatile("" ::: "memory")
#define WAIT_VM(n) asm volatile("s_waitcnt vmcnt(" #n ")" ::: "memory")
#define WAIT_VMI(n) asm volatile("s_waitcnt vmcnt(%0)" :: "i"(n) : "memory")
#define WAIT_LGKM() asm volatile("s_waitcnt lgkmcnt(0)" ::: "memory")
#define WAIT_VM0() do { CFENCE(); __builtin_amdgcn_s_waitcnt(0x0F70); WAIT_VM(0); } while (0)

__device__ __forceinline__ u32 wave_umax(u32 v) {
  v = max(v, (u32)__builtin_amdgcn_update_dpp(0, (int)v, 0x111, 0xf, 0xf, true));
  v = max(v, (u32)__builtin_amdgcn_update_dpp(0, (int)v, 0x112, 0xf, 0xf, true));
  v = max(v, (u32)__builtin_amdgcn_update_dpp(0, (int)v, 0x114, 0xf, 0xf, true));
  v = max(v, (u32)__builtin_amdgcn_update_dpp(0, (int)v, 0x118, 0xf, 0xf, true));
  v = max(v, (u32)__builtin_amdgcn_update_dpp(0, (int)v, 0x142, 0xa, 0xf, false));
  v = max(v, (u32)__builtin_amdgcn_update_dpp(0, (int)v, 0x143, 0xc, 0xf, false));
  return (u32)__builtin_amdgcn_readlane((int)v, 63);
}

#define STAMP(k) do {} while (0)

__device__ __forceinline__ void wait_vm_dyn(u32 n) {
  asm volatile(
      "s_getpc_b64 vcc\n\t"
      "s_add_u32 vcc_lo, vcc_lo, %0\n\t"
      "s_addc_u32 vcc_hi, vcc_hi, 0\n\t"
      "s_setpc_b64 vcc\n\t"
      "s_waitcnt vmcnt(0)\n\ts_branch 1f\n\t"
      "s_waitcnt vmcnt(1)\n\ts_branch 1f\n\t"
      "s_waitcnt vmcnt(2)\n\ts_branch 1f\n\t"
      "s_waitcnt vmcnt(3)\n\ts_branch 1f\n\t"
      "s_waitcnt vmcnt(4)\n\ts_branch 1f\n\t"
      "s_waitcnt vmcnt(5)\n\ts_branch 1f\n\t"
      "s_waitcnt vmcnt(6)\n\ts_branch 1f\n\t"
      "s_waitcnt vmcnt(7)\n\ts_branch 1f\n\t"
      "s_waitcnt vmcnt(8)\n\ts_branch 1f\n\t"
      "s_waitcnt vmcnt(9)\n\ts_branch 1f\n\t"
      "s_waitcnt vmcnt(10)\n\ts_branch 1f\n\t"
      "s_waitcnt vmcnt(11)\n\ts_branch 1f\n\t"
      "s_waitcnt vmcnt(12)\n\ts_branch 1f\n\t"
      "s_waitcnt vmcnt(13)\n\ts_branch 1f\n\t"
      "s_waitcnt vmcnt(14)\n\ts_branch 1f\n\t"
      "s_waitcnt vmcnt(15)\n\ts_branch 1f\n\t"
      "s_waitcnt vmcnt(16)\n\ts_branch 1f\n\t"
      "s_waitcnt vmcnt(17)\n\ts_branch 1f\n\t"
      "s_waitcnt vmcnt(18)\n\ts_branch 1f\n\t"
      "s_waitcnt vmcnt(19)\n\ts_branch 1f\n\t"
      "s_waitcnt vmcnt(20)\n\ts_branch 1f\n\t"
      "s_waitcnt vmcnt(21)\n\ts_branch 1f\n\t"
      "s_waitcnt vmcnt(22)\n\ts_branch 1f\n\t"
      "s_waitcnt vmcnt(23)\n\ts_branch 1f\n\t"
      "s_waitcnt vmcnt(24)\n\t"
      "1:"
      :: "s"(min(n, 24u) * 8 + 12) : "memory", "scc", "vcc");
}
#define BARRIER() asm volatile("s_barrier" ::: "memory")

__global__ void __launch_bounds__(NTHREADS) fused_gemm_ar_q8(Params P) {
  __shared__ __attribute__((aligned(16))) char smem[LDS_BYTES];
  LAS char* L = (LAS char*)smem;
  LAS u32* ctl = (LAS u32*)(L + CTL_OFF);
  const u32 tid = threadIdx.x, lane = tid & 63;
  const u32 wave = __builtin_amdgcn_readfirstlane(tid >> 6);
  u32 xcc;
  asm volatile("s_getreg_b32 %0, hwreg(HW_REG_XCC_ID, 0, 3)" : "=s"(xcc));
  const u32 g = xcc & 7u, idx = blockIdx.x >> 3, bid = idx * 8 + g;
  const u32 rank = (u32)P.rank & 7u;
  const u32 dest = (P.peers >> (g * 4)) & 7u;
  char* const myws = P.ws[rank];
  char* const dws = P.ws[dest];
  u32* const tmo = (u32*)(myws + WS_TMO);
  const u32 lane16 = lane * 16;
  const u64 t0 = now();
  const u32 lx = (lane & 7) * 1024 + (lane >> 3) * 16;
  const u32 lw = (((lane >> 3) & 1) * 8 + ((lane >> 1) & 3)) * 32768 + ((lane & 1) * 4 + (lane >> 4)) * 16;
  u32 call = 0;
  bool bad = false;
  if (wave == 0) STAMP(0);

  if (idx >= NCOMMWG) {
    const u32 ti = idx - NCOMMWG;
    const u32 rb = (g & 1) * 13 + (ti >> 1);
    u32 cb = (g >> 1) * 2 + (ti & 1);
    const u32 tile = g * NRB + ti;
    const __amdgpu_buffer_rsrc_t rs_ws = mkrsrc(myws, WS_END);
    char* flp = P.ws[lane & 7] + WS_FL + rank * 256 + g * 4;
    const u32 lane0 = lane ? 0x80000000u : 0u;
    auto signal_wg = [&](u32 kind, u32 inc) {
      const u32 old = __builtin_amdgcn_raw_ptr_buffer_atomic_add_i32(inc, rs_ws, lane0, WS_FC + kind * 2048 + g * 256, 16);
      if (__builtin_amdgcn_readfirstlane(old) + inc == call * NRB && lane < 8)
        st_sys((u32*)(flp + kind * 2048), call);
      CFENCE();
    };
    auto signal_all = [&](u32 kind, u32 inc = 1) {
      u32 old = 0;
      if (lane == 0) old = __hip_atomic_fetch_add(ctl + C_CNT + kind, 1u, __ATOMIC_RELAXED, __HIP_MEMORY_SCOPE_WORKGROUP);
      if (__builtin_amdgcn_readfirstlane(old) == 7) signal_wg(kind, inc);
    };
    {
      const u32 pc = g ? 2 * U_A + (tile - NRB) * 5 + wave : ti * 8 + wave;
      const bool pok = (wave < 5 || !g) && pc < 1024;
      const u32x4 e0 = __builtin_amdgcn_raw_buffer_load_b128(mkrsrc(P.x, X_BYTES), lx,
                                                             pok ? (pc & 511) * 8192 + (pc >> 9) * 128 : 0x80000000u, 0);
      CFENCE();
      if (wave == 7 && lane < 16) ctl[lane] = 0u;
      __builtin_amdgcn_raw_buffer_store_b128(e0, rs_ws, lane16, pok ? WS_EX + pc * 1024 : 0x80000000u, 17);
      asm volatile("" : "+v"(flp));
      WAIT_VM0();
      WAIT_LGKM();
      BARRIER();
      if (wave == 0) {
        const u32 old = __builtin_amdgcn_readfirstlane(__builtin_amdgcn_raw_ptr_buffer_atomic_add_i32(1, rs_ws, lane0, WS_FC + g * 256, 16));
        call = old / NRB + 1;
        if (old + 1 == call * NRB && lane < 8) {
          st_sys((u32*)flp, call);
        }
        if (lane == 0) lds_st(ctl + C_CALL, call);
        WAIT_LGKM();
      }
      if (wave == 6 && ti == 0) __builtin_amdgcn_fence(__ATOMIC_ACQUIRE, "");
      BARRIER();
      call = lds_ld(ctl + C_CALL);
      for (u32 r = 0; r < 3; ++r) {
        const u32 slot = r * 8 + wave, e = slot * NTILE + tile;
        u32 kc, k, so, eo;
        bool isw = true;
        if (e < 4352) {
          kc = (e + 768) / 640; k = e + 768 - kc * 640;
          if (kc == 1) k -= 128;
          isw = k < 512;
          k &= 511;
          eo = isw ? WS_EW + ((k >> 6) * 8 + kc) * (8 * WBLK) + (k & 63) * 4096 : WS_EX + kc * (512u << 10) + k * 4096;
        } else {
          kc = (e - 4352) >> 4; k = ((e >> 1) & 7) * 64 + 62 + (e & 1);
          eo = WS_ER + (call & 1) * (512u << 10) + (e - 4352) * 4096;
        }
        if (e < 4480) {
          so = isw ? (kc * 64 + (k & 1) * 32) * 32768 + (k >> 1) * 128 : k * 32768 + kc * 128;
          const __amdgpu_buffer_rsrc_t rs = mkrsrc(isw ? P.w : P.x, isw ? W_BYTES : X_BYTES);
          const u32 lo = isw ? lw : lx, s1 = isw ? 4 * 32768 : 8192, s2 = isw ? 16 * 32768 : 16384;
          while (now() - t0 < EXWAIT + slot * EXD) __builtin_amdgcn_s_sleep(2);
          const u32x4 e0 = __builtin_amdgcn_raw_buffer_load_b128(rs, lo, so, 0);
          const u32x4 e1 = __builtin_amdgcn_raw_buffer_load_b128(rs, lo, so + s1, 0);
          const u32x4 e2 = __builtin_amdgcn_raw_buffer_load_b128(rs, lo, so + s2, 0);
          const u32x4 e3 = __builtin_amdgcn_raw_buffer_load_b128(rs, lo, so + s1 + s2, 0);
          __builtin_amdgcn_raw_buffer_store_b128(e0, rs_ws, lane16, eo, 17);
          __builtin_amdgcn_raw_buffer_store_b128(e1, rs_ws, lane16, eo + 1024, 17);
          __builtin_amdgcn_raw_buffer_store_b128(e2, rs_ws, lane16, eo + 2048, 17);
          __builtin_amdgcn_raw_buffer_store_b128(e3, rs_ws, lane16, eo + 3072, 17);
        }
        if (r != 1) {
          WAIT_VM0();
          signal_all(1 + (r >> 1));
        }
      }
      BARRIER();
    }
    const u32 cv = call * FULL;
    const u32 wm = wave >> 2, wn = wave & 3;
    const u32 lds0 = (u32)(size_t)L;
    const u32 xa = WBLK + wm * 10240 + ((lane >> 3) & 1) * 1024 + (lane >> 5) * 256 + ((lane >> 4) & 1) * 128 + (lane & 7) * 16;
    const u32 h = lane >> 4, q = (lane >> 2) & 3, p = lane & 3;
    const u32 wa0 = wn * 8192 + (h >> 1) * 2048 + (p & 1) * 512 + (h & 1) * 128 + q * 32 + (p >> 1) * 16 + (p & 1) * 8;
    const u32 wa1 = wa0 ^ 8;
    const u32 xpo = wave < 4 ? wave * 3072 : 12288 + (wave - 4) * 2048;
    u32 pollo;
    f32x4 acc[4][5];
    u64 kx = 0, kw;
    u64 ready, taken;
    u32 issued, consumed, nis = 0;
    u32 islot, cslot, par = 0;
    u64 markD = 0;
    float inv = 0.f;
    auto sched = [&]() {
      u64 avail = ready & ~taken;
      while (avail && issued - consumed < NSLOT) {
        const u32 ord = __builtin_ctzll(avail);
        avail &= avail - 1;
        taken |= 1ull << ord;
        const u32 sw = WS_W + (ord * NWB + cb) * WBLK + wave * 4096;
        const u32 sx = WS_X + (ord * NRB + rb) * XBLK + xpo;
        const u32 mw = lds0 + islot + wave * 4096, mx = lds0 + islot + WBLK + xpo;
        u32 keep;
        asm volatile(
            "s_mov_b32 %0, m0\n\t"
            "s_mov_b32 m0, %3\n\t"
            "s_nop 0\n\t"
            "buffer_load_dwordx4 %1, %2, %5 offen lds\n\t"
            "buffer_load_dwordx4 %1, %2, %5 offen offset:1024 lds\n\t"
            "buffer_load_dwordx4 %1, %2, %5 offen offset:2048 lds\n\t"
            "buffer_load_dwordx4 %1, %2, %5 offen offset:3072 lds\n\t"
            "s_mov_b32 m0, %4\n\t"
            "s_nop 0\n\t"
            "buffer_load_dwordx4 %1, %2, %6 offen lds\n\t"
            "buffer_load_dwordx4 %1, %2, %6 offen offset:1024 lds\n\t"
            "s_cmp_gt_u32 %7, 3\n\t"
            "s_cbranch_scc1 1f\n\t"
            "buffer_load_dwordx4 %1, %2, %6 offen offset:2048 lds\n\t"
            "1:\n\t"
            "s_mov_b32 m0, %0"
            : "=&s"(keep)
            : "v"(lane16), "s"(rs_ws), "s"(mw), "s"(mx), "s"(sw), "s"(sx), "s"(wave)
            : "memory", "scc");
        nis += wave < 4 ? 7 : 6;
        const u32 sh = (issued & 3) * 16;
        markD = (markD & ~(0xffffull << sh)) | ((u64)(nis & 0xffff) << sh);
        issued += 1;
        islot = islot == 2 * SLOT_BYTES ? 0 : islot + SLOT_BYTES;
      }
    };
    auto poll = [&]() {
      if (wave == 0 && (kx & kw) != ~0ull) {
        const u64 kn = lane < 16 ? kx : kw;
        const bool act = lane < 32 && ((u32)(kn >> ((lane & 15) * 4)) & 15u) != 15u;
        const u32 po = act ? pollo : 0x80000000u;
        u32 keep;
        asm volatile("s_mov_b32 %0, m0\n\ts_mov_b32 m0, %3\n\ts_nop 0\n\t"
                     "buffer_load_dwordx4 %1, %2, 0 offen sc0 sc1 lds\n\ts_mov_b32 m0, %0"
                     : "=&s"(keep) : "v"(po), "s"(rs_ws), "s"(lds0 + SNAP_OFF) : "memory");
        nis += 1;
      }
    };
    LAS u32* const snap = (LAS u32*)(L + SNAP_OFF) + lane;
    auto seen = [&](u32 fx, u32 fw) {
      kx |= __ballot((int)(fx - cv) >= 0);
      kw |= __ballot((int)(fw - cv) >= 0);
    };
    auto tell = [&]() { if (lane == 0) *(LAS u64*)(L + CTL_OFF + C_MASK * 4 + par) = kx & kw; };
    auto told = [&]() -> u64 {
      const u64 m = *(LAS u64*)(L + CTL_OFF + C_MASK * 4 + par);
      par ^= 8;
      return m;
    };
    auto uni = [&](u64 m) -> u64 {
      return (u64)(u32)__builtin_amdgcn_readfirstlane((u32)m) | ((u64)(u32)__builtin_amdgcn_readfirstlane((u32)(m >> 32)) << 32);
    };
    auto wait_step = [&]() { wait_vm_dyn((nis - (u32)(markD >> ((consumed & 3) * 16))) & 0xffff); };
#define SB() __builtin_amdgcn_sched_barrier(0)
#define PIECE(M, S, OFF) do { \
        asm volatile("s_cmp_lg_u32 %2, -1\n\ts_cbranch_scc0 1f\n\ts_mov_b32 m0, %2\n\ts_nop 0\n\t" \
                     "buffer_load_dwordx4 %0, %1, %3 offen offset:" #OFF " lds\n\t1:" \
                     :: "v"(lane16), "s"(rs_ws), "s"(M), "s"(S) : "memory", "scc"); \
        SB(); } while (0)
#define MMA4(kb, i) do { \
        _Pragma("unroll") for (int n = 0; n < 4; ++n) \
          acc[n][i] = __builtin_amdgcn_mfma_f32_16x16x32_bf16(wf[kb][n], xf[kb][i], acc[n][i], 0, 0, 0); \
        SB(); } while (0)
    bf16x8 xf[2][5], wf[2][4];
    u32 sp = 0;
    u64 cyc0 = 0;
    for (u32 pass = 0;; ++pass) {
    pollo = lane < 16 ? WS_XF + rb * FSTRIDE + lane16 : WS_WF + cb * FSTRIDE + lane16 - 256;
    kw = taken = 0;
    issued = consumed = islot = cslot = 0;
#pragma unroll
    for (int a = 0; a < 4; ++a)
#pragma unroll
      for (int b = 0; b < 5; ++b) acc[a][b] = f32x4{0.f, 0.f, 0.f, 0.f};
#pragma unroll
    for (int n = 0; n < 5; ++n) {
      xf[1][n] = __builtin_bit_cast(bf16x8, u32x4{0u, 0u, 0u, 0u});
      if (n < 4) wf[1][n] = __builtin_bit_cast(bf16x8, u32x4{0u, 0u, 0u, 0u});
    }
    poll();
    while (consumed < NSTEP) {
      if (issued == consumed) {
        if (wave == 0) {
          while ((kx & kw & ~taken) == 0) {
            poll();
            WAIT_VM(1);
            seen(snap[0], snap[64]);
            if ((++sp & 63) == 0 && now() - t0 > DEADLINE) {
              st_sys(tmo, call);
              WAIT_VM(0);
              lds_st(ctl + C_BAD, 1u);
              kx = kw = ~0ull;
            }
          }
          tell();
          WAIT_LGKM();
        }
        BARRIER();
        ready = uni(told());
        sched();
        poll();
        wait_step();
        if (wave == 0) {
          seen(snap[0], snap[64]);
          tell();
          WAIT_LGKM();
        }
      }
      BARRIER();
      const u64 rdy = told();
      SB();
      LAS char* base = L + cslot;
      auto frags = [&](int kb) {
#pragma unroll
        for (int n = 0; n < 5; ++n) {
          if (n < 4) {
            LAS char* a = base + ((n & 1) ? wa1 : wa0) + kb * 4096 + (n >> 1) * 256;
            s16x4 lo = __builtin_amdgcn_ds_read_tr16_b64_v4i16((LAS s16x4*)a);
            s16x4 hi = __builtin_amdgcn_ds_read_tr16_b64_v4i16((LAS s16x4*)(a + 1024));
            wf[kb][n] = __builtin_bit_cast(bf16x8, __builtin_shufflevector(lo, hi, 0, 1, 2, 3, 4, 5, 6, 7));
          }
          xf[kb][n] = *(LAS bf16x8*)(base + xa + n * 2048 + kb * 512);
        }
        SB();
      };
      MMA4(1, 0);
      frags(0);
      poll();
      SB();
      MMA4(1, 1);
      ready = uni(rdy);
      const u64 av = ready & ~taken;
      const bool go = av != 0 && issued - consumed < NSLOT;
      const u32 ord = __builtin_ctzll(av | (1ull << 63));
      const u32 sw = WS_W + (ord * NWB + cb) * WBLK + wave * 4096;
      const u32 sx = WS_X + (ord * NRB + rb) * XBLK + xpo;
      u32 mw = ~0u, mx = ~0u, mx3 = ~0u;
      if (go) {
        mw = lds0 + islot + wave * 4096;
        mx = lds0 + islot + WBLK + xpo;
        if (wave < 4) mx3 = mx;
        taken |= 1ull << ord;
        nis += wave < 4 ? 7 : 6;
        const u32 sh = (issued & 3) * 16;
        markD = (markD & ~(0xffffull << sh)) | ((u64)(nis & 0xffff) << sh);
        issued += 1;
        islot = islot == 2 * SLOT_BYTES ? 0 : islot + SLOT_BYTES;
      }
      SB();
      MMA4(1, 2);
      PIECE(mw, sw, 0);
      MMA4(1, 3);
      PIECE(mw, sw, 1024);
      MMA4(1, 4);
      frags(1);
      PIECE(mw, sw, 2048);
      MMA4(0, 0);
      PIECE(mw, sw, 3072);
      PIECE(mx, sx, 0);
      MMA4(0, 1);
      PIECE(mx, sx, 1024);
      PIECE(mx3, sx, 2048);
      MMA4(0, 2);
      sched();
      poll();
      cslot = cslot == 2 * SLOT_BYTES ? 0 : cslot + SLOT_BYTES;
      consumed += 1;
      SB();
      MMA4(0, 3);
      if (issued != consumed) wait_step();
      u32 fx = cv - 1, fw = cv - 1;
      if (wave == 0) { fx = snap[0]; fw = snap[64]; }
      SB();
      MMA4(0, 4);
      if (wave == 0) { seen(fx, fw); tell(); }
      WAIT_LGKM();
    }
    MMA4(1, 0); MMA4(1, 1); MMA4(1, 2); MMA4(1, 3); MMA4(1, 4);
    if (pass == 0) {
      if (wave == 0) STAMP(2);
      const u32 row0 = rb * 160 + wm * 80;
      float m = 0.f;
#pragma unroll
      for (int b = 0; b < 5; ++b) {
        float mb = 0.f;
#pragma unroll
        for (int a = 0; a < 4; ++a)
#pragma unroll
          for (int e = 0; e < 4; ++e) mb = fmaxf(mb, fabsf(acc[a][b][e]));
        if (row0 + b * 16 < 4096) m = fmaxf(m, mb);
      }
      const u32 mb = wave_umax(__builtin_bit_cast(u32, m));
      if (lane == 0) __hip_atomic_fetch_max(ctl + C_AMAX, mb, __ATOMIC_RELAXED, __HIP_MEMORY_SCOPE_WORKGROUP);
      WAIT_LGKM();
      BARRIER();
      if (wave == 0) {
        const u32 wmax = lds_ld(ctl + C_AMAX);
        if (lane == 0) st_sys64((u64*)(myws + WS_G1) + tile, ((u64)call << 32) | wmax);
        STAMP(3);
        const u64* g2 = (const u64*)(myws + WS_G2 + g * G2S) + (lane & 7);
        u64 val;
        for (;;) {
          val = ld_sys64(g2);
          if (__all((u32)(val >> 32) == call)) break;
          if ((++sp & 63) == 0 && (ld_sys(tmo) == call || now() - t0 > DEADLINE)) { st_sys(tmo, call); bad = true; break; }
        }
        const u32 gmax = wave_umax((u32)val);
        if (lane == 0) {
          lds_st(ctl + C_GMAX, gmax);
          if (bad) lds_st(ctl + C_BAD, 1u);
        }
        WAIT_LGKM();
      }
    }
    BARRIER();
    if (pass == 0) {
      const float amax = __builtin_bit_cast(float, lds_ld(ctl + C_GMAX));
      const float scale = amax / 127.0f;
      inv = amax > 0.f ? 1.0f / scale : 0.f;
      if (wave == 0) STAMP(4);
    }
    bad = lds_ld(ctl + C_BAD) != 0;
    const bool odd = (lane >> 4) & 1;
    u32x4 qv[5];
#pragma unroll
    for (int i = 0; i < 5; ++i) {
      u32 d[4];
#pragma unroll
      for (int n = 0; n < 4; ++n) {
        u32 b0 = __builtin_bit_cast(u32, __builtin_fmaf(acc[n][i][0], inv, 12582912.0f));
        u32 b1 = __builtin_bit_cast(u32, __builtin_fmaf(acc[n][i][1], inv, 12582912.0f));
        u32 b2 = __builtin_bit_cast(u32, __builtin_fmaf(acc[n][i][2], inv, 12582912.0f));
        u32 b3 = __builtin_bit_cast(u32, __builtin_fmaf(acc[n][i][3], inv, 12582912.0f));
        u32 lo = __builtin_amdgcn_perm(b1, b0, 0x0c0c0400u);
        u32 hi = __builtin_amdgcn_perm(b3, b2, 0x04000c0cu);
        d[n] = bad ? 0x80808080u : (lo | hi);
      }
      qv[i] = u32x4{odd ? d[1] : d[0], odd ? d[0] : d[1], odd ? d[3] : d[2], odd ? d[2] : d[3]};
    }
    const u64 tq = now();
    if (!pass && !(cb & 1) && wm == 0) {
      const __amdgpu_buffer_rsrc_t rs_p = mkrsrc(dws, WS_END);
#pragma unroll
      for (int i = 0; i < 4; ++i) {
        if (i && i % PUSHD == 0) WAIT_VM(0);
        if (wave == 0 && i == PUSHD) STAMP(10);
        __builtin_amdgcn_raw_buffer_store_b128(qv[i], rs_p, lane16, WS_L + ((g * 13 + (ti >> 1)) * 16 + wn * 4 + i) * 1024, 17);
        CFENCE();
      }
      WAIT_VM0();
      if (wave == 0) STAMP(11);
      const u32 old = __builtin_amdgcn_raw_ptr_buffer_atomic_add_i32(1, rs_ws, lane0, WS_FC + 5 * 2048 + g * 256, 16);
      if (__builtin_amdgcn_readfirstlane(old) + 1 == call * 52 && lane == 0) st_sys((u32*)(dws + WS_FL + 5 * 2048) + g, call);
      CFENCE();
    }
    {
      LAS char* tw = L + (wm * 80 + (lane & 15)) * TPITCH + wn * 64 + (lane >> 4) * 16;
#pragma unroll
      for (int i = 0; i < 5; ++i) *(LAS u32x4*)(tw + i * 16 * TPITCH) = qv[i];
    }
    WAIT_LGKM();
    BARRIER();
    LAS char* tr = L + (wave * 4 + (lane >> 4)) * TPITCH + (lane & 15) * 16;
#pragma unroll
    for (int i = 0; i < 5; ++i) qv[i] = *(LAS u32x4*)(tr + i * 32 * TPITCH);
    const __amdgpu_buffer_rsrc_t rs_o = mkrsrc(P.out[rank], OUT_BYTES);
    const u32 orow = (rb * 160 + wave * 4 + (lane >> 4)) * 16384u;
    if (pass) {
      const u32 piece = (cb - 8) * 4 + ((lane >> 2) & 3);
      const u32 oo = piece == rank ? 0x80000000u : orow + piece * 2048 + 1984 + (lane & 3) * 16;
#pragma unroll
      for (int i = 0; i < 5; ++i) __builtin_amdgcn_raw_buffer_store_b128(qv[i], rs_o, oo, i * 32 * 16384, 17);
      break;
    }
    const u32 pitch = cb == 7 ? 192 : 256;
    const bool rep = (lane & 15) * 16 >= pitch;
    const u32 qo = rep ? 0x80000000u : (wave * 4 + (lane >> 4)) * pitch + (lane & 15) * 16;
    const u32 oo = orow + rank * 2048 + cb * 256 + (lane & 15) * 16;
    const u32 qa = WS_Q + cb * QSTRIDE + rb * 64 * pitch, qb = WS_Q + cb * QSTRIDE + (1664 + rb * 96) * pitch;
    if (cb & 1) {
      __builtin_amdgcn_raw_buffer_store_b128(qv[0], rs_ws, qo, qa, 17);
      __builtin_amdgcn_raw_buffer_store_b128(qv[1], rs_ws, qo, qa + 32 * pitch, 17);
      WAIT_VM0();
      signal_all(3, 2);
      if (wave == 0) STAMP(9);
    }
    while (now() - tq < QWAIT + wave * QD) __builtin_amdgcn_s_sleep(2);
    if (!(cb & 1)) {
      __builtin_amdgcn_raw_buffer_store_b128(qv[0], rs_ws, qo, qa, 17);
      __builtin_amdgcn_raw_buffer_store_b128(qv[1], rs_ws, qo, qa + 32 * pitch, 17);
    }
#pragma unroll
    for (int i = 2; i < 5; ++i) __builtin_amdgcn_raw_buffer_store_b128(qv[i], rs_ws, qo, qb + (i - 2) * 32 * pitch, 17);
    WAIT_VM0();
    signal_all(4);
    if (cb == 7) {
#pragma unroll
      for (int i = 0; i < 5; ++i) __builtin_amdgcn_raw_buffer_store_b128(qv[i], rs_o, rep ? oo : 0x80000000u, i * 32 * 16384, 17);
    }
    if (((ti >> 1) & 3) != (g >> 1)) break;
    cb = 8 + (ti & 1);
    WAIT_VM0();
    WAIT_LGKM();
    BARRIER();
    }
    WAIT_VM(0);
    if (wave == 0) STAMP(5);
    return;
  }

  const u32 vid = idx * NPW + wave;
  const __amdgpu_buffer_rsrc_t rs_s = mkrsrc(dws, WS_END);
  const u32 lane0 = lane ? 0x80000000u : 0u;
  static_assert(NPW == 4 && NCOMMWG == 6, "wave roles");
  if (wave >= NPW && wave < 7) {
    const u32 ph = (wave - NPW) * NCOMMWG + idx;
    const __amdgpu_buffer_rsrc_t rs_w = mkrsrc(P.w, W_BYTES);
    const u32 so = ph < 16 ? dest * 4096 + ph * 256 : 0x80000000u;
    const u32 to = ph < 16 ? WS_W + rank * NWB * WBLK + ph * 16384 : 0x80000000u;
    u32x4 pv[16];
#pragma unroll
    for (int i = 0; i < 16; ++i) {
      pv[i] = __builtin_amdgcn_raw_buffer_load_b128(rs_w, lw, so + (i >> 3) * 128 + (((i >> 1) & 3) * 16 + (i & 1) * 4) * 32768, 0);
      if (i == PLD - 1) BARRIER();
    }
    CFENCE();
#pragma unroll
    for (int i = 0; i < 16; ++i) {
      if (i >= PW1) WAIT_VMI(PW1 - 1);
      __builtin_amdgcn_raw_buffer_store_b128(pv[i], rs_s, lane16, to + i * 1024, 17);
      CFENCE();
    }
    WAIT_VM(0);
    if (wave == NPW) STAMP(10);
    if (ph < 16) __builtin_amdgcn_raw_ptr_buffer_atomic_add_i32(FULL / 2, rs_s, lane0, WS_WF + (ph >> 1) * FSTRIDE + rank * 4, 16);
    return;
  }
  if (wave == 7) {
    ctl[lane] = 0u;
    WAIT_LGKM();
  }
  BARRIER();
  if (wave == 7) {
    u32* const cptr = (u32*)(myws + WS_CALL + bid * 256);
    call = __builtin_amdgcn_readfirstlane(ld_sys(cptr)) + 1;
    if (lane == 0) { lds_st(ctl + C_CALL, call); st_sys(cptr, call); }
    u32 sp = 0;
    auto expired = [&]() -> bool {
      if ((++sp & 63) == 0 && (ld_sys(tmo) == call || now() - t0 > DEADLINE)) {
        st_sys(tmo, call);
        if (lane == 0) { lds_st(ctl + C_BAD, 1u); lds_st(ctl + C_STAGE, 7u); }
        bad = true;
      }
      return bad;
    };
    auto watch = [&](u32 kind, u32 row) {
      const u32* fp = (const u32*)(myws + WS_FL + kind * 2048 + row * 256) + (lane & 7);
      while (!bad) {
        const u32 f = ld_sys(fp);
        const u64 ok = __ballot((int)(f - call) >= 0);
        if (ok == ~0ull) break;
        if (kind == 0 && (ok & 1) && lane == 0) lds_st(ctl + C_STAGE, 1u);
        __builtin_amdgcn_s_sleep(1);
        expired();
      }
      if (lane == 0 && !bad) lds_st(ctl + C_STAGE, kind + 2);
      CFENCE();
    };
    watch(0, dest);
    STAMP(11);
    watch(1, dest);
    watch(2, dest);
    if (bid == 0) {
      const u64* g1 = (const u64*)(myws + WS_G1);
      u64* g2 = (u64*)(P.ws[lane & 7] + WS_G2 + (lane >> 3) * G2S) + rank;
      asm volatile("" : "+v"(g2));
      u32 gm = 0;
      while (!bad) {
        u64 a0 = ld_sys64(g1 + lane), a1 = ld_sys64(g1 + 64 + lane), a2 = ld_sys64(g1 + 128 + lane);
        u64 a3 = ld_sys64(g1 + 192 + (lane & 15));
        const bool ok = (u32)(a0 >> 32) == call && (u32)(a1 >> 32) == call && (u32)(a2 >> 32) == call && (u32)(a3 >> 32) == call;
        gm = max(max((u32)a0, (u32)a1), max((u32)a2, (u32)a3));
        const u64 okm = __ballot(ok);
        if (okm == ~0ull) break;
        if (okm == 0) __builtin_amdgcn_s_sleep(8);
        expired();
      }
      gm = wave_umax(gm);
      if (!bad) st_sys64(g2, ((u64)call << 32) | gm);
    }
    watch(3, dest);
    STAMP(9);
    watch(4, dest);
    if (g == 0) watch(5, 0);
    return;
  }
  u32 stage = 0;
  auto need = [&](u32 s) {
    while (stage < s) {
      stage = lds_ld(ctl + C_STAGE);
      if (stage < s) __builtin_amdgcn_s_sleep(1);
    }
    CFENCE();
  };
  need(1);
  if (wave == 0) STAMP(3);
  call = lds_ld(ctl + C_CALL);
  const __amdgpu_buffer_rsrc_t rs_m = mkrsrc(myws, WS_Q);
  const __amdgpu_buffer_rsrc_t rs_o = mkrsrc(P.out[rank], OUT_BYTES);
  const u32 rot = (g * 26 / 7 % 26) * 10;
  constexpr u32 U_L = U_Q + 3968 - 104, uq2 = U_Q + 3 * 208 + 156;
  const u32 nu = g ? U_L : U_L + 832;
  struct Unit { u32 so, to, fo, fw; };
  u32 qc = 0, qbase = U_Q;
  auto uoff = [&](u32 u) -> Unit {
    Unit d;
    if (u < U_R) {
      const u32 kc = (u + 128) / 384, ord = kc * 8 + dest;
      u32 k = u + 128 - kc * 384;
      if (kc == 1) k = k < 256 ? k + 128 : k - 256;
      if (k < 128) {
        d.so = WS_EW + (rank * 8 + kc) * (8 * WBLK) + k * 2048;
        d.to = WS_W + ord * NWB * WBLK + k * 2048;
        d.fo = WS_WF + (k >> 4) * FSTRIDE + ord * 4;
        d.fw = 15;
      } else {
        const u32 kk = (k - 128 + (kc ? rot : 0)) & 255, r = kk / 10;
        d.so = WS_EX + kc * (512u << 10) + kk * 2048;
        d.to = WS_X + ord * NRB * XBLK + kk * 2048;
        d.fo = WS_XF + r * FSTRIDE + ord * 4;
        d.fw = r == NRB - 1 ? 40 : 24;
      }
    } else if (u < U_Q) {
      const u32 k = u - U_R, kc = k / 28, j = k - kc * 28, ord = kc * 8 + dest;
      const u32 jj = j + ((j >> 2) >= rank ? 4 : 0);
      d.so = WS_ER + (call & 1) * (512u << 10) + kc * 65536 + jj * 2048;
      d.to = WS_W + (ord * NWB + 8) * WBLK + jj * 2048;
      d.fo = WS_WF + (8 + (jj >> 4)) * FSTRIDE + ord * 4;
      d.fw = (jj >> 4) == (rank >> 2) ? 20 : 15;
    } else if (u < U_L) {
      u32 c, na, n;
      for (;;) {
        c = (qc < 8 ? (g & 6) + 2 + qc * 2 + (qc < 4) : g + qc) & 7;
        na = c == 7 ? 156 : 208;
        n = qc < 7 ? na : qc < 8 ? 104 : c == 7 ? 228 : 304;
        if (u - qbase < n) break;
        qbase += n;
        qc += 1;
      }
      const u32 k = u - qbase + (qc >= 8 ? na : qc == 7 && !(g & 1) ? 104 : 0);
      d.so = WS_Q + c * QSTRIDE + k * 2048;
      d.to = k * 128;
      d.fo = c;
      d.fw = 0;
    } else {
      const u32 k = u - U_L, gs = k / 104, r = k - gs * 104;
      d.so = u < nu ? WS_L + k * 2048 : 0x80000000u;
      d.to = ((gs & 1) * 2080 + (r >> 3) * 160 + (r & 1) * 32) * 16384 + ((P.peers >> ((gs & 7) * 4)) & 7u) * 2048 + (gs & 6) * 256 + (r & 6) * 32;
      d.fo = 8;
      d.fw = 0;
    }
    return d;
  };
  u32x4 v0, v1;
  u32 u0 = vid, u1 = NPULL + vid, tk = vid + (call - 1) * nu - NPULL;
  Unit d0 = uoff(u0), d1 = uoff(u1);
  v0 = __builtin_amdgcn_raw_buffer_load_b128(rs_s, lane16, d0.so, 17);
  v1 = __builtin_amdgcn_raw_buffer_load_b128(rs_s, lane16, d0.so + 1024, 17);
  u32 to0 = lane16, to1 = lane16 + 1024, tso = d0.to, lo1 = lane16;
  __amdgpu_buffer_rsrc_t rs_t = rs_m;
  const u32 cv = call * FULL;
  u32 cnt = 0, pfo = 0, pfw = 0;
  auto publish = [&]() {
    if (pfw && __builtin_amdgcn_readfirstlane(cnt) + pfw == cv)
      asm volatile("s_nop 4\n\tbuffer_store_dword %0, %1, %2, %3 offen sc0 sc1" :: "v"(cv), "v"(lane0), "s"(rs_m), "s"(pfo) : "memory");
    pfw = 0;
  };
  WAIT_VM0();
  while (u0 < nu) {
    if (bad && !d0.fw) v0 = v1 = u32x4{0x80808080u, 0x80808080u, 0x80808080u, 0x80808080u};
    __builtin_amdgcn_raw_buffer_store_b128(v0, rs_t, to0, tso, 17);
    __builtin_amdgcn_raw_buffer_store_b128(v1, rs_t, to1, tso, 17);
    CFENCE();
    const u32 ns = u1 >= nu ? 0 : u1 >= U_L ? 7 : u1 >= uq2 ? 6 : u1 >= U_Q ? 5 : u1 >= U_C ? 4 : u1 >= U_B ? 3 : u1 >= U_A ? 2 : 1;
    if (stage < ns) {
      stage = lds_ld(ctl + C_STAGE);
      if (stage < ns) {
        publish();
        WAIT_VM0();
        if (d0.fw) {
          cnt = __builtin_amdgcn_raw_ptr_buffer_atomic_add_i32(d0.fw, rs_m, lane0, d0.fo + CNT_OFF, 16);
          pfo = d0.fo; pfw = d0.fw;
          publish();
        }
        d0.fw = 0;
        need(ns);
      }
      bad = lds_ld(ctl + C_BAD) != 0;
    }
    v0 = __builtin_amdgcn_raw_buffer_load_b128(rs_s, lo1, d1.so, 17);
    v1 = __builtin_amdgcn_raw_buffer_load_b128(rs_s, lo1, d1.so + 1024, 17);
    CFENCE();
    const u32 u2 = 3 * NPULL + (__builtin_amdgcn_readfirstlane(tk) - (call - 1) * nu);
    tk = __builtin_amdgcn_raw_ptr_buffer_atomic_add_i32(1, rs_m, lane0, WS_TK + g * 128, 16);
    CFENCE();
    publish();
    const Unit d2 = uoff(u2);
    tso = d1.to;
    if (u2 >= U_L) lo1 = (lane & 3) * 256 + (lane >> 2) * 16;
    if (u1 >= U_L) {
      rs_t = rs_o;
      to0 = (lane >> 2) * 16384 + (lane & 3) * 16;
      to1 = to0 + 16 * 16384;
    } else if (!d1.fw) {
      rs_t = rs_o;
      tso = dest * 2048 + d1.fo * 256;
      const u32 mul = d1.fo == 7 ? 43691u : 32768u, w16 = d1.fo == 7 ? 12 : 16;
#pragma unroll
      for (int e = 0; e < 2; ++e) {
        const u32 i16 = d1.to + e * 64 + lane;
        const u32 sr = (i16 * mul) >> 19, col = i16 - sr * w16;
        const u32 ra = (sr >> 6) * 96 + sr, rc = (((sr - 1664) * 43691u) >> 22) * 64 + sr - 1600;
        (e ? to1 : to0) = (sr < 1664 ? ra : rc) * 16384 + col * 16;
      }
    }
    CFENCE();
    WAIT_VM(3);
    cnt = __builtin_amdgcn_raw_ptr_buffer_atomic_add_i32(d0.fw, rs_m, lane0, d0.fw ? d0.fo + CNT_OFF : 0x40000000u, 16);
    pfo = d0.fo; pfw = d0.fw;
    CFENCE();
    u0 = u1; u1 = u2;
    d0 = d1; d1 = d2;
  }
  publish();
  WAIT_VM(0);
  if (wave == 0) STAMP(5);
}

extern "C" void kernel_launch(int rank, int nranks, ncclComm_t comm, hipStream_t stream,
                              void* const* d_in, const int* in_sizes, int n_in,
                              void* d_out, int out_size,
                              const int* in_dtypes, int out_dtype,
                              void* const* d_out_all,
                              void* const* d_ws_all, size_t ws_bytes) {
  if (nranks != NR || ws_bytes < WS_END || n_in < 2) { fprintf(stderr, "unsupported configuration\n"); return; }
  Params P;
  P.x = (const char*)d_in[0];
  P.w = (const char*)d_in[1];
  for (int p = 0; p < NR; ++p) { P.out[p] = (char*)d_out_all[p]; P.ws[p] = (char*)d_ws_all[p]; }
  P.rank = rank;
  const int phys[8] = {0, 2, 3, 6, 1, 4, 5, 7}, logi[8] = {0, 4, 1, 2, 5, 6, 3, 7};
  P.peers = 0;
  for (int g = 0; g < NR; ++g) P.peers |= (u32)phys[logi[rank & 7] ^ g] << (g * 4);
  hipLaunchKernelGGL(fused_gemm_ar_q8, dim3(256), dim3(NTHREADS), 0, stream, P);
  HIP_CHECK(hipGetLastError());
}
```

```cpp
#define XINFO
#include <hip/hip_runtime.h>
#include <rccl/rccl.h>
#include <cstdio>
#include <stdint.h>

#define HIP_CHECK(expr) do { hipError_t _e = (expr); if (_e != hipSuccess) { \
    fprintf(stderr, "HIP error %s at %s:%d\n", hipGetErrorString(_e), __FILE__, __LINE__); \
    return; } } while (0)

typedef unsigned u32;
typedef unsigned long long u64;
typedef u32 u32x4 __attribute__((ext_vector_type(4)));
typedef u32 u32x2 __attribute__((ext_vector_type(2)));
typedef float f32x4 __attribute__((ext_vector_type(4)));
typedef short s16x4 __attribute__((ext_vector_type(4)));
typedef short s16x8 __attribute__((ext_vector_type(8)));
typedef __bf16 bf16x8 __attribute__((ext_vector_type(8)));
#define LAS __attribute__((address_space(3)))
#define GAS __attribute__((address_space(1)))

constexpr int NR = 8;
constexpr u32 OUT_BYTES = 4096u * 16384u;
constexpr u32 X_BYTES = 4096u * 512u * 2u;
constexpr u32 W_BYTES = 512u * 16384u * 2u;

constexpr u32 NRB = 26;
constexpr u32 NCB = 8;
constexpr u32 XBLK = 20480, WBLK = 32768;
constexpr u32 NWB = NCB + 2;
constexpr u32 WS_W = 0;
constexpr u32 WS_X = 20u << 20;
constexpr u32 WS_CTL = 53u << 20;
constexpr u32 FSTRIDE = 4352;
constexpr u32 WS_XF = WS_CTL;
constexpr u32 WS_WF = WS_XF + NRB * FSTRIDE;
constexpr u32 CNT_OFF = 512u << 10;
constexpr u32 WS_C2 = WS_CTL + (160u << 10);
constexpr u32 WS_G1 = WS_C2 + 4096;
constexpr u32 WS_G2 = WS_C2 + 8192 + 256;
constexpr u32 G2S = 512;
constexpr u32 WS_TK = WS_C2 + 13312;
constexpr u32 WS_TMO = WS_C2 + 20480;
constexpr u32 WS_DBG = WS_C2 + 32768;
constexpr u32 WS_FL = WS_C2 + 65536;
constexpr u32 WS_FC = WS_FL + 16384;
constexpr u32 WS_CALL = WS_FC + 16384;
constexpr u32 WS_Q = 54u << 20;
constexpr u32 QSTRIDE = 4160 * 256;
constexpr u32 WS_EX = 63u << 20;
constexpr u32 WS_ER = 67u << 20;
constexpr u32 WS_EW = 68u << 20;
constexpr u32 WS_L = 84u << 20;
constexpr u32 WS_END = 86u << 20;
static_assert(64u * NWB * WBLK <= WS_X && WS_X + 64u * NRB * XBLK <= WS_CTL && WS_WF + NWB * FSTRIDE <= WS_C2 &&
              WS_CALL + 16384 <= WS_CTL + CNT_OFF && WS_C2 + CNT_OFF <= WS_Q && WS_Q + NCB * QSTRIDE <= WS_EX, "layout");
constexpr u32 FULL = 240;

constexpr u32 SLOT_BYTES = WBLK + XBLK;
constexpr u32 NSLOT = 3;
constexpr u32 TPITCH = 272;
constexpr u32 SNAP_OFF = SLOT_BYTES * NSLOT;
constexpr u32 CTL_OFF = SNAP_OFF + 1024;
constexpr u32 LDS_BYTES = CTL_OFF + 1024;
static_assert(LDS_BYTES <= 163840, "lds");
constexpr int C_AMAX = 0, C_GMAX = 1, C_CALL = 2, C_BAD = 3, C_STAGE = 4, C_CNT = 8, C_MASK = 16;

#define NPW 4
#define EXWAIT 300
#define EXD 60
#define PW1 4
#define PLD 4
#define POLL2 1
#define QWAIT 500
#define QD 100
#define PUSHD 2
constexpr u32 NCOMMWG = 32 - NRB;
constexpr u32 NTILE = NRB * NCB;
constexpr u32 NPULL = NCOMMWG * NPW;
constexpr u32 U_A = 104, U_B = 512, U_C = 1152;
constexpr u32 U_R = 256 + 7 * 384;
constexpr u32 U_Q = U_R + 224;
constexpr int NTHREADS = 512;
constexpr int NSTEP = 64;
constexpr u64 DEADLINE = 50000000ull;

struct Params {
  const char* x;
  const char* w;
  char* out[NR];
  char* ws[NR];
  int rank;
  u32 peers;
};

__device__ __forceinline__ __amdgpu_buffer_rsrc_t mkrsrc(const void* p, u32 bytes) {
  return __builtin_amdgcn_make_buffer_rsrc((void*)p, 0, bytes, 0x00020000);
}
__device__ __forceinline__ u64 now() { return __builtin_amdgcn_s_memrealtime(); }
__device__ __forceinline__ u32 ld_sys(const u32* p) { return __hip_atomic_load((GAS const u32*)p, __ATOMIC_RELAXED, __HIP_MEMORY_SCOPE_SYSTEM); }
__device__ __forceinline__ u64 ld_sys64(const u64* p) { return __hip_atomic_load((GAS const u64*)p, __ATOMIC_RELAXED, __HIP_MEMORY_SCOPE_SYSTEM); }
__device__ __forceinline__ void st_sys(u32* p, u32 v) { __hip_atomic_store((GAS u32*)p, v, __ATOMIC_RELAXED, __HIP_MEMORY_SCOPE_SYSTEM); }
__device__ __forceinline__ void st_sys64(u64* p, u64 v) { __hip_atomic_store((GAS u64*)p, v, __ATOMIC_RELAXED, __HIP_MEMORY_SCOPE_SYSTEM); }
__device__ __forceinline__ u32 lds_ld(LAS u32* p) { return __builtin_amdgcn_readfirstlane(__hip_atomic_load(p, __ATOMIC_RELAXED, __HIP_MEMORY_SCOPE_WORKGROUP)); }
__device__ __forceinline__ void lds_st(LAS u32* p, u32 v) { __hip_atomic_store(p, v, __ATOMIC_RELAXED, __HIP_MEMORY_SCOPE_WORKGROUP); }
__device__ __forceinline__ void lds_add1(LAS u32* p, u32 lane) {
  if (lane == 0) __hip_atomic_fetch_add(p, 1u, __ATOMIC_RELAXED, __HIP_MEMORY_SCOPE_WORKGROUP);
}
#define CFENCE() asm volatile("" ::: "memory")
#define WAIT_VM(n) asm volatile("s_waitcnt vmcnt(" #n ")" ::: "memory")
#define WAIT_VMI(n) asm volatile("s_waitcnt vmcnt(%0)" :: "i"(n) : "memory")
#define WAIT_LGKM() asm volatile("s_waitcnt lgkmcnt(0)" ::: "memory")
#define WAIT_VM0() do { CFENCE(); __builtin_amdgcn_s_waitcnt(0x0F70); WAIT_VM(0); } while (0)

__device__ __forceinline__ u32 wave_umax(u32 v) {
  v = max(v, (u32)__builtin_amdgcn_update_dpp(0, (int)v, 0x111, 0xf, 0xf, true));
  v = max(v, (u32)__builtin_amdgcn_update_dpp(0, (int)v, 0x112, 0xf, 0xf, true));
  v = max(v, (u32)__builtin_amdgcn_update_dpp(0, (int)v, 0x114, 0xf, 0xf, true));
  v = max(v, (u32)__builtin_amdgcn_update_dpp(0, (int)v, 0x118, 0xf, 0xf, true));
  v = max(v, (u32)__builtin_amdgcn_update_dpp(0, (int)v, 0x142, 0xa, 0xf, false));
  v = max(v, (u32)__builtin_amdgcn_update_dpp(0, (int)v, 0x143, 0xc, 0xf, false));
  return (u32)__builtin_amdgcn_readlane((int)v, 63);
}

#define STAMP(k) do {} while (0)

__device__ __forceinline__ void wait_vm_dyn(u32 n) {
  asm volatile(
      "s_getpc_b64 vcc\n\t"
      "s_add_u32 vcc_lo, vcc_lo, %0\n\t"
      "s_addc_u32 vcc_hi, vcc_hi, 0\n\t"
      "s_setpc_b64 vcc\n\t"
      "s_waitcnt vmcnt(0)\n\ts_branch 1f\n\t"
      "s_waitcnt vmcnt(1)\n\ts_branch 1f\n\t"
      "s_waitcnt vmcnt(2)\n\ts_branch 1f\n\t"
      "s_waitcnt vmcnt(3)\n\ts_branch 1f\n\t"
      "s_waitcnt vmcnt(4)\n\ts_branch 1f\n\t"
      "s_waitcnt vmcnt(5)\n\ts_branch 1f\n\t"
      "s_waitcnt vmcnt(6)\n\ts_branch 1f\n\t"
      "s_waitcnt vmcnt(7)\n\ts_branch 1f\n\t"
      "s_waitcnt vmcnt(8)\n\ts_branch 1f\n\t"
      "s_waitcnt vmcnt(9)\n\ts_branch 1f\n\t"
      "s_waitcnt vmcnt(10)\n\ts_branch 1f\n\t"
      "s_waitcnt vmcnt(11)\n\ts_branch 1f\n\t"
      "s_waitcnt vmcnt(12)\n\ts_branch 1f\n\t"
      "s_waitcnt vmcnt(13)\n\ts_branch 1f\n\t"
      "s_waitcnt vmcnt(14)\n\ts_branch 1f\n\t"
      "s_waitcnt vmcnt(15)\n\ts_branch 1f\n\t"
      "s_waitcnt vmcnt(16)\n\ts_branch 1f\n\t"
      "s_waitcnt vmcnt(17)\n\ts_branch 1f\n\t"
      "s_waitcnt vmcnt(18)\n\ts_branch 1f\n\t"
      "s_waitcnt vmcnt(19)\n\ts_branch 1f\n\t"
      "s_waitcnt vmcnt(20)\n\ts_branch 1f\n\t"
      "s_waitcnt vmcnt(21)\n\ts_branch 1f\n\t"
      "s_waitcnt vmcnt(22)\n\ts_branch 1f\n\t"
      "s_waitcnt vmcnt(23)\n\ts_branch 1f\n\t"
      "s_waitcnt vmcnt(24)\n\t"
      "1:"
      :: "s"(min(n, 24u) * 8 + 12) : "memory", "scc", "vcc");
}
#define BARRIER() asm volatile("s_barrier" ::: "memory")

__global__ void __launch_bounds__(NTHREADS) fused_gemm_ar_q8(Params P) {
  __shared__ __attribute__((aligned(16))) char smem[LDS_BYTES];
  LAS char* L = (LAS char*)smem;
  LAS u32* ctl = (LAS u32*)(L + CTL_OFF);
  const u32 tid = threadIdx.x, lane = tid & 63;
  const u32 wave = __builtin_amdgcn_readfirstlane(tid >> 6);
  u32 xcc;
  asm volatile("s_getreg_b32 %0, hwreg(HW_REG_XCC_ID, 0, 3)" : "=s"(xcc));
  const u32 g = xcc & 7u, idx = blockIdx.x >> 3, bid = idx * 8 + g;
  const u32 rank = (u32)P.rank & 7u;
  const u32 dest = (P.peers >> (g * 4)) & 7u;
  char* const myws = P.ws[rank];
  char* const dws = P.ws[dest];
  u32* const tmo = (u32*)(myws + WS_TMO);
  const u32 lane16 = lane * 16;
  const u64 t0 = now();
  const u32 lx = (lane & 7) * 1024 + (lane >> 3) * 16;
  const u32 lw = (((lane >> 3) & 1) * 8 + ((lane >> 1) & 3)) * 32768 + ((lane & 1) * 4 + (lane >> 4)) * 16;
  u32 call = 0;
  bool bad = false;
  if (wave == 0) STAMP(0);

  if (idx >= NCOMMWG) {
    const u32 ti = idx - NCOMMWG;
    const u32 rb = (g & 1) * 13 + (ti >> 1);
    u32 cb = (g >> 1) * 2 + (ti & 1);
    const u32 tile = g * NRB + ti;
    const __amdgpu_buffer_rsrc_t rs_ws = mkrsrc(myws, WS_END);
    char* flp = P.ws[lane & 7] + WS_FL + rank * 256 + g * 4;
    const u32 lane0 = lane ? 0x80000000u : 0u;
    auto signal_wg = [&](u32 kind, u32 inc) {
      const u32 old = __builtin_amdgcn_raw_ptr_buffer_atomic_add_i32(inc, rs_ws, lane0, WS_FC + kind * 2048 + g * 256, 16);
      if (__builtin_amdgcn_readfirstlane(old) + inc == call * NRB && lane < 8)
        st_sys((u32*)(flp + kind * 2048), call);
      CFENCE();
    };
    auto signal_all = [&](u32 kind, u32 inc = 1) {
      u32 old = 0;
      if (lane == 0) old = __hip_atomic_fetch_add(ctl + C_CNT + kind, 1u, __ATOMIC_RELAXED, __HIP_MEMORY_SCOPE_WORKGROUP);
      if (__builtin_amdgcn_readfirstlane(old) == 7) signal_wg(kind, inc);
    };
    {
      const u32 pc = g ? 2 * U_A + (tile - NRB) * 5 + wave : ti * 8 + wave;
      const bool pok = (wave < 5 || !g) && pc < 1024;
      const u32x4 e0 = __builtin_amdgcn_raw_buffer_load_b128(mkrsrc(P.x, X_BYTES), lx,
                                                             pok ? (pc & 511) * 8192 + (pc >> 9) * 128 : 0x80000000u, 0);
      CFENCE();
      if (wave == 7 && lane < 16) ctl[lane] = 0u;
      __builtin_amdgcn_raw_buffer_store_b128(e0, rs_ws, lane16, pok ? WS_EX + pc * 1024 : 0x80000000u, 17);
      asm volatile("" : "+v"(flp));
      WAIT_VM0();
      WAIT_LGKM();
      BARRIER();
      if (wave == 0) {
        const u32 old = __builtin_amdgcn_readfirstlane(__builtin_amdgcn_raw_ptr_buffer_atomic_add_i32(1, rs_ws, lane0, WS_FC + g * 256, 16));
        call = old / NRB + 1;
        if (old + 1 == call * NRB && lane < 8) {
          st_sys((u32*)flp, call);
        }
        if (lane == 0) lds_st(ctl + C_CALL, call);
        WAIT_LGKM();
      }
      if (wave == 6 && ti == 0) __builtin_amdgcn_fence(__ATOMIC_ACQUIRE, "");
      BARRIER();
      call = lds_ld(ctl + C_CALL);
      for (u32 r = 0; r < 3; ++r) {
        const u32 slot = r * 8 + wave, e = slot * NTILE + tile;
        u32 kc, k, so, eo;
        bool isw = true;
        if (e < 4352) {
          kc = (e + 768) / 640; k = e + 768 - kc * 640;
          if (kc == 1) k -= 128;
          isw = k < 512;
          k &= 511;
          eo = isw ? WS_EW + ((k >> 6) * 8 + kc) * (8 * WBLK) + (k & 63) * 4096 : WS_EX + kc * (512u << 10) + k * 4096;
        } else {
          kc = (e - 4352) >> 4; k = ((e >> 1) & 7) * 64 + 62 + (e & 1);
          eo = WS_ER + (call & 1) * (512u << 10) + (e - 4352) * 4096;
        }
        if (e < 4480) {
          so = isw ? (kc * 64 + (k & 1) * 32) * 32768 + (k >> 1) * 128 : k * 32768 + kc * 128;
          const __amdgpu_buffer_rsrc_t rs = mkrsrc(isw ? P.w : P.x, isw ? W_BYTES : X_BYTES);
          const u32 lo = isw ? lw : lx, s1 = isw ? 4 * 32768 : 8192, s2 = isw ? 16 * 32768 : 16384;
          while (now() - t0 < EXWAIT + slot * EXD) __builtin_amdgcn_s_sleep(2);
          const u32x4 e0 = __builtin_amdgcn_raw_buffer_load_b128(rs, lo, so, 0);
          const u32x4 e1 = __builtin_amdgcn_raw_buffer_load_b128(rs, lo, so + s1, 0);
          const u32x4 e2 = __builtin_amdgcn_raw_buffer_load_b128(rs, lo, so + s2, 0);
          const u32x4 e3 = __builtin_amdgcn_raw_buffer_load_b128(rs, lo, so + s1 + s2, 0);
          __builtin_amdgcn_raw_buffer_store_b128(e0, rs_ws, lane16, eo, 17);
          __builtin_amdgcn_raw_buffer_store_b128(e1, rs_ws, lane16, eo + 1024, 17);
          __builtin_amdgcn_raw_buffer_store_b128(e2, rs_ws, lane16, eo + 2048, 17);
          __builtin_amdgcn_raw_buffer_store_b128(e3, rs_ws, lane16, eo + 3072, 17);
        }
        if (r != 1) {
          WAIT_VM0();
          signal_all(1 + (r >> 1));
        }
      }
      BARRIER();
    }
    const u32 cv = call * FULL;
    const u32 wm = wave >> 2, wn = wave & 3;
    const u32 lds0 = (u32)(size_t)L;
    const u32 xa = WBLK + wm * 10240 + ((lane >> 3) & 1) * 1024 + (lane >> 5) * 256 + ((lane >> 4) & 1) * 128 + (lane & 7) * 16;
    const u32 h = lane >> 4, q = (lane >> 2) & 3, p = lane & 3;
    const u32 wa0 = wn * 8192 + (h >> 1) * 2048 + (p & 1) * 512 + (h & 1) * 128 + q * 32 + (p >> 1) * 16 + (p & 1) * 8;
    const u32 wa1 = wa0 ^ 8;
    const u32 xpo = wave < 4 ? wave * 3072 : 12288 + (wave - 4) * 2048;
    u32 pollo;
    f32x4 acc[4][5];
    u64 kx = 0, kw;
    u64 ready, taken;
    u32 issued, consumed, nis = 0;
    u32 islot, cslot, par = 0;
    u64 markD = 0;
    float inv = 0.f;
    auto sched = [&]() {
      u64 avail = ready & ~taken;
      while (avail && issued - consumed < NSLOT) {
        const u32 ord = __builtin_ctzll(avail);
        avail &= avail - 1;
        taken |= 1ull << ord;
        const u32 sw = WS_W + (ord * NWB + cb) * WBLK + wave * 4096;
        const u32 sx = WS_X + (ord * NRB + rb) * XBLK + xpo;
        const u32 mw = lds0 + islot + wave * 4096, mx = lds0 + islot + WBLK + xpo;
        u32 keep;
        asm volatile(
            "s_mov_b32 %0, m0\n\t"
            "s_mov_b32 m0, %3\n\t"
            "s_nop 0\n\t"
            "buffer_load_dwordx4 %1, %2, %5 offen lds\n\t"
            "buffer_load_dwordx4 %1, %2, %5 offen offset:1024 lds\n\t"
            "buffer_load_dwordx4 %1, %2, %5 offen offset:2048 lds\n\t"
            "buffer_load_dwordx4 %1, %2, %5 offen offset:3072 lds\n\t"
            "s_mov_b32 m0, %4\n\t"
            "s_nop 0\n\t"
            "buffer_load_dwordx4 %1, %2, %6 offen lds\n\t"
            "buffer_load_dwordx4 %1, %2, %6 offen offset:1024 lds\n\t"
            "s_cmp_gt_u32 %7, 3\n\t"
            "s_cbranch_scc1 1f\n\t"
            "buffer_load_dwordx4 %1, %2, %6 offen offset:2048 lds\n\t"
            "1:\n\t"
            "s_mov_b32 m0, %0"
            : "=&s"(keep)
            : "v"(lane16), "s"(rs_ws), "s"(mw), "s"(mx), "s"(sw), "s"(sx), "s"(wave)
            : "memory", "scc");
        nis += wave < 4 ? 7 : 6;
        const u32 sh = (issued & 3) * 16;
        markD = (markD & ~(0xffffull << sh)) | ((u64)(nis & 0xffff) << sh);
        issued += 1;
        islot = islot == 2 * SLOT_BYTES ? 0 : islot + SLOT_BYTES;
      }
    };
    auto poll = [&]() {
      if (wave == 0 && (kx & kw) != ~0ull) {
        const u64 kn = lane < 16 ? kx : kw;
        const bool act = lane < 32 && ((u32)(kn >> ((lane & 15) * 4)) & 15u) != 15u;
        const u32 po = act ? pollo : 0x80000000u;
        u32 keep;
        asm volatile("s_mov_b32 %0, m0\n\ts_mov_b32 m0, %3\n\ts_nop 0\n\t"
                     "buffer_load_dwordx4 %1, %2, 0 offen sc0 sc1 lds\n\ts_mov_b32 m0, %0"
                     : "=&s"(keep) : "v"(po), "s"(rs_ws), "s"(lds0 + SNAP_OFF) : "memory");
        nis += 1;
      }
    };
    LAS u32* const snap = (LAS u32*)(L + SNAP_OFF) + lane;
    auto seen = [&](u32 fx, u32 fw) {
      kx |= __ballot((int)(fx - cv) >= 0);
      kw |= __ballot((int)(fw - cv) >= 0);
    };
    auto tell = [&]() { if (lane == 0) *(LAS u64*)(L + CTL_OFF + C_MASK * 4 + par) = kx & kw; };
    auto told = [&]() -> u64 {
      const u64 m = *(LAS u64*)(L + CTL_OFF + C_MASK * 4 + par);
      par ^= 8;
      return m;
    };
    auto uni = [&](u64 m) -> u64 {
      return (u64)(u32)__builtin_amdgcn_readfirstlane((u32)m) | ((u64)(u32)__builtin_amdgcn_readfirstlane((u32)(m >> 32)) << 32);
    };
    auto wait_step = [&]() { wait_vm_dyn((nis - (u32)(markD >> ((consumed & 3) * 16))) & 0xffff); };
#define SB() __builtin_amdgcn_sched_barrier(0)
#define PIECE(M, S, OFF) do { \
        asm volatile("s_cmp_lg_u32 %2, -1\n\ts_cbranch_scc0 1f\n\ts_mov_b32 m0, %2\n\ts_nop 0\n\t" \
                     "buffer_load_dwordx4 %0, %1, %3 offen offset:" #OFF " lds\n\t1:" \
                     :: "v"(lane16), "s"(rs_ws), "s"(M), "s"(S) : "memory", "scc"); \
        SB(); } while (0)
#define MMA4(kb, i) do { \
        _Pragma("unroll") for (int n = 0; n < 4; ++n) \
          acc[n][i] = __builtin_amdgcn_mfma_f32_16x16x32_bf16(wf[kb][n], xf[kb][i], acc[n][i], 0, 0, 0); \
        SB(); } while (0)
    bf16x8 xf[2][5], wf[2][4];
    u32 sp = 0;
    u64 cyc0 = 0;
    for (u32 pass = 0;; ++pass) {
    pollo = lane < 16 ? WS_XF + rb * FSTRIDE + lane16 : WS_WF + cb * FSTRIDE + lane16 - 256;
    kw = taken = 0;
    issued = consumed = islot = cslot = 0;
#pragma unroll
    for (int a = 0; a < 4; ++a)
#pragma unroll
      for (int b = 0; b < 5; ++b) acc[a][b] = f32x4{0.f, 0.f, 0.f, 0.f};
#pragma unroll
    for (int n = 0; n < 5; ++n) {
      xf[1][n] = __builtin_bit_cast(bf16x8, u32x4{0u, 0u, 0u, 0u});
      if (n < 4) wf[1][n] = __builtin_bit_cast(bf16x8, u32x4{0u, 0u, 0u, 0u});
    }
    poll();
    while (consumed < NSTEP) {
      if (issued == consumed) {
        if (wave == 0) {
          while ((kx & kw & ~taken) == 0) {
            poll();
            WAIT_VM(1);
            seen(snap[0], snap[64]);
            if ((++sp & 63) == 0 && now() - t0 > DEADLINE) {
              st_sys(tmo, call);
              WAIT_VM(0);
              lds_st(ctl + C_BAD, 1u);
              kx = kw = ~0ull;
            }
          }
          tell();
          WAIT_LGKM();
        }
        BARRIER();
        ready = uni(told());
        sched();
        poll();
        wait_step();
        if (wave == 0) {
          seen(snap[0], snap[64]);
          tell();
          WAIT_LGKM();
        }
      }
      BARRIER();
      const u64 rdy = told();
      SB();
      LAS char* base = L + cslot;
      auto frags = [&](int kb) {
#pragma unroll
        for (int n = 0; n < 5; ++n) {
          if (n < 4) {
            LAS char* a = base + ((n & 1) ? wa1 : wa0) + kb * 4096 + (n >> 1) * 256;
            s16x4 lo = __builtin_amdgcn_ds_read_tr16_b64_v4i16((LAS s16x4*)a);
            s16x4 hi = __builtin_amdgcn_ds_read_tr16_b64_v4i16((LAS s16x4*)(a + 1024));
            wf[kb][n] = __builtin_bit_cast(bf16x8, __builtin_shufflevector(lo, hi, 0, 1, 2, 3, 4, 5, 6, 7));
          }
          xf[kb][n] = *(LAS bf16x8*)(base + xa + n * 2048 + kb * 512);
        }
        SB();
      };
      MMA4(1, 0);
      frags(0);
      poll();
      SB();
      MMA4(1, 1);
      ready = uni(rdy);
      const u64 av = ready & ~taken;
      const bool go = av != 0 && issued - consumed < NSLOT;
      const u32 ord = __builtin_ctzll(av | (1ull << 63));
      const u32 sw = WS_W + (ord * NWB + cb) * WBLK + wave * 4096;
      const u32 sx = WS_X + (ord * NRB + rb) * XBLK + xpo;
      u32 mw = ~0u, mx = ~0u, mx3 = ~0u;
      if (go) {
        mw = lds0 + islot + wave * 4096;
        mx = lds0 + islot + WBLK + xpo;
        if (wave < 4) mx3 = mx;
        taken |= 1ull << ord;
        nis += wave < 4 ? 7 : 6;
        const u32 sh = (issued & 3) * 16;
        markD = (markD & ~(0xffffull << sh)) | ((u64)(nis & 0xffff) << sh);
        issued += 1;
        islot = islot == 2 * SLOT_BYTES ? 0 : islot + SLOT_BYTES;
      }
      SB();
      MMA4(1, 2);
      PIECE(mw, sw, 0);
      MMA4(1, 3);
      PIECE(mw, sw, 1024);
      MMA4(1, 4);
      frags(1);
      PIECE(mw, sw, 2048);
      MMA4(0, 0);
      PIECE(mw, sw, 3072);
      PIECE(mx, sx, 0);
      MMA4(0, 1);
      PIECE(mx, sx, 1024);
      PIECE(mx3, sx, 2048);
      MMA4(0, 2);
      sched();
      poll();
      cslot = cslot == 2 * SLOT_BYTES ? 0 : cslot + SLOT_BYTES;
      consumed += 1;
      SB();
      MMA4(0, 3);
      if (issued != consumed) wait_step();
      u32 fx = cv - 1, fw = cv - 1;
      if (wave == 0) { fx = snap[0]; fw = snap[64]; }
      SB();
      MMA4(0, 4);
      if (wave == 0) { seen(fx, fw); tell(); }
      WAIT_LGKM();
    }
    MMA4(1, 0); MMA4(1, 1); MMA4(1, 2); MMA4(1, 3); MMA4(1, 4);
    if (pass == 0) {
      if (wave == 0) STAMP(2);
      const u32 row0 = rb * 160 + wm * 80;
      float m = 0.f;
#pragma unroll
      for (int b = 0; b < 5; ++b) {
        float mb = 0.f;
#pragma unroll
        for (int a = 0; a < 4; ++a)
#pragma unroll
          for (int e = 0; e < 4; ++e) mb = fmaxf(mb, fabsf(acc[a][b][e]));
        if (row0 + b * 16 < 4096) m = fmaxf(m, mb);
      }
      const u32 mb = wave_umax(__builtin_bit_cast(u32, m));
      if (lane == 0) __hip_atomic_fetch_max(ctl + C_AMAX, mb, __ATOMIC_RELAXED, __HIP_MEMORY_SCOPE_WORKGROUP);
      WAIT_LGKM();
      BARRIER();
      if (wave == 0) {
        const u32 wmax = lds_ld(ctl + C_AMAX);
        if (lane == 0) st_sys64((u64*)(myws + WS_G1) + tile, ((u64)call << 32) | wmax);
        STAMP(3);
        const u64* g2 = (const u64*)(myws + WS_G2 + g * G2S) + (lane & 7);
        u64 val;
        for (;;) {
          val = ld_sys64(g2);
          if (__all((u32)(val >> 32) == call)) break;
          if ((++sp & 63) == 0 && (ld_sys(tmo) == call || now() - t0 > DEADLINE)) { st_sys(tmo, call); bad = true; break; }
        }
        const u32 gmax = wave_umax((u32)val);
        if (lane == 0) {
          lds_st(ctl + C_GMAX, gmax);
          if (bad) lds_st(ctl + C_BAD, 1u);
        }
        WAIT_LGKM();
      }
    }
    BARRIER();
    if (pass == 0) {
      const float amax = __builtin_bit_cast(float, lds_ld(ctl + C_GMAX));
      const float scale = amax / 127.0f;
      inv = amax > 0.f ? 1.0f / scale : 0.f;
      if (wave == 0) STAMP(4);
    }
    bad = lds_ld(ctl + C_BAD) != 0;
    const bool odd = (lane >> 4) & 1;
    u32x4 qv[5];
#pragma unroll
    for (int i = 0; i < 5; ++i) {
      u32 d[4];
#pragma unroll
      for (int n = 0; n < 4; ++n) {
        u32 b0 = __builtin_bit_cast(u32, __builtin_fmaf(acc[n][i][0], inv, 12582912.0f));
        u32 b1 = __builtin_bit_cast(u32, __builtin_fmaf(acc[n][i][1], inv, 12582912.0f));
        u32 b2 = __builtin_bit_cast(u32, __builtin_fmaf(acc[n][i][2], inv, 12582912.0f));
        u32 b3 = __builtin_bit_cast(u32, __builtin_fmaf(acc[n][i][3], inv, 12582912.0f));
        u32 lo = __builtin_amdgcn_perm(b1, b0, 0x0c0c0400u);
        u32 hi = __builtin_amdgcn_perm(b3, b2, 0x04000c0cu);
        d[n] = bad ? 0x80808080u : (lo | hi);
      }
      qv[i] = u32x4{odd ? d[1] : d[0], odd ? d[0] : d[1], odd ? d[3] : d[2], odd ? d[2] : d[3]};
    }
    const u64 tq = now();
    if (!pass && !(cb & 1) && wm == 0) {
      const __amdgpu_buffer_rsrc_t rs_p = mkrsrc(dws, WS_END);
#pragma unroll
      for (int i = 0; i < 4; ++i) {
        if (i && i % PUSHD == 0) WAIT_VM(0);
        if (wave == 0 && i == PUSHD) STAMP(10);
        __builtin_amdgcn_raw_buffer_store_b128(qv[i], rs_p, lane16, WS_L + ((g * 13 + (ti >> 1)) * 16 + wn * 4 + i) * 1024, 17);
        CFENCE();
      }
      WAIT_VM0();
      if (wave == 0) STAMP(11);
      const u32 old = __builtin_amdgcn_raw_ptr_buffer_atomic_add_i32(1, rs_ws, lane0, WS_FC + 5 * 2048 + g * 256, 16);
      if (__builtin_amdgcn_readfirstlane(old) + 1 == call * 52 && lane == 0) st_sys((u32*)(dws + WS_FL + 5 * 2048) + g, call);
      CFENCE();
    }
    {
      LAS char* tw = L + (wm * 80 + (lane & 15)) * TPITCH + wn * 64 + (lane >> 4) * 16;
#pragma unroll
      for (int i = 0; i < 5; ++i) *(LAS u32x4*)(tw + i * 16 * TPITCH) = qv[i];
    }
    WAIT_LGKM();
    BARRIER();
    LAS char* tr = L + (wave * 4 + (lane >> 4)) * TPITCH + (lane & 15) * 16;
#pragma unroll
    for (int i = 0; i < 5; ++i) qv[i] = *(LAS u32x4*)(tr + i * 32 * TPITCH);
    const __amdgpu_buffer_rsrc_t rs_o = mkrsrc(P.out[rank], OUT_BYTES);
    const u32 orow = (rb * 160 + wave * 4 + (lane >> 4)) * 16384u;
    if (pass) {
      const u32 piece = (cb - 8) * 4 + ((lane >> 2) & 3);
      const u32 oo = piece == rank ? 0x80000000u : orow + piece * 2048 + 1984 + (lane & 3) * 16;
#pragma unroll
      for (int i = 0; i < 5; ++i) __builtin_amdgcn_raw_buffer_store_b128(qv[i], rs_o, oo, i * 32 * 16384, 17);
      break;
    }
    const u32 pitch = cb == 7 ? 192 : 256;
    const bool rep = (lane & 15) * 16 >= pitch;
    const u32 qo = rep ? 0x80000000u : (wave * 4 + (lane >> 4)) * pitch + (lane & 15) * 16;
    const u32 oo = orow + rank * 2048 + cb * 256 + (lane & 15) * 16;
    const u32 qa = WS_Q + cb * QSTRIDE + rb * 64 * pitch, qb = WS_Q + cb * QSTRIDE + (1664 + rb * 96) * pitch;
    if (cb & 1) {
      __builtin_amdgcn_raw_buffer_store_b128(qv[0], rs_ws, qo, qa, 17);
      __builtin_amdgcn_raw_buffer_store_b128(qv[1], rs_ws, qo, qa + 32 * pitch, 17);
      WAIT_VM0();
      signal_all(3, 2);
      if (wave == 0) STAMP(9);
    }
    while (now() - tq < QWAIT + wave * QD) __builtin_amdgcn_s_sleep(2);
    if (!(cb & 1)) {
      __builtin_amdgcn_raw_buffer_store_b128(qv[0], rs_ws, qo, qa, 17);
      __builtin_amdgcn_raw_buffer_store_b128(qv[1], rs_ws, qo, qa + 32 * pitch, 17);
    }
#pragma unroll
    for (int i = 2; i < 5; ++i) __builtin_amdgcn_raw_buffer_store_b128(qv[i], rs_ws, qo, qb + (i - 2) * 32 * pitch, 17);
    WAIT_VM0();
    signal_all(4);
    if (cb == 7) {
#pragma unroll
      for (int i = 0; i < 5; ++i) __builtin_amdgcn_raw_buffer_store_b128(qv[i], rs_o, rep ? oo : 0x80000000u, i * 32 * 16384, 17);
    }
    if (((ti >> 1) & 3) != (g >> 1)) break;
    cb = 8 + (ti & 1);
    WAIT_VM0();
    WAIT_LGKM();
    BARRIER();
    }
    WAIT_VM(0);
    if (wave == 0) STAMP(5);
    return;
  }

  const u32 vid = idx * NPW + wave;
  const __amdgpu_buffer_rsrc_t rs_s = mkrsrc(dws, WS_END);
  const u32 lane0 = lane ? 0x80000000u : 0u;
  static_assert(NPW == 4 && NCOMMWG == 6, "wave roles");
  if (wave >= NPW && wave < 7) {
    const u32 ph = (wave - NPW) * NCOMMWG + idx;
    const __amdgpu_buffer_rsrc_t rs_w = mkrsrc(P.w, W_BYTES);
    const u32 so = ph < 16 ? dest * 4096 + ph * 256 : 0x80000000u;
    const u32 to = ph < 16 ? WS_W + rank * NWB * WBLK + ph * 16384 : 0x80000000u;
    u32x4 pv[16];
#pragma unroll
    for (int i = 0; i < 16; ++i) {
      pv[i] = __builtin_amdgcn_raw_buffer_load_b128(rs_w, lw, so + (i >> 3) * 128 + (((i >> 1) & 3) * 16 + (i & 1) * 4) * 32768, 0);
      if (i == PLD - 1) BARRIER();
    }
    CFENCE();
#pragma unroll
    for (int i = 0; i < 16; ++i) {
      if (i >= PW1) WAIT_VMI(PW1 - 1);
      __builtin_amdgcn_raw_buffer_store_b128(pv[i], rs_s, lane16, to + i * 1024, 17);
      CFENCE();
    }
    WAIT_VM(0);
    if (wave == NPW) STAMP(10);
    if (ph < 16) __builtin_amdgcn_raw_ptr_buffer_atomic_add_i32(FULL / 2, rs_s, lane0, WS_WF + (ph >> 1) * FSTRIDE + rank * 4, 16);
    return;
  }
  if (wave == 7) {
    ctl[lane] = 0u;
    WAIT_LGKM();
  }
  BARRIER();
  if (wave == 7) {
    u32* const cptr = (u32*)(myws + WS_CALL + bid * 256);
    call = __builtin_amdgcn_readfirstlane(ld_sys(cptr)) + 1;
    if (lane == 0) { lds_st(ctl + C_CALL, call); st_sys(cptr, call); }
    u32 sp = 0;
    auto expired = [&]() -> bool {
      if ((++sp & 63) == 0 && (ld_sys(tmo) == call || now() - t0 > DEADLINE)) {
        st_sys(tmo, call);
        if (lane == 0) { lds_st(ctl + C_BAD, 1u); lds_st(ctl + C_STAGE, 7u); }
        bad = true;
      }
      return bad;
    };
    auto watch = [&](u32 kind, u32 row) {
      const u32* fp = (const u32*)(myws + WS_FL + kind * 2048 + row * 256) + (lane & 7);
      while (!bad) {
        const u32 f = ld_sys(fp);
        const u64 ok = __ballot((int)(f - call) >= 0);
        if (ok == ~0ull) break;
        if (kind == 0 && (ok & 1) && lane == 0) lds_st(ctl + C_STAGE, 1u);
        __builtin_amdgcn_s_sleep(1);
        expired();
      }
      if (lane == 0 && !bad) lds_st(ctl + C_STAGE, kind + 2);
      CFENCE();
    };
    watch(0, dest);
    STAMP(11);
    watch(1, dest);
    watch(2, dest);
    if (bid == 0) {
      const u64* g1 = (const u64*)(myws + WS_G1);
      u64* g2 = (u64*)(P.ws[lane & 7] + WS_G2 + (lane >> 3) * G2S) + rank;
      asm volatile("" : "+v"(g2));
      u32 gm = 0;
      while (!bad) {
        u64 a0 = ld_sys64(g1 + lane), a1 = ld_sys64(g1 + 64 + lane), a2 = ld_sys64(g1 + 128 + lane);
        u64 a3 = ld_sys64(g1 + 192 + (lane & 15));
        const bool ok = (u32)(a0 >> 32) == call && (u32)(a1 >> 32) == call && (u32)(a2 >> 32) == call && (u32)(a3 >> 32) == call;
        gm = max(max((u32)a0, (u32)a1), max((u32)a2, (u32)a3));
        const u64 okm = __ballot(ok);
        if (okm == ~0ull) break;
        if (okm == 0) __builtin_amdgcn_s_sleep(8);
        expired();
      }
      gm = wave_umax(gm);
      if (!bad) st_sys64(g2, ((u64)call << 32) | gm);
    }
    watch(3, dest);
    STAMP(9);
    watch(4, dest);
    if (g == 0) watch(5, 0);
    return;
  }
  u32 stage = 0;
  auto need = [&](u32 s) {
    while (stage < s) {
      stage = lds_ld(ctl + C_STAGE);
      if (stage < s) __builtin_amdgcn_s_sleep(1);
    }
    CFENCE();
  };
  need(1);
  if (wave == 0) STAMP(3);
  call = lds_ld(ctl + C_CALL);
  const __amdgpu_buffer_rsrc_t rs_m = mkrsrc(myws, WS_Q);
  const __amdgpu_buffer_rsrc_t rs_o = mkrsrc(P.out[rank], OUT_BYTES);
  const u32 rot = (g * 26 / 7 % 26) * 10;
  constexpr u32 U_L = U_Q + 3968 - 104, uq2 = U_Q + 3 * 208 + 156;
  const u32 nu = g ? U_L : U_L + 832;
  struct Unit { u32 so, to, fo, fw; };
  u32 qc = 0, qbase = U_Q;
  auto uoff = [&](u32 u) -> Unit {
    Unit d;
    if (u < U_R) {
      const u32 kc = (u + 128) / 384, ord = kc * 8 + dest;
      u32 k = u + 128 - kc * 384;
      if (kc == 1) k = k < 256 ? k + 128 : k - 256;
      if (k < 128) {
        d.so = WS_EW + (rank * 8 + kc) * (8 * WBLK) + k * 2048;
        d.to = WS_W + ord * NWB * WBLK + k * 2048;
        d.fo = WS_WF + (k >> 4) * FSTRIDE + ord * 4;
        d.fw = 15;
      } else {
        const u32 kk = (k - 128 + (kc ? rot : 0)) & 255, r = kk / 10;
        d.so = WS_EX + kc * (512u << 10) + kk * 2048;
        d.to = WS_X + ord * NRB * XBLK + kk * 2048;
        d.fo = WS_XF + r * FSTRIDE + ord * 4;
        d.fw = r == NRB - 1 ? 40 : 24;
      }
    } else if (u < U_Q) {
      const u32 k = u - U_R, kc = k / 28, j = k - kc * 28, ord = kc * 8 + dest;
      const u32 jj = j + ((j >> 2) >= rank ? 4 : 0);
      d.so = WS_ER + (call & 1) * (512u << 10) + kc * 65536 + jj * 2048;
      d.to = WS_W + (ord * NWB + 8) * WBLK + jj * 2048;
      d.fo = WS_WF + (8 + (jj >> 4)) * FSTRIDE + ord * 4;
      d.fw = (jj >> 4) == (rank >> 2) ? 20 : 15;
    } else if (u < U_L) {
      u32 c, na, n;
      for (;;) {
        c = (qc < 8 ? (g & 6) + 2 + qc * 2 + (qc < 4) : g + qc) & 7;
        na = c == 7 ? 156 : 208;
        n = qc < 7 ? na : qc < 8 ? 104 : c == 7 ? 228 : 304;
        if (u - qbase < n) break;
        qbase += n;
        qc += 1;
      }
      const u32 k = u - qbase + (qc >= 8 ? na : qc == 7 && !(g & 1) ? 104 : 0);
      d.so = WS_Q + c * QSTRIDE + k * 2048;
      d.to = k * 128;
      d.fo = c;
      d.fw = 0;
    } else {
      const u32 k = u - U_L, gs = k / 104, r = k - gs * 104;
      d.so = u < nu ? WS_L + k * 2048 : 0x80000000u;
      d.to = ((gs & 1) * 2080 + (r >> 3) * 160 + (r & 1) * 32) * 16384 + ((P.peers >> ((gs & 7) * 4)) & 7u) * 2048 + (gs & 6) * 256 + (r & 6) * 32;
      d.fo = 8;
      d.fw = 0;
    }
    return d;
  };
  u32x4 v0, v1;
  u32 u0 = vid, u1 = NPULL + vid, tk = vid + (call - 1) * nu - NPULL;
  Unit d0 = uoff(u0), d1 = uoff(u1);
  v0 = __builtin_amdgcn_raw_buffer_load_b128(rs_s, lane16, d0.so, 17);
  v1 = __builtin_amdgcn_raw_buffer_load_b128(rs_s, lane16, d0.so + 1024, 17);
  u32 to0 = lane16, to1 = lane16 + 1024, tso = d0.to, lo1 = lane16;
  __amdgpu_buffer_rsrc_t rs_t = rs_m;
  const u32 cv = call * FULL;
  u32 cnt = 0, pfo = 0, pfw = 0;
  auto publish = [&]() {
    if (pfw && __builtin_amdgcn_readfirstlane(cnt) + pfw == cv)
      asm volatile("s_nop 4\n\tbuffer_store_dword %0, %1, %2, %3 offen sc0 sc1" :: "v"(cv), "v"(lane0), "s"(rs_m), "s"(pfo) : "memory");
    pfw = 0;
  };
  WAIT_VM0();
  while (u0 < nu) {
    if (bad && !d0.fw) v0 = v1 = u32x4{0x80808080u, 0x80808080u, 0x80808080u, 0x80808080u};
    __builtin_amdgcn_raw_buffer_store_b128(v0, rs_t, to0, tso, 17);
    __builtin_amdgcn_raw_buffer_store_b128(v1, rs_t, to1, tso, 17);
    CFENCE();
    const u32 ns = u1 >= nu ? 0 : u1 >= U_L ? 7 : u1 >= uq2 ? 6 : u1 >= U_Q ? 5 : u1 >= U_C ? 4 : u1 >= U_B ? 3 : u1 >= U_A ? 2 : 1;
    if (stage < ns) {
      stage = lds_ld(ctl + C_STAGE);
      if (stage < ns) {
        publish();
        WAIT_VM0();
        if (d0.fw) {
          cnt = __builtin_amdgcn_raw_ptr_buffer_atomic_add_i32(d0.fw, rs_m, lane0, d0.fo + CNT_OFF, 16);
          pfo = d0.fo; pfw = d0.fw;
          publish();
        }
        d0.fw = 0;
        need(ns);
      }
      bad = lds_ld(ctl + C_BAD) != 0;
    }
    v0 = __builtin_amdgcn_raw_buffer_load_b128(rs_s, lo1, d1.so, 17);
    v1 = __builtin_amdgcn_raw_buffer_load_b128(rs_s, lo1, d1.so + 1024, 17);
    CFENCE();
    const u32 u2 = 3 * NPULL + (__builtin_amdgcn_readfirstlane(tk) - (call - 1) * nu);
    tk = __builtin_amdgcn_raw_ptr_buffer_atomic_add_i32(1, rs_m, lane0, WS_TK + g * 128, 16);
    CFENCE();
    publish();
    const Unit d2 = uoff(u2);
    tso = d1.to;
    if (u2 >= U_L) lo1 = (lane & 3) * 256 + (lane >> 2) * 16;
    if (u1 >= U_L) {
      rs_t = rs_o;
      to0 = (lane >> 2) * 16384 + (lane & 3) * 16;
      to1 = to0 + 16 * 16384;
    } else if (!d1.fw) {
      rs_t = rs_o;
      tso = dest * 2048 + d1.fo * 256;
      const u32 mul = d1.fo == 7 ? 43691u : 32768u, w16 = d1.fo == 7 ? 12 : 16;
#pragma unroll
      for (int e = 0; e < 2; ++e) {
        const u32 i16 = d1.to + e * 64 + lane;
        const u32 sr = (i16 * mul) >> 19, col = i16 - sr * w16;
        const u32 ra = (sr >> 6) * 96 + sr, rc = (((sr - 1664) * 43691u) >> 22) * 64 + sr - 1600;
        (e ? to1 : to0) = (sr < 1664 ? ra : rc) * 16384 + col * 16;
      }
    }
    CFENCE();
    WAIT_VM(3);
    cnt = __builtin_amdgcn_raw_ptr_buffer_atomic_add_i32(d0.fw, rs_m, lane0, d0.fw ? d0.fo + CNT_OFF : 0x40000000u, 16);
    pfo = d0.fo; pfw = d0.fw;
    CFENCE();
    u0 = u1; u1 = u2;
    d0 = d1; d1 = d2;
  }
  publish();
  WAIT_VM(0);
  if (wave == 0) STAMP(5);
}

extern "C" void kernel_launch(int rank, int nranks, ncclComm_t comm, hipStream_t stream,
                              void* const* d_in, const int* in_sizes, int n_in,
                              void* d_out, int out_size,
                              const int* in_dtypes, int out_dtype,
                              void* const* d_out_all,
                              void* const* d_ws_all, size_t ws_bytes) {
  if (nranks != NR || ws_bytes < WS_END || n_in < 2) { fprintf(stderr, "unsupported configuration\n"); return; }
  Params P;
  P.x = (const char*)d_in[0];
  P.w = (const char*)d_in[1];
  for (int p = 0; p < NR; ++p) { P.out[p] = (char*)d_out_all[p]; P.ws[p] = (char*)d_ws_all[p]; }
  P.rank = rank;
  const int phys[8] = {0, 2, 3, 6, 1, 4, 5, 7}, logi[8] = {0, 4, 1, 2, 5, 6, 3, 7};
  P.peers = 0;
  for (int g = 0; g < NR; ++g) P.peers |= (u32)phys[logi[rank & 7] ^ g] << (g * 4);
  if (rank == 0) {
    char line[2048]; int o = snprintf(line, sizeof line, "XINFO");
    for (int d = 0; d < NR; ++d) {
      char bus[64] = {0}, path[128], val[256] = {0};
      (void)hipDeviceGetPCIBusId(bus, 64, d);
      snprintf(path, sizeof path, "/sys/bus/pci/devices/%s/xgmi_physical_id", bus);
      FILE* f = fopen(path, "r"); int ph = -1; if (f) { if (fscanf(f, "%d", &ph) != 1) ph = -2; fclose(f); }
      snprintf(path, sizeof path, "/sys/bus/pci/devices/%s/xgmi_port_num", bus);
      f = fopen(path, "r"); if (f) { size_t n = fread(val, 1, 200, f); val[n] = 0; fclose(f); for (char* c = val; *c; ++c) if (*c == '\n') *c = ';'; }
      o += snprintf(line + o, sizeof line - o, " | %s ph%d %s", bus, ph, d < 2 ? val : "");
    }
    fprintf(stderr, "%s\n", line);
  }
  hipLaunchKernelGGL(fused_gemm_ar_q8, dim3(256), dim3(NTHREADS), 0, stream, P);
  HIP_CHECK(hipGetLastError());
}
```

```cpp
#include <hip/hip_runtime.h>
#include <rccl/rccl.h>
#include <cstdio>
#include <stdint.h>

#define HIP_CHECK(expr) do { hipError_t _e = (expr); if (_e != hipSuccess) { \
    fprintf(stderr, "HIP error %s at %s:%d\n", hipGetErrorString(_e), __FILE__, __LINE__); \
    return; } } while (0)

typedef unsigned u32;
typedef unsigned long long u64;
typedef u32 u32x4 __attribute__((ext_vector_type(4)));
typedef u32 u32x2 __attribute__((ext_vector_type(2)));
typedef float f32x4 __attribute__((ext_vector_type(4)));
typedef short s16x4 __attribute__((ext_vector_type(4)));
typedef short s16x8 __attribute__((ext_vector_type(8)));
typedef __bf16 bf16x8 __attribute__((ext_vector_type(8)));
#define LAS __attribute__((address_space(3)))
#define GAS __attribute__((address_space(1)))

constexpr int NR = 8;
constexpr u32 OUT_BYTES = 4096u * 16384u;
constexpr u32 X_BYTES = 4096u * 512u * 2u;
constexpr u32 W_BYTES = 512u * 16384u * 2u;

constexpr u32 NRB = 26;
constexpr u32 NCB = 8;
constexpr u32 XBLK = 20480, WBLK = 32768;
constexpr u32 NWB = NCB + 2;
constexpr u32 WS_W = 0;
constexpr u32 WS_X = 20u << 20;
constexpr u32 WS_CTL = 53u << 20;
constexpr u32 FSTRIDE = 4352;
constexpr u32 WS_XF = WS_CTL;
constexpr u32 WS_WF = WS_XF + NRB * FSTRIDE;
constexpr u32 CNT_OFF = 512u << 10;
constexpr u32 WS_C2 = WS_CTL + (160u << 10);
constexpr u32 WS_G1 = WS_C2 + 4096;
constexpr u32 WS_G2 = WS_C2 + 8192 + 256;
constexpr u32 G2S = 512;
constexpr u32 WS_TK = WS_C2 + 13312;
constexpr u32 WS_TMO = WS_C2 + 20480;
constexpr u32 WS_DBG = WS_C2 + 32768;
constexpr u32 WS_FL = WS_C2 + 65536;
constexpr u32 WS_FC = WS_FL + 16384;
constexpr u32 WS_CALL = WS_FC + 16384;
constexpr u32 WS_Q = 54u << 20;
constexpr u32 QSTRIDE = 4160 * 256;
constexpr u32 WS_EX = 63u << 20;
constexpr u32 WS_ER = 67u << 20;
constexpr u32 WS_EW = 68u << 20;
constexpr u32 WS_L = 84u << 20;
constexpr u32 WS_END = 86u << 20;
static_assert(64u * NWB * WBLK <= WS_X && WS_X + 64u * NRB * XBLK <= WS_CTL && WS_WF + NWB * FSTRIDE <= WS_C2 &&
              WS_CALL + 16384 <= WS_CTL + CNT_OFF && WS_C2 + CNT_OFF <= WS_Q && WS_Q + NCB * QSTRIDE <= WS_EX, "layout");
constexpr u32 FULL = 240;

constexpr u32 SLOT_BYTES = WBLK + XBLK;
constexpr u32 NSLOT = 3;
constexpr u32 TPITCH = 272;
constexpr u32 SNAP_OFF = SLOT_BYTES * NSLOT;
constexpr u32 CTL_OFF = SNAP_OFF + 1024;
constexpr u32 LDS_BYTES = CTL_OFF + 1024;
static_assert(LDS_BYTES <= 163840, "lds");
constexpr int C_AMAX = 0, C_GMAX = 1, C_CALL = 2, C_BAD = 3, C_STAGE = 4, C_GS = 5, C_GS8 = 6, C_CNT = 8, C_MASK = 16;

#define NPW 4
#define EXWAIT 300
#define EXD 60
#define PW1 4
#define PLD 4
#define POLL2 1
#define QWAIT 500
#define QD 100
#define PUSHD 2
constexpr u32 NCOMMWG = 32 - NRB;
constexpr u32 NTILE = NRB * NCB;
constexpr u32 NPULL = NCOMMWG * NPW;
constexpr u32 U_A = 104, U_B = 512, U_C = 1152;
constexpr u32 U_R = 256 + 7 * 384;
constexpr u32 U_Q = U_R + 224;
constexpr int NTHREADS = 512;
constexpr int NSTEP = 64;
constexpr u64 DEADLINE = 50000000ull;

struct Params {
  const char* x;
  const char* w;
  char* out[NR];
  char* ws[NR];
  int rank;
  u32 peers;
};

__device__ __forceinline__ __amdgpu_buffer_rsrc_t mkrsrc(const void* p, u32 bytes) {
  return __builtin_amdgcn_make_buffer_rsrc((void*)p, 0, bytes, 0x00020000);
}
__device__ __forceinline__ u64 now() { return __builtin_amdgcn_s_memrealtime(); }
__device__ __forceinline__ u32 ld_sys(const u32* p) { return __hip_atomic_load((GAS const u32*)p, __ATOMIC_RELAXED, __HIP_MEMORY_SCOPE_SYSTEM); }
__device__ __forceinline__ u64 ld_sys64(const u64* p) { return __hip_atomic_load((GAS const u64*)p, __ATOMIC_RELAXED, __HIP_MEMORY_SCOPE_SYSTEM); }
__device__ __forceinline__ void st_sys(u32* p, u32 v) { __hip_atomic_store((GAS u32*)p, v, __ATOMIC_RELAXED, __HIP_MEMORY_SCOPE_SYSTEM); }
__device__ __forceinline__ void st_sys64(u64* p, u64 v) { __hip_atomic_store((GAS u64*)p, v, __ATOMIC_RELAXED, __HIP_MEMORY_SCOPE_SYSTEM); }
__device__ __forceinline__ u32 lds_ld(LAS u32* p) { return __builtin_amdgcn_readfirstlane(__hip_atomic_load(p, __ATOMIC_RELAXED, __HIP_MEMORY_SCOPE_WORKGROUP)); }
__device__ __forceinline__ void lds_st(LAS u32* p, u32 v) { __hip_atomic_store(p, v, __ATOMIC_RELAXED, __HIP_MEMORY_SCOPE_WORKGROUP); }
__device__ __forceinline__ void lds_add1(LAS u32* p, u32 lane) {
  if (lane == 0) __hip_atomic_fetch_add(p, 1u, __ATOMIC_RELAXED, __HIP_MEMORY_SCOPE_WORKGROUP);
}
#define CFENCE() asm volatile("" ::: "memory")
#define WAIT_VM(n) asm volatile("s_waitcnt vmcnt(" #n ")" ::: "memory")
#define WAIT_VMI(n) asm volatile("s_waitcnt vmcnt(%0)" :: "i"(n) : "memory")
#define WAIT_LGKM() asm volatile("s_waitcnt lgkmcnt(0)" ::: "memory")
#define WAIT_VM0() do { CFENCE(); __builtin_amdgcn_s_waitcnt(0x0F70); WAIT_VM(0); } while (0)

__device__ __forceinline__ u32 wave_umax(u32 v) {
  v = max(v, (u32)__builtin_amdgcn_update_dpp(0, (int)v, 0x111, 0xf, 0xf, true));
  v = max(v, (u32)__builtin_amdgcn_update_dpp(0, (int)v, 0x112, 0xf, 0xf, true));
  v = max(v, (u32)__builtin_amdgcn_update_dpp(0, (int)v, 0x114, 0xf, 0xf, true));
  v = max(v, (u32)__builtin_amdgcn_update_dpp(0, (int)v, 0x118, 0xf, 0xf, true));
  v = max(v, (u32)__builtin_amdgcn_update_dpp(0, (int)v, 0x142, 0xa, 0xf, false));
  v = max(v, (u32)__builtin_amdgcn_update_dpp(0, (int)v, 0x143, 0xc, 0xf, false));
  return (u32)__builtin_amdgcn_readlane((int)v, 63);
}

#define STAMP(k) do {} while (0)

__device__ __forceinline__ void wait_vm_dyn(u32 n) {
  asm volatile(
      "s_getpc_b64 vcc\n\t"
      "s_add_u32 vcc_lo, vcc_lo, %0\n\t"
      "s_addc_u32 vcc_hi, vcc_hi, 0\n\t"
      "s_setpc_b64 vcc\n\t"
      "s_waitcnt vmcnt(0)\n\ts_branch 1f\n\t"
      "s_waitcnt vmcnt(1)\n\ts_branch 1f\n\t"
      "s_waitcnt vmcnt(2)\n\ts_branch 1f\n\t"
      "s_waitcnt vmcnt(3)\n\ts_branch 1f\n\t"
      "s_waitcnt vmcnt(4)\n\ts_branch 1f\n\t"
      "s_waitcnt vmcnt(5)\n\ts_branch 1f\n\t"
      "s_waitcnt vmcnt(6)\n\ts_branch 1f\n\t"
      "s_waitcnt vmcnt(7)\n\ts_branch 1f\n\t"
      "s_waitcnt vmcnt(8)\n\ts_branch 1f\n\t"
      "s_waitcnt vmcnt(9)\n\ts_branch 1f\n\t"
      "s_waitcnt vmcnt(10)\n\ts_branch 1f\n\t"
      "s_waitcnt vmcnt(11)\n\ts_branch 1f\n\t"
      "s_waitcnt vmcnt(12)\n\ts_branch 1f\n\t"
      "s_waitcnt vmcnt(13)\n\ts_branch 1f\n\t"
      "s_waitcnt vmcnt(14)\n\ts_branch 1f\n\t"
      "s_waitcnt vmcnt(15)\n\ts_branch 1f\n\t"
      "s_waitcnt vmcnt(16)\n\ts_branch 1f\n\t"
      "s_waitcnt vmcnt(17)\n\ts_branch 1f\n\t"
      "s_waitcnt vmcnt(18)\n\ts_branch 1f\n\t"
      "s_waitcnt vmcnt(19)\n\ts_branch 1f\n\t"
      "s_waitcnt vmcnt(20)\n\ts_branch 1f\n\t"
      "s_waitcnt vmcnt(21)\n\ts_branch 1f\n\t"
      "s_waitcnt vmcnt(22)\n\ts_branch 1f\n\t"
      "s_waitcnt vmcnt(23)\n\ts_branch 1f\n\t"
      "s_waitcnt vmcnt(24)\n\t"
      "1:"
      :: "s"(min(n, 24u) * 8 + 12) : "memory", "scc", "vcc");
}
#define BARRIER() asm volatile("s_barrier" ::: "memory")

__global__ void __launch_bounds__(NTHREADS) fused_gemm_ar_q8(Params P) {
  __shared__ __attribute__((aligned(16))) char smem[LDS_BYTES];
  LAS char* L = (LAS char*)smem;
  LAS u32* ctl = (LAS u32*)(L + CTL_OFF);
  const u32 tid = threadIdx.x, lane = tid & 63;
  const u32 wave = __builtin_amdgcn_readfirstlane(tid >> 6);
  u32 xcc;
  asm volatile("s_getreg_b32 %0, hwreg(HW_REG_XCC_ID, 0, 3)" : "=s"(xcc));
  const u32 g = xcc & 7u, idx = blockIdx.x >> 3, bid = idx * 8 + g;
  const u32 rank = (u32)P.rank & 7u;
  const u32 dest = (P.peers >> (g * 4)) & 7u;
  char* const myws = P.ws[rank];
  char* const dws = P.ws[dest];
  u32* const tmo = (u32*)(myws + WS_TMO);
  const u32 lane16 = lane * 16;
  const u64 t0 = now();
  const u32 lx = (lane & 7) * 1024 + (lane >> 3) * 16;
  const u32 lw = (((lane >> 3) & 1) * 8 + ((lane >> 1) & 3)) * 32768 + ((lane & 1) * 4 + (lane >> 4)) * 16;
  u32 call = 0;
  bool bad = false;
  if (wave == 0) STAMP(0);

  if (idx >= NCOMMWG) {
    const u32 ti = idx - NCOMMWG;
    const u32 rb = (g & 1) * 13 + (ti >> 1);
    u32 cb = (g >> 1) * 2 + (ti & 1);
    const u32 tile = g * NRB + ti;
    const __amdgpu_buffer_rsrc_t rs_ws = mkrsrc(myws, WS_END);
    char* flp = P.ws[lane & 7] + WS_FL + rank * 256 + g * 8;
    const u32 lane0 = lane ? 0x80000000u : 0u;
    auto signal_wg = [&](u32 kind, u32 inc) {
      const u32 old = __builtin_amdgcn_raw_ptr_buffer_atomic_add_i32(inc, rs_ws, lane0, WS_FC + kind * 2048 + g * 256, 16);
      if (__builtin_amdgcn_readfirstlane(old) + inc == call * NRB && lane < 8)
        st_sys64((u64*)(flp + kind * 2048), call);
      CFENCE();
    };
    auto signal_all = [&](u32 kind, u32 inc = 1) {
      u32 old = 0;
      if (lane == 0) old = __hip_atomic_fetch_add(ctl + C_CNT + kind, 1u, __ATOMIC_RELAXED, __HIP_MEMORY_SCOPE_WORKGROUP);
      if (__builtin_amdgcn_readfirstlane(old) == 7) signal_wg(kind, inc);
    };
    {
      const u32 pc = g ? 2 * U_A + (tile - NRB) * 5 + wave : ti * 8 + wave;
      const bool pok = (wave < 5 || !g) && pc < 1024;
      const u32x4 e0 = __builtin_amdgcn_raw_buffer_load_b128(mkrsrc(P.x, X_BYTES), lx,
                                                             pok ? (pc & 511) * 8192 + (pc >> 9) * 128 : 0x80000000u, 0);
      CFENCE();
      if (wave == 7 && lane < 16) ctl[lane] = 0u;
      __builtin_amdgcn_raw_buffer_store_b128(e0, rs_ws, lane16, pok ? WS_EX + pc * 1024 : 0x80000000u, 17);
      asm volatile("" : "+v"(flp));
      WAIT_VM0();
      WAIT_LGKM();
      BARRIER();
      if (wave == 0) {
        const u32 old = __builtin_amdgcn_readfirstlane(__builtin_amdgcn_raw_ptr_buffer_atomic_add_i32(1, rs_ws, lane0, WS_FC + g * 256, 16));
        call = old / NRB + 1;
        if (old + 1 == call * NRB && lane < 8) {
          st_sys64((u64*)flp, ((u64)P.peers << 32) | call);
        }
        if (lane == 0) lds_st(ctl + C_CALL, call);
        WAIT_LGKM();
      }
      if (wave == 6 && ti == 0) __builtin_amdgcn_fence(__ATOMIC_ACQUIRE, "");
      BARRIER();
      call = lds_ld(ctl + C_CALL);
      for (u32 r = 0; r < 3; ++r) {
        const u32 slot = r * 8 + wave, e = slot * NTILE + tile;
        u32 kc, k, so, eo;
        bool isw = true;
        if (e < 4352) {
          kc = (e + 768) / 640; k = e + 768 - kc * 640;
          if (kc == 1) k -= 128;
          isw = k < 512;
          k &= 511;
          eo = isw ? WS_EW + ((k >> 6) * 8 + kc) * (8 * WBLK) + (k & 63) * 4096 : WS_EX + kc * (512u << 10) + k * 4096;
        } else {
          kc = (e - 4352) >> 4; k = ((e >> 1) & 7) * 64 + 62 + (e & 1);
          eo = WS_ER + (call & 1) * (512u << 10) + (e - 4352) * 4096;
        }
        if (e < 4480) {
          so = isw ? (kc * 64 + (k & 1) * 32) * 32768 + (k >> 1) * 128 : k * 32768 + kc * 128;
          const __amdgpu_buffer_rsrc_t rs = mkrsrc(isw ? P.w : P.x, isw ? W_BYTES : X_BYTES);
          const u32 lo = isw ? lw : lx, s1 = isw ? 4 * 32768 : 8192, s2 = isw ? 16 * 32768 : 16384;
          while (now() - t0 < EXWAIT + slot * EXD) __builtin_amdgcn_s_sleep(2);
          const u32x4 e0 = __builtin_amdgcn_raw_buffer_load_b128(rs, lo, so, 0);
          const u32x4 e1 = __builtin_amdgcn_raw_buffer_load_b128(rs, lo, so + s1, 0);
          const u32x4 e2 = __builtin_amdgcn_raw_buffer_load_b128(rs, lo, so + s2, 0);
          const u32x4 e3 = __builtin_amdgcn_raw_buffer_load_b128(rs, lo, so + s1 + s2, 0);
          __builtin_amdgcn_raw_buffer_store_b128(e0, rs_ws, lane16, eo, 17);
          __builtin_amdgcn_raw_buffer_store_b128(e1, rs_ws, lane16, eo + 1024, 17);
          __builtin_amdgcn_raw_buffer_store_b128(e2, rs_ws, lane16, eo + 2048, 17);
          __builtin_amdgcn_raw_buffer_store_b128(e3, rs_ws, lane16, eo + 3072, 17);
        }
        if (r != 1) {
          WAIT_VM0();
          signal_all(1 + (r >> 1));
        }
      }
      BARRIER();
    }
    const u32 cv = call * FULL;
    const u32 wm = wave >> 2, wn = wave & 3;
    const u32 lds0 = (u32)(size_t)L;
    const u32 xa = WBLK + wm * 10240 + ((lane >> 3) & 1) * 1024 + (lane >> 5) * 256 + ((lane >> 4) & 1) * 128 + (lane & 7) * 16;
    const u32 h = lane >> 4, q = (lane >> 2) & 3, p = lane & 3;
    const u32 wa0 = wn * 8192 + (h >> 1) * 2048 + (p & 1) * 512 + (h & 1) * 128 + q * 32 + (p >> 1) * 16 + (p & 1) * 8;
    const u32 wa1 = wa0 ^ 8;
    const u32 xpo = wave < 4 ? wave * 3072 : 12288 + (wave - 4) * 2048;
    u32 pollo;
    f32x4 acc[4][5];
    u64 kx = 0, kw;
    u64 ready, taken;
    u32 issued, consumed, nis = 0;
    u32 islot, cslot, par = 0;
    u64 markD = 0;
    float inv = 0.f;
    auto sched = [&]() {
      u64 avail = ready & ~taken;
      while (avail && issued - consumed < NSLOT) {
        const u32 ord = __builtin_ctzll(avail);
        avail &= avail - 1;
        taken |= 1ull << ord;
        const u32 sw = WS_W + (ord * NWB + cb) * WBLK + wave * 4096;
        const u32 sx = WS_X + (ord * NRB + rb) * XBLK + xpo;
        const u32 mw = lds0 + islot + wave * 4096, mx = lds0 + islot + WBLK + xpo;
        u32 keep;
        asm volatile(
            "s_mov_b32 %0, m0\n\t"
            "s_mov_b32 m0, %3\n\t"
            "s_nop 0\n\t"
            "buffer_load_dwordx4 %1, %2, %5 offen lds\n\t"
            "buffer_load_dwordx4 %1, %2, %5 offen offset:1024 lds\n\t"
            "buffer_load_dwordx4 %1, %2, %5 offen offset:2048 lds\n\t"
            "buffer_load_dwordx4 %1, %2, %5 offen offset:3072 lds\n\t"
            "s_mov_b32 m0, %4\n\t"
            "s_nop 0\n\t"
            "buffer_load_dwordx4 %1, %2, %6 offen lds\n\t"
            "buffer_load_dwordx4 %1, %2, %6 offen offset:1024 lds\n\t"
            "s_cmp_gt_u32 %7, 3\n\t"
            "s_cbranch_scc1 1f\n\t"
            "buffer_load_dwordx4 %1, %2, %6 offen offset:2048 lds\n\t"
            "1:\n\t"
            "s_mov_b32 m0, %0"
            : "=&s"(keep)
            : "v"(lane16), "s"(rs_ws), "s"(mw), "s"(mx), "s"(sw), "s"(sx), "s"(wave)
            : "memory", "scc");
        nis += wave < 4 ? 7 : 6;
        const u32 sh = (issued & 3) * 16;
        markD = (markD & ~(0xffffull << sh)) | ((u64)(nis & 0xffff) << sh);
        issued += 1;
        islot = islot == 2 * SLOT_BYTES ? 0 : islot + SLOT_BYTES;
      }
    };
    auto poll = [&]() {
      if (wave == 0 && (kx & kw) != ~0ull) {
        const u64 kn = lane < 16 ? kx : kw;
        const bool act = lane < 32 && ((u32)(kn >> ((lane & 15) * 4)) & 15u) != 15u;
        const u32 po = act ? pollo : 0x80000000u;
        u32 keep;
        asm volatile("s_mov_b32 %0, m0\n\ts_mov_b32 m0, %3\n\ts_nop 0\n\t"
                     "buffer_load_dwordx4 %1, %2, 0 offen sc0 sc1 lds\n\ts_mov_b32 m0, %0"
                     : "=&s"(keep) : "v"(po), "s"(rs_ws), "s"(lds0 + SNAP_OFF) : "memory");
        nis += 1;
      }
    };
    LAS u32* const snap = (LAS u32*)(L + SNAP_OFF) + lane;
    auto seen = [&](u32 fx, u32 fw) {
      kx |= __ballot((int)(fx - cv) >= 0);
      kw |= __ballot((int)(fw - cv) >= 0);
    };
    auto tell = [&]() { if (lane == 0) *(LAS u64*)(L + CTL_OFF + C_MASK * 4 + par) = kx & kw; };
    auto told = [&]() -> u64 {
      const u64 m = *(LAS u64*)(L + CTL_OFF + C_MASK * 4 + par);
      par ^= 8;
      return m;
    };
    auto uni = [&](u64 m) -> u64 {
      return (u64)(u32)__builtin_amdgcn_readfirstlane((u32)m) | ((u64)(u32)__builtin_amdgcn_readfirstlane((u32)(m >> 32)) << 32);
    };
    auto wait_step = [&]() { wait_vm_dyn((nis - (u32)(markD >> ((consumed & 3) * 16))) & 0xffff); };
#define SB() __builtin_amdgcn_sched_barrier(0)
#define PIECE(M, S, OFF) do { \
        asm volatile("s_cmp_lg_u32 %2, -1\n\ts_cbranch_scc0 1f\n\ts_mov_b32 m0, %2\n\ts_nop 0\n\t" \
                     "buffer_load_dwordx4 %0, %1, %3 offen offset:" #OFF " lds\n\t1:" \
                     :: "v"(lane16), "s"(rs_ws), "s"(M), "s"(S) : "memory", "scc"); \
        SB(); } while (0)
#define MMA4(kb, i) do { \
        _Pragma("unroll") for (int n = 0; n < 4; ++n) \
          acc[n][i] = __builtin_amdgcn_mfma_f32_16x16x32_bf16(wf[kb][n], xf[kb][i], acc[n][i], 0, 0, 0); \
        SB(); } while (0)
    bf16x8 xf[2][5], wf[2][4];
    u32 sp = 0;
    u64 cyc0 = 0;
    for (u32 pass = 0;; ++pass) {
    pollo = lane < 16 ? WS_XF + rb * FSTRIDE + lane16 : WS_WF + cb * FSTRIDE + lane16 - 256;
    kw = taken = 0;
    issued = consumed = islot = cslot = 0;
#pragma unroll
    for (int a = 0; a < 4; ++a)
#pragma unroll
      for (int b = 0; b < 5; ++b) acc[a][b] = f32x4{0.f, 0.f, 0.f, 0.f};
#pragma unroll
    for (int n = 0; n < 5; ++n) {
      xf[1][n] = __builtin_bit_cast(bf16x8, u32x4{0u, 0u, 0u, 0u});
      if (n < 4) wf[1][n] = __builtin_bit_cast(bf16x8, u32x4{0u, 0u, 0u, 0u});
    }
    poll();
    while (consumed < NSTEP) {
      if (issued == consumed) {
        if (wave == 0) {
          while ((kx & kw & ~taken) == 0) {
            poll();
            WAIT_VM(1);
            seen(snap[0], snap[64]);
            if ((++sp & 63) == 0 && now() - t0 > DEADLINE) {
              st_sys(tmo, call);
              WAIT_VM(0);
              lds_st(ctl + C_BAD, 1u);
              kx = kw = ~0ull;
            }
          }
          tell();
          WAIT_LGKM();
        }
        BARRIER();
        ready = uni(told());
        sched();
        poll();
        wait_step();
        if (wave == 0) {
          seen(snap[0], snap[64]);
          tell();
          WAIT_LGKM();
        }
      }
      BARRIER();
      const u64 rdy = told();
      SB();
      LAS char* base = L + cslot;
      auto frags = [&](int kb) {
#pragma unroll
        for (int n = 0; n < 5; ++n) {
          if (n < 4) {
            LAS char* a = base + ((n & 1) ? wa1 : wa0) + kb * 4096 + (n >> 1) * 256;
            s16x4 lo = __builtin_amdgcn_ds_read_tr16_b64_v4i16((LAS s16x4*)a);
            s16x4 hi = __builtin_amdgcn_ds_read_tr16_b64_v4i16((LAS s16x4*)(a + 1024));
            wf[kb][n] = __builtin_bit_cast(bf16x8, __builtin_shufflevector(lo, hi, 0, 1, 2, 3, 4, 5, 6, 7));
          }
          xf[kb][n] = *(LAS bf16x8*)(base + xa + n * 2048 + kb * 512);
        }
        SB();
      };
      MMA4(1, 0);
      frags(0);
      poll();
      SB();
      MMA4(1, 1);
      ready = uni(rdy);
      const u64 av = ready & ~taken;
      const bool go = av != 0 && issued - consumed < NSLOT;
      const u32 ord = __builtin_ctzll(av | (1ull << 63));
      const u32 sw = WS_W + (ord * NWB + cb) * WBLK + wave * 4096;
      const u32 sx = WS_X + (ord * NRB + rb) * XBLK + xpo;
      u32 mw = ~0u, mx = ~0u, mx3 = ~0u;
      if (go) {
        mw = lds0 + islot + wave * 4096;
        mx = lds0 + islot + WBLK + xpo;
        if (wave < 4) mx3 = mx;
        taken |= 1ull << ord;
        nis += wave < 4 ? 7 : 6;
        const u32 sh = (issued & 3) * 16;
        markD = (markD & ~(0xffffull << sh)) | ((u64)(nis & 0xffff) << sh);
        issued += 1;
        islot = islot == 2 * SLOT_BYTES ? 0 : islot + SLOT_BYTES;
      }
      SB();
      MMA4(1, 2);
      PIECE(mw, sw, 0);
      MMA4(1, 3);
      PIECE(mw, sw, 1024);
      MMA4(1, 4);
      frags(1);
      PIECE(mw, sw, 2048);
      MMA4(0, 0);
      PIECE(mw, sw, 3072);
      PIECE(mx, sx, 0);
      MMA4(0, 1);
      PIECE(mx, sx, 1024);
      PIECE(mx3, sx, 2048);
      MMA4(0, 2);
      sched();
      poll();
      cslot = cslot == 2 * SLOT_BYTES ? 0 : cslot + SLOT_BYTES;
      consumed += 1;
      SB();
      MMA4(0, 3);
      if (issued != consumed) wait_step();
      u32 fx = cv - 1, fw = cv - 1;
      if (wave == 0) { fx = snap[0]; fw = snap[64]; }
      SB();
      MMA4(0, 4);
      if (wave == 0) { seen(fx, fw); tell(); }
      WAIT_LGKM();
    }
    MMA4(1, 0); MMA4(1, 1); MMA4(1, 2); MMA4(1, 3); MMA4(1, 4);
    if (pass == 0) {
      if (wave == 0) STAMP(2);
      const u32 row0 = rb * 160 + wm * 80;
      float m = 0.f;
#pragma unroll
      for (int b = 0; b < 5; ++b) {
        float mb = 0.f;
#pragma unroll
        for (int a = 0; a < 4; ++a)
#pragma unroll
          for (int e = 0; e < 4; ++e) mb = fmaxf(mb, fabsf(acc[a][b][e]));
        if (row0 + b * 16 < 4096) m = fmaxf(m, mb);
      }
      const u32 mb = wave_umax(__builtin_bit_cast(u32, m));
      if (lane == 0) __hip_atomic_fetch_max(ctl + C_AMAX, mb, __ATOMIC_RELAXED, __HIP_MEMORY_SCOPE_WORKGROUP);
      WAIT_LGKM();
      BARRIER();
      if (wave == 0) {
        const u32 wmax = lds_ld(ctl + C_AMAX);
        if (lane == 0) st_sys64((u64*)(myws + WS_G1) + tile, ((u64)call << 32) | wmax);
        STAMP(3);
        const u64* g2 = (const u64*)(myws + WS_G2 + g * G2S) + (lane & 7);
        u64 val;
        for (;;) {
          val = ld_sys64(g2);
          if (__all((u32)(val >> 32) == call)) break;
          if ((++sp & 63) == 0 && (ld_sys(tmo) == call || now() - t0 > DEADLINE)) { st_sys(tmo, call); bad = true; break; }
        }
        const u32 gmax = wave_umax((u32)val);
        if (lane == 0) {
          lds_st(ctl + C_GMAX, gmax);
          if (bad) lds_st(ctl + C_BAD, 1u);
        }
        WAIT_LGKM();
      }
    }
    BARRIER();
    if (pass == 0) {
      const float amax = __builtin_bit_cast(float, lds_ld(ctl + C_GMAX));
      const float scale = amax / 127.0f;
      inv = amax > 0.f ? 1.0f / scale : 0.f;
      if (wave == 0) STAMP(4);
    }
    bad = lds_ld(ctl + C_BAD) != 0;
    const bool odd = (lane >> 4) & 1;
    u32x4 qv[5];
#pragma unroll
    for (int i = 0; i < 5; ++i) {
      u32 d[4];
#pragma unroll
      for (int n = 0; n < 4; ++n) {
        u32 b0 = __builtin_bit_cast(u32, __builtin_fmaf(acc[n][i][0], inv, 12582912.0f));
        u32 b1 = __builtin_bit_cast(u32, __builtin_fmaf(acc[n][i][1], inv, 12582912.0f));
        u32 b2 = __builtin_bit_cast(u32, __builtin_fmaf(acc[n][i][2], inv, 12582912.0f));
        u32 b3 = __builtin_bit_cast(u32, __builtin_fmaf(acc[n][i][3], inv, 12582912.0f));
        u32 lo = __builtin_amdgcn_perm(b1, b0, 0x0c0c0400u);
        u32 hi = __builtin_amdgcn_perm(b3, b2, 0x04000c0cu);
        d[n] = bad ? 0x80808080u : (lo | hi);
      }
      qv[i] = u32x4{odd ? d[1] : d[0], odd ? d[0] : d[1], odd ? d[3] : d[2], odd ? d[2] : d[3]};
    }
    const u64 tq = now();
    if (!pass && !(cb & 1) && wm == 0) {
      const __amdgpu_buffer_rsrc_t rs_p = mkrsrc(dws, WS_END);
#pragma unroll
      for (int i = 0; i < 4; ++i) {
        if (i && i % PUSHD == 0) WAIT_VM(0);
        if (wave == 0 && i == PUSHD) STAMP(10);
        __builtin_amdgcn_raw_buffer_store_b128(qv[i], rs_p, lane16, WS_L + ((rank * 13 + (ti >> 1)) * 16 + wn * 4 + i) * 1024, 17);
        CFENCE();
      }
      WAIT_VM0();
      if (wave == 0) STAMP(11);
      const u32 old = __builtin_amdgcn_raw_ptr_buffer_atomic_add_i32(1, rs_ws, lane0, WS_FC + 5 * 2048 + g * 256, 16);
      if (__builtin_amdgcn_readfirstlane(old) + 1 == call * 52 && lane == 0) st_sys64((u64*)(dws + WS_FL + 5 * 2048) + rank, ((u64)g << 32) | call);
      CFENCE();
    }
    {
      LAS char* tw = L + (wm * 80 + (lane & 15)) * TPITCH + wn * 64 + (lane >> 4) * 16;
#pragma unroll
      for (int i = 0; i < 5; ++i) *(LAS u32x4*)(tw + i * 16 * TPITCH) = qv[i];
    }
    WAIT_LGKM();
    BARRIER();
    LAS char* tr = L + (wave * 4 + (lane >> 4)) * TPITCH + (lane & 15) * 16;
#pragma unroll
    for (int i = 0; i < 5; ++i) qv[i] = *(LAS u32x4*)(tr + i * 32 * TPITCH);
    const __amdgpu_buffer_rsrc_t rs_o = mkrsrc(P.out[rank], OUT_BYTES);
    const u32 orow = (rb * 160 + wave * 4 + (lane >> 4)) * 16384u;
    if (pass) {
      const u32 piece = (cb - 8) * 4 + ((lane >> 2) & 3);
      const u32 oo = piece == rank ? 0x80000000u : orow + piece * 2048 + 1984 + (lane & 3) * 16;
#pragma unroll
      for (int i = 0; i < 5; ++i) __builtin_amdgcn_raw_buffer_store_b128(qv[i], rs_o, oo, i * 32 * 16384, 17);
      break;
    }
    const u32 pitch = cb == 7 ? 192 : 256;
    const bool rep = (lane & 15) * 16 >= pitch;
    const u32 qo = rep ? 0x80000000u : (wave * 4 + (lane >> 4)) * pitch + (lane & 15) * 16;
    const u32 oo = orow + rank * 2048 + cb * 256 + (lane & 15) * 16;
    const u32 qa = WS_Q + cb * QSTRIDE + rb * 64 * pitch, qb = WS_Q + cb * QSTRIDE + (1664 + rb * 96) * pitch;
    if (cb & 1) {
      __builtin_amdgcn_raw_buffer_store_b128(qv[0], rs_ws, qo, qa, 17);
      __builtin_amdgcn_raw_buffer_store_b128(qv[1], rs_ws, qo, qa + 32 * pitch, 17);
      WAIT_VM0();
      signal_all(3, 2);
      if (wave == 0) STAMP(9);
    }
    while (now() - tq < QWAIT + wave * QD) __builtin_amdgcn_s_sleep(2);
    if (!(cb & 1)) {
      __builtin_amdgcn_raw_buffer_store_b128(qv[0], rs_ws, qo, qa, 17);
      __builtin_amdgcn_raw_buffer_store_b128(qv[1], rs_ws, qo, qa + 32 * pitch, 17);
    }
#pragma unroll
    for (int i = 2; i < 5; ++i) __builtin_amdgcn_raw_buffer_store_b128(qv[i], rs_ws, qo, qb + (i - 2) * 32 * pitch, 17);
    WAIT_VM0();
    signal_all(4);
    if (cb == 7) {
#pragma unroll
      for (int i = 0; i < 5; ++i) __builtin_amdgcn_raw_buffer_store_b128(qv[i], rs_o, rep ? oo : 0x80000000u, i * 32 * 16384, 17);
    }
    if (((ti >> 1) & 3) != (g >> 1)) break;
    cb = 8 + (ti & 1);
    WAIT_VM0();
    WAIT_LGKM();
    BARRIER();
    }
    WAIT_VM(0);
    if (wave == 0) STAMP(5);
    return;
  }

  const u32 vid = idx * NPW + wave;
  const __amdgpu_buffer_rsrc_t rs_s = mkrsrc(dws, WS_END);
  const u32 lane0 = lane ? 0x80000000u : 0u;
  static_assert(NPW == 4 && NCOMMWG == 6, "wave roles");
  if (wave >= NPW && wave < 7) {
    const u32 ph = (wave - NPW) * NCOMMWG + idx;
    const __amdgpu_buffer_rsrc_t rs_w = mkrsrc(P.w, W_BYTES);
    const u32 so = ph < 16 ? dest * 4096 + ph * 256 : 0x80000000u;
    const u32 to = ph < 16 ? WS_W + rank * NWB * WBLK + ph * 16384 : 0x80000000u;
    u32x4 pv[16];
#pragma unroll
    for (int i = 0; i < 16; ++i) {
      pv[i] = __builtin_amdgcn_raw_buffer_load_b128(rs_w, lw, so + (i >> 3) * 128 + (((i >> 1) & 3) * 16 + (i & 1) * 4) * 32768, 0);
      if (i == PLD - 1) BARRIER();
    }
    CFENCE();
#pragma unroll
    for (int i = 0; i < 16; ++i) {
      if (i >= PW1) WAIT_VMI(PW1 - 1);
      __builtin_amdgcn_raw_buffer_store_b128(pv[i], rs_s, lane16, to + i * 1024, 17);
      CFENCE();
    }
    WAIT_VM(0);
    if (wave == NPW) STAMP(10);
    if (ph < 16) __builtin_amdgcn_raw_ptr_buffer_atomic_add_i32(FULL / 2, rs_s, lane0, WS_WF + (ph >> 1) * FSTRIDE + rank * 4, 16);
    return;
  }
  if (wave == 7) {
    ctl[lane] = 0u;
    WAIT_LGKM();
  }
  BARRIER();
  if (wave == 7) {
    u32* const cptr = (u32*)(myws + WS_CALL + bid * 256);
    call = __builtin_amdgcn_readfirstlane(ld_sys(cptr)) + 1;
    if (lane == 0) { lds_st(ctl + C_CALL, call); st_sys(cptr, call); }
    u32 sp = 0;
    auto expired = [&]() -> bool {
      if ((++sp & 63) == 0 && (ld_sys(tmo) == call || now() - t0 > DEADLINE)) {
        st_sys(tmo, call);
        if (lane == 0) { lds_st(ctl + C_BAD, 1u); lds_st(ctl + C_STAGE, 7u); }
        bad = true;
      }
      return bad;
    };
    auto watch = [&](u32 kind, u32 row) {
      const u64* fp = (const u64*)(myws + WS_FL + kind * 2048 + row * 256) + (lane & 7);
      while (!bad) {
        const u64 f = ld_sys64(fp);
        const u64 ok = __ballot((int)((u32)f - call) >= 0);
        if (ok == ~0ull) {
          const u32 hi = (u32)(f >> 32), sh = (lane & 7) * 4;
          if (kind == 0 && lane < 8 && ((hi >> sh) & 7u) == rank) lds_st(ctl + C_GS, lane);
          if (kind == 5 && lane < 8) __hip_atomic_fetch_or(ctl + C_GS8, (hi & 7u) << sh, __ATOMIC_RELAXED, __HIP_MEMORY_SCOPE_WORKGROUP);
          break;
        }
        if (kind == 0 && (ok & 1) && lane == 0) lds_st(ctl + C_STAGE, 1u);
        __builtin_amdgcn_s_sleep(1);
        expired();
      }
      if (lane == 0 && !bad) lds_st(ctl + C_STAGE, kind + 2);
      CFENCE();
    };
    watch(0, dest);
    STAMP(11);
    watch(1, dest);
    watch(2, dest);
    if (bid == 0) {
      const u64* g1 = (const u64*)(myws + WS_G1);
      u64* g2 = (u64*)(P.ws[lane & 7] + WS_G2 + (lane >> 3) * G2S) + rank;
      asm volatile("" : "+v"(g2));
      u32 gm = 0;
      while (!bad) {
        u64 a0 = ld_sys64(g1 + lane), a1 = ld_sys64(g1 + 64 + lane), a2 = ld_sys64(g1 + 128 + lane);
        u64 a3 = ld_sys64(g1 + 192 + (lane & 15));
        const bool ok = (u32)(a0 >> 32) == call && (u32)(a1 >> 32) == call && (u32)(a2 >> 32) == call && (u32)(a3 >> 32) == call;
        gm = max(max((u32)a0, (u32)a1), max((u32)a2, (u32)a3));
        const u64 okm = __ballot(ok);
        if (okm == ~0ull) break;
        if (okm == 0) __builtin_amdgcn_s_sleep(8);
        expired();
      }
      gm = wave_umax(gm);
      if (!bad) st_sys64(g2, ((u64)call << 32) | gm);
    }
    watch(3, dest);
    STAMP(9);
    watch(4, dest);
    if (g == 0) watch(5, 0);
    return;
  }
  u32 stage = 0;
  auto need = [&](u32 s) {
    while (stage < s) {
      stage = lds_ld(ctl + C_STAGE);
      if (stage < s) __builtin_amdgcn_s_sleep(1);
    }
    CFENCE();
  };
  need(1);
  if (wave == 0) STAMP(3);
  call = lds_ld(ctl + C_CALL);
  const __amdgpu_buffer_rsrc_t rs_m = mkrsrc(myws, WS_Q);
  const __amdgpu_buffer_rsrc_t rs_o = mkrsrc(P.out[rank], OUT_BYTES);
  const u32 rot = (g * 26 / 7 % 26) * 10;
  constexpr u32 U_L = U_Q + 3968 - 104, uq2 = U_Q + 3 * 208 + 156;
  const u32 nu = g ? U_L : U_L + 832;
  struct Unit { u32 so, to, fo, fw; };
  u32 qc = 0, qbase = U_Q;
  u32 gs = 8;
  auto uoff = [&](u32 u) -> Unit {
    Unit d;
    if (u < U_R) {
      const u32 kc = (u + 128) / 384, ord = kc * 8 + dest;
      u32 k = u + 128 - kc * 384;
      if (kc == 1) k = k < 256 ? k + 128 : k - 256;
      if (k < 128) {
        d.so = WS_EW + (rank * 8 + kc) * (8 * WBLK) + k * 2048;
        d.to = WS_W + ord * NWB * WBLK + k * 2048;
        d.fo = WS_WF + (k >> 4) * FSTRIDE + ord * 4;
        d.fw = 15;
      } else {
        const u32 kk = (k - 128 + (kc ? rot : 0)) & 255, r = kk / 10;
        d.so = WS_EX + kc * (512u << 10) + kk * 2048;
        d.to = WS_X + ord * NRB * XBLK + kk * 2048;
        d.fo = WS_XF + r * FSTRIDE + ord * 4;
        d.fw = r == NRB - 1 ? 40 : 24;
      }
    } else if (u < U_Q) {
      const u32 k = u - U_R, kc = k / 28, j = k - kc * 28, ord = kc * 8 + dest;
      const u32 jj = j + ((j >> 2) >= rank ? 4 : 0);
      d.so = WS_ER + (call & 1) * (512u << 10) + kc * 65536 + jj * 2048;
      d.to = WS_W + (ord * NWB + 8) * WBLK + jj * 2048;
      d.fo = WS_WF + (8 + (jj >> 4)) * FSTRIDE + ord * 4;
      d.fw = (jj >> 4) == (rank >> 2) ? 20 : 15;
    } else if (u < U_L) {
      u32 c, na, n;
      if (gs == 8) {
        need(2);
        gs = lds_ld(ctl + C_GS);
      }
      for (;;) {
        c = (qc < 8 ? (gs & 6) + 2 + qc * 2 + (qc < 4) : gs + qc) & 7;
        na = c == 7 ? 156 : 208;
        n = qc < 7 ? na : qc < 8 ? 104 : c == 7 ? 228 : 304;
        if (u - qbase < n) break;
        qbase += n;
        qc += 1;
      }
      const u32 k = u - qbase + (qc >= 8 ? na : qc == 7 && !(gs & 1) ? 104 : 0);
      d.so = WS_Q + c * QSTRIDE + k * 2048;
      d.to = k * 128;
      d.fo = c;
      d.fw = 0;
    } else {
      const u32 k = u - U_L, src = k / 104, r = k - src * 104;
      d.so = u < nu ? WS_L + k * 2048 : 0x80000000u;
      d.to = ((r >> 3) * 160 + (r & 1) * 32) * 16384 + src * 2048 + (r & 6) * 32;
      d.fo = src & 7;
      d.fw = 0;
    }
    return d;
  };
  u32x4 v0, v1;
  u32 u0 = vid, u1 = NPULL + vid, tk = vid + (call - 1) * nu - NPULL;
  Unit d0 = uoff(u0), d1 = uoff(u1);
  v0 = __builtin_amdgcn_raw_buffer_load_b128(rs_s, lane16, d0.so, 17);
  v1 = __builtin_amdgcn_raw_buffer_load_b128(rs_s, lane16, d0.so + 1024, 17);
  u32 to0 = lane16, to1 = lane16 + 1024, tso = d0.to, lo1 = lane16;
  __amdgpu_buffer_rsrc_t rs_t = rs_m;
  const u32 cv = call * FULL;
  u32 cnt = 0, pfo = 0, pfw = 0;
  auto publish = [&]() {
    if (pfw && __builtin_amdgcn_readfirstlane(cnt) + pfw == cv)
      asm volatile("s_nop 4\n\tbuffer_store_dword %0, %1, %2, %3 offen sc0 sc1" :: "v"(cv), "v"(lane0), "s"(rs_m), "s"(pfo) : "memory");
    pfw = 0;
  };
  WAIT_VM0();
  while (u0 < nu) {
    if (bad && !d0.fw) v0 = v1 = u32x4{0x80808080u, 0x80808080u, 0x80808080u, 0x80808080u};
    __builtin_amdgcn_raw_buffer_store_b128(v0, rs_t, to0, tso, 17);
    __builtin_amdgcn_raw_buffer_store_b128(v1, rs_t, to1, tso, 17);
    CFENCE();
    const u32 ns = u1 >= nu ? 0 : u1 >= U_L ? 7 : u1 >= uq2 ? 6 : u1 >= U_Q ? 5 : u1 >= U_C ? 4 : u1 >= U_B ? 3 : u1 >= U_A ? 2 : 1;
    if (stage < ns) {
      stage = lds_ld(ctl + C_STAGE);
      if (stage < ns) {
        publish();
        WAIT_VM0();
        if (d0.fw) {
          cnt = __builtin_amdgcn_raw_ptr_buffer_atomic_add_i32(d0.fw, rs_m, lane0, d0.fo + CNT_OFF, 16);
          pfo = d0.fo; pfw = d0.fw;
          publish();
        }
        d0.fw = 0;
        need(ns);
      }
      bad = lds_ld(ctl + C_BAD) != 0;
    }
    v0 = __builtin_amdgcn_raw_buffer_load_b128(rs_s, lo1, d1.so, 17);
    v1 = __builtin_amdgcn_raw_buffer_load_b128(rs_s, lo1, d1.so + 1024, 17);
    CFENCE();
    const u32 u2 = 3 * NPULL + (__builtin_amdgcn_readfirstlane(tk) - (call - 1) * nu);
    tk = __builtin_amdgcn_raw_ptr_buffer_atomic_add_i32(1, rs_m, lane0, WS_TK + g * 128, 16);
    CFENCE();
    publish();
    const Unit d2 = uoff(u2);
    tso = d1.to;
    if (u2 >= U_L) lo1 = (lane & 3) * 256 + (lane >> 2) * 16;
    if (u1 >= U_L) {
      const u32 gl = lds_ld(ctl + C_GS8) >> (d1.fo * 4);
      tso += (gl & 1) * (2080 * 16384) + (gl & 6) * 256;
      rs_t = rs_o;
      to0 = (lane >> 2) * 16384 + (lane & 3) * 16;
      to1 = to0 + 16 * 16384;
    } else if (!d1.fw) {
      rs_t = rs_o;
      tso = dest * 2048 + d1.fo * 256;
      const u32 mul = d1.fo == 7 ? 43691u : 32768u, w16 = d1.fo == 7 ? 12 : 16;
#pragma unroll
      for (int e = 0; e < 2; ++e) {
        const u32 i16 = d1.to + e * 64 + lane;
        const u32 sr = (i16 * mul) >> 19, col = i16 - sr * w16;
        const u32 ra = (sr >> 6) * 96 + sr, rc = (((sr - 1664) * 43691u) >> 22) * 64 + sr - 1600;
        (e ? to1 : to0) = (sr < 1664 ? ra : rc) * 16384 + col * 16;
      }
    }
    CFENCE();
    WAIT_VM(3);
    cnt = __builtin_amdgcn_raw_ptr_buffer_atomic_add_i32(d0.fw, rs_m, lane0, d0.fw ? d0.fo + CNT_OFF : 0x40000000u, 16);
    pfo = d0.fo; pfw = d0.fw;
    CFENCE();
    u0 = u1; u1 = u2;
    d0 = d1; d1 = d2;
  }
  publish();
  WAIT_VM(0);
  if (wave == 0) STAMP(5);
}

extern "C" void kernel_launch(int rank, int nranks, ncclComm_t comm, hipStream_t stream,
                              void* const* d_in, const int* in_sizes, int n_in,
                              void* d_out, int out_size,
                              const int* in_dtypes, int out_dtype,
                              void* const* d_out_all,
                              void* const* d_ws_all, size_t ws_bytes) {
  if (nranks != NR || ws_bytes < WS_END || n_in < 2) { fprintf(stderr, "unsupported configuration\n"); return; }
  Params P;
  P.x = (const char*)d_in[0];
  P.w = (const char*)d_in[1];
  for (int p = 0; p < NR; ++p) { P.out[p] = (char*)d_out_all[p]; P.ws[p] = (char*)d_ws_all[p]; }
  P.rank = rank;
  int logi[NR] = {0, 4, 1, 2, 5, 6, 3, 7}, phys[NR], grp[NR], ndev = 0, sum = 0;
  bool known = hipGetDeviceCount(&ndev) == hipSuccess && ndev == NR;
  for (int d = 0; d < NR && known; ++d) {
    char bus[32] = {0}, path[96];
    int pos = -1;
    known = hipDeviceGetPCIBusId(bus, sizeof bus, d) == hipSuccess;
    snprintf(path, sizeof path, "/sys/bus/pci/devices/%s/xgmi_physical_id", bus);
    FILE* f = known ? fopen(path, "r") : nullptr;
    known = f && fscanf(f, "%d", &pos) == 1 && pos >= 0;
    if (f) fclose(f);
    grp[d] = (pos ^ (pos >> 1)) & 1;
    sum += grp[d];
  }
  (void)hipGetLastError();
  if (known && sum == NR / 2)
    for (int d = 0, n[2] = {0, NR / 2}; d < NR; ++d) logi[d] = n[grp[d]]++;
  for (int d = 0; d < NR; ++d) phys[logi[d]] = d;
  P.peers = 0;
  for (int g = 0; g < NR; ++g) P.peers |= (u32)phys[logi[rank & 7] ^ g] << (g * 4);
  hipLaunchKernelGGL(fused_gemm_ar_q8, dim3(256), dim3(NTHREADS), 0, stream, P);
  HIP_CHECK(hipGetLastError());
}
```

```cpp
#include <hip/hip_runtime.h>
#include <rccl/rccl.h>
#include <cstdio>
#include <stdint.h>

#define HIP_CHECK(expr) do { hipError_t _e = (expr); if (_e != hipSuccess) { \
    fprintf(stderr, "HIP error %s at %s:%d\n", hipGetErrorString(_e), __FILE__, __LINE__); \
    return; } } while (0)

typedef unsigned u32;
typedef unsigned long long u64;
typedef u32 u32x4 __attribute__((ext_vector_type(4)));
typedef u32 u32x2 __attribute__((ext_vector_type(2)));
typedef float f32x4 __attribute__((ext_vector_type(4)));
typedef short s16x4 __attribute__((ext_vector_type(4)));
typedef short s16x8 __attribute__((ext_vector_type(8)));
typedef __bf16 bf16x8 __attribute__((ext_vector_type(8)));
#define LAS __attribute__((address_space(3)))
#define GAS __attribute__((address_space(1)))

constexpr int NR = 8;
constexpr u32 OUT_BYTES = 4096u * 16384u;
constexpr u32 X_BYTES = 4096u * 512u * 2u;
constexpr u32 W_BYTES = 512u * 16384u * 2u;

constexpr u32 NRB = 26;
constexpr u32 NCB = 8;
constexpr u32 XBLK = 20480, WBLK = 32768;
constexpr u32 NWB = NCB + 2;
constexpr u32 WS_W = 0;
constexpr u32 WS_X = 20u << 20;
constexpr u32 WS_CTL = 53u << 20;
constexpr u32 FSTRIDE = 4352;
constexpr u32 WS_XF = WS_CTL;
constexpr u32 WS_WF = WS_XF + NRB * FSTRIDE;
constexpr u32 CNT_OFF = 512u << 10;
constexpr u32 WS_C2 = WS_CTL + (160u << 10);
constexpr u32 WS_G1 = WS_C2 + 4096;
constexpr u32 WS_G2 = WS_C2 + 8192 + 256;
constexpr u32 G2S = 512;
constexpr u32 WS_TK = WS_C2 + 13312;
constexpr u32 WS_TMO = WS_C2 + 20480;
constexpr u32 WS_DBG = WS_C2 + 32768;
constexpr u32 WS_FL = WS_C2 + 65536;
constexpr u32 WS_FC = WS_FL + 16384;
constexpr u32 WS_CALL = WS_FC + 16384;
constexpr u32 WS_Q = 54u << 20;
constexpr u32 QSTRIDE = 4160 * 256;
constexpr u32 WS_EX = 63u << 20;
constexpr u32 WS_ER = 67u << 20;
constexpr u32 WS_EW = 68u << 20;
constexpr u32 WS_L = 84u << 20;
constexpr u32 WS_END = 86u << 20;
static_assert(64u * NWB * WBLK <= WS_X && WS_X + 64u * NRB * XBLK <= WS_CTL && WS_WF + NWB * FSTRIDE <= WS_C2 &&
              WS_CALL + 16384 <= WS_CTL + CNT_OFF && WS_C2 + CNT_OFF <= WS_Q && WS_Q + NCB * QSTRIDE <= WS_EX, "layout");
constexpr u32 FULL = 240;

constexpr u32 SLOT_BYTES = WBLK + XBLK;
constexpr u32 NSLOT = 3;
constexpr u32 TPITCH = 272;
constexpr u32 SNAP_OFF = SLOT_BYTES * NSLOT;
constexpr u32 CTL_OFF = SNAP_OFF + 1024;
constexpr u32 LDS_BYTES = CTL_OFF + 1024;
static_assert(LDS_BYTES <= 163840, "lds");
constexpr int C_AMAX = 0, C_GMAX = 1, C_CALL = 2, C_BAD = 3, C_STAGE = 4, C_GS = 5, C_GS8 = 6, C_CNT = 8, C_MASK = 16;

#define NPW 4
#define EXWAIT 300
#define EXD 60
#define PW1 4
#define PLD 4
#define POLL2 1
#define PUSHG 0
#define AMSC " sc1"
#define QWAIT 500
#define QD 100
#define PUSHD 2
constexpr u32 NCOMMWG = 32 - NRB;
constexpr u32 NTILE = NRB * NCB;
constexpr u32 NPULL = NCOMMWG * NPW;
constexpr u32 U_B = 256, U_C = 1024;
constexpr u32 U_R = 256 + 7 * 384;
constexpr u32 U_Q = U_R + 224;
constexpr int NTHREADS = 512;
constexpr int NSTEP = 64;
constexpr u64 DEADLINE = 50000000ull;

struct Params {
  const char* x;
  const char* w;
  char* myws;
  char* out;
  u32 rank;
  u32 peers;
  u32 pad[6];
  char* dws[NR];
  char* ws[NR];
};

__device__ __forceinline__ __amdgpu_buffer_rsrc_t mkrsrc(const void* p, u32 bytes) {
  return __builtin_amdgcn_make_buffer_rsrc((void*)p, 0, bytes, 0x00020000);
}
__device__ __forceinline__ u64 now() { return __builtin_amdgcn_s_memrealtime(); }
__device__ __forceinline__ u32 ld_sys(const u32* p) { return __hip_atomic_load((GAS const u32*)p, __ATOMIC_RELAXED, __HIP_MEMORY_SCOPE_SYSTEM); }
__device__ __forceinline__ u64 ld_sys64(const u64* p) { return __hip_atomic_load((GAS const u64*)p, __ATOMIC_RELAXED, __HIP_MEMORY_SCOPE_SYSTEM); }
__device__ __forceinline__ void st_sys(u32* p, u32 v) { __hip_atomic_store((GAS u32*)p, v, __ATOMIC_RELAXED, __HIP_MEMORY_SCOPE_SYSTEM); }
__device__ __forceinline__ void st_sys64(u64* p, u64 v) { __hip_atomic_store((GAS u64*)p, v, __ATOMIC_RELAXED, __HIP_MEMORY_SCOPE_SYSTEM); }
__device__ __forceinline__ u32 lds_ld(LAS u32* p) { return __builtin_amdgcn_readfirstlane(__hip_atomic_load(p, __ATOMIC_RELAXED, __HIP_MEMORY_SCOPE_WORKGROUP)); }
__device__ __forceinline__ void lds_st(LAS u32* p, u32 v) { __hip_atomic_store(p, v, __ATOMIC_RELAXED, __HIP_MEMORY_SCOPE_WORKGROUP); }
__device__ __forceinline__ void lds_add1(LAS u32* p, u32 lane) {
  if (lane == 0) __hip_atomic_fetch_add(p, 1u, __ATOMIC_RELAXED, __HIP_MEMORY_SCOPE_WORKGROUP);
}
#define CFENCE() asm volatile("" ::: "memory")
#define WAIT_VM(n) asm volatile("s_waitcnt vmcnt(" #n ")" ::: "memory")
#define WAIT_VMI(n) asm volatile("s_waitcnt vmcnt(%0)" :: "i"(n) : "memory")
#define WAIT_LGKM() asm volatile("s_waitcnt lgkmcnt(0)" ::: "memory")
#define WAIT_VM0() do { CFENCE(); __builtin_amdgcn_s_waitcnt(0x0F70); WAIT_VM(0); } while (0)

__device__ __forceinline__ u32 wave_umax(u32 v) {
  v = max(v, (u32)__builtin_amdgcn_update_dpp(0, (int)v, 0x111, 0xf, 0xf, true));
  v = max(v, (u32)__builtin_amdgcn_update_dpp(0, (int)v, 0x112, 0xf, 0xf, true));
  v = max(v, (u32)__builtin_amdgcn_update_dpp(0, (int)v, 0x114, 0xf, 0xf, true));
  v = max(v, (u32)__builtin_amdgcn_update_dpp(0, (int)v, 0x118, 0xf, 0xf, true));
  v = max(v, (u32)__builtin_amdgcn_update_dpp(0, (int)v, 0x142, 0xa, 0xf, false));
  v = max(v, (u32)__builtin_amdgcn_update_dpp(0, (int)v, 0x143, 0xc, 0xf, false));
  return (u32)__builtin_amdgcn_readlane((int)v, 63);
}

#define STAMP(k) do {} while (0)

__device__ __forceinline__ void wait_vm_dyn(u32 n) {
  asm volatile(
      "s_getpc_b64 vcc\n\t"
      "s_add_u32 vcc_lo, vcc_lo, %0\n\t"
      "s_addc_u32 vcc_hi, vcc_hi, 0\n\t"
      "s_setpc_b64 vcc\n\t"
      "s_waitcnt vmcnt(0)\n\ts_branch 1f\n\t"
      "s_waitcnt vmcnt(1)\n\ts_branch 1f\n\t"
      "s_waitcnt vmcnt(2)\n\ts_branch 1f\n\t"
      "s_waitcnt vmcnt(3)\n\ts_branch 1f\n\t"
      "s_waitcnt vmcnt(4)\n\ts_branch 1f\n\t"
      "s_waitcnt vmcnt(5)\n\ts_branch 1f\n\t"
      "s_waitcnt vmcnt(6)\n\ts_branch 1f\n\t"
      "s_waitcnt vmcnt(7)\n\ts_branch 1f\n\t"
      "s_waitcnt vmcnt(8)\n\ts_branch 1f\n\t"
      "s_waitcnt vmcnt(9)\n\ts_branch 1f\n\t"
      "s_waitcnt vmcnt(10)\n\ts_branch 1f\n\t"
      "s_waitcnt vmcnt(11)\n\ts_branch 1f\n\t"
      "s_waitcnt vmcnt(12)\n\ts_branch 1f\n\t"
      "s_waitcnt vmcnt(13)\n\ts_branch 1f\n\t"
      "s_waitcnt vmcnt(14)\n\ts_branch 1f\n\t"
      "s_waitcnt vmcnt(15)\n\ts_branch 1f\n\t"
      "s_waitcnt vmcnt(16)\n\ts_branch 1f\n\t"
      "s_waitcnt vmcnt(17)\n\ts_branch 1f\n\t"
      "s_waitcnt vmcnt(18)\n\ts_branch 1f\n\t"
      "s_waitcnt vmcnt(19)\n\ts_branch 1f\n\t"
      "s_waitcnt vmcnt(20)\n\ts_branch 1f\n\t"
      "s_waitcnt vmcnt(21)\n\ts_branch 1f\n\t"
      "s_waitcnt vmcnt(22)\n\ts_branch 1f\n\t"
      "s_waitcnt vmcnt(23)\n\ts_branch 1f\n\t"
      "s_waitcnt vmcnt(24)\n\t"
      "1:"
      :: "s"(min(n, 24u) * 8 + 12) : "memory", "scc", "vcc");
}
#define BARRIER() asm volatile("s_barrier" ::: "memory")

__global__ void __launch_bounds__(NTHREADS) fused_gemm_ar_q8(Params P) {
  __shared__ __attribute__((aligned(16))) char smem[LDS_BYTES];
  LAS char* L = (LAS char*)smem;
  LAS u32* ctl = (LAS u32*)(L + CTL_OFF);
  const u32 tid = threadIdx.x, lane = tid & 63;
  const u32 wave = __builtin_amdgcn_readfirstlane(tid >> 6);
  u32 xcc;
  asm volatile("s_getreg_b32 %0, hwreg(HW_REG_XCC_ID, 0, 3)" : "=s"(xcc));
  const u32 g = xcc & 7u, idx = blockIdx.x >> 3, bid = idx * 8 + g;
  char* const myws = P.myws;
  char* const dws = P.dws[g];
  const u64 t0 = now();
  asm volatile("" :: "s"(P.x), "s"(P.w), "s"(myws), "s"(P.out), "s"(P.rank), "s"(P.peers), "s"(dws), "s"(t0));
  const u32 rank = P.rank & 7u;
  const u32 dest = (P.peers >> (g * 4)) & 7u;
  u32* const tmo = (u32*)(myws + WS_TMO);
  const u32 lane16 = lane * 16;
  const u32 lx = (lane & 7) * 1024 + (lane >> 3) * 16;
  const u32 lw = (((lane >> 3) & 1) * 8 + ((lane >> 1) & 3)) * 32768 + ((lane & 1) * 4 + (lane >> 4)) * 16;
  u32 call = 0;
  bool bad = false;
  if (wave == 0) STAMP(0);

  if (idx >= NCOMMWG) {
    const u32 ti = idx - NCOMMWG;
    const u32 rb = (g & 1) * 13 + (ti >> 1);
    u32 cb = (g >> 1) * 2 + (ti & 1);
    const u32 tile = g * NRB + ti;
    const __amdgpu_buffer_rsrc_t rs_ws = mkrsrc(myws, WS_END);
    char* flp = P.ws[lane & 7] + WS_FL + rank * 256 + g * 8;
    const u32 lane0 = lane ? 0x80000000u : 0u;
    auto signal_wg = [&](u32 kind, u32 inc) {
      const u32 old = __builtin_amdgcn_raw_ptr_buffer_atomic_add_i32(inc, rs_ws, lane0, WS_FC + kind * 2048 + g * 256, 16);
      if (__builtin_amdgcn_readfirstlane(old) + inc == call * NRB && lane < 8)
        st_sys64((u64*)(flp + kind * 2048), call);
      CFENCE();
    };
    auto signal_all = [&](u32 kind, u32 inc = 1) {
      u32 old = 0;
      if (lane == 0) old = __hip_atomic_fetch_add(ctl + C_CNT + kind, 1u, __ATOMIC_RELAXED, __HIP_MEMORY_SCOPE_WORKGROUP);
      if (__builtin_amdgcn_readfirstlane(old) == 7) signal_wg(kind, inc);
    };
    {
      const bool first = ti - 16 < 8;
      if (first) {
        const u32 pc = (g * 8 + ti - 16) * 8 + wave;
        const u32x4 e0 = __builtin_amdgcn_raw_buffer_load_b128(mkrsrc(P.x, X_BYTES), lx, pc * 8192, 0);
        __builtin_amdgcn_raw_buffer_store_b128(e0, rs_ws, lane16, WS_EX + pc * 1024, 17);
        asm volatile("" : "+v"(flp));
        WAIT_VM0();
        BARRIER();
      }
      if (wave == 0) {
        const u32 old = __builtin_amdgcn_readfirstlane(
            __builtin_amdgcn_raw_ptr_buffer_atomic_add_i32(1, rs_ws, lane0, WS_FC + (first ? 0 : 6 * 2048) + g * 256, 16));
        call = (first ? old >> 3 : old / (NRB - 8)) + 1;
        if (first && (old & 7) == 7 && lane < 8) {
          st_sys64((u64*)flp, ((u64)P.peers << 32) | call);
        }
        if (lane < 16) ctl[lane] = lane == C_CALL ? call : 0u;
        WAIT_LGKM();
      }
      if (wave == 6 && ti == 25) __builtin_amdgcn_fence(__ATOMIC_ACQUIRE, "");
      BARRIER();
      call = lds_ld(ctl + C_CALL);
      for (u32 r = 0; r < 3; ++r) {
        const u32 slot = r * 8 + wave, e = slot * NTILE + tile;
        u32 kc, k, so, eo;
        bool isw = true;
        if (e < 4480) {
          kc = e / 640; k = e - kc * 640;
          kc += 1;
          isw = k < 512;
          k &= 511;
          eo = isw ? WS_EW + ((k >> 6) * 8 + kc) * (8 * WBLK) + (k & 63) * 4096 : WS_EX + kc * (512u << 10) + k * 4096;
        } else {
          kc = (e - 4480) >> 4; k = ((e >> 1) & 7) * 64 + 62 + (e & 1);
          eo = WS_ER + (call & 1) * (512u << 10) + (e - 4480) * 4096;
        }
        if (e < 4608) {
          so = isw ? (kc * 64 + (k & 1) * 32) * 32768 + (k >> 1) * 128 : k * 32768 + kc * 128;
          const __amdgpu_buffer_rsrc_t rs = mkrsrc(isw ? P.w : P.x, isw ? W_BYTES : X_BYTES);
          const u32 lo = isw ? lw : lx, s1 = isw ? 4 * 32768 : 8192, s2 = isw ? 16 * 32768 : 16384;
          while (now() - t0 < EXWAIT + slot * EXD) __builtin_amdgcn_s_sleep(2);
          const u32x4 e0 = __builtin_amdgcn_raw_buffer_load_b128(rs, lo, so, 0);
          const u32x4 e1 = __builtin_amdgcn_raw_buffer_load_b128(rs, lo, so + s1, 0);
          const u32x4 e2 = __builtin_amdgcn_raw_buffer_load_b128(rs, lo, so + s2, 0);
          const u32x4 e3 = __builtin_amdgcn_raw_buffer_load_b128(rs, lo, so + s1 + s2, 0);
          __builtin_amdgcn_raw_buffer_store_b128(e0, rs_ws, lane16, eo, 17);
          __builtin_amdgcn_raw_buffer_store_b128(e1, rs_ws, lane16, eo + 1024, 17);
          __builtin_amdgcn_raw_buffer_store_b128(e2, rs_ws, lane16, eo + 2048, 17);
          __builtin_amdgcn_raw_buffer_store_b128(e3, rs_ws, lane16, eo + 3072, 17);
        }
        if (r != 1) {
          WAIT_VM0();
          signal_all(1 + (r >> 1));
        }
      }
      BARRIER();
    }
    const u32 cv = call * FULL;
    const u32 wm = wave >> 2, wn = wave & 3;
    const u32 lds0 = (u32)(size_t)L;
    const u32 xa = WBLK + wm * 10240 + ((lane >> 3) & 1) * 1024 + (lane >> 5) * 256 + ((lane >> 4) & 1) * 128 + (lane & 7) * 16;
    const u32 h = lane >> 4, q = (lane >> 2) & 3, p = lane & 3;
    const u32 wa0 = wn * 8192 + (h >> 1) * 2048 + (p & 1) * 512 + (h & 1) * 128 + q * 32 + (p >> 1) * 16 + (p & 1) * 8;
    const u32 wa1 = wa0 ^ 8;
    const u32 xpo = wave < 4 ? wave * 3072 : 12288 + (wave - 4) * 2048;
    u32 pollo;
    f32x4 acc[4][5];
    u64 kx = 0, kw;
    u64 ready, taken;
    u32 issued, consumed, nis = 0;
    u32 islot, cslot, par = 0;
    u64 markD = 0;
    float inv = 0.f;
    auto sched = [&]() {
      u64 avail = ready & ~taken;
      while (avail && issued - consumed < NSLOT) {
        const u32 ord = __builtin_ctzll(avail);
        avail &= avail - 1;
        taken |= 1ull << ord;
        const u32 sw = WS_W + (ord * NWB + cb) * WBLK + wave * 4096;
        const u32 sx = WS_X + (ord * NRB + rb) * XBLK + xpo;
        const u32 mw = lds0 + islot + wave * 4096, mx = lds0 + islot + WBLK + xpo;
        u32 keep;
        asm volatile(
            "s_mov_b32 %0, m0\n\t"
            "s_mov_b32 m0, %3\n\t"
            "s_nop 0\n\t"
            "buffer_load_dwordx4 %1, %2, %5 offen lds\n\t"
            "buffer_load_dwordx4 %1, %2, %5 offen offset:1024 lds\n\t"
            "buffer_load_dwordx4 %1, %2, %5 offen offset:2048 lds\n\t"
            "buffer_load_dwordx4 %1, %2, %5 offen offset:3072 lds\n\t"
            "s_mov_b32 m0, %4\n\t"
            "s_nop 0\n\t"
            "buffer_load_dwordx4 %1, %2, %6 offen lds\n\t"
            "buffer_load_dwordx4 %1, %2, %6 offen offset:1024 lds\n\t"
            "s_cmp_gt_u32 %7, 3\n\t"
            "s_cbranch_scc1 1f\n\t"
            "buffer_load_dwordx4 %1, %2, %6 offen offset:2048 lds\n\t"
            "1:\n\t"
            "s_mov_b32 m0, %0"
            : "=&s"(keep)
            : "v"(lane16), "s"(rs_ws), "s"(mw), "s"(mx), "s"(sw), "s"(sx), "s"(wave)
            : "memory", "scc");
        nis += wave < 4 ? 7 : 6;
        const u32 sh = (issued & 3) * 16;
        markD = (markD & ~(0xffffull << sh)) | ((u64)(nis & 0xffff) << sh);
        issued += 1;
        islot = islot == 2 * SLOT_BYTES ? 0 : islot + SLOT_BYTES;
      }
    };
    auto poll = [&]() {
      if (wave == 0 && (kx & kw) != ~0ull) {
        const u64 kn = lane < 16 ? kx : kw;
        const bool act = lane < 32 && ((u32)(kn >> ((lane & 15) * 4)) & 15u) != 15u;
        const u32 po = act ? pollo : 0x80000000u;
        u32 keep;
        asm volatile("s_mov_b32 %0, m0\n\ts_mov_b32 m0, %3\n\ts_nop 0\n\t"
                     "buffer_load_dwordx4 %1, %2, 0 offen sc0 sc1 lds\n\ts_mov_b32 m0, %0"
                     : "=&s"(keep) : "v"(po), "s"(rs_ws), "s"(lds0 + SNAP_OFF) : "memory");
        nis += 1;
      }
    };
    LAS u32* const snap = (LAS u32*)(L + SNAP_OFF) + lane;
    auto seen = [&](u32 fx, u32 fw) {
      kx |= __ballot((int)(fx - cv) >= 0);
      kw |= __ballot((int)(fw - cv) >= 0);
    };
    auto tell = [&]() { if (lane == 0) *(LAS u64*)(L + CTL_OFF + C_MASK * 4 + par) = kx & kw; };
    auto told = [&]() -> u64 {
      const u64 m = *(LAS u64*)(L + CTL_OFF + C_MASK * 4 + par);
      par ^= 8;
      return m;
    };
    auto uni = [&](u64 m) -> u64 {
      return (u64)(u32)__builtin_amdgcn_readfirstlane((u32)m) | ((u64)(u32)__builtin_amdgcn_readfirstlane((u32)(m >> 32)) << 32);
    };
    auto wait_step = [&]() { wait_vm_dyn((nis - (u32)(markD >> ((consumed & 3) * 16))) & 0xffff); };
#define SB() __builtin_amdgcn_sched_barrier(0)
#define PIECE(M, S, OFF) do { \
        asm volatile("s_cmp_lg_u32 %2, -1\n\ts_cbranch_scc0 1f\n\ts_mov_b32 m0, %2\n\ts_nop 0\n\t" \
                     "buffer_load_dwordx4 %0, %1, %3 offen offset:" #OFF " lds\n\t1:" \
                     :: "v"(lane16), "s"(rs_ws), "s"(M), "s"(S) : "memory", "scc"); \
        SB(); } while (0)
#define MMA4(kb, i) do { \
        _Pragma("unroll") for (int n = 0; n < 4; ++n) \
          acc[n][i] = __builtin_amdgcn_mfma_f32_16x16x32_bf16(wf[kb][n], xf[kb][i], acc[n][i], 0, 0, 0); \
        SB(); } while (0)
    bf16x8 xf[2][5], wf[2][4];
    u32 sp = 0;
    u64 cyc0 = 0;
    for (u32 pass = 0;; ++pass) {
    pollo = lane < 16 ? WS_XF + rb * FSTRIDE + lane16 : WS_WF + cb * FSTRIDE + lane16 - 256;
    kw = taken = 0;
    issued = consumed = islot = cslot = 0;
#pragma unroll
    for (int a = 0; a < 4; ++a)
#pragma unroll
      for (int b = 0; b < 5; ++b) acc[a][b] = f32x4{0.f, 0.f, 0.f, 0.f};
#pragma unroll
    for (int n = 0; n < 5; ++n) {
      xf[1][n] = __builtin_bit_cast(bf16x8, u32x4{0u, 0u, 0u, 0u});
      if (n < 4) wf[1][n] = __builtin_bit_cast(bf16x8, u32x4{0u, 0u, 0u, 0u});
    }
    poll();
    while (consumed < NSTEP) {
      if (issued == consumed) {
        if (wave == 0) {
          while ((kx & kw & ~taken) == 0) {
            poll();
            WAIT_VM(1);
            seen(snap[0], snap[64]);
            if ((++sp & 63) == 0 && now() - t0 > DEADLINE) {
              st_sys(tmo, call);
              WAIT_VM(0);
              lds_st(ctl + C_BAD, 1u);
              kx = kw = ~0ull;
            }
          }
          tell();
          WAIT_LGKM();
        }
        BARRIER();
        ready = uni(told());
        sched();
        poll();
        wait_step();
        if (wave == 0) {
          seen(snap[0], snap[64]);
          tell();
          WAIT_LGKM();
        }
      }
      BARRIER();
      const u64 rdy = told();
      SB();
      LAS char* base = L + cslot;
      auto frags = [&](int kb) {
#pragma unroll
        for (int n = 0; n < 5; ++n) {
          if (n < 4) {
            LAS char* a = base + ((n & 1) ? wa1 : wa0) + kb * 4096 + (n >> 1) * 256;
            s16x4 lo = __builtin_amdgcn_ds_read_tr16_b64_v4i16((LAS s16x4*)a);
            s16x4 hi = __builtin_amdgcn_ds_read_tr16_b64_v4i16((LAS s16x4*)(a + 1024));
            wf[kb][n] = __builtin_bit_cast(bf16x8, __builtin_shufflevector(lo, hi, 0, 1, 2, 3, 4, 5, 6, 7));
          }
          xf[kb][n] = *(LAS bf16x8*)(base + xa + n * 2048 + kb * 512);
        }
        SB();
      };
      MMA4(1, 0);
      frags(0);
      poll();
      SB();
      MMA4(1, 1);
      ready = uni(rdy);
      const u64 av = ready & ~taken;
      const bool go = av != 0 && issued - consumed < NSLOT;
      const u32 ord = __builtin_ctzll(av | (1ull << 63));
      const u32 sw = WS_W + (ord * NWB + cb) * WBLK + wave * 4096;
      const u32 sx = WS_X + (ord * NRB + rb) * XBLK + xpo;
      u32 mw = ~0u, mx = ~0u, mx3 = ~0u;
      if (go) {
        mw = lds0 + islot + wave * 4096;
        mx = lds0 + islot + WBLK + xpo;
        if (wave < 4) mx3 = mx;
        taken |= 1ull << ord;
        nis += wave < 4 ? 7 : 6;
        const u32 sh = (issued & 3) * 16;
        markD = (markD & ~(0xffffull << sh)) | ((u64)(nis & 0xffff) << sh);
        issued += 1;
        islot = islot == 2 * SLOT_BYTES ? 0 : islot + SLOT_BYTES;
      }
      SB();
      MMA4(1, 2);
      PIECE(mw, sw, 0);
      MMA4(1, 3);
      PIECE(mw, sw, 1024);
      MMA4(1, 4);
      frags(1);
      PIECE(mw, sw, 2048);
      MMA4(0, 0);
      PIECE(mw, sw, 3072);
      PIECE(mx, sx, 0);
      MMA4(0, 1);
      PIECE(mx, sx, 1024);
      PIECE(mx3, sx, 2048);
      MMA4(0, 2);
      sched();
      poll();
      cslot = cslot == 2 * SLOT_BYTES ? 0 : cslot + SLOT_BYTES;
      consumed += 1;
      SB();
      MMA4(0, 3);
      if (issued != consumed) wait_step();
      u32 fx = cv - 1, fw = cv - 1;
      if (wave == 0) { fx = snap[0]; fw = snap[64]; }
      SB();
      MMA4(0, 4);
      if (wave == 0) { seen(fx, fw); tell(); }
      WAIT_LGKM();
    }
    MMA4(1, 0); MMA4(1, 1); MMA4(1, 2); MMA4(1, 3); MMA4(1, 4);
    if (pass == 0) {
      if (wave == 0) STAMP(2);
      const u32 row0 = rb * 160 + wm * 80;
      float m = 0.f;
#pragma unroll
      for (int b = 0; b < 5; ++b) {
        float mb = 0.f;
#pragma unroll
        for (int a = 0; a < 4; ++a)
#pragma unroll
          for (int e = 0; e < 4; ++e) mb = fmaxf(mb, fabsf(acc[a][b][e]));
        if (row0 + b * 16 < 4096) m = fmaxf(m, mb);
      }
      const u32 mb = wave_umax(__builtin_bit_cast(u32, m));
      if (lane == 0) __hip_atomic_fetch_max(ctl + C_AMAX, mb, __ATOMIC_RELAXED, __HIP_MEMORY_SCOPE_WORKGROUP);
      WAIT_LGKM();
      BARRIER();
      if (wave == 0) {
        const u32 wmax = lds_ld(ctl + C_AMAX);
        u64 ret = 1;
        if (lane == 0)
          asm volatile("s_nop 4\n\tglobal_atomic_umax %1, %2, %3 offset:4" AMSC "\n\tglobal_atomic_add_x2 %0, %1, %0, %3 sc0" AMSC "\n\ts_waitcnt vmcnt(0)"
                       : "+v"(ret) : "v"(0u), "v"(wmax), "s"(myws + WS_G1 + g * 256) : "memory");
        ret = uni(ret);
        if ((u32)ret == NRB - 1) {
          st_sys64((u64*)(flp + (int)((lane >> 3) * G2S + WS_G2 - WS_FL - rank * 192)), ((u64)call << 32) | (u32)(ret >> 32));
          if (lane == 0) asm volatile("s_nop 4\n\tglobal_store_dwordx2 %0, %1, %2" AMSC :: "v"(0u), "v"(0ull), "s"(myws + WS_G1 + g * 256) : "memory");
        }
        STAMP(3);
        const u64* g2 = (const u64*)(myws + WS_G2 + g * G2S) + lane;
        u64 val;
        for (;;) {
          val = ld_sys64(g2);
          if (__all((u32)(val >> 32) == call)) break;
          if ((++sp & 63) == 0 && (ld_sys(tmo) == call || now() - t0 > DEADLINE)) { st_sys(tmo, call); bad = true; break; }
        }
        const u32 gmax = wave_umax((u32)val);
        if (lane == 0) {
          lds_st(ctl + C_GMAX, gmax);
          if (bad) lds_st(ctl + C_BAD, 1u);
        }
        WAIT_LGKM();
      }
    }
    BARRIER();
    if (pass == 0) {
      const float amax = __builtin_bit_cast(float, lds_ld(ctl + C_GMAX));
      const float scale = amax / 127.0f;
      inv = amax > 0.f ? 1.0f / scale : 0.f;
      if (wave == 0) STAMP(4);
    }
    bad = lds_ld(ctl + C_BAD) != 0;
    const bool odd = (lane >> 4) & 1;
    u32x4 qv[5];
#pragma unroll
    for (int i = 0; i < 5; ++i) {
      u32 d[4];
#pragma unroll
      for (int n = 0; n < 4; ++n) {
        u32 b0 = __builtin_bit_cast(u32, __builtin_fmaf(acc[n][i][0], inv, 12582912.0f));
        u32 b1 = __builtin_bit_cast(u32, __builtin_fmaf(acc[n][i][1], inv, 12582912.0f));
        u32 b2 = __builtin_bit_cast(u32, __builtin_fmaf(acc[n][i][2], inv, 12582912.0f));
        u32 b3 = __builtin_bit_cast(u32, __builtin_fmaf(acc[n][i][3], inv, 12582912.0f));
        u32 lo = __builtin_amdgcn_perm(b1, b0, 0x0c0c0400u);
        u32 hi = __builtin_amdgcn_perm(b3, b2, 0x04000c0cu);
        d[n] = bad ? 0x80808080u : (lo | hi);
      }
      qv[i] = u32x4{odd ? d[1] : d[0], odd ? d[0] : d[1], odd ? d[3] : d[2], odd ? d[2] : d[3]};
    }
    const u64 tq = now();
    if (!pass && !(cb & 1) && wm == 0) {
      const __amdgpu_buffer_rsrc_t rs_p = mkrsrc(dws, WS_END);
#pragma unroll
      for (int i = 0; i < 4; ++i) {
        if (i && i % PUSHD == 0) WAIT_VM(0);
        if (wave == 0 && i == PUSHD) STAMP(10);
        __builtin_amdgcn_raw_buffer_store_b128(qv[i], rs_p, lane16, WS_L + ((rank * 13 + (ti >> 1)) * 16 + wn * 4 + i) * 1024, 17);
        CFENCE();
      }
      WAIT_VM0();
      if (wave == 0) STAMP(11);
      const u32 old = __builtin_amdgcn_raw_ptr_buffer_atomic_add_i32(1, rs_ws, lane0, WS_FC + 5 * 2048 + g * 256, 16);
      if (__builtin_amdgcn_readfirstlane(old) + 1 == call * 52 && lane == 0) st_sys64((u64*)(dws + WS_FL + 5 * 2048) + rank, ((u64)g << 32) | call);
      CFENCE();
    }
    {
      LAS char* tw = L + (wm * 80 + (lane & 15)) * TPITCH + wn * 64 + (lane >> 4) * 16;
#pragma unroll
      for (int i = 0; i < 5; ++i) *(LAS u32x4*)(tw + i * 16 * TPITCH) = qv[i];
    }
    WAIT_LGKM();
    BARRIER();
    LAS char* tr = L + (wave * 4 + (lane >> 4)) * TPITCH + (lane & 15) * 16;
#pragma unroll
    for (int i = 0; i < 5; ++i) qv[i] = *(LAS u32x4*)(tr + i * 32 * TPITCH);
    const __amdgpu_buffer_rsrc_t rs_o = mkrsrc(P.out, OUT_BYTES);
    const u32 orow = (rb * 160 + wave * 4 + (lane >> 4)) * 16384u;
    if (pass) {
      const u32 piece = (cb - 8) * 4 + ((lane >> 2) & 3);
      const u32 oo = piece == rank ? 0x80000000u : orow + piece * 2048 + 1984 + (lane & 3) * 16;
#pragma unroll
      for (int i = 0; i < 5; ++i) __builtin_amdgcn_raw_buffer_store_b128(qv[i], rs_o, oo, i * 32 * 16384, 17);
      break;
    }
    const u32 pitch = cb == 7 ? 192 : 256;
    const bool rep = (lane & 15) * 16 >= pitch;
    const u32 qo = rep ? 0x80000000u : (wave * 4 + (lane >> 4)) * pitch + (lane & 15) * 16;
    const u32 oo = orow + rank * 2048 + cb * 256 + (lane & 15) * 16;
    const u32 qa = WS_Q + cb * QSTRIDE + rb * 64 * pitch, qb = WS_Q + cb * QSTRIDE + (1664 + rb * 96) * pitch;
    if (cb & 1) {
      __builtin_amdgcn_raw_buffer_store_b128(qv[0], rs_ws, qo, qa, 17);
      __builtin_amdgcn_raw_buffer_store_b128(qv[1], rs_ws, qo, qa + 32 * pitch, 17);
      WAIT_VM0();
      signal_all(3, 2);
      if (wave == 0) STAMP(9);
    }
    while (now() - tq < QWAIT + wave * QD) __builtin_amdgcn_s_sleep(2);
    if (!(cb & 1)) {
      __builtin_amdgcn_raw_buffer_store_b128(qv[0], rs_ws, qo, qa, 17);
      __builtin_amdgcn_raw_buffer_store_b128(qv[1], rs_ws, qo, qa + 32 * pitch, 17);
    }
#pragma unroll
    for (int i = 2; i < 5; ++i) __builtin_amdgcn_raw_buffer_store_b128(qv[i], rs_ws, qo, qb + (i - 2) * 32 * pitch, 17);
    WAIT_VM0();
    signal_all(4);
    if (cb == 7) {
#pragma unroll
      for (int i = 0; i < 5; ++i) __builtin_amdgcn_raw_buffer_store_b128(qv[i], rs_o, rep ? oo : 0x80000000u, i * 32 * 16384, 17);
    }
    if (((ti >> 1) & 3) != (g >> 1)) break;
    cb = 8 + (ti & 1);
    WAIT_VM0();
    WAIT_LGKM();
    BARRIER();
    }
    WAIT_VM(0);
    if (wave == 0) STAMP(5);
    return;
  }

  const u32 vid = idx * NPW + wave;
  const __amdgpu_buffer_rsrc_t rs_s = mkrsrc(dws, WS_END);
  const u32 lane0 = lane ? 0x80000000u : 0u;
  static_assert(NPW == 4 && NCOMMWG == 6, "wave roles");
  if (wave >= NPW && wave < 7) {
    const u32 ph = (wave - NPW) * NCOMMWG + idx;
    const __amdgpu_buffer_rsrc_t rs_w = mkrsrc(P.w, W_BYTES);
    const u32 so = ph < 16 ? dest * 4096 + ph * 256 : 0x80000000u;
    const u32 to = ph < 16 ? WS_W + rank * NWB * WBLK + ph * 16384 : 0x80000000u;
    u32x4 pv[16];
#pragma unroll
    for (int i = 0; i < 16; ++i) {
      pv[i] = __builtin_amdgcn_raw_buffer_load_b128(rs_w, lw, so + (i >> 3) * 128 + (((i >> 1) & 3) * 16 + (i & 1) * 4) * 32768, 0);
      if (i == PLD - 1) BARRIER();
    }
    CFENCE();
#pragma unroll
    for (int i = 0; i < 16; ++i) {
      if (i >= PW1) WAIT_VMI(PW1 - 1);
      __builtin_amdgcn_raw_buffer_store_b128(pv[i], rs_s, lane16, to + i * 1024, 17);
      CFENCE();
    }
    WAIT_VM(0);
    if (wave == NPW) STAMP(10);
    if (ph < 16) __builtin_amdgcn_raw_ptr_buffer_atomic_add_i32(FULL / 2, rs_s, lane0, WS_WF + (ph >> 1) * FSTRIDE + rank * 4, 16);
    return;
  }
  if (wave == 7) {
    ctl[lane] = 0u;
    WAIT_LGKM();
  }
  BARRIER();
  if (wave == 7) {
    u32* const cptr = (u32*)(myws + WS_CALL + bid * 256);
    call = __builtin_amdgcn_readfirstlane(ld_sys(cptr)) + 1;
    if (lane == 0) { lds_st(ctl + C_CALL, call); st_sys(cptr, call); }
    u32 sp = 0;
    auto expired = [&]() -> bool {
      if ((++sp & 63) == 0 && (ld_sys(tmo) == call || now() - t0 > DEADLINE)) {
        st_sys(tmo, call);
        if (lane == 0) { lds_st(ctl + C_BAD, 1u); lds_st(ctl + C_STAGE, 6u); }
        bad = true;
      }
      return bad;
    };
    auto watch = [&](u32 kind, u32 row) {
      const u64* fp = (const u64*)(myws + WS_FL + kind * 2048 + row * 256) + (lane & 7);
      while (!bad) {
        const u64 f = ld_sys64(fp);
        const u64 ok = __ballot((int)((u32)f - call) >= 0);
        if (ok == ~0ull) {
          const u32 hi = (u32)(f >> 32), sh = (lane & 7) * 4;
          if (kind == 0 && lane < 8 && ((hi >> sh) & 7u) == rank) lds_st(ctl + C_GS, lane);
          if (kind == 5 && lane < 8) __hip_atomic_fetch_or(ctl + C_GS8, (hi & 7u) << sh, __ATOMIC_RELAXED, __HIP_MEMORY_SCOPE_WORKGROUP);
          break;
        }
        __builtin_amdgcn_s_sleep(1);
        expired();
      }
      if (lane == 0 && !bad) lds_st(ctl + C_STAGE, kind + 1);
      CFENCE();
    };
    watch(0, dest);
    STAMP(11);
    watch(1, dest);
    watch(2, dest);
    watch(3, dest);
    STAMP(9);
    watch(4, dest);
    if (g == 0) watch(5, 0);
    return;
  }
  u32 stage = 0;
  auto need = [&](u32 s) {
    while (stage < s) {
      stage = lds_ld(ctl + C_STAGE);
      if (stage < s) __builtin_amdgcn_s_sleep(1);
    }
    CFENCE();
  };
  need(1);
  if (wave == 0) STAMP(3);
  call = lds_ld(ctl + C_CALL);
  const __amdgpu_buffer_rsrc_t rs_m = mkrsrc(myws, WS_Q);
  const __amdgpu_buffer_rsrc_t rs_o = mkrsrc(P.out, OUT_BYTES);
  const u32 rot = (g * 26 / 7 % 26) * 10;
  constexpr u32 U_L = U_Q + 3968 - 104, uq2 = U_Q + 3 * 208 + 156;
  const u32 nu = g ? U_L : U_L + 832;
  struct Unit { u32 so, to, fo, fw; };
  u32 qc = 0, qbase = U_Q;
  const u32 gs = lds_ld(ctl + C_GS);
  auto uoff = [&](u32 u) -> Unit {
    Unit d;
    if (u < U_R) {
      const u32 kc = (u + 128) / 384, ord = kc * 8 + dest;
      const u32 k = u + 128 - kc * 384;
      if (k < 128) {
        d.so = WS_EW + (rank * 8 + kc) * (8 * WBLK) + k * 2048;
        d.to = WS_W + ord * NWB * WBLK + k * 2048;
        d.fo = WS_WF + (k >> 4) * FSTRIDE + ord * 4;
        d.fw = 15;
      } else {
        const u32 kk = (k - 128 + rot) & 255, r = kk / 10;
        d.so = WS_EX + kc * (512u << 10) + kk * 2048;
        d.to = WS_X + ord * NRB * XBLK + kk * 2048;
        d.fo = WS_XF + r * FSTRIDE + ord * 4;
        d.fw = r == NRB - 1 ? 40 : 24;
      }
    } else if (u < U_Q) {
      const u32 k = u - U_R, kc = k / 28, j = k - kc * 28, ord = kc * 8 + dest;
      const u32 jj = j + ((j >> 2) >= rank ? 4 : 0);
      d.so = WS_ER + (call & 1) * (512u << 10) + kc * 65536 + jj * 2048;
      d.to = WS_W + (ord * NWB + 8) * WBLK + jj * 2048;
      d.fo = WS_WF + (8 + (jj >> 4)) * FSTRIDE + ord * 4;
      d.fw = (jj >> 4) == (rank >> 2) ? 20 : 15;
    } else if (u < U_L) {
      u32 c, na, n;
      for (;;) {
        c = (qc < 8 ? (gs & 6) + 2 + qc * 2 + (qc < 4) : gs + qc) & 7;
        na = c == 7 ? 156 : 208;
        n = qc < 7 ? na : qc < 8 ? 104 : c == 7 ? 228 : 304;
        if (u - qbase < n) break;
        qbase += n;
        qc += 1;
      }
      const u32 k = u - qbase + (qc >= 8 ? na : qc == 7 && !(gs & 1) ? 104 : 0);
      d.so = WS_Q + c * QSTRIDE + k * 2048;
      d.to = k * 128;
      d.fo = c;
      d.fw = 0;
    } else {
      const u32 k = u - U_L, src = k / 104, r = k - src * 104;
      d.so = u < nu ? WS_L + k * 2048 : 0x80000000u;
      d.to = ((r >> 3) * 160 + (r & 1) * 32) * 16384 + src * 2048 + (r & 6) * 32;
      d.fo = src & 7;
      d.fw = 0;
    }
    return d;
  };
  u32x4 v0, v1;
  u32 u0 = vid, u1 = NPULL + vid, tk = vid + (call - 1) * nu - NPULL;
  Unit d0 = uoff(u0), d1 = uoff(u1);
  v0 = __builtin_amdgcn_raw_buffer_load_b128(rs_s, lane16, d0.so, 17);
  v1 = __builtin_amdgcn_raw_buffer_load_b128(rs_s, lane16, d0.so + 1024, 17);
  u32 to0 = lane16, to1 = lane16 + 1024, tso = d0.to, lo1 = lane16;
  __amdgpu_buffer_rsrc_t rs_t = rs_m;
  const u32 cv = call * FULL;
  u32 cnt = 0, pfo = 0, pfw = 0;
  auto publish = [&]() {
    if (pfw && __builtin_amdgcn_readfirstlane(cnt) + pfw == cv)
      asm volatile("s_nop 4\n\tbuffer_store_dword %0, %1, %2, %3 offen sc0 sc1" :: "v"(cv), "v"(lane0), "s"(rs_m), "s"(pfo) : "memory");
    pfw = 0;
  };
  WAIT_VM0();
  while (u0 < nu) {
    if (bad && !d0.fw) v0 = v1 = u32x4{0x80808080u, 0x80808080u, 0x80808080u, 0x80808080u};
    __builtin_amdgcn_raw_buffer_store_b128(v0, rs_t, to0, tso, 17);
    __builtin_amdgcn_raw_buffer_store_b128(v1, rs_t, to1, tso, 17);
    CFENCE();
    const u32 ns = u1 >= nu ? 0 : u1 >= U_L ? 6 : u1 >= uq2 ? 5 : u1 >= U_Q ? 4 : u1 >= U_C ? 3 : u1 >= U_B ? 2 : 1;
    if (stage < ns) {
      stage = lds_ld(ctl + C_STAGE);
      if (stage < ns) {
        publish();
        WAIT_VM0();
        if (d0.fw) {
          cnt = __builtin_amdgcn_raw_ptr_buffer_atomic_add_i32(d0.fw, rs_m, lane0, d0.fo + CNT_OFF, 16);
          pfo = d0.fo; pfw = d0.fw;
          publish();
        }
        d0.fw = 0;
        need(ns);
      }
      bad = lds_ld(ctl + C_BAD) != 0;
    }
    v0 = __builtin_amdgcn_raw_buffer_load_b128(rs_s, lo1, d1.so, 17);
    v1 = __builtin_amdgcn_raw_buffer_load_b128(rs_s, lo1, d1.so + 1024, 17);
    CFENCE();
    const u32 u2 = 3 * NPULL + (__builtin_amdgcn_readfirstlane(tk) - (call - 1) * nu);
    tk = __builtin_amdgcn_raw_ptr_buffer_atomic_add_i32(1, rs_m, lane0, WS_TK + g * 128, 16);
    CFENCE();
    publish();
    const Unit d2 = uoff(u2);
    tso = d1.to;
    if (u2 >= U_L) lo1 = (lane & 3) * 256 + (lane >> 2) * 16;
    if (u1 >= U_L) {
      const u32 gl = lds_ld(ctl + C_GS8) >> (d1.fo * 4);
      tso += (gl & 1) * (2080 * 16384) + (gl & 6) * 256;
      rs_t = rs_o;
      to0 = (lane >> 2) * 16384 + (lane & 3) * 16;
      to1 = to0 + 16 * 16384;
    } else if (!d1.fw) {
      rs_t = rs_o;
      tso = dest * 2048 + d1.fo * 256;
      const u32 mul = d1.fo == 7 ? 43691u : 32768u, w16 = d1.fo == 7 ? 12 : 16;
#pragma unroll
      for (int e = 0; e < 2; ++e) {
        const u32 i16 = d1.to + e * 64 + lane;
        const u32 sr = (i16 * mul) >> 19, col = i16 - sr * w16;
        const u32 ra = (sr >> 6) * 96 + sr, rc = (((sr - 1664) * 43691u) >> 22) * 64 + sr - 1600;
        (e ? to1 : to0) = (sr < 1664 ? ra : rc) * 16384 + col * 16;
      }
    }
    CFENCE();
    WAIT_VM(3);
    cnt = __builtin_amdgcn_raw_ptr_buffer_atomic_add_i32(d0.fw, rs_m, lane0, d0.fw ? d0.fo + CNT_OFF : 0x40000000u, 16);
    pfo = d0.fo; pfw = d0.fw;
    CFENCE();
    u0 = u1; u1 = u2;
    d0 = d1; d1 = d2;
  }
  publish();
  WAIT_VM(0);
  if (wave == 0) STAMP(5);
}

extern "C" void kernel_launch(int rank, int nranks, ncclComm_t comm, hipStream_t stream,
                              void* const* d_in, const int* in_sizes, int n_in,
                              void* d_out, int out_size,
                              const int* in_dtypes, int out_dtype,
                              void* const* d_out_all,
                              void* const* d_ws_all, size_t ws_bytes) {
  if (nranks != NR || ws_bytes < WS_END || n_in < 2) { fprintf(stderr, "unsupported configuration\n"); return; }
  Params P = {};
  P.x = (const char*)d_in[0];
  P.w = (const char*)d_in[1];
  for (int p = 0; p < NR; ++p) P.ws[p] = (char*)d_ws_all[p];
  P.myws = P.ws[rank & 7];
  P.out = (char*)d_out;
  P.rank = rank;
  int logi[NR] = {0, 4, 1, 2, 5, 6, 3, 7}, phys[NR], grp[NR], ndev = 0, sum = 0;
  bool known = hipGetDeviceCount(&ndev) == hipSuccess && ndev == NR;
  for (int d = 0; d < NR && known; ++d) {
    char bus[32] = {0}, path[96];
    int pos = -1;
    known = hipDeviceGetPCIBusId(bus, sizeof bus, d) == hipSuccess;
    snprintf(path, sizeof path, "/sys/bus/pci/devices/%s/xgmi_physical_id", bus);
    FILE* f = known ? fopen(path, "r") : nullptr;
    known = f && fscanf(f, "%d", &pos) == 1 && pos >= 0;
    if (f) fclose(f);
    grp[d] = (pos ^ (pos >> 1)) & 1;
    sum += grp[d];
  }
  (void)hipGetLastError();
  if (known && sum == NR / 2)
    for (int d = 0, n[2] = {0, NR / 2}; d < NR; ++d) logi[d] = n[grp[d]]++;
  for (int d = 0; d < NR; ++d) phys[logi[d]] = d;
  P.peers = 0;
  for (int g = 0; g < NR; ++g) P.peers |= (u32)phys[logi[rank & 7] ^ g] << (g * 4);
  for (int g = 0; g < NR; ++g) P.dws[g] = P.ws[(P.peers >> (g * 4)) & 7];
  hipLaunchKernelGGL(fused_gemm_ar_q8, dim3(256), dim3(NTHREADS), 0, stream, P);
  HIP_CHECK(hipGetLastError());
}
```

```cpp
#include <hip/hip_runtime.h>
#include <rccl/rccl.h>
#include <cstdio>
#include <stdint.h>

#define HIP_CHECK(expr) do { hipError_t _e = (expr); if (_e != hipSuccess) { \
    fprintf(stderr, "HIP error %s at %s:%d\n", hipGetErrorString(_e), __FILE__, __LINE__); \
    return; } } while (0)

typedef unsigned u32;
typedef unsigned long long u64;
typedef u32 u32x4 __attribute__((ext_vector_type(4)));
typedef u32 u32x2 __attribute__((ext_vector_type(2)));
typedef float f32x4 __attribute__((ext_vector_type(4)));
typedef short s16x4 __attribute__((ext_vector_type(4)));
typedef short s16x8 __attribute__((ext_vector_type(8)));
typedef __bf16 bf16x8 __attribute__((ext_vector_type(8)));
#define LAS __attribute__((address_space(3)))
#define GAS __attribute__((address_space(1)))

constexpr int NR = 8;
constexpr u32 OUT_BYTES = 4096u * 16384u;
constexpr u32 X_BYTES = 4096u * 512u * 2u;
constexpr u32 W_BYTES = 512u * 16384u * 2u;

constexpr u32 NRB = 26;
constexpr u32 NCB = 8;
constexpr u32 XBLK = 20480, WBLK = 32768;
constexpr u32 NWB = NCB + 2;
constexpr u32 WS_W = 0;
constexpr u32 WS_X = 20u << 20;
constexpr u32 WS_CTL = 53u << 20;
constexpr u32 FSTRIDE = 4352;
constexpr u32 WS_XF = WS_CTL;
constexpr u32 WS_WF = WS_XF + NRB * FSTRIDE;
constexpr u32 CNT_OFF = 512u << 10;
constexpr u32 WS_C2 = WS_CTL + (160u << 10);
constexpr u32 WS_G1 = WS_C2 + 4096;
constexpr u32 WS_G2 = WS_C2 + 8192 + 256;
constexpr u32 G2S = 512;
constexpr u32 WS_TK = WS_C2 + 13312;
constexpr u32 WS_TMO = WS_C2 + 20480;
constexpr u32 WS_DBG = WS_C2 + 32768;
constexpr u32 WS_FL = WS_C2 + 65536;
constexpr u32 WS_FC = WS_FL + 16384;
constexpr u32 WS_CALL = WS_FC + 16384;
constexpr u32 WS_Q = 54u << 20;
constexpr u32 QSTRIDE = 4160 * 256;
constexpr u32 WS_EX = 63u << 20;
constexpr u32 WS_ER = 67u << 20;
constexpr u32 WS_EW = 68u << 20;
constexpr u32 WS_L = 84u << 20;
constexpr u32 WS_RL = 86u << 20;
constexpr u32 WS_END = 97u << 20;
static_assert(64u * NWB * WBLK <= WS_X && WS_X + 64u * NRB * XBLK <= WS_CTL && WS_WF + NWB * FSTRIDE <= WS_C2 &&
              WS_CALL + 16384 <= WS_CTL + CNT_OFF && WS_C2 + CNT_OFF <= WS_Q && WS_Q + NCB * QSTRIDE <= WS_EX, "layout");
constexpr u32 FULL = 240;

constexpr u32 SLOT_BYTES = WBLK + XBLK;
constexpr u32 NSLOT = 3;
constexpr u32 TPITCH = 272;
constexpr u32 SNAP_OFF = SLOT_BYTES * NSLOT;
constexpr u32 CTL_OFF = SNAP_OFF + 1024;
constexpr u32 LDS_BYTES = CTL_OFF + 1024;
static_assert(LDS_BYTES <= 163840, "lds");
constexpr int C_AMAX = 0, C_GMAX = 1, C_CALL = 2, C_BAD = 3, C_STAGE = 4, C_GS = 5, C_GS8 = 6, C_X = 7, C_CNT = 8, C_FULL = 16, C_PROG = 24, C_QUEUE = 64, C_FLP = 208;

#define NPW 4
#define EXWAIT 300
#define EXD 60
#define PW1 4
#define PLD 4
#define POLL2 1
#define PUSHG 0
#define AMSC " sc1"
#define QWAIT 500
#define QD 100
#define SSTART 1750
#define SFAST 56
#define SPL 24
#define SPD 1
#define SPS 4
#define PRE 1
#define DIRECT 0
#define PUSHD 1
constexpr u32 NCOMMWG = 32 - NRB;
constexpr u32 NTILE = NRB * NCB;
constexpr u32 NPULL = NCOMMWG * NPW;
constexpr u32 U_B = 256, U_C = 640;
constexpr u32 PU = 96;
constexpr u32 NPOOL = 7 * PU / NCOMMWG;
constexpr u32 U_P1 = 256 + 7 * 384;
constexpr u32 U_R = U_P1 + NPOOL;
constexpr u32 U_Q = U_R + 224;
constexpr u32 U_L = U_Q + 3968 - 104;
constexpr u32 U_P3 = U_L + 832;
static_assert(WS_RL + 56 * PU * 2048 <= WS_END && 7 * PU % NCOMMWG == 0 && WS_TK + 4096 <= WS_TMO, "layout");
constexpr int NTHREADS = 768;
constexpr int NSTEP = 64;
constexpr u64 DEADLINE = 50000000ull;

struct Params {
  const char* x;
  const char* w;
  char* myws;
  char* out;
  u32 rank;
  u32 peers;
  u32 pad[6];
  char* dws[NR];
  char* ws[NR];
};

__device__ __forceinline__ __amdgpu_buffer_rsrc_t mkrsrc(const void* p, u32 bytes) {
  return __builtin_amdgcn_make_buffer_rsrc((void*)p, 0, bytes, 0x00020000);
}
__device__ __forceinline__ u64 now() { return __builtin_amdgcn_s_memrealtime(); }
__device__ __forceinline__ u32 ld_sys(const u32* p) { return __hip_atomic_load((GAS const u32*)p, __ATOMIC_RELAXED, __HIP_MEMORY_SCOPE_SYSTEM); }
__device__ __forceinline__ u64 ld_sys64(const u64* p) { return __hip_atomic_load((GAS const u64*)p, __ATOMIC_RELAXED, __HIP_MEMORY_SCOPE_SYSTEM); }
__device__ __forceinline__ void st_sys(u32* p, u32 v) { __hip_atomic_store((GAS u32*)p, v, __ATOMIC_RELAXED, __HIP_MEMORY_SCOPE_SYSTEM); }
__device__ __forceinline__ void st_sys64(u64* p, u64 v) { __hip_atomic_store((GAS u64*)p, v, __ATOMIC_RELAXED, __HIP_MEMORY_SCOPE_SYSTEM); }
__device__ __forceinline__ u32 lds_ld(LAS u32* p) { return __builtin_amdgcn_readfirstlane(__hip_atomic_load(p, __ATOMIC_RELAXED, __HIP_MEMORY_SCOPE_WORKGROUP)); }
__device__ __forceinline__ void lds_st(LAS u32* p, u32 v) { __hip_atomic_store(p, v, __ATOMIC_RELAXED, __HIP_MEMORY_SCOPE_WORKGROUP); }
__device__ __forceinline__ void lds_add1(LAS u32* p, u32 lane) {
  asm volatile("ds_add_u32 %0, %1" :: "v"((u32)(size_t)p), "v"(lane ? 0u : 1u) : "memory");
}
#define CFENCE() asm volatile("" ::: "memory")
#define WAIT_VM(n) asm volatile("s_waitcnt vmcnt(" #n ")" ::: "memory")
#define WAIT_VMI(n) asm volatile("s_waitcnt vmcnt(%0)" :: "i"(n) : "memory")
#define WAIT_LGKM() asm volatile("s_waitcnt lgkmcnt(0)" ::: "memory")
#define WAIT_VM0() do { CFENCE(); __builtin_amdgcn_s_waitcnt(0x0F70); WAIT_VM(0); } while (0)

__device__ __forceinline__ u32 wave_umax(u32 v) {
  v = max(v, (u32)__builtin_amdgcn_update_dpp(0, (int)v, 0x111, 0xf, 0xf, true));
  v = max(v, (u32)__builtin_amdgcn_update_dpp(0, (int)v, 0x112, 0xf, 0xf, true));
  v = max(v, (u32)__builtin_amdgcn_update_dpp(0, (int)v, 0x114, 0xf, 0xf, true));
  v = max(v, (u32)__builtin_amdgcn_update_dpp(0, (int)v, 0x118, 0xf, 0xf, true));
  v = max(v, (u32)__builtin_amdgcn_update_dpp(0, (int)v, 0x142, 0xa, 0xf, false));
  v = max(v, (u32)__builtin_amdgcn_update_dpp(0, (int)v, 0x143, 0xc, 0xf, false));
  return (u32)__builtin_amdgcn_readlane((int)v, 63);
}

#define STAMP(k) do {} while (0)

#define BARRIER() asm volatile("s_barrier" ::: "memory")

__global__ void __launch_bounds__(NTHREADS) fused_gemm_ar_q8(Params P) {
  __shared__ __attribute__((aligned(16))) char smem[LDS_BYTES];
  LAS char* L = (LAS char*)smem;
  LAS u32* ctl = (LAS u32*)(L + CTL_OFF);
  const u32 tid = threadIdx.x;
  u32 lane = tid & 63;
  const u32 wave = __builtin_amdgcn_readfirstlane(tid >> 6);
  u32 xcc;
  asm volatile("s_getreg_b32 %0, hwreg(HW_REG_XCC_ID, 0, 3)" : "=s"(xcc));
  const u32 g = xcc & 7u, idx = blockIdx.x >> 3, bid = idx * 8 + g;
  char* const myws = P.myws;
  char* const dws = P.dws[g];
  const u64 t0 = now();
  asm volatile("" :: "s"(P.x), "s"(P.w), "s"(myws), "s"(P.out), "s"(P.rank), "s"(P.peers), "s"(dws), "s"(t0));
  const u32 rank = P.rank & 7u;
  const u32 dest = (P.peers >> (g * 4)) & 7u;
  u32* const tmo = (u32*)(myws + WS_TMO);
  u32 lane16 = lane * 16;
  const u32 lx = (lane & 7) * 1024 + (lane >> 3) * 16;
  const u32 lw = (((lane >> 3) & 1) * 8 + ((lane >> 1) & 3)) * 32768 + ((lane & 1) * 4 + (lane >> 4)) * 16;
  u32 call = 0;
  bool bad = false;
  if (wave == 0) STAMP(0);

  if (idx >= NCOMMWG) {
    const u32 ti = idx - NCOMMWG;
    const u32 rb = (g & 1) * 13 + (ti >> 1);
    u32 cb = (g >> 1) * 2 + (ti & 1);
    const u32 tile = g * NRB + ti;
    const __amdgpu_buffer_rsrc_t rs_ws = mkrsrc(myws, WS_END);
    char* flp = P.ws[lane & 7] + WS_FL + rank * 256 + g * 8;
    u32 lane0 = lane ? 0x80000000u : 0u;
    auto signal_wg = [&](u32 kind, u32 inc) {
      const u32 old = __builtin_amdgcn_raw_ptr_buffer_atomic_add_i32(inc, rs_ws, lane0, WS_FC + kind * 2048 + g * 256, 16);
      if (__builtin_amdgcn_readfirstlane(old) + inc == call * NRB && lane < 8)
        st_sys64((u64*)(flp + kind * 2048), call);
      CFENCE();
    };
    auto signal_all = [&](u32 kind, u32 inc = 1) {
      u32 old = 0;
      if (lane == 0) old = __hip_atomic_fetch_add(ctl + C_CNT + kind, 1u, __ATOMIC_RELAXED, __HIP_MEMORY_SCOPE_WORKGROUP);
      if (__builtin_amdgcn_readfirstlane(old) == 7) signal_wg(kind, inc);
    };
    {
      const bool first = ti - 16 < 8;
      if (first) {
        if (wave < 8) {
          const u32 pc = (g * 8 + ti - 16) * 8 + wave;
          const u32x4 e0 = __builtin_amdgcn_raw_buffer_load_b128(mkrsrc(P.x, X_BYTES), lx, pc * 8192, 0);
          __builtin_amdgcn_raw_buffer_store_b128(e0, rs_ws, lane16, WS_EX + pc * 1024, 17);
          asm volatile("" : "+v"(flp));
          WAIT_VM0();
        }
        BARRIER();
      }
      if (wave == 0) {
        const u32 old = __builtin_amdgcn_readfirstlane(
            __builtin_amdgcn_raw_ptr_buffer_atomic_add_i32(1, rs_ws, lane0, WS_FC + (first ? 0 : 6 * 2048) + g * 256, 16));
        call = (first ? old >> 3 : old / (NRB - 8)) + 1;
        if (first && (old & 7) == 7 && lane < 8) {
          st_sys64((u64*)flp, ((u64)P.peers << 32) | call);
        }
        if (lane < 16) ctl[lane] = lane == C_CALL ? call : 0u;
        WAIT_LGKM();
      }
      if (wave == 6 && ti == 25) __builtin_amdgcn_fence(__ATOMIC_ACQUIRE, "");
      if (wave == 8) {
        ctl[16 + lane] = ctl[80 + lane] = ctl[144 + lane] = 0u;
        ((LAS u64*)(ctl + C_FLP))[lane & 7] = (u64)flp;
        WAIT_LGKM();
      }
      BARRIER();
      call = lds_ld(ctl + C_CALL);
      if (wave >= 8) {
        const bool two = ((ti >> 1) & 3) == (g >> 1);
        const u32 lds0 = (u32)(size_t)L, cv = call * FULL;
        __builtin_amdgcn_s_setprio(1);
        if (wave < 11) {
          const u32 mb = lds0 + (wave - 8) * SLOT_BYTES;
          for (u32 n = wave - 8; n < (two ? 2 * NSTEP : NSTEP); n += NSLOT) {
            if (n >= NSTEP && cb < 8) {
              BARRIER(); BARRIER(); BARRIER(); BARRIER();
              cb = 8 + (ti & 1);
            }
            LAS u32* const wp = ctl + (lane < 8 ? C_PROG + lane : C_QUEUE + n);
            const u32 mn = lane < 8 ? n - 2 : 1;
            u32 v;
            while (v = __hip_atomic_load(wp, __ATOMIC_RELAXED, __HIP_MEMORY_SCOPE_WORKGROUP), __ballot((int)(v - mn) >= 0) != ~0ull)
              __builtin_amdgcn_s_sleep(1);
            const u32 ord = __builtin_amdgcn_readlane(v, 8) - 1;
            const u32 sw = WS_W + (ord * NWB + cb) * WBLK, sx = WS_X + (ord * NRB + rb) * XBLK;
#pragma unroll
            for (u32 i = 0; i < 13; ++i)
              asm volatile("s_mov_b32 m0, %2\n\ts_nop 0\n\t"
                           "buffer_load_dwordx4 %0, %1, %3 offen lds\n\t"
                           "buffer_load_dwordx4 %0, %1, %3 offen offset:1024 lds\n\t"
                           "buffer_load_dwordx4 %0, %1, %3 offen offset:2048 lds\n\t"
                           "buffer_load_dwordx4 %0, %1, %3 offen offset:3072 lds"
                           :: "v"(lane16), "s"(rs_ws), "s"(i < 5 ? mb + WBLK + i * 4096 : mb + (i - 5) * 4096),
                              "s"(i < 5 ? sx + i * 4096 : sw + (i - 5) * 4096) : "memory");
            WAIT_VM(0);
            lds_st(ctl + C_FULL + (wave - 8), n + 1);
          }
          return;
        }
        LAS u32* const snap = (LAS u32*)(L + SNAP_OFF) + lane;
        u64 kx = 0;
        u32 qn = 0, sp = 0;
        while (now() - t0 < SSTART) __builtin_amdgcn_s_sleep(8);
        for (;;) {
          const u32 pollo = lane < 16 ? WS_XF + rb * FSTRIDE + lane16 : WS_WF + cb * FSTRIDE + lane16 - 256;
          u64 kw = 0, taken = 0;
          auto poll = [&]() {
            const u64 kn = lane < 16 ? kx : kw;
            const bool act = lane < 32 && ((u32)(kn >> ((lane & 15) * 4)) & 15u) != 15u;
            asm volatile("s_mov_b32 m0, %2\n\ts_nop 0\n\tbuffer_load_dwordx4 %0, %1, 0 offen sc0 sc1 lds"
                         :: "v"(act ? pollo : 0x80000000u), "s"(rs_ws), "s"(lds0 + SNAP_OFF) : "memory");
          };
          poll();
          WAIT_VM(0);
          for (;;) {
            u32 fx = snap[0], fw = snap[64];
            kx |= __ballot((int)(fx - cv) >= 0);
            kw |= __ballot((int)(fw - cv) >= 0);
            const u64 av = kx & kw & ~taken;
            if (av) {
              const u32 pos = __builtin_amdgcn_mbcnt_hi((u32)(av >> 32), __builtin_amdgcn_mbcnt_lo((u32)av, qn));
              if ((av >> lane) & 1) lds_st(ctl + C_QUEUE + pos, lane + 1);
              qn += __builtin_popcountll(av);
              taken |= av;
              if (taken == ~0ull) break;
            }
            poll();
            WAIT_VMI(SPD);
            __builtin_amdgcn_s_sleep(SPS);
            if (qn - SFAST >= (u32)NSTEP - SFAST) __builtin_amdgcn_s_sleep(SPL);
            if ((++sp & 63) == 0 && now() - t0 > DEADLINE) {
              st_sys(tmo, call);
              lds_st(ctl + C_BAD, 1u);
              kx = kw = ~0ull;
            }
          }
          if (cb < 8) STAMP(1);
          WAIT_VM0();
          WAIT_LGKM();
          if (cb >= 8 || !two) return;
          BARRIER(); BARRIER(); BARRIER(); BARRIER();
          cb = 8 + (ti & 1);
        }
      }
      for (u32 r = 0; r < 3; ++r) {
        const u32 slot = r * 8 + wave, e = slot * NTILE + tile;
        u32 kc, k, so, eo;
        bool isw = true;
        if (e - 512 < 896) {
          kc = (e - 384) >> 7; k = e & 127;
          isw = false;
          eo = WS_EX + kc * (512u << 10) + k * 4096;
        } else if (e < 4480) {
          k = (e < 512 ? e : e - 384) & 511;
          kc = 1 + ((((k >> 6) - rank - 1) & 7) + (e < 512 ? 1 : (e - 384) >> 9)) % 7;
          eo = WS_EW + ((k >> 6) * 8 + kc) * (8 * WBLK) + (k & 63) * 4096;
        } else {
          kc = (e - 4480) >> 4; k = ((e >> 1) & 7) * 64 + 62 + (e & 1);
          eo = WS_ER + (call & 1) * (512u << 10) + (e - 4480) * 4096;
        }
        if (e < 4608) {
          so = isw ? (kc * 64 + (k & 1) * 32) * 32768 + (k >> 1) * 128 : k * 32768 + kc * 128;
          const __amdgpu_buffer_rsrc_t rs = mkrsrc(isw ? P.w : P.x, isw ? W_BYTES : X_BYTES);
          const u32 lo = isw ? lw : lx, s1 = isw ? 4 * 32768 : 8192, s2 = isw ? 16 * 32768 : 16384;
          while (now() - t0 < EXWAIT + slot * EXD) __builtin_amdgcn_s_sleep(2);
          const u32x4 e0 = __builtin_amdgcn_raw_buffer_load_b128(rs, lo, so, 0);
          const u32x4 e1 = __builtin_amdgcn_raw_buffer_load_b128(rs, lo, so + s1, 0);
          const u32x4 e2 = __builtin_amdgcn_raw_buffer_load_b128(rs, lo, so + s2, 0);
          const u32x4 e3 = __builtin_amdgcn_raw_buffer_load_b128(rs, lo, so + s1 + s2, 0);
          __builtin_amdgcn_raw_buffer_store_b128(e0, rs_ws, lane16, eo, 17);
          __builtin_amdgcn_raw_buffer_store_b128(e1, rs_ws, lane16, eo + 1024, 17);
          __builtin_amdgcn_raw_buffer_store_b128(e2, rs_ws, lane16, eo + 2048, 17);
          __builtin_amdgcn_raw_buffer_store_b128(e3, rs_ws, lane16, eo + 3072, 17);
        }
        if (r != 1) {
          WAIT_VM0();
          signal_all(1 + (r >> 1));
        }
      }
    }
    const u32 wm = wave >> 2, wn = wave & 3;
    const u32 xa = WBLK + wm * 10240 + ((lane >> 3) & 1) * 1024 + (lane >> 5) * 256 + ((lane >> 4) & 1) * 128 + (lane & 7) * 16;
    const u32 h = lane >> 4, q = (lane >> 2) & 3, p = lane & 3;
    const u32 wa0 = wn * 8192 + (h >> 1) * 2048 + (p & 1) * 512 + (h & 1) * 128 + q * 32 + (p >> 1) * 16 + (p & 1) * 8;
    const u32 wa1 = wa0 ^ 8;
    f32x4 acc[4][5];
    auto uni = [&](u64 m) -> u64 {
      return (u64)(u32)__builtin_amdgcn_readfirstlane((u32)m) | ((u64)(u32)__builtin_amdgcn_readfirstlane((u32)(m >> 32)) << 32);
    };
#define SB() __builtin_amdgcn_sched_barrier(0)
#define MMA4(kb, i) do { \
        _Pragma("unroll") for (int n = 0; n < 4; ++n) \
          acc[n][i] = __builtin_amdgcn_mfma_f32_16x16x32_bf16(wf[kb][n], xf[kb][i], acc[n][i], 0, 0, 0); \
        SB(); } while (0)
    bf16x8 xf[2][5], wf[2][4];
    auto frags = [&](int kb, u32 slot) {
      LAS char* base = L + slot * SLOT_BYTES;
#pragma unroll
      for (int n = 0; n < 5; ++n) {
        if (n < 4) {
          LAS char* a = base + ((n & 1) ? wa1 : wa0) + kb * 4096 + (n >> 1) * 256;
          s16x4 lo = __builtin_amdgcn_ds_read_tr16_b64_v4i16((LAS s16x4*)a);
          s16x4 hi = __builtin_amdgcn_ds_read_tr16_b64_v4i16((LAS s16x4*)(a + 1024));
          wf[kb][n] = __builtin_bit_cast(bf16x8, __builtin_shufflevector(lo, hi, 0, 1, 2, 3, 4, 5, 6, 7));
        }
        if (n == 0) xf[kb][0] = *(LAS bf16x8*)(base + xa + kb * 512);
      }
#pragma unroll
      for (int n = 1; n < 5; ++n) xf[kb][n] = *(LAS bf16x8*)(base + xa + n * 2048 + kb * 512);
      SB();
    };
    u32 sp = 0, step = 0, slot = 0;
    u64 cyc0 = 0;
    for (u32 pass = 0;; ++pass) {
#pragma unroll
    for (int a = 0; a < 4; ++a)
#pragma unroll
      for (int b = 0; b < 5; ++b) acc[a][b] = f32x4{0.f, 0.f, 0.f, 0.f};
#pragma unroll
    for (int n = 0; n < 5; ++n) {
      xf[1][n] = __builtin_bit_cast(bf16x8, u32x4{0u, 0u, 0u, 0u});
      if (n < 4) wf[1][n] = __builtin_bit_cast(bf16x8, u32x4{0u, 0u, 0u, 0u});
    }
    bool pre = false;
    do {
      if (!pre) {
        MMA4(1, 1); MMA4(1, 2); MMA4(1, 3); MMA4(1, 4);
        while (lds_ld(ctl + C_FULL + slot) <= step) __builtin_amdgcn_s_sleep(1);
      }
      CFENCE();
      frags(0, slot);
      if (pre) { MMA4(1, 1); MMA4(1, 2); MMA4(1, 3); MMA4(1, 4); }
      frags(1, slot);
      slot = slot == NSLOT - 1 ? 0 : slot + 1;
      step += 1;
      MMA4(0, 0);
      MMA4(0, 1);
      MMA4(0, 2);
      MMA4(0, 3);
      const u32 nf = __hip_atomic_load(ctl + C_FULL + slot, __ATOMIC_RELAXED, __HIP_MEMORY_SCOPE_WORKGROUP);
      SB();
      MMA4(0, 4);
      WAIT_LGKM();
      lds_st(ctl + C_PROG + wave, step);
      SB();
      MMA4(1, 0);
      pre = PRE && __builtin_amdgcn_readfirstlane(nf) > step;
    } while (step & (NSTEP - 1));
    MMA4(1, 1); MMA4(1, 2); MMA4(1, 3); MMA4(1, 4);
    asm volatile("v_mbcnt_lo_u32_b32 %0, -1, 0\n\tv_mbcnt_hi_u32_b32 %0, -1, %0" : "=v"(lane));
    lane16 = lane * 16;
    lane0 = lane ? 0x80000000u : 0u;
    flp = (char*)((LAS u64*)(ctl + C_FLP))[lane & 7];
    asm volatile("" : "+s"(call));
    if (pass == 0) {
      if (wave == 0) STAMP(2);
      const u32 row0 = rb * 160 + wm * 80;
      float m = 0.f;
#pragma unroll
      for (int b = 0; b < 5; ++b) {
        float mb = 0.f;
#pragma unroll
        for (int a = 0; a < 4; ++a)
#pragma unroll
          for (int e = 0; e < 4; ++e) mb = fmaxf(mb, fabsf(acc[a][b][e]));
        if (row0 + b * 16 < 4096) m = fmaxf(m, mb);
      }
      const u32 mb = wave_umax(__builtin_bit_cast(u32, m));
      if (lane == 0) __hip_atomic_fetch_max(ctl + C_AMAX, mb, __ATOMIC_RELAXED, __HIP_MEMORY_SCOPE_WORKGROUP);
      WAIT_LGKM();
      BARRIER();
      if (wave == 0) {
        const u32 wmax = lds_ld(ctl + C_AMAX);
        u64 ret = 1;
        if (lane == 0)
          asm volatile("s_nop 4\n\tglobal_atomic_umax %1, %2, %3 offset:4" AMSC "\n\tglobal_atomic_add_x2 %0, %1, %0, %3 sc0" AMSC "\n\ts_waitcnt vmcnt(0)"
                       : "+v"(ret) : "v"(0u), "v"(wmax), "s"(myws + WS_G1 + g * 256) : "memory");
        ret = uni(ret);
        if ((u32)ret == NRB - 1) {
          st_sys64((u64*)(flp + (int)((lane >> 3) * G2S + WS_G2 - WS_FL - rank * 192)), ((u64)call << 32) | (u32)(ret >> 32));
          if (lane == 0) asm volatile("s_nop 4\n\tglobal_store_dwordx2 %0, %1, %2" AMSC :: "v"(0u), "v"(0ull), "s"(myws + WS_G1 + g * 256) : "memory");
        }
        STAMP(3);
        const u64* g2 = (const u64*)(myws + WS_G2 + g * G2S) + lane;
        u64 val;
        for (;;) {
          val = ld_sys64(g2);
          if (__all((u32)(val >> 32) == call)) break;
          if ((++sp & 63) == 0 && (ld_sys(tmo) == call || now() - t0 > DEADLINE)) { st_sys(tmo, call); bad = true; break; }
        }
        const u32 gmax = wave_umax((u32)val);
        if (lane == 0) {
          lds_st(ctl + C_GMAX, gmax);
          if (bad) lds_st(ctl + C_BAD, 1u);
        }
        WAIT_LGKM();
      }
    }
    BARRIER();
    const float amax = __builtin_bit_cast(float, lds_ld(ctl + C_GMAX));
    const float inv = amax > 0.f ? 1.0f / (amax / 127.0f) : 0.f;
    if (wave == 0 && pass == 0) STAMP(4);
    bad = lds_ld(ctl + C_BAD) != 0;
    const bool odd = (lane >> 4) & 1;
    u32x4 qv[5];
#pragma unroll
    for (int i = 0; i < 5; ++i) {
      u32 d[4];
#pragma unroll
      for (int n = 0; n < 4; ++n) {
        u32 b0 = __builtin_bit_cast(u32, __builtin_fmaf(acc[n][i][0], inv, 12582912.0f));
        u32 b1 = __builtin_bit_cast(u32, __builtin_fmaf(acc[n][i][1], inv, 12582912.0f));
        u32 b2 = __builtin_bit_cast(u32, __builtin_fmaf(acc[n][i][2], inv, 12582912.0f));
        u32 b3 = __builtin_bit_cast(u32, __builtin_fmaf(acc[n][i][3], inv, 12582912.0f));
        u32 lo = __builtin_amdgcn_perm(b1, b0, 0x0c0c0400u);
        u32 hi = __builtin_amdgcn_perm(b3, b2, 0x04000c0cu);
        d[n] = bad ? 0x80808080u : (lo | hi);
      }
      qv[i] = u32x4{odd ? d[1] : d[0], odd ? d[0] : d[1], odd ? d[3] : d[2], odd ? d[2] : d[3]};
    }
    const u64 tq = now();
    if (!pass && !(cb & 1) && wm == 0) {
      const __amdgpu_buffer_rsrc_t rs_p = mkrsrc(dws, WS_END);
#pragma unroll
      for (int i = 0; i < 4; ++i) {
        if (i && i % PUSHD == 0) WAIT_VM(0);
        if (wave == 0 && i == PUSHD) STAMP(10);
        __builtin_amdgcn_raw_buffer_store_b128(qv[i], rs_p, lane16, WS_L + ((rank * 13 + (ti >> 1)) * 16 + wn * 4 + i) * 1024, 17);
        CFENCE();
      }
      WAIT_VM0();
      if (wave == 0) STAMP(11);
      const u32 old = __builtin_amdgcn_raw_ptr_buffer_atomic_add_i32(1, rs_ws, lane0, WS_FC + 5 * 2048 + g * 256, 16);
      if (__builtin_amdgcn_readfirstlane(old) + 1 == call * 52 && lane == 0) st_sys64((u64*)(dws + WS_FL + 5 * 2048) + rank, ((u64)g << 32) | call);
      CFENCE();
    }
    {
      LAS char* tw = L + (wm * 80 + (lane & 15)) * TPITCH + wn * 64 + (lane >> 4) * 16;
#pragma unroll
      for (int i = 0; i < 5; ++i) *(LAS u32x4*)(tw + i * 16 * TPITCH) = qv[i];
    }
    WAIT_LGKM();
    BARRIER();
    LAS char* tr = L + (wave * 4 + (lane >> 4)) * TPITCH + (lane & 15) * 16;
#pragma unroll
    for (int i = 0; i < 5; ++i) qv[i] = *(LAS u32x4*)(tr + i * 32 * TPITCH);
    const __amdgpu_buffer_rsrc_t rs_o = mkrsrc(P.out, OUT_BYTES);
    const u32 orow = (rb * 160 + wave * 4 + (lane >> 4)) * 16384u;
    if (pass) {
      const u32 piece = (cb - 8) * 4 + ((lane >> 2) & 3);
      const u32 oo = piece == rank ? 0x80000000u : orow + piece * 2048 + 1984 + (lane & 3) * 16;
#pragma unroll
      for (int i = 0; i < 5; ++i) __builtin_amdgcn_raw_buffer_store_b128(qv[i], rs_o, oo, i * 32 * 16384, 17);
      break;
    }
    const u32 pitch = cb == 7 ? 192 : 256;
    const bool rep = (lane & 15) * 16 >= pitch;
    const u32 qo = rep ? 0x80000000u : (wave * 4 + (lane >> 4)) * pitch + (lane & 15) * 16;
    const u32 oo = orow + rank * 2048 + cb * 256 + (lane & 15) * 16;
    const u32 qa = WS_Q + cb * QSTRIDE + rb * 64 * pitch, qb = WS_Q + cb * QSTRIDE + (1664 + rb * 96) * pitch;
    if (cb & 1) {
      __builtin_amdgcn_raw_buffer_store_b128(qv[0], rs_ws, qo, qa, 17);
      __builtin_amdgcn_raw_buffer_store_b128(qv[1], rs_ws, qo, qa + 32 * pitch, 17);
      WAIT_VM0();
      signal_all(3, 2);
      if (wave == 0) STAMP(9);
    }
    while (now() - tq < QWAIT + wave * QD) __builtin_amdgcn_s_sleep(2);
    if (!(cb & 1)) {
      __builtin_amdgcn_raw_buffer_store_b128(qv[0], rs_ws, qo, qa, 17);
      __builtin_amdgcn_raw_buffer_store_b128(qv[1], rs_ws, qo, qa + 32 * pitch, 17);
    }
#pragma unroll
    for (int i = 2; i < 5; ++i) __builtin_amdgcn_raw_buffer_store_b128(qv[i], rs_ws, qo, qb + (i - 2) * 32 * pitch, 17);
    WAIT_VM0();
    signal_all(4);
    if (cb == 7) {
#pragma unroll
      for (int i = 0; i < 5; ++i) __builtin_amdgcn_raw_buffer_store_b128(qv[i], rs_o, rep ? oo : 0x80000000u, i * 32 * 16384, 17);
    }
    if (((ti >> 1) & 3) != (g >> 1)) break;
    cb = 8 + (ti & 1);
    WAIT_VM0();
    WAIT_LGKM();
    BARRIER();
    }
    WAIT_VM(0);
    if (wave == 0) STAMP(5);
    return;
  }

  const u32 vid = idx * NPW + wave;
  const __amdgpu_buffer_rsrc_t rs_s = mkrsrc(dws, WS_END);
  const u32 lane0 = lane ? 0x80000000u : 0u;
  static_assert(NPW == 4 && NCOMMWG == 6, "wave roles");
  if (wave >= 8) return;
  if (wave >= NPW && wave < 7) {
    const u32 ph = (wave - NPW) * NCOMMWG + idx;
    const __amdgpu_buffer_rsrc_t rs_w = mkrsrc(P.w, W_BYTES);
    const u32 so = ph < 16 ? dest * 4096 + ph * 256 : 0x80000000u;
    const u32 to = ph < 16 ? WS_W + rank * NWB * WBLK + ph * 16384 : 0x80000000u;
    u32x4 pv[16];
#pragma unroll
    for (int i = 0; i < 16; ++i) {
      pv[i] = __builtin_amdgcn_raw_buffer_load_b128(rs_w, lw, so + (i >> 3) * 128 + (((i >> 1) & 3) * 16 + (i & 1) * 4) * 32768, 0);
      if (i == PLD - 1) BARRIER();
    }
    CFENCE();
#pragma unroll
    for (int i = 0; i < 16; ++i) {
      if (i >= PW1) WAIT_VMI(PW1 - 1);
      __builtin_amdgcn_raw_buffer_store_b128(pv[i], rs_s, lane16, to + i * 1024, 17);
      CFENCE();
    }
    WAIT_VM(0);
    if (wave == NPW) STAMP(10);
    if (ph < 16) __builtin_amdgcn_raw_ptr_buffer_atomic_add_i32(FULL / 2, rs_s, lane0, WS_WF + (ph >> 1) * FSTRIDE + rank * 4, 16);
    return;
  }
  if (wave == 7) {
    ctl[lane] = 0u;
    WAIT_LGKM();
  }
  BARRIER();
  if (wave == 7) {
    u32* const cptr = (u32*)(myws + WS_CALL + bid * 256);
    call = __builtin_amdgcn_readfirstlane(ld_sys(cptr)) + 1;
    if (lane == 0) { lds_st(ctl + C_CALL, call); st_sys(cptr, call); }
    u32 sp = 0;
    auto expired = [&]() -> bool {
      if ((++sp & 63) == 0 && (ld_sys(tmo) == call || now() - t0 > DEADLINE)) {
        st_sys(tmo, call);
        if (lane == 0) { lds_st(ctl + C_BAD, 1u); lds_st(ctl + C_STAGE, 7u); }
        bad = true;
      }
      return bad;
    };
    u32 nx = 0;
    auto relay = [&]() {
      if (nx < 2 && lds_ld(ctl + C_X) >= NPW * (nx + 1)) {
        const u32 old = __builtin_amdgcn_raw_ptr_buffer_atomic_add_i32(1, mkrsrc(myws, WS_Q), lane0, WS_FC + 7 * 2048 + g * 256 + nx * 128, 16);
        if (__builtin_amdgcn_readfirstlane(old) + 1 == call * NCOMMWG && lane < 8)
          st_sys64((u64*)(P.ws[lane] + WS_FL + (6 + nx) * 2048 + rank * 256 + g * 8), call);
        nx += 1;
      }
    };
    auto watch = [&](u32 kind, u32 row, u32 st) {
      const u64* fp = (const u64*)(myws + WS_FL + kind * 2048 + row * 256) + (lane & 7);
      while (!bad) {
        const u64 f = ld_sys64(fp);
        const u64 ok = __ballot((int)((u32)f - call) >= 0);
        if (ok == ~0ull) {
          const u32 hi = (u32)(f >> 32), sh = (lane & 7) * 4;
          if (kind == 0 && lane < 8 && ((hi >> sh) & 7u) == rank) lds_st(ctl + C_GS, lane);
          if (kind == 5 && lane < 8) __hip_atomic_fetch_or(ctl + C_GS8, (hi & 7u) << sh, __ATOMIC_RELAXED, __HIP_MEMORY_SCOPE_WORKGROUP);
          break;
        }
        __builtin_amdgcn_s_sleep(1);
        if (kind > 2) relay();
        expired();
      }
      if (lane == 0 && !bad) lds_st(ctl + C_STAGE, st);
      CFENCE();
    };
    watch(0, dest, 1);
    STAMP(11);
    watch(1, dest, 2);
    watch(2, dest, 3);
    if (g) watch(6, dest, 4);
    watch(3, dest, 5);
    STAMP(9);
    watch(4, dest, 6);
    watch(g ? 7 : 5, g ? dest : 0, 7);
    while (nx < 2 && !expired()) {
      __builtin_amdgcn_s_sleep(8);
      relay();
    }
    return;
  }
  u32 stage = 0;
  auto need = [&](u32 s) {
    while (stage < s) {
      stage = lds_ld(ctl + C_STAGE);
      if (stage < s) __builtin_amdgcn_s_sleep(1);
    }
    CFENCE();
  };
  const __amdgpu_buffer_rsrc_t rs_m = mkrsrc(myws, WS_END);
  const __amdgpu_buffer_rsrc_t rs_o = mkrsrc(P.out, OUT_BYTES);
  const u32 jd = (rank - dest - 1) & 7;
  struct Unit { u32 so, to, fo, fw, rl; };
  auto xunit = [&](u32 src, u32 j, u32 kc, u32 k) -> Unit {
    const u32 kk = (k + (j * 26 / 7 % 26) * 10) & 255, r = kk / 10, ord = kc * 8 + src;
    return Unit{WS_EX + kc * (512u << 10) + kk * 2048, WS_X + ord * NRB * XBLK + kk * 2048, WS_XF + r * FSTRIDE + ord * 4 + (DIRECT && kc == 1 + j ? 0 : CNT_OFF), r == NRB - 1 ? 40u : 24u, 0x80000000u};
  };
  Unit d0 = xunit(dest, jd, 0, vid), d1 = xunit(dest, jd, 0, NPULL + vid);
  asm volatile("" : "+s"(d0.so), "+s"(d0.to), "+s"(d0.fo), "+s"(d0.fw), "+s"(d1.so), "+s"(d1.to), "+s"(d1.fo), "+s"(d1.fw));
  need(1);
  u32x4 v0 = __builtin_amdgcn_raw_buffer_load_b128(rs_s, lane16, d0.so, 17);
  u32x4 v1 = __builtin_amdgcn_raw_buffer_load_b128(rs_s, lane16, d0.so + 1024, 17);
  CFENCE();
  if (wave == 0) STAMP(3);
  call = lds_ld(ctl + C_CALL);
  constexpr u32 uq2 = U_Q + 3 * 208 + 156;
  const u32 nu = g ? U_P3 + NPOOL : U_P3;
  u32 qc = 0, qbase = U_Q;
  const u32 gs = lds_ld(ctl + C_GS);
  auto uoff = [&](u32 u, bool s3) -> Unit {
    Unit d;
    d.rl = 0x80000000u;
    if (!s3 && u < U_P1) {
      const u32 pos = (u + 128) / 384, k = u + 128 - pos * 384;
      const u32 kc = pos ? 1 + (jd + pos) % 7 : 0, ord = kc * 8 + dest;
      if (k < 128) {
        d.so = WS_EW + (rank * 8 + kc) * (8 * WBLK) + k * 2048;
        d.to = WS_W + ord * NWB * WBLK + k * 2048;
        d.fo = WS_WF + CNT_OFF + (k >> 4) * FSTRIDE + ord * 4;
        d.fw = 15;
      } else {
        d = xunit(dest, jd, kc, k - 128);
      }
    } else if (!s3 && u < U_R) {
      const u32 q = (u - U_P1) * NCOMMWG + idx, k = q / 7, j = q - k * 7;
      d = xunit((rank - 1 - j) & 7, j, 1 + j, 256 - PU + k);
      d.so = d.to;
    } else if (!s3 && u < U_Q) {
      const u32 k = u - U_R, kc = k / 28, j = k - kc * 28, ord = kc * 8 + dest;
      const u32 jj = j + ((j >> 2) >= rank ? 4 : 0);
      d.so = WS_ER + (call & 1) * (512u << 10) + kc * 65536 + jj * 2048;
      d.to = WS_W + (ord * NWB + 8) * WBLK + jj * 2048;
      d.fo = WS_WF + CNT_OFF + (8 + (jj >> 4)) * FSTRIDE + ord * 4;
      d.fw = (jj >> 4) == (rank >> 2) ? 20 : 15;
    } else if (u < U_L) {
      u32 c, na, n;
      for (;;) {
        c = (qc < 8 ? (gs & 6) + 2 + qc * 2 + (qc < 4) : jd + 1 + qc) & 7;
        na = c == 7 ? 156 : 208;
        n = qc < 7 ? na : qc < 8 ? 104 : c == 7 ? 228 : 304;
        if (u - qbase < n) break;
        qbase += n;
        qc += 1;
      }
      const u32 k = u - qbase + (qc >= 8 ? na : qc == 7 && !(gs & 1) ? 104 : 0);
      d.so = WS_Q + c * QSTRIDE + k * 2048;
      d.to = k * 128;
      d.fo = dest * 8 + c;
      d.fw = 0;
      if (k >= 512 - PU) d.rl = WS_RL + ((dest * 7 + c) * PU + k - (512 - PU)) * 2048;
    } else if (u < U_P3) {
      const u32 k = u - U_L, src = k / 104, r = k - src * 104;
      d.so = WS_L + k * 2048;
      d.to = ((r >> 3) * 160 + (r & 1) * 32) * 16384 + src * 2048 + (r & 6) * 32;
      d.fo = src & 7;
      d.fw = 0;
    } else {
      const u32 q = (u - U_P3) * NCOMMWG + idx, k = q / 7, j = q - k * 7, src = (rank - 1 - j) & 7;
      d.so = u < nu ? WS_RL + ((src * 7 + j) * PU + k) * 2048 : 0x80000000u;
      d.to = (512 - PU + k) * 128;
      d.fo = src * 8 + j;
      d.fw = 0;
    }
    return d;
  };
  u32 lim = g ? U_P1 - PU - NPULL : U_P1, tko = WS_TK + g * 128, tkb = 3 * NPULL - (call - 1) * (lim - 2 * NPULL);
  u32 u0 = vid, u1 = NPULL + vid, tk = 2 * NPULL + vid - tkb;
  u32 xb = g ? U_P1 - 384 : U_P1;
  u32 to0 = lane16, to1 = lane16 + 1024, tso = d0.to, lo1 = lane16;
  u32 ns = 1;
  const u32 cv = call * FULL;
  u32 cnt = 0, pfo = 0, pfw = 0;
  auto publish = [&]() {
    if (pfw && __builtin_amdgcn_readfirstlane(cnt) + pfw == cv)
      asm volatile("s_nop 4\n\tbuffer_store_dword %0, %1, %2, %3 offen sc0 sc1" :: "v"(cv), "v"(lane0), "s"(rs_m), "s"(pfo) : "memory");
    pfw = 0;
  };
  WAIT_VM0();
  auto iter = [&](const bool s3) {
    if (s3 && bad) v0 = v1 = u32x4{0x80808080u, 0x80808080u, 0x80808080u, 0x80808080u};
    __builtin_amdgcn_raw_buffer_store_b128(v0, s3 ? rs_o : rs_m, to0, tso, 17);
    __builtin_amdgcn_raw_buffer_store_b128(v1, s3 ? rs_o : rs_m, to1, tso, 17);
    CFENCE();
    if (s3) {
      __builtin_amdgcn_raw_buffer_store_b128(v0, rs_m, lane16, d0.rl, 17);
      __builtin_amdgcn_raw_buffer_store_b128(v1, rs_m, lane16, d0.rl + 1024, 17);
      CFENCE();
    }
    if (stage < ns) {
      stage = lds_ld(ctl + C_STAGE);
      if (stage < ns) {
        if (!s3) {
          publish();
          WAIT_VM0();
          cnt = __builtin_amdgcn_raw_ptr_buffer_atomic_add_i32(d0.fw, rs_m, lane0, d0.fo, 16);
          pfo = d0.fo - CNT_OFF; pfw = d0.fo < WS_CTL + CNT_OFF ? 0 : d0.fw;
          WAIT_VM0();
          publish();
          d0.fw = 0;
        }
        need(ns);
      }
      bad = lds_ld(ctl + C_BAD) != 0;
    }
    v0 = __builtin_amdgcn_raw_buffer_load_b128(rs_s, lo1, d1.so, 17);
    v1 = __builtin_amdgcn_raw_buffer_load_b128(rs_s, lo1, d1.so + 1024, 17);
    CFENCE();
    u32 u2 = __builtin_amdgcn_readfirstlane(tk) + tkb;
    if (u2 >= lim) {
      if (lim == nu || !lim) {
        u2 = nu;
        lim = 0;
        tko = 0x40000000u;
      } else if (lim == U_P1 || lim == U_R) {
        u2 = U_R + vid;
        lim = g ? U_L - PU - NPULL : nu;
        tko = WS_TK + 2048 + g * 128; tkb = U_R + NPULL - (call - 1) * (lim - U_R);
      } else {
        const u32 up = lim < U_P1 ? U_P1 : U_P3;
        tko = WS_TK + (lim < U_P1 ? 1024 : 3072) + idx * 128; tkb = up - (call - 1) * (NPOOL + 7 * NPW);
        lim = up + NPOOL;
      }
    }
    tk = __builtin_amdgcn_raw_ptr_buffer_atomic_add_i32(1, rs_m, lane0, tko, 16);
    CFENCE();
    if (s3 && u0 >= xb) {
      lds_add1(ctl + C_X, lane);
      xb = ~0u;
    }
    const Unit d2 = uoff(u2, s3);
    ns = u2 >= nu ? 0 : u2 >= U_L ? 7 : u2 >= uq2 ? 6 : u2 >= U_Q ? 5 : u2 >= U_R ? 3 : u2 >= U_P1 ? 4 : u2 >= U_C ? 3 : u2 >= U_B ? 2 : 1;
    tso = d1.to;
    if (u2 - U_L < U_P3 - U_L) lo1 = (lane & 3) * 256 + (lane >> 2) * 16;
    if (u1 - U_L < U_P3 - U_L) {
      const u32 gl = lds_ld(ctl + C_GS8) >> (d1.fo * 4);
      tso += (gl & 1) * (2080 * 16384) + (gl & 6) * 256;
      to0 = (lane >> 2) * 16384 + (lane & 3) * 16;
      to1 = to0 + 16 * 16384;
    } else if (s3 || u1 >= U_Q) {
      tso = d1.fo * 256;
      const u32 mul = (d1.fo & 7) == 7 ? 43691u : 32768u, w16 = (d1.fo & 7) == 7 ? 12 : 16;
#pragma unroll
      for (int e = 0; e < 2; ++e) {
        const u32 i16 = d1.to + e * 64 + lane;
        const u32 sr = (i16 * mul) >> 19, col = i16 - sr * w16;
        const u32 ra = (sr >> 6) * 96 + sr, rc = (((sr - 1664) * 43691u) >> 22) * 64 + sr - 1600;
        (e ? to1 : to0) = (sr < 1664 ? ra : rc) * 16384 + col * 16;
      }
    }
    tso = __builtin_amdgcn_readfirstlane(tso);
    CFENCE();
    if (!s3) {
      if (pfw && __builtin_amdgcn_readfirstlane(cnt) + pfw == cv)
        asm volatile("s_nop 4\n\tbuffer_store_dword %0, %1, %2, %3 offen sc0 sc1\n\ts_waitcnt vmcnt(4)" :: "v"(cv), "v"(lane0), "s"(rs_m), "s"(pfo) : "memory");
      else
        WAIT_VM(3);
      cnt = __builtin_amdgcn_raw_ptr_buffer_atomic_add_i32(d0.fw, rs_m, lane0, d0.fw ? d0.fo : 0x40000000u, 16);
      pfo = d0.fo - CNT_OFF; pfw = d0.fo < WS_CTL + CNT_OFF ? 0 : d0.fw;
      if (u1 >= xb) {
        lds_add1(ctl + C_X, lane);
        xb = ~0u;
      }
      CFENCE();
    }
    u0 = u1; u1 = u2;
    d0 = d1; d1 = d2;
  };
  while (u0 < U_Q) iter(false);
  publish();
  xb = g ? U_L - 304 : U_L;
  while (u0 < nu) iter(true);
  WAIT_VM(0);
  if (xb != ~0u) lds_add1(ctl + C_X, lane);
  if (wave == 0) STAMP(5);
}

extern "C" void kernel_launch(int rank, int nranks, ncclComm_t comm, hipStream_t stream,
                              void* const* d_in, const int* in_sizes, int n_in,
                              void* d_out, int out_size,
                              const int* in_dtypes, int out_dtype,
                              void* const* d_out_all,
                              void* const* d_ws_all, size_t ws_bytes) {
  if (nranks != NR || ws_bytes < WS_END || n_in < 2) { fprintf(stderr, "unsupported configuration\n"); return; }
  Params P = {};
  P.x = (const char*)d_in[0];
  P.w = (const char*)d_in[1];
  for (int p = 0; p < NR; ++p) P.ws[p] = (char*)d_ws_all[p];
  P.myws = P.ws[rank & 7];
  P.out = (char*)d_out;
  P.rank = rank;
  int logi[NR] = {0, 4, 1, 2, 5, 6, 3, 7}, phys[NR], grp[NR], ndev = 0, sum = 0;
  bool known = hipGetDeviceCount(&ndev) == hipSuccess && ndev == NR;
  for (int d = 0; d < NR && known; ++d) {
    char bus[32] = {0}, path[96];
    int pos = -1;
    known = hipDeviceGetPCIBusId(bus, sizeof bus, d) == hipSuccess;
    snprintf(path, sizeof path, "/sys/bus/pci/devices/%s/xgmi_physical_id", bus);
    FILE* f = known ? fopen(path, "r") : nullptr;
    known = f && fscanf(f, "%d", &pos) == 1 && pos >= 0;
    if (f) fclose(f);
    grp[d] = (pos ^ (pos >> 1)) & 1;
    sum += grp[d];
  }
  (void)hipGetLastError();
  if (known && sum == NR / 2)
    for (int d = 0, n[2] = {0, NR / 2}; d < NR; ++d) logi[d] = n[grp[d]]++;
  for (int d = 0; d < NR; ++d) phys[logi[d]] = d;
  P.peers = 0;
  for (int g = 0; g < NR; ++g) P.peers |= (u32)phys[logi[rank & 7] ^ g] << (g * 4);
  for (int g = 0; g < NR; ++g) P.dws[g] = P.ws[(P.peers >> (g * 4)) & 7];
  hipLaunchKernelGGL(fused_gemm_ar_q8, dim3(256), dim3(NTHREADS), 0, stream, P);
  HIP_CHECK(hipGetLastError());
}
```

```cpp
#include <hip/hip_runtime.h>
#include <rccl/rccl.h>
#include <cstdio>
#include <stdint.h>

#define HIP_CHECK(expr) do { hipError_t _e = (expr); if (_e != hipSuccess) { \
    fprintf(stderr, "HIP error %s at %s:%d\n", hipGetErrorString(_e), __FILE__, __LINE__); \
    return; } } while (0)

typedef unsigned u32;
typedef unsigned long long u64;
typedef u32 u32x4 __attribute__((ext_vector_type(4)));
typedef u32 u32x2 __attribute__((ext_vector_type(2)));
typedef float f32x4 __attribute__((ext_vector_type(4)));
typedef short s16x4 __attribute__((ext_vector_type(4)));
typedef short s16x8 __attribute__((ext_vector_type(8)));
typedef __bf16 bf16x8 __attribute__((ext_vector_type(8)));
#define LAS __attribute__((address_space(3)))
#define GAS __attribute__((address_space(1)))

constexpr int NR = 8;
constexpr u32 OUT_BYTES = 4096u * 16384u;
constexpr u32 X_BYTES = 4096u * 512u * 2u;
constexpr u32 W_BYTES = 512u * 16384u * 2u;

constexpr u32 NRB = 26;
constexpr u32 NCB = 8;
constexpr u32 XBLK = 20480, WBLK = 32768;
constexpr u32 NWB = NCB + 2;
constexpr u32 WS_W = 0;
constexpr u32 WS_X = 20u << 20;
constexpr u32 WS_CTL = 53u << 20;
constexpr u32 FSTRIDE = 4352;
constexpr u32 WS_XF = WS_CTL;
constexpr u32 WS_WF = WS_XF + NRB * FSTRIDE;
constexpr u32 CNT_OFF = 512u << 10;
constexpr u32 WS_C2 = WS_CTL + (160u << 10);
constexpr u32 WS_G1 = WS_C2 + 4096;
constexpr u32 WS_G2 = WS_C2 + 8192 + 256;
constexpr u32 G2S = 512;
constexpr u32 WS_TK = WS_C2 + 13312;
constexpr u32 WS_TMO = WS_C2 + 20480;
constexpr u32 WS_DBG = WS_C2 + 32768;
constexpr u32 WS_FL = WS_C2 + 65536;
constexpr u32 WS_FC = WS_FL + 16384;
constexpr u32 WS_CALL = WS_FC + 16384;
constexpr u32 WS_Q = 54u << 20;
constexpr u32 QSTRIDE = 4160 * 256;
constexpr u32 WS_EX = 63u << 20;
constexpr u32 WS_ER = 67u << 20;
constexpr u32 WS_EW = 68u << 20;
constexpr u32 WS_L = 84u << 20;
constexpr u32 WS_RL = 86u << 20;
constexpr u32 WS_END = 97u << 20;
static_assert(64u * NWB * WBLK <= WS_X && WS_X + 64u * NRB * XBLK <= WS_CTL && WS_WF + NWB * FSTRIDE <= WS_C2 &&
              WS_CALL + 16384 <= WS_CTL + CNT_OFF && WS_C2 + CNT_OFF <= WS_Q && WS_Q + NCB * QSTRIDE <= WS_EX, "layout");
constexpr u32 FULL = 240;

constexpr u32 SLOT_BYTES = WBLK + XBLK;
constexpr u32 NSLOT = 3;
constexpr u32 TPITCH = 272;
constexpr u32 SNAP_OFF = SLOT_BYTES * NSLOT;
constexpr u32 CTL_OFF = SNAP_OFF + 1024;
constexpr u32 LDS_BYTES = CTL_OFF + 1024;
static_assert(LDS_BYTES <= 163840, "lds");
constexpr int C_AMAX = 0, C_GMAX = 1, C_CALL = 2, C_BAD = 3, C_STAGE = 4, C_GS = 5, C_GS8 = 6, C_X = 7, C_CNT = 8, C_THR = 15, C_FULL = 16, C_PROG = 24, C_QUEUE = 64, C_FLP = 208;

#define NPW 4
#define EXWAIT 300
#define EXD 60
#define PW1 4
#define PLD 4
#define POLL2 1
#define PUSHG 0
#define AMSC " sc1"
#define QWAIT 500
#define QD 100
#define SSTART 1750
#define SFAST 56
#define SPL 24
#define SPD 1
#define SPS 4
#define PRE 1
#define DIRECT 0
#define PTH 1
#define THR 0
#define PTT 0
#define PTS 64
#define PUSHD 1
constexpr u32 NCOMMWG = 32 - NRB;
constexpr u32 NTILE = NRB * NCB;
constexpr u32 NPULL = NCOMMWG * NPW;
constexpr u32 U_B = 256, U_C = 640;
constexpr u32 PU = 96;
constexpr u32 NPOOL = 7 * PU / NCOMMWG;
constexpr u32 U_P1 = 256 + 7 * 384;
constexpr u32 U_R = U_P1 + NPOOL;
constexpr u32 U_Q = U_R + 224;
constexpr u32 U_L = U_Q + 3968 - 104;
constexpr u32 U_P3 = U_L + 832;
static_assert(WS_RL + 56 * PU * 2048 <= WS_END && 7 * PU % NCOMMWG == 0 && WS_TK + 4096 <= WS_TMO, "layout");
constexpr int NTHREADS = 768;
constexpr int NSTEP = 64;
constexpr u64 DEADLINE = 50000000ull;

struct Params {
  const char* x;
  const char* w;
  char* myws;
  char* out;
  u32 rank;
  u32 peers;
  u32 pad[6];
  char* dws[NR];
  char* ws[NR];
};

__device__ __forceinline__ __amdgpu_buffer_rsrc_t mkrsrc(const void* p, u32 bytes) {
  return __builtin_amdgcn_make_buffer_rsrc((void*)p, 0, bytes, 0x00020000);
}
__device__ __forceinline__ u64 now() { return __builtin_amdgcn_s_memrealtime(); }
__device__ __forceinline__ u32 ld_sys(const u32* p) { return __hip_atomic_load((GAS const u32*)p, __ATOMIC_RELAXED, __HIP_MEMORY_SCOPE_SYSTEM); }
__device__ __forceinline__ u64 ld_sys64(const u64* p) { return __hip_atomic_load((GAS const u64*)p, __ATOMIC_RELAXED, __HIP_MEMORY_SCOPE_SYSTEM); }
__device__ __forceinline__ void st_sys(u32* p, u32 v) { __hip_atomic_store((GAS u32*)p, v, __ATOMIC_RELAXED, __HIP_MEMORY_SCOPE_SYSTEM); }
__device__ __forceinline__ void st_sys64(u64* p, u64 v) { __hip_atomic_store((GAS u64*)p, v, __ATOMIC_RELAXED, __HIP_MEMORY_SCOPE_SYSTEM); }
__device__ __forceinline__ u32 lds_ld(LAS u32* p) { return __builtin_amdgcn_readfirstlane(__hip_atomic_load(p, __ATOMIC_RELAXED, __HIP_MEMORY_SCOPE_WORKGROUP)); }
__device__ __forceinline__ void lds_st(LAS u32* p, u32 v) { __hip_atomic_store(p, v, __ATOMIC_RELAXED, __HIP_MEMORY_SCOPE_WORKGROUP); }
__device__ __forceinline__ void lds_add1(LAS u32* p, u32 lane) {
  asm volatile("ds_add_u32 %0, %1" :: "v"((u32)(size_t)p), "v"(lane ? 0u : 1u) : "memory");
}
#define CFENCE() asm volatile("" ::: "memory")
#define WAIT_VM(n) asm volatile("s_waitcnt vmcnt(" #n ")" ::: "memory")
#define WAIT_VMI(n) asm volatile("s_waitcnt vmcnt(%0)" :: "i"(n) : "memory")
#define WAIT_LGKM() asm volatile("s_waitcnt lgkmcnt(0)" ::: "memory")
#define WAIT_VM0() do { CFENCE(); __builtin_amdgcn_s_waitcnt(0x0F70); WAIT_VM(0); } while (0)

__device__ __forceinline__ u32 wave_umax(u32 v) {
  v = max(v, (u32)__builtin_amdgcn_update_dpp(0, (int)v, 0x111, 0xf, 0xf, true));
  v = max(v, (u32)__builtin_amdgcn_update_dpp(0, (int)v, 0x112, 0xf, 0xf, true));
  v = max(v, (u32)__builtin_amdgcn_update_dpp(0, (int)v, 0x114, 0xf, 0xf, true));
  v = max(v, (u32)__builtin_amdgcn_update_dpp(0, (int)v, 0x118, 0xf, 0xf, true));
  v = max(v, (u32)__builtin_amdgcn_update_dpp(0, (int)v, 0x142, 0xa, 0xf, false));
  v = max(v, (u32)__builtin_amdgcn_update_dpp(0, (int)v, 0x143, 0xc, 0xf, false));
  return (u32)__builtin_amdgcn_readlane((int)v, 63);
}

#define STAMP(k) do {} while (0)

#define BARRIER() asm volatile("s_barrier" ::: "memory")

__global__ void __launch_bounds__(NTHREADS) fused_gemm_ar_q8(Params P) {
  __shared__ __attribute__((aligned(16))) char smem[LDS_BYTES];
  LAS char* L = (LAS char*)smem;
  LAS u32* ctl = (LAS u32*)(L + CTL_OFF);
  const u32 tid = threadIdx.x;
  u32 lane = tid & 63;
  const u32 wave = __builtin_amdgcn_readfirstlane(tid >> 6);
  u32 xcc;
  asm volatile("s_getreg_b32 %0, hwreg(HW_REG_XCC_ID, 0, 3)" : "=s"(xcc));
  const u32 g = xcc & 7u, idx = blockIdx.x >> 3, bid = idx * 8 + g;
  char* const myws = P.myws;
  char* const dws = P.dws[g];
  const u64 t0 = now();
  asm volatile("" :: "s"(P.x), "s"(P.w), "s"(myws), "s"(P.out), "s"(P.rank), "s"(P.peers), "s"(dws), "s"(t0));
  const u32 rank = P.rank & 7u;
  const u32 dest = (P.peers >> (g * 4)) & 7u;
  u32* const tmo = (u32*)(myws + WS_TMO);
  u32 lane16 = lane * 16;
  const u32 lx = (lane & 7) * 1024 + (lane >> 3) * 16;
  const u32 lw = (((lane >> 3) & 1) * 8 + ((lane >> 1) & 3)) * 32768 + ((lane & 1) * 4 + (lane >> 4)) * 16;
  u32 call = 0;
  bool bad = false;
  if (wave == 0) STAMP(0);

  if (idx >= NCOMMWG) {
    const u32 ti = idx - NCOMMWG;
    const u32 rb = (g & 1) * 13 + (ti >> 1);
    u32 cb = (g >> 1) * 2 + (ti & 1);
    const u32 tile = g * NRB + ti;
    const __amdgpu_buffer_rsrc_t rs_ws = mkrsrc(myws, WS_END);
    char* flp = P.ws[lane & 7] + WS_FL + rank * 256 + g * 8;
    u32 lane0 = lane ? 0x80000000u : 0u;
    auto signal_wg = [&](u32 kind, u32 inc) {
      const u32 old = __builtin_amdgcn_raw_ptr_buffer_atomic_add_i32(inc, rs_ws, lane0, WS_FC + kind * 2048 + g * 256, 16);
      if (__builtin_amdgcn_readfirstlane(old) + inc == call * NRB && lane < 8)
        st_sys64((u64*)(flp + kind * 2048), call);
      CFENCE();
    };
    auto signal_all = [&](u32 kind, u32 inc = 1) {
      u32 old = 0;
      if (lane == 0) old = __hip_atomic_fetch_add(ctl + C_CNT + kind, 1u, __ATOMIC_RELAXED, __HIP_MEMORY_SCOPE_WORKGROUP);
      if (__builtin_amdgcn_readfirstlane(old) == 7) signal_wg(kind, inc);
    };
    {
      const bool first = ti - 16 < 8;
      if (first) {
        if (wave < 8) {
          const u32 pc = (g * 8 + ti - 16) * 8 + wave;
          const u32x4 e0 = __builtin_amdgcn_raw_buffer_load_b128(mkrsrc(P.x, X_BYTES), lx, pc * 8192, 0);
          __builtin_amdgcn_raw_buffer_store_b128(e0, rs_ws, lane16, WS_EX + pc * 1024, 17);
          asm volatile("" : "+v"(flp));
          WAIT_VM0();
        }
        BARRIER();
      }
      if (wave == 0) {
        const u32 old = __builtin_amdgcn_readfirstlane(
            __builtin_amdgcn_raw_ptr_buffer_atomic_add_i32(1, rs_ws, lane0, WS_FC + (first ? 0 : 6 * 2048) + g * 256, 16));
        call = (first ? old >> 3 : old / (NRB - 8)) + 1;
        if (first && (old & 7) == 7 && lane < 8) {
          st_sys64((u64*)flp, ((u64)P.peers << 32) | call);
        }
        if (lane < 16) ctl[lane] = lane == C_CALL ? call : 0u;
        WAIT_LGKM();
      }
      if (wave == 6 && ti == 25) __builtin_amdgcn_fence(__ATOMIC_ACQUIRE, "");
      if (wave == 8) {
        ctl[16 + lane] = ctl[80 + lane] = ctl[144 + lane] = 0u;
        ((LAS u64*)(ctl + C_FLP))[lane & 7] = (u64)flp;
        WAIT_LGKM();
      }
      BARRIER();
      call = lds_ld(ctl + C_CALL);
      if (wave >= 8) {
        const bool two = ((ti >> 1) & 3) == (g >> 1);
        const u32 lds0 = (u32)(size_t)L, cv = call * FULL;
        __builtin_amdgcn_s_setprio(1);
        if (wave < 11) {
          const u32 mb = lds0 + (wave - 8) * SLOT_BYTES;
          for (u32 n = wave - 8; n < (two ? 2 * NSTEP : NSTEP); n += NSLOT) {
            if (n >= NSTEP && cb < 8) {
              BARRIER(); BARRIER(); BARRIER(); BARRIER();
              cb = 8 + (ti & 1);
            }
            LAS u32* const wp = ctl + (lane < 8 ? C_PROG + lane : C_QUEUE + n);
            const u32 mn = lane < 8 ? n - 2 : 1;
            u32 v;
            while (v = __hip_atomic_load(wp, __ATOMIC_RELAXED, __HIP_MEMORY_SCOPE_WORKGROUP), __ballot((int)(v - mn) >= 0) != ~0ull)
              __builtin_amdgcn_s_sleep(1);
            const u32 ord = __builtin_amdgcn_readlane(v, 8) - 1;
            const u32 sw = WS_W + (ord * NWB + cb) * WBLK, sx = WS_X + (ord * NRB + rb) * XBLK;
#pragma unroll
            for (u32 i = 0; i < 13; ++i)
              asm volatile("s_mov_b32 m0, %2\n\ts_nop 0\n\t"
                           "buffer_load_dwordx4 %0, %1, %3 offen lds\n\t"
                           "buffer_load_dwordx4 %0, %1, %3 offen offset:1024 lds\n\t"
                           "buffer_load_dwordx4 %0, %1, %3 offen offset:2048 lds\n\t"
                           "buffer_load_dwordx4 %0, %1, %3 offen offset:3072 lds"
                           :: "v"(lane16), "s"(rs_ws), "s"(i < 5 ? mb + WBLK + i * 4096 : mb + (i - 5) * 4096),
                              "s"(i < 5 ? sx + i * 4096 : sw + (i - 5) * 4096) : "memory");
            WAIT_VM(0);
            lds_st(ctl + C_FULL + (wave - 8), n + 1);
          }
          return;
        }
        LAS u32* const snap = (LAS u32*)(L + SNAP_OFF) + lane;
        u64 kx = 0;
        u32 qn = 0, sp = 0;
        while (now() - t0 < SSTART) __builtin_amdgcn_s_sleep(8);
        for (;;) {
          const u32 pollo = lane < 16 ? WS_XF + rb * FSTRIDE + lane16 : WS_WF + cb * FSTRIDE + lane16 - 256;
          u64 kw = 0, taken = 0;
          auto poll = [&]() {
            const u64 kn = lane < 16 ? kx : kw;
            const bool act = lane < 32 && ((u32)(kn >> ((lane & 15) * 4)) & 15u) != 15u;
            asm volatile("s_mov_b32 m0, %2\n\ts_nop 0\n\tbuffer_load_dwordx4 %0, %1, 0 offen sc0 sc1 lds"
                         :: "v"(act ? pollo : 0x80000000u), "s"(rs_ws), "s"(lds0 + SNAP_OFF) : "memory");
          };
          poll();
          WAIT_VM(0);
          for (;;) {
            u32 fx = snap[0], fw = snap[64];
            kx |= __ballot((int)(fx - cv) >= 0);
            kw |= __ballot((int)(fw - cv) >= 0);
            const u64 av = kx & kw & ~taken;
            if (av) {
              const u32 pos = __builtin_amdgcn_mbcnt_hi((u32)(av >> 32), __builtin_amdgcn_mbcnt_lo((u32)av, qn));
              if ((av >> lane) & 1) lds_st(ctl + C_QUEUE + pos, lane + 1);
              qn += __builtin_popcountll(av);
              taken |= av;
              if (taken == ~0ull) break;
            }
            poll();
            WAIT_VMI(SPD);
            __builtin_amdgcn_s_sleep(SPS);
            if (qn - SFAST >= (u32)NSTEP - SFAST) __builtin_amdgcn_s_sleep(SPL);
            if ((++sp & 63) == 0 && now() - t0 > DEADLINE) {
              st_sys(tmo, call);
              lds_st(ctl + C_BAD, 1u);
              kx = kw = ~0ull;
            }
          }
          if (cb < 8) STAMP(1);
          WAIT_VM0();
          WAIT_LGKM();
          if (cb >= 8 || !two) return;
          BARRIER(); BARRIER(); BARRIER(); BARRIER();
          cb = 8 + (ti & 1);
        }
      }
      for (u32 r = 0; r < 3; ++r) {
        const u32 slot = r * 8 + wave, e = slot * NTILE + tile;
        u32 kc, k, so, eo;
        bool isw = true;
        if (e - 512 < 896) {
          kc = (e - 384) >> 7; k = e & 127;
          isw = false;
          eo = WS_EX + kc * (512u << 10) + k * 4096;
        } else if (e < 4480) {
          k = (e < 512 ? e : e - 384) & 511;
          kc = 1 + ((((k >> 6) - rank - 1) & 7) + (e < 512 ? 1 : (e - 384) >> 9)) % 7;
          eo = WS_EW + ((k >> 6) * 8 + kc) * (8 * WBLK) + (k & 63) * 4096;
        } else {
          kc = (e - 4480) >> 4; k = ((e >> 1) & 7) * 64 + 62 + (e & 1);
          eo = WS_ER + (call & 1) * (512u << 10) + (e - 4480) * 4096;
        }
        if (e < 4608) {
          so = isw ? (kc * 64 + (k & 1) * 32) * 32768 + (k >> 1) * 128 : k * 32768 + kc * 128;
          const __amdgpu_buffer_rsrc_t rs = mkrsrc(isw ? P.w : P.x, isw ? W_BYTES : X_BYTES);
          const u32 lo = isw ? lw : lx, s1 = isw ? 4 * 32768 : 8192, s2 = isw ? 16 * 32768 : 16384;
          while (now() - t0 < EXWAIT + slot * EXD) __builtin_amdgcn_s_sleep(2);
          const u32x4 e0 = __builtin_amdgcn_raw_buffer_load_b128(rs, lo, so, 0);
          const u32x4 e1 = __builtin_amdgcn_raw_buffer_load_b128(rs, lo, so + s1, 0);
          const u32x4 e2 = __builtin_amdgcn_raw_buffer_load_b128(rs, lo, so + s2, 0);
          const u32x4 e3 = __builtin_amdgcn_raw_buffer_load_b128(rs, lo, so + s1 + s2, 0);
          __builtin_amdgcn_raw_buffer_store_b128(e0, rs_ws, lane16, eo, 17);
          __builtin_amdgcn_raw_buffer_store_b128(e1, rs_ws, lane16, eo + 1024, 17);
          __builtin_amdgcn_raw_buffer_store_b128(e2, rs_ws, lane16, eo + 2048, 17);
          __builtin_amdgcn_raw_buffer_store_b128(e3, rs_ws, lane16, eo + 3072, 17);
        }
        if (r != 1) {
          WAIT_VM0();
          signal_all(1 + (r >> 1));
        }
      }
    }
    const u32 wm = wave >> 2, wn = wave & 3;
    const u32 xa = WBLK + wm * 10240 + ((lane >> 3) & 1) * 1024 + (lane >> 5) * 256 + ((lane >> 4) & 1) * 128 + (lane & 7) * 16;
    const u32 h = lane >> 4, q = (lane >> 2) & 3, p = lane & 3;
    const u32 wa0 = wn * 8192 + (h >> 1) * 2048 + (p & 1) * 512 + (h & 1) * 128 + q * 32 + (p >> 1) * 16 + (p & 1) * 8;
    const u32 wa1 = wa0 ^ 8;
    f32x4 acc[4][5];
    auto uni = [&](u64 m) -> u64 {
      return (u64)(u32)__builtin_amdgcn_readfirstlane((u32)m) | ((u64)(u32)__builtin_amdgcn_readfirstlane((u32)(m >> 32)) << 32);
    };
#define SB() __builtin_amdgcn_sched_barrier(0)
#define MMA4(kb, i) do { \
        _Pragma("unroll") for (int n = 0; n < 4; ++n) \
          acc[n][i] = __builtin_amdgcn_mfma_f32_16x16x32_bf16(wf[kb][n], xf[kb][i], acc[n][i], 0, 0, 0); \
        SB(); } while (0)
    bf16x8 xf[2][5], wf[2][4];
    auto frags = [&](int kb, u32 slot) {
      LAS char* base = L + slot * SLOT_BYTES;
#pragma unroll
      for (int n = 0; n < 5; ++n) {
        if (n < 4) {
          LAS char* a = base + ((n & 1) ? wa1 : wa0) + kb * 4096 + (n >> 1) * 256;
          s16x4 lo = __builtin_amdgcn_ds_read_tr16_b64_v4i16((LAS s16x4*)a);
          s16x4 hi = __builtin_amdgcn_ds_read_tr16_b64_v4i16((LAS s16x4*)(a + 1024));
          wf[kb][n] = __builtin_bit_cast(bf16x8, __builtin_shufflevector(lo, hi, 0, 1, 2, 3, 4, 5, 6, 7));
        }
        if (n == 0) xf[kb][0] = *(LAS bf16x8*)(base + xa + kb * 512);
      }
#pragma unroll
      for (int n = 1; n < 5; ++n) xf[kb][n] = *(LAS bf16x8*)(base + xa + n * 2048 + kb * 512);
      SB();
    };
    u32 sp = 0, step = 0, slot = 0;
    u64 cyc0 = 0;
    for (u32 pass = 0;; ++pass) {
#pragma unroll
    for (int a = 0; a < 4; ++a)
#pragma unroll
      for (int b = 0; b < 5; ++b) acc[a][b] = f32x4{0.f, 0.f, 0.f, 0.f};
#pragma unroll
    for (int n = 0; n < 5; ++n) {
      xf[1][n] = __builtin_bit_cast(bf16x8, u32x4{0u, 0u, 0u, 0u});
      if (n < 4) wf[1][n] = __builtin_bit_cast(bf16x8, u32x4{0u, 0u, 0u, 0u});
    }
    bool pre = false;
    do {
      if (!pre) {
        MMA4(1, 1); MMA4(1, 2); MMA4(1, 3); MMA4(1, 4);
        while (lds_ld(ctl + C_FULL + slot) <= step) __builtin_amdgcn_s_sleep(1);
      }
      CFENCE();
      frags(0, slot);
      if (pre) { MMA4(1, 1); MMA4(1, 2); MMA4(1, 3); MMA4(1, 4); }
      frags(1, slot);
      slot = slot == NSLOT - 1 ? 0 : slot + 1;
      step += 1;
      MMA4(0, 0);
      MMA4(0, 1);
      MMA4(0, 2);
      MMA4(0, 3);
      const u32 nf = __hip_atomic_load(ctl + C_FULL + slot, __ATOMIC_RELAXED, __HIP_MEMORY_SCOPE_WORKGROUP);
      SB();
      MMA4(0, 4);
      WAIT_LGKM();
      lds_st(ctl + C_PROG + wave, step);
      SB();
      MMA4(1, 0);
      pre = PRE && __builtin_amdgcn_readfirstlane(nf) > step;
    } while (step & (NSTEP - 1));
    MMA4(1, 1); MMA4(1, 2); MMA4(1, 3); MMA4(1, 4);
    asm volatile("v_mbcnt_lo_u32_b32 %0, -1, 0\n\tv_mbcnt_hi_u32_b32 %0, -1, %0" : "=v"(lane));
    lane16 = lane * 16;
    lane0 = lane ? 0x80000000u : 0u;
    flp = (char*)((LAS u64*)(ctl + C_FLP))[lane & 7];
    asm volatile("" : "+s"(call));
    if (pass == 0) {
      if (wave == 0) STAMP(2);
      const u32 row0 = rb * 160 + wm * 80;
      float m = 0.f;
#pragma unroll
      for (int b = 0; b < 5; ++b) {
        float mb = 0.f;
#pragma unroll
        for (int a = 0; a < 4; ++a)
#pragma unroll
          for (int e = 0; e < 4; ++e) mb = fmaxf(mb, fabsf(acc[a][b][e]));
        if (row0 + b * 16 < 4096) m = fmaxf(m, mb);
      }
      const u32 mb = wave_umax(__builtin_bit_cast(u32, m));
      if (lane == 0) __hip_atomic_fetch_max(ctl + C_AMAX, mb, __ATOMIC_RELAXED, __HIP_MEMORY_SCOPE_WORKGROUP);
      WAIT_LGKM();
      BARRIER();
      if (wave == 0) {
        const u32 wmax = lds_ld(ctl + C_AMAX);
        u64 ret = 1;
        if (lane == 0)
          asm volatile("s_nop 4\n\tglobal_atomic_umax %1, %2, %3 offset:4" AMSC "\n\tglobal_atomic_add_x2 %0, %1, %0, %3 sc0" AMSC "\n\ts_waitcnt vmcnt(0)"
                       : "+v"(ret) : "v"(0u), "v"(wmax), "s"(myws + WS_G1 + g * 256) : "memory");
        ret = uni(ret);
        if ((u32)ret == NRB - 1) {
          st_sys64((u64*)(flp + (int)((lane >> 3) * G2S + WS_G2 - WS_FL - rank * 192)), ((u64)call << 32) | (u32)(ret >> 32));
          if (lane == 0) asm volatile("s_nop 4\n\tglobal_store_dwordx2 %0, %1, %2" AMSC :: "v"(0u), "v"(0ull), "s"(myws + WS_G1 + g * 256) : "memory");
        }
        STAMP(3);
        const u64* g2 = (const u64*)(myws + WS_G2 + g * G2S) + lane;
        u64 val;
        for (;;) {
          val = ld_sys64(g2);
          if (__all((u32)(val >> 32) == call)) break;
          if ((++sp & 63) == 0 && (ld_sys(tmo) == call || now() - t0 > DEADLINE)) { st_sys(tmo, call); bad = true; break; }
        }
        const u32 gmax = wave_umax((u32)val);
        if (lane == 0) {
          lds_st(ctl + C_GMAX, gmax);
          if (bad) lds_st(ctl + C_BAD, 1u);
        }
        WAIT_LGKM();
      }
    }
    BARRIER();
    const float amax = __builtin_bit_cast(float, lds_ld(ctl + C_GMAX));
    const float inv = amax > 0.f ? 1.0f / (amax / 127.0f) : 0.f;
    if (wave == 0 && pass == 0) STAMP(4);
    bad = lds_ld(ctl + C_BAD) != 0;
    const bool odd = (lane >> 4) & 1;
    u32x4 qv[5];
#pragma unroll
    for (int i = 0; i < 5; ++i) {
      u32 d[4];
#pragma unroll
      for (int n = 0; n < 4; ++n) {
        u32 b0 = __builtin_bit_cast(u32, __builtin_fmaf(acc[n][i][0], inv, 12582912.0f));
        u32 b1 = __builtin_bit_cast(u32, __builtin_fmaf(acc[n][i][1], inv, 12582912.0f));
        u32 b2 = __builtin_bit_cast(u32, __builtin_fmaf(acc[n][i][2], inv, 12582912.0f));
        u32 b3 = __builtin_bit_cast(u32, __builtin_fmaf(acc[n][i][3], inv, 12582912.0f));
        u32 lo = __builtin_amdgcn_perm(b1, b0, 0x0c0c0400u);
        u32 hi = __builtin_amdgcn_perm(b3, b2, 0x04000c0cu);
        d[n] = bad ? 0x80808080u : (lo | hi);
      }
      qv[i] = u32x4{odd ? d[1] : d[0], odd ? d[0] : d[1], odd ? d[3] : d[2], odd ? d[2] : d[3]};
    }
    const u64 tq = now();
    if (!pass && !(cb & 1) && wm == 0) {
      const __amdgpu_buffer_rsrc_t rs_p = mkrsrc(dws, WS_END);
#pragma unroll
      for (int i = 0; i < 4; ++i) {
        if (i && i % PUSHD == 0) WAIT_VM(0);
        if (wave == 0 && i == PUSHD) STAMP(10);
        __builtin_amdgcn_raw_buffer_store_b128(qv[i], rs_p, lane16, WS_L + ((rank * 13 + (ti >> 1)) * 16 + wn * 4 + i) * 1024, 17);
        CFENCE();
      }
      WAIT_VM0();
      if (wave == 0) STAMP(11);
      const u32 old = __builtin_amdgcn_raw_ptr_buffer_atomic_add_i32(1, rs_ws, lane0, WS_FC + 5 * 2048 + g * 256, 16);
      if (__builtin_amdgcn_readfirstlane(old) + 1 == call * 52 && lane == 0) st_sys64((u64*)(dws + WS_FL + 5 * 2048) + rank, ((u64)g << 32) | call);
      CFENCE();
    }
    {
      LAS char* tw = L + (wm * 80 + (lane & 15)) * TPITCH + wn * 64 + (lane >> 4) * 16;
#pragma unroll
      for (int i = 0; i < 5; ++i) *(LAS u32x4*)(tw + i * 16 * TPITCH) = qv[i];
    }
    WAIT_LGKM();
    BARRIER();
    LAS char* tr = L + (wave * 4 + (lane >> 4)) * TPITCH + (lane & 15) * 16;
#pragma unroll
    for (int i = 0; i < 5; ++i) qv[i] = *(LAS u32x4*)(tr + i * 32 * TPITCH);
    const __amdgpu_buffer_rsrc_t rs_o = mkrsrc(P.out, OUT_BYTES);
    const u32 orow = (rb * 160 + wave * 4 + (lane >> 4)) * 16384u;
    if (pass) {
      const u32 piece = (cb - 8) * 4 + ((lane >> 2) & 3);
      const u32 oo = piece == rank ? 0x80000000u : orow + piece * 2048 + 1984 + (lane & 3) * 16;
#pragma unroll
      for (int i = 0; i < 5; ++i) __builtin_amdgcn_raw_buffer_store_b128(qv[i], rs_o, oo, i * 32 * 16384, 17);
      break;
    }
    const u32 pitch = cb == 7 ? 192 : 256;
    const bool rep = (lane & 15) * 16 >= pitch;
    const u32 qo = rep ? 0x80000000u : (wave * 4 + (lane >> 4)) * pitch + (lane & 15) * 16;
    const u32 oo = orow + rank * 2048 + cb * 256 + (lane & 15) * 16;
    const u32 qa = WS_Q + cb * QSTRIDE + rb * 64 * pitch, qb = WS_Q + cb * QSTRIDE + (1664 + rb * 96) * pitch;
    if (cb & 1) {
      __builtin_amdgcn_raw_buffer_store_b128(qv[0], rs_ws, qo, qa, 17);
      __builtin_amdgcn_raw_buffer_store_b128(qv[1], rs_ws, qo, qa + 32 * pitch, 17);
      WAIT_VM0();
      signal_all(3, 2);
      if (wave == 0) STAMP(9);
    }
    while (now() - tq < QWAIT + wave * QD) __builtin_amdgcn_s_sleep(2);
    if (!(cb & 1)) {
      __builtin_amdgcn_raw_buffer_store_b128(qv[0], rs_ws, qo, qa, 17);
      __builtin_amdgcn_raw_buffer_store_b128(qv[1], rs_ws, qo, qa + 32 * pitch, 17);
    }
#pragma unroll
    for (int i = 2; i < 5; ++i) __builtin_amdgcn_raw_buffer_store_b128(qv[i], rs_ws, qo, qb + (i - 2) * 32 * pitch, 17);
    WAIT_VM0();
    signal_all(4);
    if (cb == 7) {
#pragma unroll
      for (int i = 0; i < 5; ++i) __builtin_amdgcn_raw_buffer_store_b128(qv[i], rs_o, rep ? oo : 0x80000000u, i * 32 * 16384, 17);
    }
    if (((ti >> 1) & 3) != (g >> 1)) break;
    cb = 8 + (ti & 1);
    WAIT_VM0();
    WAIT_LGKM();
    BARRIER();
    }
    WAIT_VM(0);
    if (wave == 0) STAMP(5);
    return;
  }

  const u32 vid = idx * NPW + wave;
  const __amdgpu_buffer_rsrc_t rs_s = mkrsrc(dws, WS_END);
  const u32 lane0 = lane ? 0x80000000u : 0u;
  static_assert(NPW == 4 && NCOMMWG == 6, "wave roles");
  if (wave >= 8) return;
  if (wave >= NPW && wave < 7) {
    const u32 ph = (wave - NPW) * NCOMMWG + idx;
    const __amdgpu_buffer_rsrc_t rs_w = mkrsrc(P.w, W_BYTES);
    const u32 so = ph < 16 ? dest * 4096 + ph * 256 : 0x80000000u;
    const u32 to = ph < 16 ? WS_W + rank * NWB * WBLK + ph * 16384 : 0x80000000u;
    u32x4 pv[16];
#pragma unroll
    for (int i = 0; i < 16; ++i) {
      pv[i] = __builtin_amdgcn_raw_buffer_load_b128(rs_w, lw, so + (i >> 3) * 128 + (((i >> 1) & 3) * 16 + (i & 1) * 4) * 32768, 0);
      if (i == PLD - 1) BARRIER();
    }
    CFENCE();
#pragma unroll
    for (int i = 0; i < 16; ++i) {
      if (i >= PW1) WAIT_VMI(PW1 - 1);
      if (i >= PTH && lds_ld(ctl + (THR ? C_THR : C_STAGE)) && now() - t0 >= PTT) {
        WAIT_VMI(PTH - 1);
        for (int k = 0; k < PTS; k += 64) __builtin_amdgcn_s_sleep(PTS < 64 ? PTS : 64);
      }
      __builtin_amdgcn_raw_buffer_store_b128(pv[i], rs_s, lane16, to + i * 1024, 17);
      asm volatile("s_nop 1" ::: "memory");
    }
    WAIT_VM(0);
    if (wave == NPW) STAMP(10);
    if (ph < 16) __builtin_amdgcn_raw_ptr_buffer_atomic_add_i32(FULL / 2, rs_s, lane0, WS_WF + (ph >> 1) * FSTRIDE + rank * 4, 16);
    return;
  }
  if (wave == 7) {
    ctl[lane] = 0u;
    WAIT_LGKM();
  }
  BARRIER();
  if (wave == 7) {
    u32* const cptr = (u32*)(myws + WS_CALL + bid * 256);
    u64 f = ld_sys64((const u64*)(myws + WS_FL + dest * 256) + (lane & 7));
    const u32* const wp = (const u32*)(myws + WS_WF + idx * FSTRIDE) + dest;
    u32 wv = THR ? ld_sys(wp) : 0;
    bool thr = !THR;
    call = __builtin_amdgcn_readfirstlane(ld_sys(cptr)) + 1;
    if (lane == 0) { lds_st(ctl + C_CALL, call); st_sys(cptr, call); }
    u32 sp = 0;
    auto expired = [&]() -> bool {
      if ((++sp & 63) == 0 && (ld_sys(tmo) == call || now() - t0 > DEADLINE)) {
        st_sys(tmo, call);
        if (lane == 0) { lds_st(ctl + C_BAD, 1u); lds_st(ctl + C_STAGE, 7u); }
        bad = true;
      }
      return bad;
    };
    u32 nx = 0;
    auto relay = [&]() {
      if (nx < 2 && lds_ld(ctl + C_X) >= NPW * (nx + 1)) {
        const u32 old = __builtin_amdgcn_raw_ptr_buffer_atomic_add_i32(1, mkrsrc(myws, WS_Q), lane0, WS_FC + 7 * 2048 + g * 256 + nx * 128, 16);
        if (__builtin_amdgcn_readfirstlane(old) + 1 == call * NCOMMWG && lane < 8)
          st_sys64((u64*)(P.ws[lane] + WS_FL + (6 + nx) * 2048 + rank * 256 + g * 8), call);
        nx += 1;
      }
    };
    auto watch = [&](u32 kind, u32 row, u32 st) {
      const u64* fp = (const u64*)(myws + WS_FL + kind * 2048 + row * 256) + (lane & 7);
      if (kind) f = ld_sys64(fp);
      while (!bad) {
        const u64 ok = __ballot((int)((u32)f - call) >= 0);
        if (kind < 2 && !thr && (kind || ok == ~0ull) && (int)(__builtin_amdgcn_readfirstlane(wv) - (call - 1) * FULL) >= THR) {
          thr = true;
          lds_st(ctl + C_THR, 1u);
        }
        if (ok == ~0ull) {
          const u32 hi = (u32)(f >> 32), sh = (lane & 7) * 4;
          if (kind == 0 && lane < 8 && ((hi >> sh) & 7u) == rank) lds_st(ctl + C_GS, lane);
          if (kind == 5 && lane < 8) __hip_atomic_fetch_or(ctl + C_GS8, (hi & 7u) << sh, __ATOMIC_RELAXED, __HIP_MEMORY_SCOPE_WORKGROUP);
          break;
        }
        __builtin_amdgcn_s_sleep(1);
        if (kind > 2) relay();
        expired();
        f = ld_sys64(fp);
        if (kind < 2 && !thr) wv = ld_sys(wp);
      }
      if (lane == 0 && !bad) lds_st(ctl + C_STAGE, st);
      CFENCE();
    };
    watch(0, dest, 1);
    STAMP(11);
    watch(1, dest, 2);
    watch(2, dest, 3);
    if (g) watch(6, dest, 4);
    watch(3, dest, 5);
    STAMP(9);
    watch(4, dest, 6);
    watch(g ? 7 : 5, g ? dest : 0, 7);
    while (nx < 2 && !expired()) {
      __builtin_amdgcn_s_sleep(8);
      relay();
    }
    return;
  }
  u32 stage = 0;
  auto need = [&](u32 s) {
    while (stage < s) {
      stage = lds_ld(ctl + C_STAGE);
      if (stage < s) __builtin_amdgcn_s_sleep(1);
    }
    CFENCE();
  };
  const __amdgpu_buffer_rsrc_t rs_m = mkrsrc(myws, WS_END);
  const __amdgpu_buffer_rsrc_t rs_o = mkrsrc(P.out, OUT_BYTES);
  const u32 jd = (rank - dest - 1) & 7;
  struct Unit { u32 so, to, fo, fw, rl; };
  auto xunit = [&](u32 src, u32 j, u32 kc, u32 k) -> Unit {
    const u32 kk = (k + (j * 26 / 7 % 26) * 10) & 255, r = kk / 10, ord = kc * 8 + src;
    return Unit{WS_EX + kc * (512u << 10) + kk * 2048, WS_X + ord * NRB * XBLK + kk * 2048, WS_XF + r * FSTRIDE + ord * 4 + (DIRECT && kc == 1 + j ? 0 : CNT_OFF), r == NRB - 1 ? 40u : 24u, 0x80000000u};
  };
  Unit d0 = xunit(dest, jd, 0, vid), d1 = xunit(dest, jd, 0, NPULL + vid);
  asm volatile("" : "+s"(d0.so), "+s"(d0.to), "+s"(d0.fo), "+s"(d0.fw), "+s"(d1.so), "+s"(d1.to), "+s"(d1.fo), "+s"(d1.fw));
  need(1);
  u32x4 v0 = __builtin_amdgcn_raw_buffer_load_b128(rs_s, lane16, d0.so, 17);
  u32x4 v1 = __builtin_amdgcn_raw_buffer_load_b128(rs_s, lane16, d0.so + 1024, 17);
  CFENCE();
  if (wave == 0) STAMP(3);
  call = lds_ld(ctl + C_CALL);
  constexpr u32 uq2 = U_Q + 3 * 208 + 156;
  const u32 nu = g ? U_P3 + NPOOL : U_P3;
  u32 qc = 0, qbase = U_Q;
  const u32 gs = lds_ld(ctl + C_GS);
  auto uoff = [&](u32 u, bool s3) -> Unit {
    Unit d;
    d.rl = 0x80000000u;
    if (!s3 && u < U_P1) {
      const u32 pos = (u + 128) / 384, k = u + 128 - pos * 384;
      const u32 kc = pos ? 1 + (jd + pos) % 7 : 0, ord = kc * 8 + dest;
      if (k < 128) {
        d.so = WS_EW + (rank * 8 + kc) * (8 * WBLK) + k * 2048;
        d.to = WS_W + ord * NWB * WBLK + k * 2048;
        d.fo = WS_WF + CNT_OFF + (k >> 4) * FSTRIDE + ord * 4;
        d.fw = 15;
      } else {
        d = xunit(dest, jd, kc, k - 128);
      }
    } else if (!s3 && u < U_R) {
      const u32 q = (u - U_P1) * NCOMMWG + idx, k = q / 7, j = q - k * 7;
      d = xunit((rank - 1 - j) & 7, j, 1 + j, 256 - PU + k);
      d.so = d.to;
    } else if (!s3 && u < U_Q) {
      const u32 k = u - U_R, kc = k / 28, j = k - kc * 28, ord = kc * 8 + dest;
      const u32 jj = j + ((j >> 2) >= rank ? 4 : 0);
      d.so = WS_ER + (call & 1) * (512u << 10) + kc * 65536 + jj * 2048;
      d.to = WS_W + (ord * NWB + 8) * WBLK + jj * 2048;
      d.fo = WS_WF + CNT_OFF + (8 + (jj >> 4)) * FSTRIDE + ord * 4;
      d.fw = (jj >> 4) == (rank >> 2) ? 20 : 15;
    } else if (u < U_L) {
      u32 c, na, n;
      for (;;) {
        c = (qc < 8 ? (gs & 6) + 2 + qc * 2 + (qc < 4) : jd + 1 + qc) & 7;
        na = c == 7 ? 156 : 208;
        n = qc < 7 ? na : qc < 8 ? 104 : c == 7 ? 228 : 304;
        if (u - qbase < n) break;
        qbase += n;
        qc += 1;
      }
      const u32 k = u - qbase + (qc >= 8 ? na : qc == 7 && !(gs & 1) ? 104 : 0);
      d.so = WS_Q + c * QSTRIDE + k * 2048;
      d.to = k * 128;
      d.fo = dest * 8 + c;
      d.fw = 0;
      if (k >= 512 - PU) d.rl = WS_RL + ((dest * 7 + c) * PU + k - (512 - PU)) * 2048;
    } else if (u < U_P3) {
      const u32 k = u - U_L, src = k / 104, r = k - src * 104;
      d.so = WS_L + k * 2048;
      d.to = ((r >> 3) * 160 + (r & 1) * 32) * 16384 + src * 2048 + (r & 6) * 32;
      d.fo = src & 7;
      d.fw = 0;
    } else {
      const u32 q = (u - U_P3) * NCOMMWG + idx, k = q / 7, j = q - k * 7, src = (rank - 1 - j) & 7;
      d.so = u < nu ? WS_RL + ((src * 7 + j) * PU + k) * 2048 : 0x80000000u;
      d.to = (512 - PU + k) * 128;
      d.fo = src * 8 + j;
      d.fw = 0;
    }
    return d;
  };
  u32 lim = g ? U_P1 - PU - NPULL : U_P1, tko = WS_TK + g * 128, tkb = 3 * NPULL - (call - 1) * (lim - 2 * NPULL);
  u32 u0 = vid, u1 = NPULL + vid, tk = 2 * NPULL + vid - tkb;
  u32 xb = g ? U_P1 - 384 : U_P1;
  u32 to0 = lane16, to1 = lane16 + 1024, tso = d0.to, lo1 = lane16;
  u32 ns = 1;
  const u32 cv = call * FULL;
  u32 cnt = 0, pfo = 0, pfw = 0;
  auto publish = [&]() {
    if (pfw && __builtin_amdgcn_readfirstlane(cnt) + pfw == cv)
      asm volatile("s_nop 4\n\tbuffer_store_dword %0, %1, %2, %3 offen sc0 sc1" :: "v"(cv), "v"(lane0), "s"(rs_m), "s"(pfo) : "memory");
    pfw = 0;
  };
  WAIT_VM0();
  auto iter = [&](const bool s3) {
    if (s3 && bad) v0 = v1 = u32x4{0x80808080u, 0x80808080u, 0x80808080u, 0x80808080u};
    __builtin_amdgcn_raw_buffer_store_b128(v0, s3 ? rs_o : rs_m, to0, tso, 17);
    __builtin_amdgcn_raw_buffer_store_b128(v1, s3 ? rs_o : rs_m, to1, tso, 17);
    CFENCE();
    if (s3) {
      __builtin_amdgcn_raw_buffer_store_b128(v0, rs_m, lane16, d0.rl, 17);
      __builtin_amdgcn_raw_buffer_store_b128(v1, rs_m, lane16, d0.rl + 1024, 17);
      CFENCE();
    }
    if (stage < ns) {
      stage = lds_ld(ctl + C_STAGE);
      if (stage < ns) {
        if (!s3) {
          publish();
          WAIT_VM0();
          cnt = __builtin_amdgcn_raw_ptr_buffer_atomic_add_i32(d0.fw, rs_m, lane0, d0.fo, 16);
          pfo = d0.fo - CNT_OFF; pfw = d0.fo < WS_CTL + CNT_OFF ? 0 : d0.fw;
          WAIT_VM0();
          publish();
          d0.fw = 0;
        }
        need(ns);
      }
      bad = lds_ld(ctl + C_BAD) != 0;
    }
    v0 = __builtin_amdgcn_raw_buffer_load_b128(rs_s, lo1, d1.so, 17);
    v1 = __builtin_amdgcn_raw_buffer_load_b128(rs_s, lo1, d1.so + 1024, 17);
    CFENCE();
    u32 u2 = __builtin_amdgcn_readfirstlane(tk) + tkb;
    if (u2 >= lim) {
      if (lim == nu || !lim) {
        u2 = nu;
        lim = 0;
        tko = 0x40000000u;
      } else if (lim == U_P1 || lim == U_R) {
        u2 = U_R + vid;
        lim = g ? U_L - PU - NPULL : nu;
        tko = WS_TK + 2048 + g * 128; tkb = U_R + NPULL - (call - 1) * (lim - U_R);
      } else {
        const u32 up = lim < U_P1 ? U_P1 : U_P3;
        tko = WS_TK + (lim < U_P1 ? 1024 : 3072) + idx * 128; tkb = up - (call - 1) * (NPOOL + 7 * NPW);
        lim = up + NPOOL;
      }
    }
    tk = __builtin_amdgcn_raw_ptr_buffer_atomic_add_i32(1, rs_m, lane0, tko, 16);
    CFENCE();
    if (s3 && u0 >= xb) {
      lds_add1(ctl + C_X, lane);
      xb = ~0u;
    }
    const Unit d2 = uoff(u2, s3);
    ns = u2 >= nu ? 0 : u2 >= U_L ? 7 : u2 >= uq2 ? 6 : u2 >= U_Q ? 5 : u2 >= U_R ? 3 : u2 >= U_P1 ? 4 : u2 >= U_C ? 3 : u2 >= U_B ? 2 : 1;
    tso = d1.to;
    if (u2 - U_L < U_P3 - U_L) lo1 = (lane & 3) * 256 + (lane >> 2) * 16;
    if (u1 - U_L < U_P3 - U_L) {
      const u32 gl = lds_ld(ctl + C_GS8) >> (d1.fo * 4);
      tso += (gl & 1) * (2080 * 16384) + (gl & 6) * 256;
      to0 = (lane >> 2) * 16384 + (lane & 3) * 16;
      to1 = to0 + 16 * 16384;
    } else if (s3 || u1 >= U_Q) {
      tso = d1.fo * 256;
      const u32 mul = (d1.fo & 7) == 7 ? 43691u : 32768u, w16 = (d1.fo & 7) == 7 ? 12 : 16;
#pragma unroll
      for (int e = 0; e < 2; ++e) {
        const u32 i16 = d1.to + e * 64 + lane;
        const u32 sr = (i16 * mul) >> 19, col = i16 - sr * w16;
        const u32 ra = (sr >> 6) * 96 + sr, rc = (((sr - 1664) * 43691u) >> 22) * 64 + sr - 1600;
        (e ? to1 : to0) = (sr < 1664 ? ra : rc) * 16384 + col * 16;
      }
    }
    tso = __builtin_amdgcn_readfirstlane(tso);
    CFENCE();
    if (!s3) {
      if (pfw && __builtin_amdgcn_readfirstlane(cnt) + pfw == cv)
        asm volatile("s_nop 4\n\tbuffer_store_dword %0, %1, %2, %3 offen sc0 sc1\n\ts_waitcnt vmcnt(4)" :: "v"(cv), "v"(lane0), "s"(rs_m), "s"(pfo) : "memory");
      else
        WAIT_VM(3);
      cnt = __builtin_amdgcn_raw_ptr_buffer_atomic_add_i32(d0.fw, rs_m, lane0, d0.fw ? d0.fo : 0x40000000u, 16);
      pfo = d0.fo - CNT_OFF; pfw = d0.fo < WS_CTL + CNT_OFF ? 0 : d0.fw;
      if (u1 >= xb) {
        lds_add1(ctl + C_X, lane);
        xb = ~0u;
      }
      CFENCE();
    }
    u0 = u1; u1 = u2;
    d0 = d1; d1 = d2;
  };
  while (u0 < U_Q) iter(false);
  publish();
  xb = g ? U_L - 304 : U_L;
  while (u0 < nu) iter(true);
  WAIT_VM(0);
  if (xb != ~0u) lds_add1(ctl + C_X, lane);
  if (wave == 0) STAMP(5);
}

extern "C" void kernel_launch(int rank, int nranks, ncclComm_t comm, hipStream_t stream,
                              void* const* d_in, const int* in_sizes, int n_in,
                              void* d_out, int out_size,
                              const int* in_dtypes, int out_dtype,
                              void* const* d_out_all,
                              void* const* d_ws_all, size_t ws_bytes) {
  if (nranks != NR || ws_bytes < WS_END || n_in < 2) { fprintf(stderr, "unsupported configuration\n"); return; }
  Params P = {};
  P.x = (const char*)d_in[0];
  P.w = (const char*)d_in[1];
  for (int p = 0; p < NR; ++p) P.ws[p] = (char*)d_ws_all[p];
  P.myws = P.ws[rank & 7];
  P.out = (char*)d_out;
  P.rank = rank;
  int logi[NR] = {0, 4, 1, 2, 5, 6, 3, 7}, phys[NR], grp[NR], ndev = 0, sum = 0;
  bool known = hipGetDeviceCount(&ndev) == hipSuccess && ndev == NR;
  for (int d = 0; d < NR && known; ++d) {
    char bus[32] = {0}, path[96];
    int pos = -1;
    known = hipDeviceGetPCIBusId(bus, sizeof bus, d) == hipSuccess;
    snprintf(path, sizeof path, "/sys/bus/pci/devices/%s/xgmi_physical_id", bus);
    FILE* f = known ? fopen(path, "r") : nullptr;
    known = f && fscanf(f, "%d", &pos) == 1 && pos >= 0;
    if (f) fclose(f);
    grp[d] = (pos ^ (pos >> 1)) & 1;
    sum += grp[d];
  }
  (void)hipGetLastError();
  if (known && sum == NR / 2)
    for (int d = 0, n[2] = {0, NR / 2}; d < NR; ++d) logi[d] = n[grp[d]]++;
  for (int d = 0; d < NR; ++d) phys[logi[d]] = d;
  P.peers = 0;
  for (int g = 0; g < NR; ++g) P.peers |= (u32)phys[logi[rank & 7] ^ g] << (g * 4);
  for (int g = 0; g < NR; ++g) P.dws[g] = P.ws[(P.peers >> (g * 4)) & 7];
  hipLaunchKernelGGL(fused_gemm_ar_q8, dim3(256), dim3(NTHREADS), 0, stream, P);
  HIP_CHECK(hipGetLastError());
}
```

```cpp
#include <hip/hip_runtime.h>
#include <rccl/rccl.h>
#include <cstdio>
#include <stdint.h>

#define HIP_CHECK(expr) do { hipError_t _e = (expr); if (_e != hipSuccess) { \
    fprintf(stderr, "HIP error %s at %s:%d\n", hipGetErrorString(_e), __FILE__, __LINE__); \
    return; } } while (0)

typedef unsigned u32;
typedef unsigned long long u64;
typedef u32 u32x4 __attribute__((ext_vector_type(4)));
typedef float f32x4 __attribute__((ext_vector_type(4)));
typedef short s16x4 __attribute__((ext_vector_type(4)));
typedef __bf16 bf16x8 __attribute__((ext_vector_type(8)));
#define LAS __attribute__((address_space(3)))
#define GAS __attribute__((address_space(1)))

constexpr int NR = 8;
constexpr u32 OUT_BYTES = 4096u * 16384u;
constexpr u32 X_BYTES = 4096u * 512u * 2u;
constexpr u32 W_BYTES = 512u * 16384u * 2u;

constexpr u32 NRB = 26;
constexpr u32 NCB = 8;
constexpr u32 XBLK = 20480, WBLK = 32768;
constexpr u32 NWB = NCB;
constexpr u32 NE = 16;
constexpr u32 NL = 12;
constexpr u32 XA = NL * 10, XR = 256 - XA;
constexpr u32 NWD = NE * 5 * NCB;
constexpr u32 QA = (NRB - NE) * 12, QB = (NRB - NE - 1) * 8;
constexpr u32 WS_W = 0;
constexpr u32 WS_X = 20u << 20;
constexpr u32 WS_CTL = 53u << 20;
constexpr u32 FSTRIDE = 4352;
constexpr u32 WS_XF = WS_CTL;
constexpr u32 WS_WF = WS_XF + NRB * FSTRIDE;
constexpr u32 CNT_OFF = 512u << 10;
constexpr u32 WS_C2 = WS_CTL + (160u << 10);
constexpr u32 WS_G1 = WS_C2 + 4096;
constexpr u32 WS_G2 = WS_C2 + 8192 + 256;
constexpr u32 G2S = 512;
constexpr u32 WS_TK = WS_C2 + 13312;
constexpr u32 WS_TMO = WS_C2 + 20480;
constexpr u32 WS_FL = WS_C2 + 65536;
constexpr u32 WS_FC = WS_FL + 16384;
constexpr u32 WS_CALL = WS_FC + 16384;
constexpr u32 WS_SS = 54u << 20;
constexpr u32 WS_Q = WS_SS + NWD * 1024;
constexpr u32 QSTRIDE = (QA + QB + 8) * 2048;
constexpr u32 WS_WD = WS_Q + NCB * QSTRIDE;
constexpr u32 WS_EX = 63u << 20;
constexpr u32 WS_EW = 68u << 20;
constexpr u32 WS_RL = 84u << 20;
constexpr u32 WS_SG = 95u << 20;
constexpr u32 WS_END = 101u << 20;
static_assert(64u * NWB * WBLK <= WS_X && WS_X + 64u * NRB * XBLK <= WS_CTL && WS_WF + NWB * FSTRIDE <= WS_C2 &&
              WS_CALL + 16384 <= WS_CTL + CNT_OFF && WS_C2 + CNT_OFF <= WS_SS && WS_WD + NWD * 7168 <= WS_EX && WS_SG + 8 * NWD * 1024 <= WS_END, "layout");
constexpr u32 FULL = 240;

constexpr u32 SLOT_BYTES = WBLK + XBLK;
constexpr u32 NSLOT = 3;
constexpr u32 TPITCH = 272;
constexpr u32 SNAP_OFF = SLOT_BYTES * NSLOT;
constexpr u32 CTL_OFF = SNAP_OFF + 1024;
constexpr u32 LDS_BYTES = CTL_OFF + 1024;
static_assert(LDS_BYTES <= 163840, "lds");
constexpr int C_AMAX = 0, C_GMAX = 1, C_CALL = 2, C_BAD = 3, C_STAGE = 4, C_X = 7, C_CNT = 8, C_FULL = 16, C_PROG = 24, C_QUEUE = 64, C_FLP = 208;

constexpr u32 NPW = 4;
constexpr u32 EXWAIT = 300, EXD = 60;
constexpr int PW1 = 4, PLD = 4;
constexpr int GOI = 26;
constexpr u32 QWAIT = 500, QD = 100;
constexpr u32 SSTART = 1750, SFAST = 40;
constexpr u32 CSP = 60;
constexpr u32 LP = 256 / NL;
constexpr u32 PU1 = 132;
#define AMSC " sc1"
constexpr u32 NCOMMWG = 32 - NRB;
constexpr u32 NTILE = NRB * NCB;
constexpr u32 NPULL = NCOMMWG * NPW;
constexpr u32 U_B = XA, U_C = XA + 640;
constexpr u32 NPOOL1 = 7 * PU1 / NCOMMWG;
constexpr u32 PU = QB / 6 * 6;
constexpr u32 NPOOL = 7 * PU / NCOMMWG;
constexpr u32 U_P1 = 512 + 6 * 384;
constexpr u32 U_Q = U_P1 + NPOOL1;
constexpr u32 U_F = U_Q + NWD / 2 + 4 * QA;
constexpr u32 U_M = U_F + 7 * QB;
constexpr u32 U_W = U_M + 2 * QA;
constexpr u32 U_X = U_W + NWD;
constexpr u32 U_P3 = U_X + 2 * QA + QB;
static_assert(PU1 <= XR && 3 * NPULL <= XA && WS_RL + 56 * PU * 2048 <= WS_SG && 7 * PU % NCOMMWG == 0 && 7 * PU1 % NCOMMWG == 0 && WS_TK + 4096 <= WS_TMO, "layout");
constexpr int NTHREADS = 768;
constexpr int NSTEP = 64;
constexpr u64 DEADLINE = 50000000ull;

struct Params {
  const char* x;
  const char* w;
  char* myws;
  char* out;
  u32 rank;
  u32 peers;
  u32 pad[6];
  char* dws[NR];
  char* ws[NR];
};

__device__ __forceinline__ __amdgpu_buffer_rsrc_t mkrsrc(const void* p, u32 bytes) {
  return __builtin_amdgcn_make_buffer_rsrc((void*)p, 0, bytes, 0x00020000);
}
__device__ __forceinline__ u64 now() { return __builtin_amdgcn_s_memrealtime(); }
__device__ __forceinline__ u32 ld_sys(const u32* p) { return __hip_atomic_load((GAS const u32*)p, __ATOMIC_RELAXED, __HIP_MEMORY_SCOPE_SYSTEM); }
__device__ __forceinline__ u64 ld_sys64(const u64* p) { return __hip_atomic_load((GAS const u64*)p, __ATOMIC_RELAXED, __HIP_MEMORY_SCOPE_SYSTEM); }
__device__ __forceinline__ void st_sys(u32* p, u32 v) { __hip_atomic_store((GAS u32*)p, v, __ATOMIC_RELAXED, __HIP_MEMORY_SCOPE_SYSTEM); }
__device__ __forceinline__ void st_sys64(u64* p, u64 v) { __hip_atomic_store((GAS u64*)p, v, __ATOMIC_RELAXED, __HIP_MEMORY_SCOPE_SYSTEM); }
__device__ __forceinline__ u32 lds_ld(LAS u32* p) { return __builtin_amdgcn_readfirstlane(__hip_atomic_load(p, __ATOMIC_RELAXED, __HIP_MEMORY_SCOPE_WORKGROUP)); }
__device__ __forceinline__ void lds_st(LAS u32* p, u32 v) { __hip_atomic_store(p, v, __ATOMIC_RELAXED, __HIP_MEMORY_SCOPE_WORKGROUP); }
__device__ __forceinline__ void lds_add1(LAS u32* p, u32 lane) {
  asm volatile("ds_add_u32 %0, %1" :: "v"((u32)(size_t)p), "v"(lane ? 0u : 1u) : "memory");
}
#define CFENCE() asm volatile("" ::: "memory")
#define WAIT_VM(n) asm volatile("s_waitcnt vmcnt(" #n ")" ::: "memory")
#define WAIT_VMI(n) asm volatile("s_waitcnt vmcnt(%0)" :: "i"(n) : "memory")
#define WAIT_LGKM() asm volatile("s_waitcnt lgkmcnt(0)" ::: "memory")
#define WAIT_VM0() do { CFENCE(); __builtin_amdgcn_s_waitcnt(0x0F70); WAIT_VM(0); } while (0)

__device__ __forceinline__ u32 wave_umax(u32 v) {
  v = max(v, (u32)__builtin_amdgcn_update_dpp(0, (int)v, 0x111, 0xf, 0xf, true));
  v = max(v, (u32)__builtin_amdgcn_update_dpp(0, (int)v, 0x112, 0xf, 0xf, true));
  v = max(v, (u32)__builtin_amdgcn_update_dpp(0, (int)v, 0x114, 0xf, 0xf, true));
  v = max(v, (u32)__builtin_amdgcn_update_dpp(0, (int)v, 0x118, 0xf, 0xf, true));
  v = max(v, (u32)__builtin_amdgcn_update_dpp(0, (int)v, 0x142, 0xa, 0xf, false));
  v = max(v, (u32)__builtin_amdgcn_update_dpp(0, (int)v, 0x143, 0xc, 0xf, false));
  return (u32)__builtin_amdgcn_readlane((int)v, 63);
}


#define UNI(x) __builtin_amdgcn_readfirstlane(x)
#define BARRIER() asm volatile("s_barrier" ::: "memory")

__global__ void __launch_bounds__(NTHREADS) fused_gemm_ar_q8(Params P) {
  __shared__ __attribute__((aligned(16))) char smem[LDS_BYTES];
  LAS char* L = (LAS char*)smem;
  LAS u32* ctl = (LAS u32*)(L + CTL_OFF);
  const u32 tid = threadIdx.x;
  u32 lane = tid & 63;
  const u32 wave = __builtin_amdgcn_readfirstlane(tid >> 6);
  u32 xcc;
  asm volatile("s_getreg_b32 %0, hwreg(HW_REG_XCC_ID, 0, 3)" : "=s"(xcc));
  const u32 g = xcc & 7u, idx = blockIdx.x >> 3, bid = idx * 8 + g;
  char* const myws = P.myws;
  char* const dws = P.dws[g];
  const u64 t0 = now();
  asm volatile("" :: "s"(P.x), "s"(P.w), "s"(myws), "s"(P.out), "s"(P.rank), "s"(P.peers), "s"(dws), "s"(t0));
  const u32 rank = P.rank & 7u;
  const u32 dest = (P.peers >> (g * 4)) & 7u;
  u32* const tmo = (u32*)(myws + WS_TMO);
  u32 lane16 = lane * 16;
  const u32 lx = (lane & 7) * 1024 + (lane >> 3) * 16;
  const u32 lw = (((lane >> 3) & 1) * 8 + ((lane >> 1) & 3)) * 32768 + ((lane & 1) * 4 + (lane >> 4)) * 16;
  u32 call = 0;
  bool bad = false;

  if (idx >= NCOMMWG) {
    const u32 ti = idx - NCOMMWG;
    const u32 rb = (g & 1) + (ti & ~1u);
    const u32 cb = (g >> 1) * 2 + (ti & 1);
    const u32 tile = g * NRB + ti;
    const __amdgpu_buffer_rsrc_t rs_ws = mkrsrc(myws, WS_END);
    char* flp = P.ws[lane & 7] + WS_FL + rank * 256 + g * 8;
    u32 lane0 = lane ? 0x80000000u : 0u;
    auto signal_wg = [&](u32 kind, u32 inc) {
      const u32 old = __builtin_amdgcn_raw_ptr_buffer_atomic_add_i32(inc, rs_ws, lane0, WS_FC + kind * 2048 + g * 256, 16);
      if (__builtin_amdgcn_readfirstlane(old) + inc == call * NRB && lane < 8)
        st_sys64((u64*)(flp + kind * 2048), call);
      CFENCE();
    };
    auto signal_all = [&](u32 kind, u32 inc = 1) {
      u32 old = 0;
      if (lane == 0) old = __hip_atomic_fetch_add(ctl + C_CNT + kind, 1u, __ATOMIC_RELAXED, __HIP_MEMORY_SCOPE_WORKGROUP);
      if (__builtin_amdgcn_readfirstlane(old) == 7) signal_wg(kind, inc);
    };
    {
      const bool first = ti - 16 < 8;
      if (first) {
        if (wave < 8) {
          const u32 pc = (g * 8 + ti - 16) * 8 + wave;
          const u32x4 e0 = __builtin_amdgcn_raw_buffer_load_b128(mkrsrc(P.x, X_BYTES), lx, pc * 8192, 0);
          __builtin_amdgcn_raw_buffer_store_b128(e0, rs_ws, lane16, WS_EX + pc * 1024, 17);
          asm volatile("" : "+v"(flp));
          WAIT_VM0();
        }
        BARRIER();
      }
      if (wave == 0) {
        const u32 old = __builtin_amdgcn_readfirstlane(
            __builtin_amdgcn_raw_ptr_buffer_atomic_add_i32(1, rs_ws, lane0, WS_FC + (first ? 0 : 6 * 2048) + g * 256, 16));
        call = (first ? old >> 3 : old / (NRB - 8)) + 1;
        if (first && (old & 7) == 7 && lane < 8) st_sys64((u64*)flp, call);
        if (lane < 16) ctl[lane] = lane == C_CALL ? call : 0u;
        WAIT_LGKM();
      }
      if (wave == 6 && ti == 25) __builtin_amdgcn_fence(__ATOMIC_ACQUIRE, "");
      if (wave == 8) {
        ctl[16 + lane] = ctl[80 + lane] = ctl[144 + lane] = 0u;
        ((LAS u64*)(ctl + C_FLP))[lane & 7] = (u64)flp;
        WAIT_LGKM();
      }
      BARRIER();
      call = lds_ld(ctl + C_CALL);
      if (wave >= 8) {
        const u32 lds0 = (u32)(size_t)L, cv = call * FULL;
        __builtin_amdgcn_s_setprio(1);
        if (wave < 11) {
          const u32 mb = lds0 + (wave - 8) * SLOT_BYTES;
          for (u32 n = wave - 8; n < NSTEP; n += NSLOT) {
            LAS u32* const wp = ctl + (lane < 8 ? C_PROG + lane : C_QUEUE + n);
            const u32 mn = lane < 8 ? n - 2 : 1;
            u32 v;
            while (v = __hip_atomic_load(wp, __ATOMIC_RELAXED, __HIP_MEMORY_SCOPE_WORKGROUP), __ballot((int)(v - mn) >= 0) != ~0ull)
              __builtin_amdgcn_s_sleep(1);
            u32 ord = __builtin_amdgcn_readlane(v, 8) - 1;
            const u32 sw = WS_W + ((ord & 63) * NWB + cb) * WBLK, sx = WS_X + ((ord & 63) * NRB + rb) * XBLK;
#pragma unroll
            for (u32 i = 0; i < 13; ++i) {
              if (i == 8)
                while (ord >= 64) {
                  __builtin_amdgcn_s_sleep(1);
                  ord = lds_ld(ctl + C_QUEUE + n) - 1;
                }
              asm volatile("s_mov_b32 m0, %2\n\ts_nop 0\n\t"
                           "buffer_load_dwordx4 %0, %1, %3 offen lds\n\t"
                           "buffer_load_dwordx4 %0, %1, %3 offen offset:1024 lds\n\t"
                           "buffer_load_dwordx4 %0, %1, %3 offen offset:2048 lds\n\t"
                           "buffer_load_dwordx4 %0, %1, %3 offen offset:3072 lds"
                           :: "v"(lane16), "s"(rs_ws), "s"(mb + i * 4096),
                              "s"(i >= 8 ? sx + (i - 8) * 4096 : sw + i * 4096) : "memory");
            }
            WAIT_VM(0);
            lds_st(ctl + C_FULL + (wave - 8), n + 1);
          }
          return;
        }
        LAS u32* const snap = (LAS u32*)(L + SNAP_OFF) + lane;
        u64 kx = 0;
        u32 qn = 0, sp = 0;
        while (now() - t0 < SSTART) __builtin_amdgcn_s_sleep(8);
        {
          const u32 pollo = lane < 16 ? WS_XF + rb * FSTRIDE + lane16 : WS_WF + cb * FSTRIDE + lane16 - 256;
          u64 kw = 0, taken = 0;
          auto poll = [&]() {
            const u64 kn = lane < 16 ? kx : kw;
            const bool act = lane < 32 && ((u32)(kn >> ((lane & 15) * 4)) & 15u) != 15u;
            asm volatile("s_mov_b32 m0, %2\n\ts_nop 0\n\tbuffer_load_dwordx4 %0, %1, 0 offen sc0 sc1 lds"
                         :: "v"(act ? pollo : 0x80000000u), "s"(rs_ws), "s"(lds0 + SNAP_OFF) : "memory");
          };
          poll();
          WAIT_VM(0);
          for (;;) {
            u32 fx = snap[0], fw = snap[64];
            kx |= __ballot((int)(fx - cv) >= 0);
            kw |= __ballot((int)(fw - cv) >= 0);
            const u64 av = kx & kw & ~taken;
            if (av) {
              const u32 pos = __builtin_amdgcn_mbcnt_hi((u32)(av >> 32), __builtin_amdgcn_mbcnt_lo((u32)av, qn));
              if ((av >> lane) & 1) lds_st(ctl + C_QUEUE + pos, lane + 1);
              qn += __builtin_popcountll(av);
              taken |= av;
              if (taken == ~0ull) break;
            }
            if ((qn & 63) == 63 && ((kw & ~taken) >> lane) & 1) lds_st(ctl + C_QUEUE + qn, lane + 65);
            poll();
            WAIT_VM(1);
            __builtin_amdgcn_s_sleep(4);
            if (qn - SFAST >= (u32)NSTEP - SFAST) __builtin_amdgcn_s_sleep(24);
            if ((++sp & 63) == 0 && now() - t0 > DEADLINE) {
              st_sys(tmo, call);
              lds_st(ctl + C_BAD, 1u);
              kx = kw = ~0ull;
            }
          }
          WAIT_VM0();
          WAIT_LGKM();
          return;
        }
      }
      for (u32 r = 0; r < 3; ++r) {
        const u32 slot = r * 8 + wave, e = slot * NTILE + tile;
        u32 kc, k, so, eo;
        bool isw = true;
        if (e < 896) {
          kc = 1 + (e >> 7); k = e & 127;
          isw = false;
          eo = WS_EX + kc * (512u << 10) + k * 4096;
        } else {
          k = (e + 128) & 511;
          kc = 1 + ((((k >> 6) - rank - 1) & 7) + ((e + 128) >> 9)) % 7;
          eo = WS_EW + ((k >> 6) * 8 + kc) * (8 * WBLK) + (k & 63) * 4096;
        }
        const u32 mark = e < 3968 ? 0u : 0x80808080u, em = e - (3968 + NWD / 4);
        if (e >= 3968) eo = WS_SS + (e - 3968) * 4096 + ((int)em < 0 ? 0 : QSTRIDE + em / (QA / 2) * (2 * QSTRIDE - QA * 2048));
        if (e < 3968 + NWD / 4 + 2 * QA) {
          so = isw ? (kc * 64 + (k & 1) * 32) * 32768 + (k >> 1) * 128 : k * 32768 + kc * 128;
          const __amdgpu_buffer_rsrc_t rs = mkrsrc(isw ? P.w : P.x, isw ? W_BYTES : X_BYTES);
          const u32 lo = isw ? lw : lx, s1 = isw ? 4 * 32768 : 8192, s2 = isw ? 16 * 32768 : 16384;
          if (e >= 3968) so = 0x80000000u;
          while (now() - t0 < EXWAIT + slot * EXD) __builtin_amdgcn_s_sleep(2);
          u32x4 e0 = __builtin_amdgcn_raw_buffer_load_b128(rs, lo, so, 0) | mark;
          u32x4 e1 = __builtin_amdgcn_raw_buffer_load_b128(rs, lo, so + s1, 0) | mark;
          u32x4 e2 = __builtin_amdgcn_raw_buffer_load_b128(rs, lo, so + s2, 0) | mark;
          u32x4 e3 = __builtin_amdgcn_raw_buffer_load_b128(rs, lo, so + s1 + s2, 0) | mark;
          asm volatile("" : "+v"(e0), "+v"(e1), "+v"(e2), "+v"(e3));
          __builtin_amdgcn_raw_buffer_store_b128(e0, rs_ws, lane16, eo, 17);
          __builtin_amdgcn_raw_buffer_store_b128(e1, rs_ws, lane16, eo + 1024, 17);
          __builtin_amdgcn_raw_buffer_store_b128(e2, rs_ws, lane16, eo + 2048, 17);
          __builtin_amdgcn_raw_buffer_store_b128(e3, rs_ws, lane16, eo + 3072, 17);
        }
        if (r != 1) {
          WAIT_VM0();
          signal_all(1 + (r >> 1));
        }
      }
    }
    const u32 wm = wave >> 2, wn = wave & 3;
    const u32 xa = WBLK + wm * 10240 + ((lane >> 3) & 1) * 1024 + (lane >> 5) * 256 + ((lane >> 4) & 1) * 128 + (lane & 7) * 16;
    const u32 h = lane >> 4, q = (lane >> 2) & 3, p = lane & 3;
    const u32 wa0 = wn * 8192 + (h >> 1) * 2048 + (p & 1) * 512 + (h & 1) * 128 + q * 32 + (p >> 1) * 16 + (p & 1) * 8;
    const u32 wa1 = wa0 ^ 8;
    f32x4 acc[4][5];
    auto uni = [&](u64 m) -> u64 {
      return (u64)(u32)__builtin_amdgcn_readfirstlane((u32)m) | ((u64)(u32)__builtin_amdgcn_readfirstlane((u32)(m >> 32)) << 32);
    };
#define SB() __builtin_amdgcn_sched_barrier(0)
#define MMA4(kb, i) do { \
        _Pragma("unroll") for (int n = 0; n < 4; ++n) \
          acc[n][i] = __builtin_amdgcn_mfma_f32_16x16x32_bf16(wf[kb][n], xf[kb][i], acc[n][i], 0, 0, 0); \
        SB(); } while (0)
    bf16x8 xf[2][5], wf[2][4];
    auto frags = [&](int kb, u32 slot) {
      LAS char* base = L + slot * SLOT_BYTES;
#pragma unroll
      for (int n = 0; n < 5; ++n) {
        if (n < 4) {
          LAS char* a = base + ((n & 1) ? wa1 : wa0) + kb * 4096 + (n >> 1) * 256;
          s16x4 lo = __builtin_amdgcn_ds_read_tr16_b64_v4i16((LAS s16x4*)a);
          s16x4 hi = __builtin_amdgcn_ds_read_tr16_b64_v4i16((LAS s16x4*)(a + 1024));
          wf[kb][n] = __builtin_bit_cast(bf16x8, __builtin_shufflevector(lo, hi, 0, 1, 2, 3, 4, 5, 6, 7));
        }
        if (n == 0) xf[kb][0] = *(LAS bf16x8*)(base + xa + kb * 512);
      }
#pragma unroll
      for (int n = 1; n < 5; ++n) xf[kb][n] = *(LAS bf16x8*)(base + xa + n * 2048 + kb * 512);
      SB();
    };
    u32 sp = 0, step = 0, slot = 0;
#pragma unroll
    for (int a = 0; a < 4; ++a)
#pragma unroll
      for (int b = 0; b < 5; ++b) acc[a][b] = f32x4{0.f, 0.f, 0.f, 0.f};
#pragma unroll
    for (int n = 0; n < 5; ++n) {
      xf[1][n] = __builtin_bit_cast(bf16x8, u32x4{0u, 0u, 0u, 0u});
      if (n < 4) wf[1][n] = __builtin_bit_cast(bf16x8, u32x4{0u, 0u, 0u, 0u});
    }
    bool pre = false;
    do {
      if (!pre) {
        MMA4(1, 1); MMA4(1, 2); MMA4(1, 3); MMA4(1, 4);
        while (lds_ld(ctl + C_FULL + slot) <= step) __builtin_amdgcn_s_sleep(1);
      }
      CFENCE();
      frags(0, slot);
      if (pre) { MMA4(1, 1); MMA4(1, 2); MMA4(1, 3); MMA4(1, 4); }
      frags(1, slot);
      slot = slot == NSLOT - 1 ? 0 : slot + 1;
      step += 1;
      MMA4(0, 0);
      MMA4(0, 1);
      MMA4(0, 2);
      MMA4(0, 3);
      const u32 nf = __hip_atomic_load(ctl + C_FULL + slot, __ATOMIC_RELAXED, __HIP_MEMORY_SCOPE_WORKGROUP);
      SB();
      MMA4(0, 4);
      WAIT_LGKM();
      lds_st(ctl + C_PROG + wave, step);
      SB();
      MMA4(1, 0);
      pre = __builtin_amdgcn_readfirstlane(nf) > step;
    } while (step < NSTEP);
    MMA4(1, 1); MMA4(1, 2); MMA4(1, 3); MMA4(1, 4);
    asm volatile("v_mbcnt_lo_u32_b32 %0, -1, 0\n\tv_mbcnt_hi_u32_b32 %0, -1, %0" : "=v"(lane));
    lane16 = lane * 16;
    lane0 = lane ? 0x80000000u : 0u;
    flp = (char*)((LAS u64*)(ctl + C_FLP))[lane & 7];
    asm volatile("" : "+s"(call));
    const bool odd = (lane >> 4) & 1;
    LAS char* const tw = L + (wm * 80 + (lane & 15)) * TPITCH + wn * 64 + (lane >> 4) * 16;
    LAS char* const tp = L + ((wave - 1) * 32 + (lane >> 4)) * TPITCH + (lane & 15) * 16;
    const u32 piece = (rb * 5 + wave - 1) * NCB + cb;
    {
      const u32 row0 = rb * 160 + wm * 80;
      float m = 0.f;
#pragma unroll
      for (int b = 0; b < 5; ++b) {
        float mb = 0.f;
#pragma unroll
        for (int a = 0; a < 4; ++a)
#pragma unroll
          for (int e = 0; e < 4; ++e) mb = fmaxf(mb, fabsf(acc[a][b][e]));
        if (row0 + b * 16 < 4096) m = fmaxf(m, mb);
      }
      const u32 mb = wave_umax(__builtin_bit_cast(u32, m));
      if (lane == 0) __hip_atomic_fetch_max(ctl + C_AMAX, mb, __ATOMIC_RELAXED, __HIP_MEMORY_SCOPE_WORKGROUP);
      WAIT_LGKM();
      BARRIER();
      if (wave == 0) {
        const u32 wmax = lds_ld(ctl + C_AMAX);
        char* const g1 = myws + WS_G1 + g * 256;
        u64 ret = 1;
        if (lane == 0)
          asm volatile("s_nop 4\n\tglobal_atomic_umax %1, %2, %3 offset:4" AMSC "\n\tglobal_atomic_add_x2 %0, %1, %0, %3 sc0" AMSC "\n\ts_waitcnt vmcnt(0)"
                       : "+v"(ret) : "v"(0u), "v"(wmax), "s"(g1) : "memory");
        ret = uni(ret);
        if ((u32)ret == NRB - 1) {
          st_sys64((u64*)(flp + (int)((lane >> 3) * G2S + WS_G2 - WS_FL - rank * 192)), ((u64)call << 32) | (u32)(ret >> 32));
          if (lane == 0) asm volatile("s_nop 4\n\tglobal_store_dwordx2 %0, %1, %2" AMSC :: "v"(0u), "v"(0ull), "s"(g1) : "memory");
        }
      }
      if (rb < NE) {
#pragma unroll
        for (int i = 0; i < 5; ++i) {
          u32 d[4];
#pragma unroll
          for (int n = 0; n < 4; ++n) {
            const u32x4 a = __builtin_bit_cast(u32x4, acc[n][i]);
            d[n] = __builtin_amdgcn_perm(a[1], a[0], 0x0c0c0703u) | __builtin_amdgcn_perm(a[3], a[2], 0x07030c0cu);
          }
          *(LAS u32x4*)(tw + i * 16 * TPITCH) = u32x4{odd ? d[1] : d[0], odd ? d[0] : d[1], odd ? d[3] : d[2], odd ? d[2] : d[3]};
        }
        WAIT_LGKM();
        BARRIER();
        if (wave - 1 < 5) {
          u32x4 sg = {0u, 0u, 0u, 0u};
#pragma unroll
          for (int i = 0; i < 8; ++i) sg |= (*(LAS u32x4*)(tp + i * 4 * TPITCH) & 0x80808080u) >> i;
          __builtin_amdgcn_raw_buffer_store_b128(sg, rs_ws, lane16, WS_SS + piece * 1024, 17);
          asm volatile("s_nop 1" :: "v"(sg) : "memory");
        }
      }
      if (wave == 0) {
        const u64* g2 = (const u64*)(myws + WS_G2 + g * G2S) + lane;
        u64 val;
        for (;;) {
          val = ld_sys64(g2);
          if (__all((u32)(val >> 32) == call)) break;
          if ((++sp & 63) == 0 && (ld_sys(tmo) == call || now() - t0 > DEADLINE)) { st_sys(tmo, call); bad = true; break; }
        }
        const u32 gmax = wave_umax((u32)val);
        if (lane == 0) {
          lds_st(ctl + C_GMAX, gmax);
          if (bad) lds_st(ctl + C_BAD, 1u);
        }
        WAIT_LGKM();
      }
    }
    BARRIER();
    const float amax = __builtin_bit_cast(float, lds_ld(ctl + C_GMAX));
    const float inv = amax > 0.f ? 1.0f / (amax / 127.0f) : 0.f;
    bad = lds_ld(ctl + C_BAD) != 0;
    u32x4 qv[8];
    const u32 qa = WS_Q + cb * QSTRIDE + (rb - NE) * 96 * 256, qb = WS_Q + cb * QSTRIDE + QA * 2048 + (rb - NE) * 64 * 256;
    const u32 q1 = (cb & 1) && rb >= NE ? qa + wm * 80 * 256 + wn * 64 : 0x80000000u;
#pragma unroll
    for (int i = 0; i < 5; ++i) {
      u32 d[4];
#pragma unroll
      for (int n = 0; n < 4; ++n) {
        u32 b0 = __builtin_bit_cast(u32, __builtin_fmaf(acc[n][i][0], inv, 12582912.0f));
        u32 b1 = __builtin_bit_cast(u32, __builtin_fmaf(acc[n][i][1], inv, 12582912.0f));
        u32 b2 = __builtin_bit_cast(u32, __builtin_fmaf(acc[n][i][2], inv, 12582912.0f));
        u32 b3 = __builtin_bit_cast(u32, __builtin_fmaf(acc[n][i][3], inv, 12582912.0f));
        u32 lo = __builtin_amdgcn_perm(b1, b0, 0x0c0c0400u);
        u32 hi = __builtin_amdgcn_perm(b3, b2, 0x04000c0cu);
        d[n] = bad ? 0x80808080u : (lo | hi);
      }
      qv[i] = u32x4{odd ? d[1] : d[0], odd ? d[0] : d[1], odd ? d[3] : d[2], odd ? d[2] : d[3]};
      __builtin_amdgcn_raw_buffer_store_b128(qv[i], rs_ws, (lane & 15) * 256 + (lane >> 4) * 16, i && wm ? 0x80000000u : q1 + i * 16 * 256, 17);
      SB();
    }
    const u64 tq = now();
#pragma unroll
    for (int i = 0; i < 5; ++i) *(LAS u32x4*)(tw + i * 16 * TPITCH) = qv[i];
    WAIT_LGKM();
    BARRIER();
    if (rb < NE) {
      if (wave - 1 < 5) {
#pragma unroll
        for (int i = 0; i < 8; ++i) qv[i] = *(LAS u32x4*)(tp + i * 4 * TPITCH);
#pragma unroll
        for (int i = 0; i < 7; ++i) qv[i] = (qv[i] & 0x7f7f7f7fu) | ((qv[7] << (7 - i)) & 0x80808080u);
        while (now() - tq < QWAIT + wave * QD) __builtin_amdgcn_s_sleep(2);
#pragma unroll
        for (int i = 0; i < 7; ++i) __builtin_amdgcn_raw_buffer_store_b128(qv[i], rs_ws, lane16, WS_WD + piece * 7168 + i * 1024, 17);
      }
    } else {
      LAS char* tr = L + (wave * 4 + (lane >> 4)) * TPITCH + (lane & 15) * 16;
#pragma unroll
      for (int i = 0; i < 5; ++i) qv[i] = *(LAS u32x4*)(tr + i * 32 * TPITCH);
      while (now() - tq < QWAIT + wave * QD) __builtin_amdgcn_s_sleep(2);
      if (!(cb & 1)) {
#pragma unroll
        for (int i = 0; i < 3; ++i) __builtin_amdgcn_raw_buffer_store_b128(qv[i], rs_ws, lane16, qa + wave * 1024 + i * 8192, 17);
      }
#pragma unroll
      for (int i = 3; i < 5; ++i) __builtin_amdgcn_raw_buffer_store_b128(qv[i], rs_ws, lane16, qb + wave * 1024 + (i - 3) * 8192, 17);
    }
    WAIT_VM0();
    signal_all(4);
    WAIT_VM(0);
    return;
  }

  const u32 vid = idx * NPW + wave;
  const __amdgpu_buffer_rsrc_t rs_s = mkrsrc(dws, WS_END);
  const u32 lane0 = lane ? 0x80000000u : 0u;
  static_assert(NPW == 4 && NCOMMWG == 6, "wave roles");
  if (wave >= 8) return;
  if (wave >= NPW && wave < 7) {
    const u32 ph = (wave - NPW) * NCOMMWG + idx;
    const __amdgpu_buffer_rsrc_t rs_w = mkrsrc(P.w, W_BYTES);
    const u32 so = ph < 16 ? dest * 4096 + ph * 256 : 0x80000000u;
    u32 kc2 = 1 + (((dest - rank - 1) & 7) + 1) % 7;
    u32 b0 = WS_W + rank * NWB * WBLK + ph * 16384, b1 = b0 + kc2 * (8 * NWB * WBLK), n = 32;
    char* tb = dws;
    u32x4 pv[32];
    const u32 mo = WS_CALL + idx * 2048 + 128;
    u32 goo = wave == NPW + 1 ? mo + rank * 256 : 0x40000000u;
    auto go = [&]() {
      asm volatile("buffer_atomic_add %0, %1, %2, %3 offen sc1" :: "v"(1u), "v"(lane0), "s"(rs_s), "s"(goo) : "memory");
      goo = 0x40000000u;
    };
#pragma unroll
    for (int i = 0; i < 16; ++i) {
      pv[i] = __builtin_amdgcn_raw_buffer_load_b128(rs_w, lw, so + (i >> 3) * 128 + (((i >> 1) & 3) * 16 + (i & 1) * 4) * 32768, 0);
      if (i == PLD - 1) BARRIER();
    }
    CFENCE();
    if (ph >= 16) return;
#pragma unroll
    for (int i = 0; i < 32; ++i) {
      if (i >= PW1) WAIT_VMI(PW1 - 1 + (i <= 16 ? PW1 : i < 16 + PW1 ? 16 + PW1 - i : 0));
      if (i && !(i & 1) && n == 32 && lds_ld(ctl + C_STAGE)) {
        if (i <= GOI) go();
        n = i;
        tb = myws;
        b0 = WS_EW + dest * (64 * WBLK) + ph * 16384;
        b1 = b0 + kc2 * (8 * WBLK);
      }
      __builtin_amdgcn_raw_buffer_store_b128(pv[i], mkrsrc(tb, WS_END), lane16, (i < 16 ? b0 : b1) + (i & 15) * 1024, 17);
      asm volatile("s_nop 1" ::: "memory");
      if (i < 16) pv[16 + i] = __builtin_amdgcn_raw_buffer_load_b128(rs_w, lw, so + kc2 * (64 * 32768) + (i >> 3) * 128 + (((i >> 1) & 3) * 16 + (i & 1) * 4) * 32768, 0);
      CFENCE();
      if (i == GOI) go();
    }
    while (!(call = lds_ld(ctl + C_CALL))) __builtin_amdgcn_s_sleep(1);
    WAIT_VM0();
    __builtin_amdgcn_raw_buffer_store_b32(call * 64 + n, rs_s, lane0, mo + rank * 256 + wave * 8, 17);
    __builtin_amdgcn_raw_ptr_buffer_atomic_add_i32((lane ? max(n, 16u) - 16 : min(n, 16u)) * 15 / 2, rs_s, lane < 2 ? lane * kc2 * 32 : 0x80000000u,
                                                   WS_WF + (ph >> 1) * FSTRIDE + rank * 4, 16);
    for (;;) {
      n = __builtin_amdgcn_readfirstlane(ld_sys((const u32*)(myws + mo + dest * 256 + wave * 8))) - call * 64;
      if ((int)n >= 0) break;
      if (now() - t0 > DEADLINE) { st_sys(tmo, call); return; }
      __builtin_amdgcn_s_sleep(16);
    }
    const __amdgpu_buffer_rsrc_t rs_m = mkrsrc(myws, WS_END);
    if (n < 32) {
      kc2 = 1 + (((rank - dest - 1) & 7) + 1) % 7;
      for (u32 i = n; i < 32; i += 2) {
        const u32 kc = i < 16 ? 0 : kc2, o = ph * 16384 + (i & 15) * 1024;
        const u32x4 e0 = __builtin_amdgcn_raw_buffer_load_b128(rs_s, lane16, WS_EW + (rank * 8 + kc) * (8 * WBLK) + o, 17);
        const u32x4 e1 = __builtin_amdgcn_raw_buffer_load_b128(rs_s, lane16, WS_EW + (rank * 8 + kc) * (8 * WBLK) + o + 1024, 17);
        __builtin_amdgcn_raw_buffer_store_b128(e0, rs_m, lane16, WS_W + (kc * 8 + dest) * NWB * WBLK + o, 17);
        __builtin_amdgcn_raw_buffer_store_b128(e1, rs_m, lane16, WS_W + (kc * 8 + dest) * NWB * WBLK + o + 1024, 17);
        asm volatile("s_nop 1" ::: "memory");
      }
      WAIT_VM0();
      __builtin_amdgcn_raw_ptr_buffer_atomic_add_i32((lane ? 32 - max(n, 16u) : 16 - min(n, 16u)) * 15 / 2, rs_m, lane < 2 ? lane * kc2 * 32 : 0x80000000u,
                                                     WS_WF + (ph >> 1) * FSTRIDE + dest * 4, 16);
    }
    return;
  }
  if (wave == 7) {
    ctl[lane] = 0u;
    WAIT_LGKM();
  }
  BARRIER();
  if (wave == 7) {
    u32* const cptr = (u32*)(myws + WS_CALL + bid * 256);
    const u64* const fp0 = lane & 8 ? (const u64*)(myws + WS_CALL + (idx * 8 + dest) * 256 + 128)
                                             : (const u64*)(myws + WS_FL + dest * 256) + (lane & 7);
    u64 f = ld_sys64(fp0);
    call = __builtin_amdgcn_readfirstlane(ld_sys(cptr)) + 1;
    if (lane == 0) { lds_st(ctl + C_CALL, call); st_sys(cptr, call); }
    u32 sp = 0;
    auto expired = [&]() -> bool {
      if ((++sp & 63) == 0 && (ld_sys(tmo) == call || now() - t0 > DEADLINE)) {
        st_sys(tmo, call);
        if (lane == 0) { lds_st(ctl + C_BAD, 1u); lds_st(ctl + C_STAGE, 8u); }
        bad = true;
      }
      return bad;
    };
    u32 nx = 0;
    auto relay = [&]() {
      if (nx < 2 && lds_ld(ctl + C_X) >= NPW * (nx + 1)) {
        const u32 old = __builtin_amdgcn_raw_ptr_buffer_atomic_add_i32(1, mkrsrc(myws, WS_SS), lane0, WS_FC + 7 * 2048 + g * 256 + nx * 128, 16);
        if (__builtin_amdgcn_readfirstlane(old) + 1 == call * NCOMMWG && lane < 8)
          st_sys64((u64*)(P.ws[lane] + WS_FL + (6 + nx) * 2048 + rank * 256 + g * 8), call);
        nx += 1;
      }
    };
    auto watch = [&](u32 kind, u32 row, u32 st) {
      const u64* fp = kind ? (const u64*)(myws + WS_FL + kind * 2048 + row * 256) + (st == 7 ? g : lane & 7) : fp0;
      if (kind) f = ld_sys64(fp);
      while (!bad) {
        const u64 ok = __ballot((int)((u32)f - call) >= 0);
        if (ok == ~0ull) break;
        __builtin_amdgcn_s_sleep(1);
        if (kind > 2) relay();
        expired();
        f = ld_sys64(fp);
      }
      if (st == 6) {
        sp = 63;
        expired();
      }
      if (lane == 0 && !bad) lds_st(ctl + C_STAGE, st);
      CFENCE();
    };
    watch(0, dest, 1);
    watch(1, dest, 2);
    watch(2, dest, 3);
    if (g) watch(6, dest, 4);
    watch(4, dest, 6);
    watch(7, rank, 7);
    if (g) watch(7, dest, 8);
    while (nx < 2 && !expired()) {
      __builtin_amdgcn_s_sleep(8);
      relay();
    }
    return;
  }
  u32 stage = 0;
  auto need = [&](u32 s) {
    while (stage < s) {
      stage = lds_ld(ctl + C_STAGE);
      if (stage < s) __builtin_amdgcn_s_sleep(1);
    }
    CFENCE();
  };
  const __amdgpu_buffer_rsrc_t rs_m = mkrsrc(myws, WS_END);
  const __amdgpu_buffer_rsrc_t rs_o = mkrsrc(P.out, OUT_BYTES);
  const u32 jd = (rank - dest - 1) & 7;
  struct Unit { u32 so, to, fo, fw, rl; };
  auto xunit = [&](u32 src, u32 j, u32 kc, u32 k0, u32 n, u32 k) -> Unit {
    u32 kk = k + j % 7 * (n + 9) / 70 * 10;
    kk += kk >= n ? k0 - n : k0;
    const u32 r = kk / 10, ord = kc * 8 + src;
    return Unit{WS_EX + kc * (512u << 10) + kk * 2048, WS_X + ord * NRB * XBLK + kk * 2048, WS_XF + r * FSTRIDE + ord * 4,
                (r == NRB - 1 ? 40u : 24u) + ord * (CSP << 8), 0x80000000u};
  };
  Unit d0 = xunit(dest, jd, 0, 0, XA, vid), d1 = xunit(dest, jd, 0, 0, XA, NPULL + vid);
  asm volatile("" : "+s"(d0.so), "+s"(d0.to), "+s"(d0.fo), "+s"(d0.fw), "+s"(d1.so), "+s"(d1.to), "+s"(d1.fo), "+s"(d1.fw));
  need(1);
  u32x4 v0 = __builtin_amdgcn_raw_buffer_load_b128(rs_s, lane16, d0.so, 17);
  u32x4 v1 = __builtin_amdgcn_raw_buffer_load_b128(rs_s, lane16, d0.so + 1024, 17);
  CFENCE();
  call = lds_ld(ctl + C_CALL);
  const u32 nu = g ? U_P3 + NPOOL : U_P3;
  u32 qc = 0, qbase = U_Q;
  auto uoff = [&](u32 u, bool s3) -> Unit {
    Unit d;
    d.rl = 0x80000000u;
    if (!s3 && u < U_P1) {
      const u32 v = u + (u < XA ? 128 : 512 - XA), pos = min(v / 384, 7u), k = v - pos * 384;
      const u32 kc = pos ? 1 + (jd + pos) % 7 : 0, ord = kc * 8 + dest;
      if (k < 128) {
        d.so = WS_EW + (rank * 8 + kc) * (8 * WBLK) + k * 2048;
        d.to = WS_W + ord * NWB * WBLK + k * 2048;
        d.fo = WS_WF + (k >> 4) * FSTRIDE + ord * 4;
        d.fw = 15 + ord * (CSP << 8);
      } else {
        const u32 b = min((k - 128) / LP, NL - 1u), o = k - 128 - b * LP;
        const bool rest = pos == 7 && o >= 10;
        d = xunit(dest, pos == 7 && !rest ? 0 : jd, rest && k < 384 ? 0 : kc, rest ? XA : 0, rest ? XR : pos ? 256 : XA,
                  pos < 7 ? k - 128 : k >= 384 ? k - 384 : rest ? k - 138 - b * 10 : b * 10 + o);
      }
    } else if (!s3 && u < U_Q) {
      const u32 q = (u - U_P1) * NCOMMWG + idx, k = q / 7, j = q - k * 7;
      d = xunit((rank - 1 - j) & 7, j, 1 + j, XA, XR, XR - PU1 + k);
      d.so = d.to;
    } else {
      u32 src = dest, c, k;
      bool b;
      if (u < U_P3) {
        for (;;) {
          b = qc - 5 < 7 || qc == 17;
          const u32 n = qc == 0 ? NWD / 2 : qc == 14 ? NWD : b ? QB : QA;
          if (u - qbase < n) break;
          qbase += n;
          qc += 1;
        }
        k = u - qbase;
        c = (b ? jd + min(qc, 12u) - 4 : (jd + qc - (qc > 14)) * 2 + (qc < 5)) & 7;
        d.so = WS_Q + c * QSTRIDE + (b ? QA + k : k) * 2048;
        if (b && c < 7 && k >= QB - PU) d.rl = WS_RL + ((dest * 7 + c) * PU + k - (QB - PU)) * 2048;
      } else {
        const u32 q = (u - U_P3) * NCOMMWG + idx;
        k = q / 7; c = q - k * 7; src = (rank - 1 - c) & 7;
        b = true;
        d.so = u < nu ? WS_RL + ((src * 7 + c) * PU + k) * 2048 : 0x80000000u;
        k += QB - PU;
      }
      u32 row = NE * 160 + k * 8 + (b ? (k >> 3) * 96 + 96 : k / 12 * 64);
      if (u - U_W < NWD) {
        k += jd * (NWD / 56 * 8);
        if (k >= NWD) k -= NWD;
        row = (k >> 3) * 32; c = k & 7;
        d.so = WS_WD + k * 7168;
        d.rl = WS_SG + (dest * NWD + k) * 1024;
      }
      d.to = row * 16384 + src * 2048 + c * 256;
      if (u - U_Q < NWD / 2) {
        d.so = WS_SS + k * 2048;
        d.rl = WS_SG + dest * NWD * 1024 + k * 2048;
        d.to = 0x80000000u;
      }
      d.fo = d.fw = 0;
    }
    return d;
  };
  u32 lim = g ? U_P1 - PU1 - NPULL : U_P1, tko = WS_TK + g * 128, tkb = 3 * NPULL - (call - 1) * (lim - 2 * NPULL);
  u32 u0 = vid, u1 = NPULL + vid, tk = 2 * NPULL + vid - tkb;
  u32 xb = g ? U_P1 - 384 - XR : U_P1;
  const u32 ov = (lane >> 4) * 16384 + (lane & 15) * 16;
  u32 ns = 1;
  const u32 cv = call * FULL;
  u32 cnt = 0, pfo = 0, pfw = 0;
  auto publish = [&]() {
    if (pfw && __builtin_amdgcn_readfirstlane(cnt) + pfw == cv)
      asm volatile("s_nop 4\n\tbuffer_store_dword %0, %1, %2, %3 offen sc0 sc1" :: "v"(cv), "v"(lane0), "s"(rs_m), "s"(__builtin_amdgcn_readfirstlane(pfo)) : "memory");
    pfw = 0;
  };
  auto arrive = [&]() {
    pfo = d0.fo; pfw = d0.fw & 255;
    cnt = __builtin_amdgcn_raw_ptr_buffer_atomic_add_i32(pfw, rs_m, lane0, pfw ? pfo + CNT_OFF + (d0.fw >> 8) : 0x40000000u, 16);
  };
  WAIT_VM0();
  auto gate = [&](const bool s3) {
    if (stage < ns) {
      stage = lds_ld(ctl + C_STAGE);
      if (stage < ns) {
        publish();
        WAIT_VM0();
        if (!s3) {
          arrive();
          WAIT_VM0();
          publish();
          d0.fw = 0;
        }
        if (u1 >= xb) {
          lds_add1(ctl + C_X, lane);
          xb = ~0u;
        }
        need(ns);
      }
      bad = lds_ld(ctl + C_BAD) != 0;
    }
  };
  auto next = [&](const bool s3) {
    u32 u2 = __builtin_amdgcn_readfirstlane(tk) + tkb;
    if (u2 >= lim) {
      if (lim == nu || !lim) {
        u2 = nu;
        lim = 0;
        tko = 0x40000000u;
      } else if (lim == U_P1 || lim == U_Q) {
        u2 = U_Q + vid;
        lim = g ? U_P3 - PU - NPULL : nu;
        tko = WS_TK + 2048 + g * 128; tkb = U_Q + NPULL - (call - 1) * (lim - U_Q);
      } else {
        const u32 up = lim < U_P1 ? U_P1 : U_P3, np = lim < U_P1 ? NPOOL1 : NPOOL;
        tko = WS_TK + (lim < U_P1 ? 1024 : 3072) + idx * 128; tkb = up - (call - 1) * (np + 7 * NPW);
        lim = up + np;
      }
    }
    tk = __builtin_amdgcn_raw_ptr_buffer_atomic_add_i32(1, rs_m, lane0, UNI(tko), 16);
    CFENCE();
    if (s3 && pfw) publish();
    if (s3 && u0 >= xb) {
      lds_add1(ctl + C_X, lane);
      xb = ~0u;
    }
    const Unit d2 = uoff(u2, s3);
    ns = u2 >= nu ? 0 : u2 >= U_P3 ? 8 : u2 >= U_W ? 7 : u2 >= (g ? U_F : U_Q) ? 6 : u2 >= U_Q ? 3 : u2 >= U_P1 ? 4 : u2 >= U_C ? 3 : u2 >= U_B ? 2 : 1;
    CFENCE();
    if (!s3) {
      if (pfw && __builtin_amdgcn_readfirstlane(cnt) + pfw == cv)
        asm volatile("s_nop 4\n\tbuffer_store_dword %0, %1, %2, %3 offen sc0 sc1\n\ts_waitcnt vmcnt(4)" :: "v"(cv), "v"(lane0), "s"(rs_m), "s"(pfo) : "memory");
      else
        WAIT_VM(3);
      arrive();
      if (u1 >= xb) {
        lds_add1(ctl + C_X, lane);
        xb = ~0u;
      }
      CFENCE();
    }
    u0 = u1; u1 = u2;
    d0 = d1; d1 = d2;
  };
  auto iter = [&](const bool s3) {
    if (s3 && u0 < U_F)
      for (u32 st = 0; st < 6;) {
        bool inv = false;
#pragma unroll
        for (int e = 0; e < 4; ++e) inv |= v0[e] == 0x80808080u || v1[e] == 0x80808080u;
        if (!__ballot(inv)) break;
        st = lds_ld(ctl + C_STAGE);
        bad = lds_ld(ctl + C_BAD) != 0;
        CFENCE();
        v0 = __builtin_amdgcn_raw_buffer_load_b128(rs_s, lane16, d0.so, 17);
        v1 = __builtin_amdgcn_raw_buffer_load_b128(rs_s, lane16, d0.so + 1024, 17);
        CFENCE();
        publish();
        WAIT_VM0();
      }
    if (s3 && bad) v0 = v1 = u32x4{0x80808080u, 0x80808080u, 0x80808080u, 0x80808080u};
    __builtin_amdgcn_raw_buffer_store_b128(v0, s3 ? rs_o : rs_m, s3 ? ov : lane16, d0.to, 17);
    __builtin_amdgcn_raw_buffer_store_b128(v1, s3 ? rs_o : rs_m, s3 ? ov : lane16, d0.to + (s3 ? 65536 : 1024), 17);
    CFENCE();
    if (s3) {
      __builtin_amdgcn_raw_buffer_store_b128(v0, rs_m, lane16, UNI(d0.rl), 17);
      __builtin_amdgcn_raw_buffer_store_b128(v1, rs_m, lane16, UNI(d0.rl) + 1024, 17);
      CFENCE();
    }
    gate(s3);
    v0 = __builtin_amdgcn_raw_buffer_load_b128(rs_s, lane16, d1.so, 17);
    v1 = __builtin_amdgcn_raw_buffer_load_b128(rs_s, lane16, d1.so + 1024, 17);
    CFENCE();
    next(s3);
  };
  while (u0 < U_Q) iter(false);
  xb = U_M;
  for (u32 ue = U_W;; ue = nu) {
    while (u0 < ue) iter(true);
    if (ue == nu) break;
    u32x4 sg = __builtin_amdgcn_raw_buffer_load_b128(rs_m, lane16, UNI(d0.rl), 17), sn;
    WAIT_VM0();
    while (u0 < U_X) {
      u32x4 pk[7], m7 = {0u, 0u, 0u, 0u};
      pk[0] = v0; pk[1] = v1;
      auto put = [&](u32x4 m, int i) {
        m |= (m + 0x7f7f7f7fu) & (sg << i) & 0x80808080u;
        if (bad) m = u32x4{0x80808080u, 0x80808080u, 0x80808080u, 0x80808080u};
        __builtin_amdgcn_raw_buffer_store_b128(m, rs_o, ov, d0.to + i * 65536, 17);
        asm volatile("s_nop 1" :: "v"(m) : "memory");
      };
#pragma unroll
      for (int i = 0; i < 7; ++i) {
        asm volatile("" :: "v"(pk[i]) : "memory");
        if (i < 5) pk[i + 2] = __builtin_amdgcn_raw_buffer_load_b128(rs_s, lane16, d0.so + (i + 2) * 1024, 17);
        if (i == 5) v0 = __builtin_amdgcn_raw_buffer_load_b128(rs_s, lane16, d1.so, 17);
        if (i == 6) v1 = __builtin_amdgcn_raw_buffer_load_b128(rs_s, lane16, d1.so + 1024, 17);
        if (i == 3) sn = __builtin_amdgcn_raw_buffer_load_b128(rs_m, lane16, UNI(d1.rl), 17);
        CFENCE();
        __builtin_amdgcn_sched_barrier(0);
        m7 |= (pk[i] & 0x80808080u) >> (7 - i);
        put(pk[i] & 0x7f7f7f7fu, i);
        if (i == 4) gate(true);
      }
      put(m7, 7);
      sg = sn;
      next(true);
    }
  }
  if (xb != ~0u) {
    WAIT_VM(0);
    lds_add1(ctl + C_X, lane);
  }
}

extern "C" void kernel_launch(int rank, int nranks, ncclComm_t comm, hipStream_t stream,
                              void* const* d_in, const int* in_sizes, int n_in,
                              void* d_out, int out_size,
                              const int* in_dtypes, int out_dtype,
                              void* const* d_out_all,
                              void* const* d_ws_all, size_t ws_bytes) {
  if (nranks != NR || ws_bytes < WS_END || n_in < 2) { fprintf(stderr, "unsupported configuration\n"); return; }
  Params P = {};
  P.x = (const char*)d_in[0];
  P.w = (const char*)d_in[1];
  for (int p = 0; p < NR; ++p) P.ws[p] = (char*)d_ws_all[p];
  P.myws = P.ws[rank & 7];
  P.out = (char*)d_out;
  P.rank = rank;
  int logi[NR] = {0, 4, 1, 2, 5, 6, 3, 7}, phys[NR], grp[NR], ndev = 0, sum = 0;
  bool known = hipGetDeviceCount(&ndev) == hipSuccess && ndev == NR;
  for (int d = 0; d < NR && known; ++d) {
    char bus[32] = {0}, path[96];
    int pos = -1;
    known = hipDeviceGetPCIBusId(bus, sizeof bus, d) == hipSuccess;
    snprintf(path, sizeof path, "/sys/bus/pci/devices/%s/xgmi_physical_id", bus);
    FILE* f = known ? fopen(path, "r") : nullptr;
    known = f && fscanf(f, "%d", &pos) == 1 && pos >= 0;
    if (f) fclose(f);
    grp[d] = (pos ^ (pos >> 1)) & 1;
    sum += grp[d];
  }
  (void)hipGetLastError();
  if (known && sum == NR / 2)
    for (int d = 0, n[2] = {0, NR / 2}; d < NR; ++d) logi[d] = n[grp[d]]++;
  for (int d = 0; d < NR; ++d) phys[logi[d]] = d;
  P.peers = 0;
  for (int g = 0; g < NR; ++g) P.peers |= (u32)phys[logi[rank & 7] ^ g] << (g * 4);
  for (int g = 0; g < NR; ++g) P.dws[g] = P.ws[(P.peers >> (g * 4)) & 7];
  hipLaunchKernelGGL(fused_gemm_ar_q8, dim3(256), dim3(NTHREADS), 0, stream, P);
  HIP_CHECK(hipGetLastError());
}
```
